```python
import numpy as np

kernel_name = "gpu_gemm_rs_m4096_k4096_n16384_f32_gelu"


ABSMAX_RTOL = 0.001

COMPUTE_DTYPE = "float32"
EPILOGUE = "gelu"
IN_QUANT = None
OUT_QUANT = None
IN_DTYPES = {"x": "float32", "w_mat": "float32"}
WORLD_SIZE = 8

def setup_inputs():

    x = np.random.randn(4096, 4096).astype(np.float32)
    w_mat = (np.random.randn(4096, 16384) / np.sqrt(4096)).astype(np.float32)
    return {"x": x, "w_mat": w_mat}

def _epilogue(y):
    c = 0.7978845608028654
    return (0.5 * y * (1.0 + np.tanh(c * (y + 0.044715 * y ** 3)))).astype(np.float32)

def reference(x, w_mat):
    y = (x @ w_mat).astype(np.float32)
    return _epilogue(y)
```

```cpp
#include <hip/hip_runtime.h>
#include <rccl/rccl.h>
#include <cstdio>

#define HIP_CHECK(expr) do { hipError_t _e = (expr); if (_e != hipSuccess) { \
    fprintf(stderr, "HIP error %s at %s:%d\n", hipGetErrorString(_e), __FILE__, __LINE__); \
    return; } } while (0)

typedef float f32x4 __attribute__((ext_vector_type(4)));
typedef unsigned u32x4 __attribute__((ext_vector_type(4)));
typedef _Float16 h16x8 __attribute__((ext_vector_type(8)));
#define LAS __attribute__((address_space(3)))
#define GAS __attribute__((address_space(1)))

constexpr int NR = 8;
constexpr int GN = 16384;
constexpr int KSH = 512;
constexpr int NBLK = 256;
constexpr int NTHR = 512;
constexpr int NSTEP = 64;
constexpr unsigned A_STEP = 32768, B_STEP = 16384, STAGE = A_STEP + B_STEP;
constexpr size_t WS_CTR = 0;
constexpr size_t WS_FLAG = 64 << 10;
constexpr size_t WS_DONE = 128 << 10;
constexpr size_t WS_TMO = 192 << 10;
constexpr size_t WS_DIAG = 256 << 10;
constexpr size_t WS_STG = 1 << 20;
constexpr size_t STG_A = 8ull * NSTEP * A_STEP;
constexpr size_t STG_B = 32ull * NSTEP * B_STEP;
constexpr size_t STG_BYTES = STG_A + STG_B;
constexpr size_t WS_NEED = WS_STG + 2 * STG_BYTES;
constexpr unsigned long long DEADLINE_TICKS = 50000000ull;

#define PUSH_WIN 1
#define OUT_WIN 1
#define TAU_IN 112
#define TAU_OUT 112
#define T_FIRST 250
#define T_OUT_ADJ 0
#define OUT_SLACK 0
#define GLDS_AUX 17

struct Params {
    const float* x;
    const float* w;
    float* out[NR];
    char* ws[NR];
    int rank;
    int pad;
};

#define STR2(x) #x
#define STR(x) STR2(x)
#define WAIT_V(n) asm volatile("s_waitcnt vmcnt(" STR(n) ")" ::: "memory")
#define WAIT_L0() do { asm volatile("s_waitcnt lgkmcnt(0)" ::: "memory"); __builtin_amdgcn_s_waitcnt(0xC07F); } while (0)
#define DRAIN_V() do { asm volatile("s_waitcnt vmcnt(0)" ::: "memory"); __builtin_amdgcn_s_waitcnt(0x0F70); } while (0)

template <int N> __device__ __forceinline__ void wait_vm() { asm volatile("s_waitcnt vmcnt(%0)" ::"i"(N) : "memory"); }

__device__ __forceinline__ void st16_sys(const void* base, unsigned voff, u32x4 v) {
    asm volatile("global_store_dwordx4 %0, %1, %2 sc0 sc1\n\ts_nop 1" ::"v"(voff), "v"(v), "s"(base) : "memory");
}

__device__ __forceinline__ unsigned now32() { return (unsigned)__builtin_amdgcn_s_memrealtime(); }
template <unsigned TAU> __device__ __forceinline__ unsigned slot_after(unsigned t, unsigned v) {
    const unsigned ph = (v * TAU) >> 6, r = t % TAU;
    return t - r + ph + (r > ph ? TAU : 0u);
}
template <unsigned TAU> __device__ __forceinline__ void pace(unsigned& tnext) {
    int late = (int)(now32() - tnext);
    while (late < 0) { __builtin_amdgcn_s_sleep(1); late = (int)(now32() - tnext); }
    tnext += TAU + ((unsigned)late / TAU) * TAU;
}

__device__ __forceinline__ u32x4 pack8(float a, float b, float c, float d, float e, float f, float g, float h) {
    h16x8 r = {(_Float16)a, (_Float16)b, (_Float16)c, (_Float16)d, (_Float16)e, (_Float16)f, (_Float16)g, (_Float16)h};
    return __builtin_bit_cast(u32x4, r);
}

__device__ __forceinline__ int rb_owner(int d, int rb) { return rb >= 6 ? d : ((((d >> 1) + 1 + (rb >> 1)) & 3) << 1) | (rb & 1); }
__device__ __forceinline__ int rb_local(int d, int rb) { return rb >= 6 ? (rb - 6) * 256 : (d & 1) * 256; }

__device__ __forceinline__ float gelu1(float y) {
    const float k1 = -2.0f * 0.7978845608028654f * 1.4426950408889634f;
    const float k3 = k1 * 0.044715f;
    float t = y * y;
    float p = __builtin_fmaf(t, k3, k1);
    float e = __builtin_amdgcn_exp2f(p * y);
    return y * __builtin_amdgcn_rcpf(1.0f + e);
}

#define STAMP(i) do {} while (0)

__global__ __launch_bounds__(NTHR, 2) void fused_gemm_rs(Params p) {
    __shared__ __attribute__((aligned(16))) unsigned char lds_raw[3 * STAGE];
    LAS unsigned char* lds = (LAS unsigned char*)lds_raw;
    const int tid = threadIdx.x, lane = tid & 63;
    const int wid = __builtin_amdgcn_readfirstlane(tid >> 6);
    const int fr = lane & 15, fq = lane >> 4;
    const int rank = p.rank & 7;
    const int xcd = blockIdx.x & 7, idx = blockIdx.x >> 3;
    char* myws = p.ws[rank];
    const unsigned t_entry = now32();
    STAMP(0);

    GAS unsigned* ctr = (GAS unsigned*)(myws + WS_CTR + (size_t)blockIdx.x * 128);
    GAS unsigned* tmo = (GAS unsigned*)(myws + WS_TMO);
    const bool pusher = idx < 8;
    unsigned call;

    if (pusher) {
        const int d = (rank + idx) & 7, j = xcd, u = j * 8 + wid;
        const GAS float* xw = (const GAS float*)p.x;
        const GAS float* ww = (const GAS float*)p.w + (size_t)(8 * fq) * GN + (d >> 1) * 4096 + j * 512 + wid * 64 + 4 * fr;
        const GAS float* xs[3];
        unsigned xo[3];
#pragma unroll
        for (int i = 0; i < 3; ++i) {
            const int f = 3 * u + i, mta = f >> 1, kk = f & 1, rb = mta >> 4, mt = mta & 15;
            xs[i] = xw + (size_t)(rb_owner(d, rb) * 512 + rb_local(d, rb) + mt * 16 + fr) * KSH + kk * 32 + 8 * fq;
            xo[i] = (unsigned)(rb * NSTEP) * A_STEP + (unsigned)(mt * 2 + kk) * 1024u + lane * 16u;
        }
        const unsigned wo = (unsigned)STG_A + (unsigned)((4 * j + (wid >> 1)) * NSTEP) * B_STEP + (unsigned)((wid & 1) * 8) * 1024u + lane * 16u;
        f32x4 rw[2][8], rx[3][2];
        const int lmt = u >> 1, lhalf = u & 1;
        const GAS float* xl = xw + (size_t)(d * 512 + lmt * 16 + fr) * KSH + lhalf * 256 + 8 * fq;
        auto ldw = [&](int c, int kk, int jj) {
            rw[kk][jj] = c < 8 ? *(const GAS f32x4*)(ww + (size_t)(c * 64 + kk * 32 + jj) * GN)
                               : *(const GAS f32x4*)(xl + (kk * 4 + (jj >> 1)) * 32 + (jj & 1) * 4);
        };
        auto ldx = [&](int c, int i, int h) { rx[i][h] = *(const GAS f32x4*)(xs[i] + c * 64 + h * 4); };
        const unsigned ctr_v = __hip_atomic_load(ctr, __ATOMIC_RELAXED, __HIP_MEMORY_SCOPE_AGENT);
#pragma unroll
        for (int q = 0; q < 16; ++q) ldw(0, q >> 3, q & 7);
#pragma unroll
        for (int q = 0; q < 6; ++q) ldx(0, q >> 1, q & 1);
        call = __builtin_amdgcn_readfirstlane(ctr_v) + 1u;
        const bool paced = idx != 0;
        unsigned tnext = slot_after<TAU_IN>(t_entry, u);
        char* stg = p.ws[d] + WS_STG + (size_t)(call & 1u) * STG_BYTES;
        GAS unsigned* flag = (GAS unsigned*)(p.ws[d] + WS_FLAG) + rank * 64 + u;
        auto chunk = [&](int c, int cn) {
            const int ks = rank * 8 + c;
            const char* sa = stg + (size_t)ks * A_STEP;
            const char* sb = stg + (size_t)ks * B_STEP;
#pragma unroll
            for (int kk = 0; kk < 2; ++kk) {
                u32x4 pk[4];
#pragma unroll
                for (int t = 0; t < 4; ++t)
                    pk[t] = pack8(rw[kk][0][t], rw[kk][1][t], rw[kk][2][t], rw[kk][3][t], rw[kk][4][t], rw[kk][5][t], rw[kk][6][t], rw[kk][7][t]);
#pragma unroll
                for (int t = 0; t < 4; ++t) {
                    if (paced) { if (kk + t) wait_vm<3 * PUSH_WIN - 1>(); pace<TAU_IN>(tnext); }
                    st16_sys(sb, wo + (unsigned)(t * 2 + kk) * 1024u, pk[t]);
                    ldw(cn, kk, 2 * t); ldw(cn, kk, 2 * t + 1);
                }
            }
            u32x4 pk[3];
#pragma unroll
            for (int i = 0; i < 3; ++i)
                pk[i] = pack8(rx[i][0][0], rx[i][0][1], rx[i][0][2], rx[i][0][3], rx[i][1][0], rx[i][1][1], rx[i][1][2], rx[i][1][3]);
#pragma unroll
            for (int i = 0; i < 3; ++i) {
                if (paced) { wait_vm<3 * PUSH_WIN - 1>(); pace<TAU_IN>(tnext); }
                st16_sys(sa, xo[i], pk[i]);
                if (cn < 8 && i < 2) { ldx(cn, (3 * i) >> 1, (3 * i) & 1); ldx(cn, (3 * i + 1) >> 1, (3 * i + 1) & 1); ldx(cn, (3 * i + 2) >> 1, (3 * i + 2) & 1); }
            }
            DRAIN_V();
            if (lane == 0) __hip_atomic_store(flag, call * 16u + (unsigned)c + 1u, __ATOMIC_RELAXED, __HIP_MEMORY_SCOPE_SYSTEM);
            STAMP(1 + c);
        };
        for (int c = 0; c < 7; ++c) chunk(c, c + 1);
        chunk(7, 8);
        {
            const unsigned lo = (unsigned)((6 + (lmt >> 4)) * NSTEP + rank * 8 + lhalf * 4) * A_STEP + (unsigned)((lmt & 15) * 2) * 1024u + lane * 16u;
#pragma unroll
            for (int i = 0; i < 8; ++i) {
                const f32x4 e = rw[i >> 2][(i & 3) * 2], o = rw[i >> 2][(i & 3) * 2 + 1];
                u32x4 v = pack8(e[0], e[1], e[2], e[3], o[0], o[1], o[2], o[3]);
                if (paced) { wait_vm<PUSH_WIN - 1>(); pace<TAU_IN>(tnext); }
                st16_sys(stg, lo + (unsigned)(i >> 1) * A_STEP + (unsigned)(i & 1) * 1024u, v);
            }
            WAIT_V(0);
            if (lane == 0) __hip_atomic_store(flag, call * 16u + 9u, __ATOMIC_RELAXED, __HIP_MEMORY_SCOPE_SYSTEM);
            STAMP(9);
        }
    } else {
        call = __builtin_amdgcn_readfirstlane(__hip_atomic_load(ctr, __ATOMIC_RELAXED, __HIP_MEMORY_SCOPE_AGENT)) + 1u;
    }

    const int rb = pusher ? 6 + (idx >> 2) : (idx - 8) >> 2;
    const int cb = xcd * 4 + (idx & 3);
    const int wm = wid >> 1, wn = wid & 1;
    const char* stgA = myws + WS_STG + (size_t)(call & 1u) * STG_BYTES + (size_t)(rb * NSTEP) * A_STEP;
    const char* stgB = myws + WS_STG + (size_t)(call & 1u) * STG_BYTES + STG_A + (size_t)(cb * NSTEP) * B_STEP;
    const GAS unsigned* flags = (const GAS unsigned*)(myws + WS_FLAG);
    const unsigned base = call * 16u;
    unsigned known = 0;

    int pi = 0, pc = 0;
    auto ensure_ready = [&]() {
        if (wid == 0) {
            const int s = (rank + pi) & 7;
            const unsigned need = pusher ? 9u : (unsigned)pc + 1u;
            if (((known >> (4 * s)) & 15u) < need) {
                const unsigned long long t0 = __builtin_amdgcn_s_memrealtime();
                unsigned got;
                for (;;) {
                    const unsigned v = __hip_atomic_load(flags + s * 64 + lane, __ATOMIC_RELAXED, __HIP_MEMORY_SCOPE_SYSTEM);
                    if (__all(v >= base + need)) {
                        got = need;
                        while (got < 9u && __all(v >= base + got + 1u)) ++got;
                        break;
                    }
                    if (__builtin_amdgcn_readfirstlane(__hip_atomic_load(tmo, __ATOMIC_RELAXED, __HIP_MEMORY_SCOPE_AGENT)) == call
                        || __builtin_amdgcn_s_memrealtime() - t0 > DEADLINE_TICKS) {
                        if (lane == 0) __hip_atomic_store(tmo, call, __ATOMIC_RELAXED, __HIP_MEMORY_SCOPE_AGENT);
                        got = 9u;
                        break;
                    }
                    __builtin_amdgcn_s_sleep(8);
                }
                known = (known & ~(15u << (4 * s))) | (got << (4 * s));
            }
        }
    };
    auto stage = [&](unsigned buf) {
        const int ks = ((rank + pi) & 7) * 8 + pc;
        const char* ga = stgA + (size_t)ks * A_STEP + wid * 4096 + lane * 16;
        const char* gb = stgB + (size_t)ks * B_STEP + wid * 2048 + lane * 16;
#pragma unroll
        for (int i = 0; i < 4; ++i)
            __builtin_amdgcn_global_load_lds((const GAS unsigned*)(ga + i * 1024), (LAS unsigned*)(lds + buf + wid * 4096 + i * 1024), 16, 0, GLDS_AUX);
#pragma unroll
        for (int i = 0; i < 2; ++i)
            __builtin_amdgcn_global_load_lds((const GAS unsigned*)(gb + i * 1024), (LAS unsigned*)(lds + buf + A_STEP + wid * 2048 + i * 1024), 16, 0, GLDS_AUX);
        if (pi == 0) { if (++pc == 8) { pc = 0; pi = 1; } }
        else if (++pi == 8) { pi = 1; ++pc; }
    };
#define BAR() do { asm volatile("" ::: "memory"); __builtin_amdgcn_s_barrier(); asm volatile("" ::: "memory"); } while (0)

    f32x4 acc[4][4];
#pragma unroll
    for (int m = 0; m < 4; ++m)
#pragma unroll
        for (int n = 0; n < 4; ++n) acc[m][n] = (f32x4){0.f, 0.f, 0.f, 0.f};
    h16x8 a0[4], b0[4], a1[4], b1[4];
    const unsigned aoff = wm * 8192 + lane * 16, boff = A_STEP + wn * 8192 + lane * 16;
#define LDF(A, B, buf, kk) do { \
        _Pragma("unroll") for (int n = 0; n < 4; ++n) B[n] = *(const LAS h16x8*)(lds + (buf) + boff + n * 2048 + (kk) * 1024); \
        _Pragma("unroll") for (int m = 0; m < 4; ++m) A[m] = *(const LAS h16x8*)(lds + (buf) + aoff + m * 2048 + (kk) * 1024); } while (0)
#define MMA(A, B) do { __builtin_amdgcn_s_setprio(1); \
        _Pragma("unroll") for (int m = 0; m < 4; ++m) _Pragma("unroll") for (int n = 0; n < 4; ++n) \
            acc[m][n] = __builtin_amdgcn_mfma_f32_16x16x32_f16(A[m], B[n], acc[m][n], 0, 0, 0); \
        __builtin_amdgcn_s_setprio(0); } while (0)

    STAMP(10);
    ensure_ready(); BAR(); stage(0);
    ensure_ready(); BAR(); stage(STAGE);
    ensure_ready(); BAR(); stage(2 * STAGE);
    WAIT_V(12); BAR();
    LDF(a0, b0, 0u, 0);
    unsigned cur = 0;
    for (int t = 0; t < NSTEP; ++t) {
        const unsigned nxt = cur == 2 * STAGE ? 0u : cur + STAGE;
        WAIT_L0();
        LDF(a1, b1, cur, 1);
        MMA(a0, b0);
        if (t + 3 < NSTEP) ensure_ready();
        WAIT_L0();
        if (t + 2 < NSTEP) WAIT_V(6); else WAIT_V(0);
        BAR();
        if (t + 3 < NSTEP) stage(cur);
        if (t + 1 < NSTEP) LDF(a0, b0, nxt, 0);
        MMA(a1, b1);
        cur = nxt;
    }

    const bool bad = __builtin_amdgcn_readfirstlane(__hip_atomic_load(tmo, __ATOMIC_RELAXED, __HIP_MEMORY_SCOPE_AGENT)) == call;
    const int owner = rb_owner(rank, rb);
    {
        const char* ob = (const char*)(p.out[owner] + (size_t)(rb_local(rank, rb) + wm * 64) * GN + (rank >> 1) * 4096 + cb * 128 + wn * 64);
        const unsigned oo = (unsigned)(fq * 4) * (GN * 4u) + fr * 16u;
        const float qnan = __builtin_nanf("");
#pragma unroll
        for (int m = 0; m < 4; ++m)
#pragma unroll
            for (int n = 0; n < 4; ++n)
#pragma unroll
                for (int j = 0; j < 4; ++j) acc[m][n][j] = bad ? qnan : gelu1(acc[m][n][j]);
        const unsigned tsched = t_entry + (unsigned)(T_FIRST + 96 * TAU_IN + T_OUT_ADJ);
        for (int ph = 0; ph < 4; ++ph) {
            if (pusher || wm == ph) {
                unsigned tnext = now32();
                if ((int)(tnext - tsched) < 0) tnext = tsched;
                tnext = slot_after<TAU_OUT>(tnext - OUT_SLACK, wn * 32 + cb);
#pragma unroll
                for (int m = 0; m < 4; ++m)
#pragma unroll
                    for (int j = 0; j < 4; ++j) {
                        const f32x4 v = {acc[m][0][j], acc[m][1][j], acc[m][2][j], acc[m][3][j]};
                        if (!pusher) { wait_vm<OUT_WIN - 1>(); pace<TAU_OUT>(tnext); }
                        st16_sys(ob + (size_t)(m * 16 + j) * (GN * 4), oo, __builtin_bit_cast(u32x4, v));
                    }
            }
            if (pusher) break;
            BAR();
        }
    }
    STAMP(19);
    WAIT_V(0);
    BAR();
    STAMP(20);
    if (tid == 0) {
        if (!pusher) __hip_atomic_store((GAS unsigned*)(p.ws[owner] + WS_DONE) + rank * 32 + cb, call, __ATOMIC_RELAXED, __HIP_MEMORY_SCOPE_SYSTEM);
        __hip_atomic_store(ctr, call, __ATOMIC_RELAXED, __HIP_MEMORY_SCOPE_AGENT);
    }
    if (blockIdx.x == 0 && wid == 0) {
        const GAS unsigned* dn = (const GAS unsigned*)(myws + WS_DONE) + lane * 4;
        const bool skip = (lane >> 4) == (rank >> 1);
        const unsigned long long t0 = __builtin_amdgcn_s_memrealtime();
        for (;;) {
            bool ok = true;
#pragma unroll
            for (int i = 0; i < 4; ++i) ok &= __hip_atomic_load(dn + i, __ATOMIC_RELAXED, __HIP_MEMORY_SCOPE_SYSTEM) >= call;
            if (__all(ok || skip)) break;
            if (__builtin_amdgcn_readfirstlane(__hip_atomic_load(tmo, __ATOMIC_RELAXED, __HIP_MEMORY_SCOPE_AGENT)) == call
                || __builtin_amdgcn_s_memrealtime() - t0 > DEADLINE_TICKS) {
                if (lane == 0) {
                    __hip_atomic_store(tmo, call, __ATOMIC_RELAXED, __HIP_MEMORY_SCOPE_AGENT);
                    __hip_atomic_store((GAS float*)p.out[rank] + (rank >> 1) * 4096 + cb * 128, __builtin_nanf(""), __ATOMIC_RELAXED, __HIP_MEMORY_SCOPE_SYSTEM);
                }
                break;
            }
            __builtin_amdgcn_s_sleep(2);
        }
        STAMP(21);
    }
}

extern "C" void kernel_launch(int rank, int nranks, ncclComm_t comm, hipStream_t stream,
                              void* const* d_in, const int* in_sizes, int n_in,
                              void* d_out, int out_size, const int* in_dtypes, int out_dtype,
                              void* const* d_out_all, void* const* d_ws_all, size_t ws_bytes) {
    (void)comm; (void)in_dtypes; (void)out_dtype; (void)d_out;
    if (nranks != NR || (unsigned)rank >= (unsigned)NR || n_in != 2 || ws_bytes < WS_NEED ||
        in_sizes[0] != 4096 * KSH || in_sizes[1] != KSH * GN || out_size != 512 * GN) {
        fprintf(stderr, "kernel_launch: unexpected problem shape\n");
        return;
    }
    Params p;
    p.x = (const float*)d_in[0];
    p.w = (const float*)d_in[1];
    for (int i = 0; i < NR; ++i) { p.out[i] = (float*)d_out_all[i]; p.ws[i] = (char*)d_ws_all[i]; }
    p.rank = rank;
    p.pad = 0;
    hipLaunchKernelGGL(fused_gemm_rs, dim3(NBLK), dim3(NTHR), 0, stream, p);
    HIP_CHECK(hipGetLastError());
}
```

```cpp
#include <hip/hip_runtime.h>
#include <rccl/rccl.h>
#include <cstdio>

#define HIP_CHECK(expr) do { hipError_t _e = (expr); if (_e != hipSuccess) { \
    fprintf(stderr, "HIP error %s at %s:%d\n", hipGetErrorString(_e), __FILE__, __LINE__); \
    return; } } while (0)

typedef float f32x4 __attribute__((ext_vector_type(4)));
typedef unsigned u32x4 __attribute__((ext_vector_type(4)));
typedef unsigned u32x2 __attribute__((ext_vector_type(2)));
typedef _Float16 h16x8 __attribute__((ext_vector_type(8)));
typedef unsigned long long u64;
#define LAS __attribute__((address_space(3)))
#define GAS __attribute__((address_space(1)))
#define RLX __ATOMIC_RELAXED
#define SC_WG __HIP_MEMORY_SCOPE_WORKGROUP
#define SC_AGENT __HIP_MEMORY_SCOPE_AGENT
#define SC_SYS __HIP_MEMORY_SCOPE_SYSTEM

constexpr int NR = 8;
constexpr int GN = 16384;
constexpr int KSH = 512;
constexpr int NBLK = 256;
constexpr int NTHR = 512;
constexpr int NSTEP = 64;
constexpr unsigned A_STEP = 32768, B_STEP = 16384, STAGE = A_STEP + B_STEP;
constexpr int NUNIT = 48;
#define RINGN 4
constexpr int RING = RINGN;
constexpr size_t WS_CTR = 0;
constexpr size_t WS_FLAG = 64 << 10;
constexpr size_t WS_MBOX = 96 << 10;
constexpr size_t WS_DONE = 128 << 10;
constexpr size_t WS_TMO = 192 << 10;
constexpr size_t WS_DIAG = 256 << 10;
constexpr size_t WS_STG = 1 << 20;
constexpr size_t STG_A = 8ull * NSTEP * A_STEP;
constexpr size_t STG_B = 32ull * NSTEP * B_STEP;
constexpr size_t STG_BYTES = STG_A + STG_B;
constexpr size_t WS_NEED = WS_STG + 2 * STG_BYTES;
constexpr u64 DEADLINE_TICKS = 50000000ull;

#define TAU 112
#define T_FIRST 150
#define CREDIT 30
#define SEND_WIN 8
#define LOAD_GAP 4
#define POLL_SLEEP 8

struct Params {
    const float* x;
    const float* w;
    float* out[NR];
    char* ws[NR];
    int rank;
    int pad;
};

#define WAIT_L0() do { asm volatile("s_waitcnt lgkmcnt(0)" ::: "memory"); __builtin_amdgcn_s_waitcnt(0xC07F); } while (0)
#define BAR() do { asm volatile("" ::: "memory"); __builtin_amdgcn_s_barrier(); asm volatile("" ::: "memory"); } while (0)
#define SB() __builtin_amdgcn_sched_barrier(0)
template <int N> __device__ __forceinline__ void wait_vm() { asm volatile("s_waitcnt vmcnt(%0)" ::"i"(N) : "memory"); }

__device__ __forceinline__ void st16_sys(const void* base, unsigned voff, u32x4 v) {
    asm volatile("global_store_dwordx4 %0, %1, %2 sc0 sc1\n\ts_nop 1" ::"v"(voff), "v"(v), "s"(base) : "memory");
}
__device__ __forceinline__ unsigned now8() { return (unsigned)__builtin_amdgcn_s_memrealtime() << 3; }
__device__ __forceinline__ void pace(unsigned& t, unsigned period) {
    int late = (int)(now8() - t);
    while (late < 0) { __builtin_amdgcn_s_sleep(1); late = (int)(now8() - t); }
    t += period + (late > CREDIT * 8 ? (unsigned)(late - CREDIT * 8) : 0u);
}
__device__ __forceinline__ u32x4 pack8(float a, float b, float c, float d, float e, float f, float g, float h) {
    h16x8 r = {(_Float16)a, (_Float16)b, (_Float16)c, (_Float16)d, (_Float16)e, (_Float16)f, (_Float16)g, (_Float16)h};
    return __builtin_bit_cast(u32x4, r);
}
__device__ __forceinline__ int rb_owner(int d, int rb) { return rb >= 6 ? d : ((((d >> 1) + 1 + (rb >> 1)) & 3) << 1) | (rb & 1); }
__device__ __forceinline__ int rb_local(int d, int rb) { return rb >= 6 ? (rb - 6) * 256 : (d & 1) * 256; }

__device__ __forceinline__ void unit_of(int k, int& c, int& id) {
    if (k < 352) {
        c = k / 44;
        const int r = k - c * 44, i = r - 12;
        id = r < 12 ? 32 + r : (i & 7) * 4 + (i >> 3);
    } else { c = (k - 352) >> 2; id = 44 + (k & 3); }
}

__device__ __forceinline__ float gelu1(float y) {
    const float k1 = -2.0f * 0.7978845608028654f * 1.4426950408889634f;
    const float k3 = k1 * 0.044715f;
    float t = y * y;
    float p = __builtin_fmaf(t, k3, k1);
    float e = __builtin_amdgcn_exp2f(p * y);
    return y * __builtin_amdgcn_rcpf(1.0f + e);
}

#define STAMP(i) do {} while (0)

__global__ __launch_bounds__(NTHR, 2) void fused_gemm_rs(Params p) {
    __shared__ __attribute__((aligned(16))) unsigned char lds_raw[3 * STAGE + 512];
    LAS unsigned char* lds = (LAS unsigned char*)lds_raw;
    LAS unsigned* ctl = (LAS unsigned*)(lds + 3 * STAGE);
    const int tid = threadIdx.x, lane = tid & 63;
    const int wid = __builtin_amdgcn_readfirstlane(tid >> 6);
    const int fr = lane & 15, fq = lane >> 4;
    const int rank = p.rank & 7;
    const int xcd = blockIdx.x & 7, idx = blockIdx.x >> 3;
    char* myws = p.ws[rank];
    const unsigned t_entry = now8();
    STAMP(0);

    GAS unsigned* ctr = (GAS unsigned*)(myws + WS_CTR + (size_t)blockIdx.x * 128);
    GAS unsigned* tmo = (GAS unsigned*)(myws + WS_TMO);
    const unsigned ctr_v = __hip_atomic_load(ctr, RLX, SC_AGENT);
    const bool pusher = idx < 8;

    if (pusher) {
        const int d = idx < 6 ? rb_owner(rank, idx) : rank ^ (7 - idx);
        if (tid < 64) ctl[tid] = 0u;
        WAIT_L0();
        if (wid < 4) {
            const int q = wid;
            const unsigned xoff = (unsigned)((q * 32 + fq) * KSH + 4 * fr) * 4u;
            const unsigned woff = (unsigned)(((q & 1) * 32 + 8 * fq) * GN + (q >> 1) * 64 + 4 * fr) * 4u;
            f32x4 r0[8], r1[8];
            auto fetch = [&](f32x4 (&r)[8], int n, bool spread) {
                int c, id; unit_of(n * 8 + xcd, c, id);
                const int rb = (id - 32) >> 1;
                const GAS char* b = id < 32 ? (const GAS char*)(p.w + (size_t)(c * 64) * GN + (d >> 1) * 4096 + id * 128) + woff
                                            : (const GAS char*)(p.x + (size_t)(rb_owner(d, rb) * 512 + rb_local(d, rb) + (id & 1) * 128) * KSH + c * 64) + xoff;
                const unsigned step = id < 32 ? GN * 4 : 4 * KSH * 4;
#pragma unroll
                for (int e = 0; e < 8; ++e) {
                    r[e] = *(const GAS f32x4*)(b + (size_t)e * step);
                    if (spread) { SB(); __builtin_amdgcn_s_sleep(LOAD_GAP); SB(); }
                }
            };
            auto put = [&](f32x4 (&r)[8], int n) {
                int c, id; unit_of(n * 8 + xcd, c, id);
                u32x4 pk[4];
                if (id < 32) {
#pragma unroll
                    for (int t = 0; t < 4; ++t) pk[t] = pack8(r[0][t], r[1][t], r[2][t], r[3][t], r[4][t], r[5][t], r[6][t], r[7][t]);
                } else {
#pragma unroll
                    for (int t = 0; t < 4; ++t) pk[t] = pack8(r[2 * t][0], r[2 * t][1], r[2 * t][2], r[2 * t][3], r[2 * t + 1][0], r[2 * t + 1][1], r[2 * t + 1][2], r[2 * t + 1][3]);
                }
                while ((int)__builtin_amdgcn_readfirstlane(__hip_atomic_load(ctl + 48, RLX, SC_WG)) + RING <= n) __builtin_amdgcn_s_sleep(4);
                asm volatile("" ::: "memory");
                LAS unsigned char* slot = lds + (unsigned)(n % RING) * 16384u;
                if (id < 32) {
#pragma unroll
                    for (int t = 0; t < 4; ++t) *(LAS u32x4*)(slot + ((q >> 1) * 8 + (q & 1) + 2 * t) * 1024 + lane * 16) = pk[t];
                } else {
                    LAS unsigned char* dst = slot + (q * 4 + (fr >> 3)) * 1024 + (((fr >> 1) & 3) * 16 + fq) * 16 + (fr & 1) * 8;
#pragma unroll
                    for (int e = 0; e < 8; ++e)
                        *(LAS u32x2*)(dst + (e >> 2) * 2048 + (e & 3) * 64) = (u32x2){pk[e >> 1][(e & 1) * 2], pk[e >> 1][(e & 1) * 2 + 1]};
                }
                WAIT_L0();
                if (lane == 0) __hip_atomic_fetch_add(ctl + n, 1u, RLX, SC_WG);
                fetch(r, n + 2 < NUNIT ? n + 2 : NUNIT - 1, true);
            };
            fetch(r0, 0, false);
            fetch(r1, 1, false);
            BAR();
            for (int n = 0; n < NUNIT; n += 2) { put(r0, n); put(r1, n + 1); }
        } else if (wid == 4) {
            BAR();
            const unsigned call = __builtin_amdgcn_readfirstlane(ctr_v) + 1u;
            const char* stg = p.ws[d] + WS_STG + (size_t)(call & 1u) * STG_BYTES;
            const unsigned period = idx == 7 ? 0u : 2u * TAU;
            unsigned tnext = t_entry + T_FIRST * 8 + xcd * (TAU / 4);
            for (int n = 0; n < NUNIT; ++n) {
                int c, id; unit_of(n * 8 + xcd, c, id);
                const int ks = rank * 8 + c;
                const char* dst = id < 32 ? stg + STG_A + (size_t)(id * NSTEP + ks) * B_STEP
                                          : stg + (size_t)(((id - 32) >> 1) * NSTEP + ks) * A_STEP + (id & 1) * 16384;
                if (__builtin_amdgcn_readfirstlane(__hip_atomic_load(ctl + n, RLX, SC_WG)) < 4u) {
                    wait_vm<0>();
                    if (lane == 0) __hip_atomic_store(ctl + 49, (unsigned)n, RLX, SC_WG);
                    while (__builtin_amdgcn_readfirstlane(__hip_atomic_load(ctl + n, RLX, SC_WG)) < 4u) __builtin_amdgcn_s_sleep(1);
                }
                asm volatile("" ::: "memory");
                u32x4 v[16];
#pragma unroll
                for (int i = 0; i < 16; ++i) v[i] = *(const LAS u32x4*)(lds + (unsigned)(n % RING) * 16384u + i * 1024 + lane * 16u);
                WAIT_L0();
                if (lane == 0) __hip_atomic_store(ctl + 48, (unsigned)n + 1u, RLX, SC_WG);
#pragma unroll
                for (int i = 0; i < 16; i += 2) {
                    wait_vm<SEND_WIN - 2>();
                    if (i == ((SEND_WIN - 1) & ~1) && lane == 0) __hip_atomic_store(ctl + 49, (unsigned)n, RLX, SC_WG);
                    if (i == 0 && n == NUNIT - 1 && lane == 0) __hip_atomic_store(ctl + 50, (tnext + 16u * TAU) | 1u, RLX, SC_WG);
                    pace(tnext, period);
                    st16_sys(dst + i * 1024, lane * 16u, v[i]);
                    st16_sys(dst + i * 1024 + 1024, lane * 16u, v[i + 1]);
                }
            }
            wait_vm<0>();
            if (lane == 0) __hip_atomic_store(ctl + 49, (unsigned)NUNIT, RLX, SC_WG);
            STAMP(9);
        } else if (wid == 5) {
            BAR();
            const unsigned call = __builtin_amdgcn_readfirstlane(ctr_v) + 1u;
            GAS unsigned* flags_d = (GAS unsigned*)(p.ws[d] + WS_FLAG);
            for (int n = 0; n < NUNIT; ++n) {
                int c, id; unit_of(n * 8 + xcd, c, id);
                if (n == NUNIT - 2 && idx < 6) {
                    unsigned th;
                    while ((th = __builtin_amdgcn_readfirstlane(__hip_atomic_load(ctl + 50, RLX, SC_WG))) == 0u) __builtin_amdgcn_s_sleep(1);
                    if (lane == 0) __hip_atomic_store((GAS u64*)(myws + WS_MBOX + (size_t)((8 + idx * 4) * 8 + xcd) * 128), ((u64)call << 32) | th, RLX, SC_AGENT);
                }
                while ((int)__builtin_amdgcn_readfirstlane(__hip_atomic_load(ctl + 49, RLX, SC_WG)) <= n) __builtin_amdgcn_s_sleep(1);
                if (lane == 0) __hip_atomic_store(flags_d + id * 64 + c * 8 + rank, call, RLX, SC_SYS);
            }
        } else {
            BAR();
        }
    }
    const unsigned call = __builtin_amdgcn_readfirstlane(ctr_v) + 1u;

    const int rb = pusher ? 6 + (idx >> 2) : (idx - 8) >> 2;
    const int cb = xcd * 4 + (idx & 3);
    const int wm = wid >> 1, wn = wid & 1;
    const char* stgA = myws + WS_STG + (size_t)(call & 1u) * STG_BYTES + (size_t)(rb * NSTEP) * A_STEP + wid * 4096;
    const char* stgB = myws + WS_STG + (size_t)(call & 1u) * STG_BYTES + STG_A + (size_t)(cb * NSTEP) * B_STEP + wid * 2048;
    const GAS unsigned* flags = (const GAS unsigned*)(myws + WS_FLAG) + lane;
    u64 todo = ~0ull, avail = 0ull;
    auto pick = [&](int slot) {
        if (wid == 0) {
            if (avail == 0ull) {
                const u64 t0 = __builtin_amdgcn_s_memrealtime();
                for (;;) {
                    const unsigned f0 = __hip_atomic_load(flags + cb * 64, RLX, SC_SYS);
                    const unsigned f1 = __hip_atomic_load(flags + (32 + rb * 2) * 64, RLX, SC_SYS);
                    const unsigned f2 = __hip_atomic_load(flags + (33 + rb * 2) * 64, RLX, SC_SYS);
                    avail = __ballot(f0 >= call && f1 >= call && f2 >= call) & todo;
                    if (avail) break;
                    if (__builtin_amdgcn_readfirstlane(__hip_atomic_load(tmo, RLX, SC_AGENT)) == call
                        || __builtin_amdgcn_s_memrealtime() - t0 > DEADLINE_TICKS) {
                        if (lane == 0) __hip_atomic_store(tmo, call, RLX, SC_AGENT);
                        avail = todo;
                        break;
                    }
                    __builtin_amdgcn_s_sleep(POLL_SLEEP);
                }
            }
            const int b = __builtin_ctzll(avail);
            avail &= avail - 1; todo &= ~(1ull << b);
            if (lane == 0) ctl[56 + slot] = (unsigned)((b & 7) * 8 + (b >> 3));
        }
    };
    const unsigned lo = lane * 16u;
    auto g1 = [&](unsigned ks, unsigned buf, int i) {
        if (i < 4) __builtin_amdgcn_global_load_lds((const GAS unsigned*)(stgA + (size_t)ks * A_STEP + i * 1024 + lo), (LAS unsigned*)(lds + buf + wid * 4096 + i * 1024), 16, 0, 17);
        else __builtin_amdgcn_global_load_lds((const GAS unsigned*)(stgB + (size_t)ks * B_STEP + (i - 4) * 1024 + lo), (LAS unsigned*)(lds + buf + A_STEP + wid * 2048 + (i - 4) * 1024), 16, 0, 17);
    };
#define GETKS(slot) ((unsigned)__builtin_amdgcn_readfirstlane(ctl[56 + (slot)]) & 63u)

    f32x4 acc[4][4];
#pragma unroll
    for (int m = 0; m < 4; ++m)
#pragma unroll
        for (int n = 0; n < 4; ++n) acc[m][n] = (f32x4){0.f, 0.f, 0.f, 0.f};
    h16x8 a0[4], b0[4], a1[4], b1[4];
    const unsigned aoff = wm * 8192 + lo, boff = A_STEP + wn * 8192 + lo;
#define RDA(A, buf, kk, m) A[m] = *(const LAS h16x8*)(lds + (buf) + aoff + (m) * 2048 + (kk) * 1024)
#define RDB(B, buf, kk, n) B[n] = *(const LAS h16x8*)(lds + (buf) + boff + (n) * 2048 + (kk) * 1024)
#define ROW(A, B, m) _Pragma("unroll") for (int n = 0; n < 4; ++n) acc[m][n] = __builtin_amdgcn_mfma_f32_16x16x32_f16(A[m], B[n], acc[m][n], 0, 0, 0)

    STAMP(10);
    unsigned kq;
    pick(0); WAIT_L0(); BAR(); kq = GETKS(0);
#pragma unroll
    for (int i = 0; i < 6; ++i) g1(kq, 0u, i);
    pick(1); WAIT_L0(); BAR(); kq = GETKS(1);
#pragma unroll
    for (int i = 0; i < 6; ++i) g1(kq, STAGE, i);
    pick(2); WAIT_L0(); BAR(); kq = GETKS(2);
#pragma unroll
    for (int i = 0; i < 3; ++i) g1(kq, 2 * STAGE, i);
    wait_vm<9>(); BAR();
#pragma unroll
    for (int i = 0; i < 4; ++i) { RDB(b0, 0u, 0, i); RDA(a0, 0u, 0, i); }
    unsigned cur = 0, prv = 2 * STAGE;
    for (int t = 0; t < NSTEP; ++t) {
        const unsigned nxt = cur == 2 * STAGE ? 0u : cur + STAGE;
        const bool m2 = t + 2 < NSTEP, m3 = t + 3 < NSTEP;
        WAIT_L0(); SB();
        RDB(b1, cur, 1, 0); RDB(b1, cur, 1, 1); SB(); ROW(a0, b0, 0); SB(); if (m2) g1(kq, prv, 3); SB();
        RDB(b1, cur, 1, 2); RDB(b1, cur, 1, 3); SB(); ROW(a0, b0, 1); SB(); if (m2) g1(kq, prv, 4); SB();
        RDA(a1, cur, 1, 0); RDA(a1, cur, 1, 1); SB(); ROW(a0, b0, 2); SB(); if (m2) g1(kq, prv, 5); SB();
        RDA(a1, cur, 1, 2); RDA(a1, cur, 1, 3); SB(); ROW(a0, b0, 3); SB();
        if (m3) pick((t + 3) & 3);
        WAIT_L0();
        if (m2) wait_vm<6>(); else wait_vm<0>();
        BAR(); SB();
        const unsigned kv = ctl[56 + ((t + 3) & 3)];
        RDB(b0, nxt, 0, 0); RDB(b0, nxt, 0, 1); SB(); ROW(a1, b1, 0); SB();
        kq = (unsigned)__builtin_amdgcn_readfirstlane(kv) & 63u;
        if (m3) g1(kq, cur, 0); SB();
        RDB(b0, nxt, 0, 2); RDB(b0, nxt, 0, 3); SB(); ROW(a1, b1, 1); SB(); if (m3) g1(kq, cur, 1); SB();
        RDA(a0, nxt, 0, 0); RDA(a0, nxt, 0, 1); SB(); ROW(a1, b1, 2); SB(); if (m3) g1(kq, cur, 2); SB();
        RDA(a0, nxt, 0, 2); RDA(a0, nxt, 0, 3); SB(); ROW(a1, b1, 3); SB();
        prv = cur; cur = nxt;
    }

    const bool bad = __builtin_amdgcn_readfirstlane(__hip_atomic_load(tmo, RLX, SC_AGENT)) == call;
    const int owner = rb_owner(rank, rb);
    const char* otile = (const char*)(p.out[owner] + (size_t)rb_local(rank, rb) * GN + (rank >> 1) * 4096 + cb * 128);
    const float qnan = __builtin_nanf("");
    if (pusher) {
        const char* ob = otile + (size_t)(wm * 64) * (GN * 4) + wn * 256;
        const unsigned oo = (unsigned)(fq * 4) * (GN * 4u) + fr * 16u;
#pragma unroll
        for (int m = 0; m < 4; ++m)
#pragma unroll
            for (int j = 0; j < 4; ++j) {
                f32x4 v = {gelu1(acc[m][0][j]), gelu1(acc[m][1][j]), gelu1(acc[m][2][j]), gelu1(acc[m][3][j])};
                if (bad) v = (f32x4){qnan, qnan, qnan, qnan};
                st16_sys(ob + (size_t)(m * 16 + j) * (GN * 4), oo, __builtin_bit_cast(u32x4, v));
            }
    } else {
        WAIT_L0(); BAR();
        LAS unsigned char* tp = lds + (wm * 64 + fq * 4) * 512 + wn * 256 + fr * 16;
#pragma unroll
        for (int m = 0; m < 4; ++m)
#pragma unroll
            for (int j = 0; j < 4; ++j) {
                f32x4 v = {gelu1(acc[m][0][j]), gelu1(acc[m][1][j]), gelu1(acc[m][2][j]), gelu1(acc[m][3][j])};
                if (bad) v = (f32x4){qnan, qnan, qnan, qnan};
                *(LAS f32x4*)(tp + (m * 16 + j) * 512) = v;
            }
        WAIT_L0(); BAR();
        STAMP(22);
        if (wid == 0) {
            const GAS u64* mb = (const GAS u64*)(myws + WS_MBOX + (size_t)blockIdx.x * 128);
            const u64 t0 = __builtin_amdgcn_s_memrealtime();
            unsigned tnext;
            for (;;) {
                const u64 m = __hip_atomic_load(mb, RLX, SC_AGENT);
                tnext = __builtin_amdgcn_readfirstlane((unsigned)m);
                if ((unsigned)__builtin_amdgcn_readfirstlane((unsigned)(m >> 32)) == call) break;
                if (__builtin_amdgcn_readfirstlane(__hip_atomic_load(tmo, RLX, SC_AGENT)) == call
                    || __builtin_amdgcn_s_memrealtime() - t0 > DEADLINE_TICKS) {
                    tnext = now8();
                    break;
                }
                __builtin_amdgcn_s_sleep(POLL_SLEEP);
            }
            STAMP(23);
            const unsigned oo = (unsigned)(lane >> 5) * (GN * 4u) + (lane & 31) * 16u;
            for (int n = 0; n < 8; ++n) {
                u32x4 v[16];
#pragma unroll
                for (int i = 0; i < 16; ++i) v[i] = *(const LAS u32x4*)(lds + (unsigned)n * 16384u + i * 1024 + lo);
                if (n == 7 && (idx & 3) != 3 && lane == 0)
                    __hip_atomic_store((GAS u64*)(myws + WS_MBOX + (size_t)(blockIdx.x + 8) * 128), ((u64)call << 32) | (tnext + 16u * TAU), RLX, SC_AGENT);
                WAIT_L0();
                const char* dst = otile + (size_t)(n * 32) * (GN * 4);
#pragma unroll
                for (int i = 0; i < 16; i += 2) {
                    wait_vm<SEND_WIN - 2>();
                    pace(tnext, 2u * TAU);
                    st16_sys(dst + (size_t)(2 * i) * (GN * 4), oo, v[i]);
                    st16_sys(dst + (size_t)(2 * i + 2) * (GN * 4), oo, v[i + 1]);
                }
            }
        }
    }
    STAMP(19);
    wait_vm<0>();
    BAR();
    STAMP(20);
    if (tid == 0) {
        if (!pusher) __hip_atomic_store((GAS unsigned*)(p.ws[owner] + WS_DONE) + (rank * 32 + cb) * 32, call, RLX, SC_SYS);
        __hip_atomic_store(ctr, call, RLX, SC_AGENT);
    }
    if (blockIdx.x == 0 && wid == 0) {
        const GAS unsigned* dn = (const GAS unsigned*)(myws + WS_DONE) + lane * 32;
        const u64 t0 = __builtin_amdgcn_s_memrealtime();
        for (;;) {
            bool ok = true;
#pragma unroll
            for (int i = 1; i < 4; ++i)
                ok &= __hip_atomic_load(dn + (((rank >> 1) + i) & 3) * 2048, RLX, SC_SYS) >= call;
            if (__all(ok)) break;
            if (__builtin_amdgcn_readfirstlane(__hip_atomic_load(tmo, RLX, SC_AGENT)) == call
                || __builtin_amdgcn_s_memrealtime() - t0 > DEADLINE_TICKS) {
                if (lane == 0) {
                    __hip_atomic_store(tmo, call, RLX, SC_AGENT);
                    __hip_atomic_store((GAS float*)p.out[rank] + (rank >> 1) * 4096 + cb * 128, __builtin_nanf(""), RLX, SC_SYS);
                }
                break;
            }
            __builtin_amdgcn_s_sleep(2);
        }
        STAMP(21);
    }
}

extern "C" void kernel_launch(int rank, int nranks, ncclComm_t comm, hipStream_t stream,
                              void* const* d_in, const int* in_sizes, int n_in,
                              void* d_out, int out_size, const int* in_dtypes, int out_dtype,
                              void* const* d_out_all, void* const* d_ws_all, size_t ws_bytes) {
    (void)comm; (void)in_dtypes; (void)out_dtype; (void)d_out;
    if (nranks != NR || (unsigned)rank >= (unsigned)NR || n_in != 2 || ws_bytes < WS_NEED ||
        in_sizes[0] != 4096 * KSH || in_sizes[1] != KSH * GN || out_size != 512 * GN) {
        fprintf(stderr, "kernel_launch: unexpected problem shape\n");
        return;
    }
    Params p;
    p.x = (const float*)d_in[0];
    p.w = (const float*)d_in[1];
    for (int i = 0; i < NR; ++i) { p.out[i] = (float*)d_out_all[i]; p.ws[i] = (char*)d_ws_all[i]; }
    p.rank = rank;
    p.pad = 0;
    hipLaunchKernelGGL(fused_gemm_rs, dim3(NBLK), dim3(NTHR), 0, stream, p);
    HIP_CHECK(hipGetLastError());
}
```

```cpp
#include <hip/hip_runtime.h>
#include <rccl/rccl.h>
#include <cstdio>

#define HIP_CHECK(expr) do { hipError_t _e = (expr); if (_e != hipSuccess) { \
    fprintf(stderr, "HIP error %s at %s:%d\n", hipGetErrorString(_e), __FILE__, __LINE__); \
    return; } } while (0)

typedef float f32x4 __attribute__((ext_vector_type(4)));
typedef unsigned u32x4 __attribute__((ext_vector_type(4)));
typedef unsigned u32x3 __attribute__((ext_vector_type(3)));
typedef unsigned short u16x2 __attribute__((ext_vector_type(2)));
typedef _Float16 h16x8 __attribute__((ext_vector_type(8)));
typedef unsigned long long u64;
#define LAS __attribute__((address_space(3)))
#define GAS __attribute__((address_space(1)))
#define RLX __ATOMIC_RELAXED
#define SC_WG __HIP_MEMORY_SCOPE_WORKGROUP
#define SC_AGENT __HIP_MEMORY_SCOPE_AGENT
#define SC_SYS __HIP_MEMORY_SCOPE_SYSTEM

#define PACK_IN 1
#define PACK_OUT 1
#define TAU 109
#define CREDIT 30
#define WIN 10
#define PROD_GAP 30
#define POLL_SLEEP 8
#define SKEW2 1

constexpr int NR = 8;
constexpr int GN = 16384;
constexpr int KSH = 512;
constexpr int NBLK = 256;
constexpr int NTHR = 512;
constexpr int NSTEP = 64;
constexpr unsigned A_STEP = 32768, B_STEP = 16384, STAGE = A_STEP + B_STEP;
constexpr int NUNIT = 48;
constexpr int RING = 8;
constexpr int OPS_IN = PACK_IN ? 15 : 16, OPS_OUT = PACK_OUT ? 14 : 16;
constexpr size_t WS_CTR = 0;
constexpr size_t WS_FLAG = 32 << 10;
constexpr size_t WS_RDY = 48 << 10;
constexpr size_t WS_ORDY = 96 << 10;
constexpr size_t WS_MBOX = 128 << 10;
constexpr size_t WS_TMO = 192 << 10;
constexpr size_t WS_DIAG = 256 << 10;
constexpr size_t WS_STG = 1 << 20;
constexpr size_t STG_A = 8ull * NSTEP * A_STEP;
constexpr size_t STG_B = 32ull * NSTEP * B_STEP;
constexpr size_t WS_EXI = WS_STG + STG_A + STG_B;
constexpr size_t EXI_BYTES = 1280ull * 16384;
constexpr size_t WS_EXO = WS_EXI + 2 * EXI_BYTES;
constexpr size_t WS_NEED = WS_EXO + (size_t)NBLK * 8 * 16384;
constexpr u64 DEADLINE_TICKS = 50000000ull;

struct Params {
    const float* x;
    const float* w;
    float* out;
    char* ws[NR];
    int rank;
    int pad;
    unsigned char order[48];
    unsigned char prod[40];
};

#define WAIT_L0() do { asm volatile("s_waitcnt lgkmcnt(0)" ::: "memory"); __builtin_amdgcn_s_waitcnt(0xC07F); } while (0)
#define BAR() do { asm volatile("" ::: "memory"); __builtin_amdgcn_s_barrier(); asm volatile("" ::: "memory"); } while (0)
#define SB() __builtin_amdgcn_sched_barrier(0)
#define UNI(x) __builtin_amdgcn_readfirstlane(x)
template <int N> __device__ __forceinline__ void wait_vm() { asm volatile("s_waitcnt vmcnt(%0)" ::"i"(N) : "memory"); }

__device__ __forceinline__ void st16(const void* base, unsigned voff, u32x4 v) {
    asm volatile("global_store_dwordx4 %0, %1, %2 sc0 sc1\n\ts_nop 1" ::"v"(voff), "v"(v), "s"(base) : "memory");
}
__device__ __forceinline__ void st12(const void* base, unsigned voff, u32x3 v) {
    asm volatile("global_store_dwordx3 %0, %1, %2 sc0 sc1\n\ts_nop 1" ::"v"(voff), "v"(v), "s"(base) : "memory");
}
__device__ __forceinline__ void st4(const void* base, unsigned voff, unsigned v) {
    asm volatile("global_store_dword %0, %1, %2 sc0 sc1" ::"v"(voff), "v"(v), "s"(base) : "memory");
}
__device__ __forceinline__ unsigned ld4(const void* base, unsigned voff) {
    unsigned r;
    asm volatile("s_nop 4\n\tglobal_load_dword %0, %1, %2 sc0 sc1\n\ts_waitcnt vmcnt(0)" : "=&v"(r) : "v"(voff), "s"(base) : "memory");
    return r;
}
template <bool TWO> __device__ __forceinline__ void dma(const void* base, unsigned voff, unsigned lds_dst) {
    unsigned keep;
    if (TWO) asm volatile("s_mov_b32 %0, m0\n\ts_mov_b32 m0, %3\n\ts_nop 4\n\tglobal_load_lds_dwordx4 %1, %2 sc0 sc1\n\tglobal_load_lds_dwordx4 %1, %2 offset:1024 sc0 sc1\n\ts_mov_b32 m0, %0"
                          : "=&s"(keep) : "v"(voff), "s"(base), "s"(lds_dst) : "memory");
    else asm volatile("s_mov_b32 %0, m0\n\ts_mov_b32 m0, %3\n\ts_nop 4\n\tglobal_load_lds_dwordx4 %1, %2 sc0 sc1\n\ts_mov_b32 m0, %0"
                      : "=&s"(keep) : "v"(voff), "s"(base), "s"(lds_dst) : "memory");
}
__device__ __forceinline__ unsigned now8() { return (unsigned)__builtin_amdgcn_s_memrealtime() << 3; }
__device__ __forceinline__ void pace(unsigned& t, unsigned period) {
    int late = (int)(now8() - t);
    while (late < 0) { __builtin_amdgcn_s_sleep(1); late = (int)(now8() - t); }
    t += period + (late > CREDIT * 8 ? (unsigned)(late - CREDIT * 8) : 0u);
}
__device__ __forceinline__ unsigned lds_get(const LAS unsigned* a) { return UNI(__hip_atomic_load(a, RLX, SC_WG)); }
__device__ __forceinline__ unsigned cvt2(float a, float b) {
    typedef _Float16 h2 __attribute__((ext_vector_type(2)));
    h2 r = {(_Float16)a, (_Float16)b};
    return __builtin_bit_cast(unsigned, r);
}
__device__ __forceinline__ int rb_owner(int d, int rb) { return rb >= 6 ? d : ((((d >> 1) + 1 + (rb >> 1)) & 3) << 1) | (rb & 1); }
__device__ __forceinline__ int rb_local(int d, int rb) { return rb >= 6 ? (rb - 6) * 256 : (d & 1) * 256; }
__device__ __forceinline__ int rb_of(int d, int o) { return ((((o >> 1) - (d >> 1) - 1) & 3) << 1) | (o & 1); }
__device__ __forceinline__ void unit_of(int e, int j, int& c, int& id) {
    if (e < 32) { c = e >> 2; id = j * 4 + (e & 3); }
    else if (e < 44) { const int u = (e - 32) * 8 + j; c = u / 12; id = 32 + u - c * 12; }
    else { const int u = (e - 44) * 8 + j; c = u >> 2; id = 44 + (u & 3); }
}
__device__ __forceinline__ int gid_of(int d, int id) {
    const int rb = (id - 32) >> 1;
    return id < 32 ? (d >> 1) * 32 + id : 128 + rb_owner(d, rb) * 4 + (rb_local(d, rb) >> 7) + (id & 1);
}
__device__ __forceinline__ size_t stage_off(int id, int ks) {
    return id < 32 ? STG_A + (size_t)(id * NSTEP + ks) * B_STEP : (size_t)(((id - 32) >> 1) * NSTEP + ks) * A_STEP + (id & 1) * 16384;
}
template <int BITS, int SH> __device__ __forceinline__ unsigned wave_max_exp(unsigned mag) {
    unsigned e = 0;
#pragma unroll
    for (int b = BITS - 1; b >= 0; --b) if (__any(mag >= ((e + (1u << b)) << SH))) e += 1u << b;
    return e;
}

__device__ __forceinline__ float gelu1(float y) {
    const float k1 = -2.0f * 0.7978845608028654f * 1.4426950408889634f;
    const float k3 = k1 * 0.044715f;
    float t = y * y;
    float p = __builtin_fmaf(t, k3, k1);
    float e = __builtin_amdgcn_exp2f(p * y);
    return y * __builtin_amdgcn_rcpf(1.0f + e);
}

#define STAMP(i) do {} while (0)
#define TIMED_OUT(t0) (UNI(__hip_atomic_load(tmo, RLX, SC_AGENT)) == call || __builtin_amdgcn_s_memrealtime() - (t0) > DEADLINE_TICKS)

__global__ __launch_bounds__(NTHR, 2) void fused_gemm_rs(Params p) {
    __shared__ __attribute__((aligned(16))) unsigned char lds_raw[3 * STAGE + 1024];
    LAS unsigned char* lds = (LAS unsigned char*)lds_raw;
    LAS unsigned* ctl = (LAS unsigned*)(lds + 3 * STAGE);
    const unsigned lds0 = (unsigned)(size_t)lds;
    const int tid = threadIdx.x, lane = tid & 63;
    const int wid = UNI(tid >> 6);
    const int fr = lane & 15, fq = lane >> 4;
    const unsigned lo = lane * 16u;
    const int rank = p.rank & 7;
    const int xcd = blockIdx.x & 7, idx = blockIdx.x >> 3;
    char* myws = p.ws[rank];
    const unsigned t_entry = now8();
    STAMP(0);

    GAS unsigned* ctr = (GAS unsigned*)(myws + WS_CTR + (size_t)blockIdx.x * 128);
    GAS unsigned* tmo = (GAS unsigned*)(myws + WS_TMO);
    const unsigned ctr_v = __hip_atomic_load(ctr, RLX, SC_AGENT);
    const bool late = idx < 8;
    GAS unsigned* gflag = (GAS unsigned*)(myws + WS_FLAG);
    if (tid < 256) ctl[tid] = 0u;
    WAIT_L0();
    BAR();
    const int q = wid & 3;

    if (!late) {
        const unsigned xoff = (unsigned)((q * 32 + fr) * KSH + 8 * fq) * 4u;
        const unsigned woff = (unsigned)(((q & 1) * 32 + 8 * fq) * GN + (q >> 1) * 64 + 4 * fr) * 4u;
        const int team = (wid >> 2) * 192 + (int)blockIdx.x - 64;
#pragma unroll 1
        for (int it = 0; it < 4; ++it) {
            const int t = it * 384 + team;
            if (t >= 1280) break;
            const int g = p.prod[t >> 5], u = t & 31;
            const bool isw = g < 32;
            const int c = isw ? g >> 2 : g - 32;
            const int gid = isw ? u * 4 + (g & 3) : 128 + u;
            { const unsigned ts = t_entry + (unsigned)(t >> 5) * (PROD_GAP * 8); while ((int)(now8() - ts) < 0) __builtin_amdgcn_s_sleep(2); }
            f32x4 r[8];
            unsigned v[16];
            if (isw) {
                const GAS char* b = (const GAS char*)(p.w + (size_t)(c * 64) * GN + gid * 128) + woff;
#pragma unroll
                for (int e = 0; e < 8; ++e) r[e] = *(const GAS f32x4*)(b + (size_t)e * (GN * 4));
#pragma unroll
                for (int i = 0; i < 16; ++i) v[i] = cvt2(r[(i & 3) * 2][i >> 2], r[(i & 3) * 2 + 1][i >> 2]);
            } else {
                const GAS char* b = (const GAS char*)(p.x + (size_t)(u * 128) * KSH + c * 64) + xoff;
#pragma unroll
                for (int e = 0; e < 8; ++e) r[e] = *(const GAS f32x4*)(b + (e >> 2) * (16 * KSH * 4) + ((e >> 1) & 1) * 128 + (e & 1) * 16);
#pragma unroll
                for (int i = 0; i < 16; ++i) v[i] = cvt2(r[i >> 1][(i & 1) * 2], r[i >> 1][(i & 1) * 2 + 1]);
            }
            const unsigned call = UNI(ctr_v) + 1u;
            const int o = u >> 2;
            const int myid = isw ? ((gid >> 5) == (rank >> 1) ? gid & 31 : -1)
                                 : (o == rank ? 44 + (u & 3) : ((o >> 1) != (rank >> 1) && ((u >> 1) & 1) == (rank & 1) ? 32 + rb_of(rank, o) * 2 + (u & 1) : -1));
            const char* ex = myws + WS_EXI + (size_t)(call & 1u) * EXI_BYTES + (size_t)(gid * 8 + c) * 16384;
            unsigned hdr = 0;
            {
                unsigned w[16];
#pragma unroll
                for (int i = 0; i < 16; ++i) w[i] = v[i];
                if (PACK_IN) {
                    u16x2 mx = __builtin_bit_cast(u16x2, w[0] & 0x7fff7fffu), mn = mx;
#pragma unroll
                    for (int i = 1; i < 16; ++i) {
                        const u16x2 m = __builtin_bit_cast(u16x2, w[i] & 0x7fff7fffu);
                        mx = __builtin_elementwise_max(mx, m); mn = __builtin_elementwise_min(mn, m);
                    }
                    const unsigned emax = wave_max_exp<5, 10>(mx.x > mx.y ? mx.x : mx.y);
                    const unsigned base = emax > 15u ? emax - 15u : 0u;
                    if (__all((unsigned)(mn.x < mn.y ? mn.x : mn.y) >= (base << 10))) {
                        const unsigned bb = base * 0x04000400u, cc = w[15] - bb;
#pragma unroll
                        for (int i = 0; i < 14; ++i) w[i] = (w[i] - bb) | ((cc << (14 - i)) & 0x40004000u);
                        w[14] = (w[14] - bb) | ((cc >> 1) & 0x40004000u);
                        hdr = base;
                    } else {
                        st4(ex + 15360 + q * 256, lane * 4u, w[15]);
                        hdr = 32u;
                    }
#pragma unroll
                    for (int i = 0; i < 3; ++i) st16(ex + q * 3840 + i * 1024, lo, (u32x4){w[4 * i], w[4 * i + 1], w[4 * i + 2], w[4 * i + 3]});
                    st12(ex + q * 3840 + 3072, lane * 12u, (u32x3){w[12], w[13], w[14]});
                } else {
#pragma unroll
                    for (int i = 0; i < 4; ++i) st16(ex + q * 4096 + i * 1024, lo, (u32x4){w[4 * i], w[4 * i + 1], w[4 * i + 2], w[4 * i + 3]});
                }
            }
            if (myid >= 0) {
                const char* sg = myws + WS_STG + stage_off(myid, rank * 8 + c) + (isw ? (q >> 1) * 8 + (q & 1) : q * 4) * 1024;
#pragma unroll
                for (int i = 0; i < 4; ++i) st16(sg + i * (isw ? 2048 : 1024), lo, (u32x4){v[4 * i], v[4 * i + 1], v[4 * i + 2], v[4 * i + 3]});
            }
            wait_vm<0>();
            unsigned tw = (1u << 24) | (hdr << (6 * q));
            if (lane == 0) tw += __hip_atomic_fetch_add(ctl + 200 + (wid >> 2) * 4 + it, tw, RLX, SC_WG);
            tw = UNI(tw);
            if ((tw >> 24) == 4u) {
                const int d = lane & 7;
                const int did = isw ? ((gid >> 5) == (d >> 1) ? gid & 31 : -1)
                                    : (o == d ? 44 + (u & 3) : ((o >> 1) != (d >> 1) && ((u >> 1) & 1) == (d & 1) ? 32 + rb_of(d, o) * 2 + (u & 1) : -1));
                if (lane < 8 && did >= 0) {
                    if (d == rank) __hip_atomic_store(gflag + did * 64 + c * 8 + rank, call, RLX, SC_SYS);
                    else __hip_atomic_store((GAS u64*)(p.ws[d] + WS_RDY) + (((call & 1u) * 8 + rank) * 48 + did) * 8 + c, ((u64)call << 32) | (tw & 0xffffffu), RLX, SC_SYS);
                }
            }
        }
        STAMP(9);
    } else if (idx < 7) {
        const int s = idx < 6 ? rb_owner(rank, idx) : rank ^ 1;
        const unsigned call = UNI(ctr_v) + 1u;
        const char* exs = p.ws[s] + WS_EXI + (size_t)(call & 1u) * EXI_BYTES;
        int uc, uid;
        unit_of(p.order[lane < NUNIT ? lane : 0], xcd, uc, uid);
        const int uslot = gid_of(rank, uid) * 8 + uc;
        if (wid < 4) {
            for (int n = 0; n < NUNIT; ++n) {
                const int c = __builtin_amdgcn_readlane(uc, n), id = __builtin_amdgcn_readlane(uid, n);
                while ((int)lds_get(ctl + 192) <= n) __builtin_amdgcn_s_sleep(1);
                asm volatile("" ::: "memory");
                const LAS unsigned char* src = lds + (unsigned)(n % RING) * 16384u + q * (PACK_IN ? 3840 : 4096);
                unsigned w[16];
#pragma unroll
                for (int i = 0; i < 3; ++i) { const u32x4 a = *(const LAS u32x4*)(src + i * 1024 + lo); w[4 * i] = a[0]; w[4 * i + 1] = a[1]; w[4 * i + 2] = a[2]; w[4 * i + 3] = a[3]; }
                unsigned h = 0;
                if (PACK_IN) {
#pragma unroll
                    for (int i = 0; i < 3; ++i) w[12 + i] = *(const LAS unsigned*)(src + 3072 + lane * 12 + i * 4);
                    h = ctl[n];
                } else { const u32x4 a = *(const LAS u32x4*)(src + 3072 + lo); w[12] = a[0]; w[13] = a[1]; w[14] = a[2]; w[15] = a[3]; }
                WAIT_L0();
                if (lane == 0) __hip_atomic_fetch_add(ctl + 64 + n, 1u, RLX, SC_WG);
                if (PACK_IN) {
                    h = (UNI(h) >> (6 * q)) & 63u;
                    if (h < 32u) {
                        const unsigned bb = h * 0x04000400u;
                        unsigned cc = (w[14] & 0x40004000u) << 1;
#pragma unroll
                        for (int i = 0; i < 14; ++i) cc |= (w[i] >> (14 - i)) & (0x00010001u << i);
#pragma unroll
                        for (int i = 0; i < 15; ++i) w[i] = (w[i] & 0xbfffbfffu) + bb;
                        w[15] = cc + bb;
                    } else {
                        w[15] = ld4(exs + (size_t)__builtin_amdgcn_readlane(uslot, n) * 16384 + 15360 + q * 256, lane * 4u);
                    }
                }
                const char* sg = myws + WS_STG + stage_off(id, s * 8 + c) + (id < 32 ? (q >> 1) * 8 + (q & 1) : q * 4) * 1024;
#pragma unroll
                for (int i = 0; i < 4; ++i) st16(sg + i * (id < 32 ? 2048 : 1024), lo, (u32x4){w[4 * i], w[4 * i + 1], w[4 * i + 2], w[4 * i + 3]});
                wait_vm<0>();
                if (lane == 0 && __hip_atomic_fetch_add(ctl + 128 + n, 1u, RLX, SC_WG) == 3u)
                    __hip_atomic_store(gflag + id * 64 + c * 8 + s, call, RLX, SC_SYS);
            }
        } else if (wid == 4) {
            unsigned tnext = t_entry + xcd * (TAU / 4);
            for (int n = 0; n < NUNIT; ++n) {
                const char* src = exs + (size_t)__builtin_amdgcn_readlane(uslot, n) * 16384;
                if (lds_get(ctl + n) == 0u || (n >= RING && lds_get(ctl + 64 + n - RING) < 4u)) {
                    wait_vm<0>();
                    if (lane == 0) __hip_atomic_store(ctl + 192, (unsigned)n, RLX, SC_WG);
                    while (lds_get(ctl + n) == 0u || (n >= RING && lds_get(ctl + 64 + n - RING) < 4u)) __builtin_amdgcn_s_sleep(1);
                }
                asm volatile("" ::: "memory");
                if (n == NUNIT - 1 && lane == 0) __hip_atomic_store(ctl + 193, (tnext + OPS_IN * TAU) | 1u, RLX, SC_WG);
#pragma unroll
                for (int i = 0; i < OPS_IN; i += 2) {
                    wait_vm<WIN - 2>();
                    if (i == WIN - 2 && lane == 0) __hip_atomic_store(ctl + 192, (unsigned)n, RLX, SC_WG);
                    pace(tnext, (i + 1 < OPS_IN ? 2u : 1u) * TAU);
                    if (i + 1 < OPS_IN) dma<true>(src + i * 1024, lo, lds0 + (unsigned)(n % RING) * 16384u + i * 1024);
                    else dma<false>(src + i * 1024, lo, lds0 + (unsigned)(n % RING) * 16384u + i * 1024);
                }
            }
            wait_vm<0>();
            if (lane == 0) __hip_atomic_store(ctl + 192, (unsigned)NUNIT, RLX, SC_WG);
            STAMP(9);
        } else if (wid == 5) {
            const GAS u64* rp = (const GAS u64*)(myws + WS_RDY) + (((call & 1u) * 8 + s) * 48 + uid) * 8 + uc;
            bool pend = lane < NUNIT;
            const u64 t0 = __builtin_amdgcn_s_memrealtime();
            for (;;) {
                if (pend) {
                    const u64 f = __hip_atomic_load(rp, RLX, SC_SYS);
                    if ((unsigned)(f >> 32) >= call) { __hip_atomic_store(ctl + lane, (unsigned)f | 0x80000000u, RLX, SC_WG); pend = false; }
                }
                if (!__any(pend)) break;
                if (TIMED_OUT(t0)) {
                    if (lane == 0) __hip_atomic_store(tmo, call, RLX, SC_AGENT);
                    wait_vm<0>();
                    if (pend) __hip_atomic_store(ctl + lane, 0x80000000u, RLX, SC_WG);
                    break;
                }
                __builtin_amdgcn_s_sleep(2);
            }
            if (idx < 6) {
                unsigned th;
                while ((th = lds_get(ctl + 193)) == 0u) __builtin_amdgcn_s_sleep(4);
                if (lane == 0) __hip_atomic_store((GAS u64*)(myws + WS_MBOX + (size_t)((8 + idx * 4) * 8 + xcd) * 128), ((u64)call << 32) | th, RLX, SC_AGENT);
            }
        }
    }
    const unsigned call = UNI(ctr_v) + 1u;

    const int rb = late ? 6 + (idx >> 2) : (idx - 8) >> 2;
    const int cb = xcd * 4 + (idx & 3);
    const int wm = wid >> 1, wn = wid & 1;
    const char* stgA = myws + WS_STG + (size_t)(rb * NSTEP) * A_STEP + wid * 4096;
    const char* stgB = myws + WS_STG + STG_A + (size_t)(cb * NSTEP) * B_STEP + wid * 2048;
    u64 todo = ~0ull, avail = 0ull;
    auto pick = [&](int slot) {
        if (wid == 0) {
            if (avail == 0ull) {
                const u64 t0 = __builtin_amdgcn_s_memrealtime();
                for (;;) {
                    const unsigned f0 = __hip_atomic_load(gflag + cb * 64 + lane, RLX, SC_SYS);
                    const unsigned f1 = __hip_atomic_load(gflag + (32 + rb * 2) * 64 + lane, RLX, SC_SYS);
                    const unsigned f2 = __hip_atomic_load(gflag + (33 + rb * 2) * 64 + lane, RLX, SC_SYS);
                    avail = __ballot(f0 >= call && f1 >= call && f2 >= call) & todo;
                    if (avail) break;
                    if (TIMED_OUT(t0)) {
                        if (lane == 0) __hip_atomic_store(tmo, call, RLX, SC_AGENT);
                        avail = todo;
                        break;
                    }
                    __builtin_amdgcn_s_sleep(POLL_SLEEP);
                }
            }
            const int b = __builtin_ctzll(avail);
            avail &= avail - 1; todo &= ~(1ull << b);
            if (lane == 0) ctl[196 + slot] = (unsigned)((b & 7) * 8 + (b >> 3));
        }
    };
    auto g1 = [&](unsigned ks, unsigned buf, int i) {
        if (i < 4) __builtin_amdgcn_global_load_lds((const GAS unsigned*)(stgA + (size_t)ks * A_STEP + i * 1024 + lo), (LAS unsigned*)(lds + buf + wid * 4096 + i * 1024), 16, 0, 17);
        else __builtin_amdgcn_global_load_lds((const GAS unsigned*)(stgB + (size_t)ks * B_STEP + (i - 4) * 1024 + lo), (LAS unsigned*)(lds + buf + A_STEP + wid * 2048 + (i - 4) * 1024), 16, 0, 17);
    };
#define GETKS(slot) ((unsigned)UNI(ctl[196 + (slot)]) & 63u)

    f32x4 acc[4][4];
#pragma unroll
    for (int m = 0; m < 4; ++m)
#pragma unroll
        for (int n = 0; n < 4; ++n) acc[m][n] = (f32x4){0.f, 0.f, 0.f, 0.f};
    h16x8 a0[4], b0[4], a1[4], b1[4];
    const unsigned aoff = wm * 8192 + lo, boff = A_STEP + wn * 8192 + lo;
#define RDA(A, buf, kk, m) A[m] = *(const LAS h16x8*)(lds + (buf) + aoff + (m) * 2048 + (kk) * 1024)
#define RDB(B, buf, kk, n) B[n] = *(const LAS h16x8*)(lds + (buf) + boff + (n) * 2048 + (kk) * 1024)
#define ROW(A, B, m) _Pragma("unroll") for (int n = 0; n < 4; ++n) acc[m][n] = __builtin_amdgcn_mfma_f32_16x16x32_f16(A[m], B[n], acc[m][n], 0, 0, 0)

    STAMP(10);
    unsigned kq;
    pick(0); WAIT_L0(); BAR(); kq = GETKS(0);
#pragma unroll
    for (int i = 0; i < 6; ++i) g1(kq, 0u, i);
    pick(1); WAIT_L0(); BAR(); kq = GETKS(1);
#pragma unroll
    for (int i = 0; i < 6; ++i) g1(kq, STAGE, i);
    pick(2); WAIT_L0(); BAR(); kq = GETKS(2);
#pragma unroll
    for (int i = 0; i < 3; ++i) g1(kq, 2 * STAGE, i);
    wait_vm<9>(); BAR();
#pragma unroll
    for (int i = 0; i < 4; ++i) { RDB(b0, 0u, 0, i); RDA(a0, 0u, 0, i); }
    unsigned cur = 0, prv = 2 * STAGE;
    for (int t = 0; t < NSTEP; ++t) {
        const unsigned nxt = cur == 2 * STAGE ? 0u : cur + STAGE;
        const bool m2 = t + 2 < NSTEP, m3 = t + 3 < NSTEP;
        WAIT_L0(); SB();
        RDB(b1, cur, 1, 0); RDB(b1, cur, 1, 1); SB(); ROW(a0, b0, 0); SB(); if (m2) g1(kq, prv, 3); SB();
        RDB(b1, cur, 1, 2); RDB(b1, cur, 1, 3); SB(); ROW(a0, b0, 1); SB(); if (m2) g1(kq, prv, 4); SB();
        RDA(a1, cur, 1, 0); RDA(a1, cur, 1, 1); SB(); ROW(a0, b0, 2); SB(); if (m2) g1(kq, prv, 5); SB();
        RDA(a1, cur, 1, 2); RDA(a1, cur, 1, 3); SB(); ROW(a0, b0, 3); SB();
        if (m3) pick((t + 3) & 3);
        WAIT_L0();
        if (m2) wait_vm<6>(); else wait_vm<0>();
        BAR(); SB();
        const unsigned kv = ctl[196 + ((t + 3) & 3)];
        RDB(b0, nxt, 0, 0); RDB(b0, nxt, 0, 1); SB(); ROW(a1, b1, 0); SB();
        kq = (unsigned)UNI(kv) & 63u;
        if (m3) g1(kq, cur, 0); SB();
        RDB(b0, nxt, 0, 2); RDB(b0, nxt, 0, 3); SB(); ROW(a1, b1, 1); SB(); if (m3) g1(kq, cur, 1); SB();
        RDA(a0, nxt, 0, 0); RDA(a0, nxt, 0, 1); SB(); ROW(a1, b1, 2); SB(); if (m3) g1(kq, cur, 2); SB();
        RDA(a0, nxt, 0, 2); RDA(a0, nxt, 0, 3); SB(); ROW(a1, b1, 3); SB();
        prv = cur; cur = nxt;
    }

    const bool bad = UNI(__hip_atomic_load(tmo, RLX, SC_AGENT)) == call;
    const unsigned oo = (unsigned)(fq * 4) * (GN * 4u) + fr * 16u;
    unsigned y[64];
#pragma unroll
    for (int i = 0; i < 64; ++i) y[i] = bad ? 0x7fc00000u : __builtin_bit_cast(unsigned, gelu1(acc[i >> 4][i & 3][(i >> 2) & 3]));
    if (late) {
        const char* ob = (const char*)(p.out + (size_t)(rb_local(rank, rb) + wm * 64) * GN + (rank >> 1) * 4096 + cb * 128 + wn * 64);
#pragma unroll
        for (int i = 0; i < 16; ++i) st16(ob + (size_t)((i >> 2) * 16 + (i & 3)) * (GN * 4), oo, (u32x4){y[4 * i], y[4 * i + 1], y[4 * i + 2], y[4 * i + 3]});
    } else {
        const int peer = rb_owner(rank, rb);
        const int mirror = (8 + rb_of(peer, rank) * 4 + (idx & 3)) * 8 + xcd;
        {
            const char* ex = myws + WS_EXO + (size_t)(blockIdx.x * 8 + wid) * 16384;
            unsigned hdr = 0;
            if (PACK_OUT) {
                unsigned mx = 0;
#pragma unroll
                for (int i = 0; i < 64; ++i) { const unsigned m = y[i] & 0x7fffffffu; mx = m > mx ? m : mx; }
                const unsigned emax = wave_max_exp<8, 23>(mx);
                const unsigned base = emax > 15u ? emax - 15u : 0u, bb = base << 23;
                hdr = base;
                unsigned z[56];
#pragma unroll
                for (int g = 0; g < 8; ++g) {
                    unsigned mn = 0x7fffffffu;
#pragma unroll
                    for (int i = 0; i < 8; ++i) { const unsigned m = y[8 * g + i] & 0x7fffffffu; mn = m < mn ? m : mn; }
                    if (__all(mn >= bb)) {
                        unsigned cc = y[8 * g + 7] - bb;
                        cc = (cc & 0x07ffffffu) | ((cc >> 4) & 0x08000000u);
#pragma unroll
                        for (int i = 0; i < 7; ++i) z[7 * g + i] = (y[8 * g + i] - bb) | ((cc << (27 - 4 * i)) & 0x78000000u);
                    } else {
#pragma unroll
                        for (int i = 0; i < 7; ++i) z[7 * g + i] = y[8 * g + i];
                        st4(ex + 14336 + g * 256, lane * 4u, y[8 * g + 7]);
                        hdr |= 256u << g;
                    }
                }
#pragma unroll
                for (int i = 0; i < 14; ++i) st16(ex + i * 1024, lo, (u32x4){z[4 * i], z[4 * i + 1], z[4 * i + 2], z[4 * i + 3]});
            } else {
#pragma unroll
                for (int i = 0; i < 16; ++i) st16(ex + i * 1024, lo, (u32x4){y[4 * i], y[4 * i + 1], y[4 * i + 2], y[4 * i + 3]});
            }
            wait_vm<0>();
            if (lane == 0) __hip_atomic_store((GAS u64*)(p.ws[peer] + WS_ORDY + (size_t)mirror * 128) + wid, ((u64)call << 32) | hdr, RLX, SC_SYS);
        }
        WAIT_L0(); BAR();
        STAMP(22);
        const char* exs = p.ws[peer] + WS_EXO + (size_t)mirror * (8 * 16384);
        if (wid == 7) {
            const GAS u64* fp = lane < 8 ? (const GAS u64*)(myws + WS_ORDY + (size_t)blockIdx.x * 128) + lane : (const GAS u64*)(myws + WS_MBOX + (size_t)blockIdx.x * 128);
            const u64 t0 = __builtin_amdgcn_s_memrealtime();
            unsigned tnext;
            for (;;) {
                const u64 f = __hip_atomic_load(fp, RLX, SC_SYS);
                if (__all((unsigned)(f >> 32) >= call)) {
                    if (lane < 8) ctl[lane] = (unsigned)f;
                    tnext = __builtin_amdgcn_readlane((unsigned)f, 8);
                    break;
                }
                if (TIMED_OUT(t0)) {
                    if (lane == 0) __hip_atomic_store(tmo, call, RLX, SC_AGENT);
                    if (lane < 8) ctl[lane] = 0x80000000u;
                    tnext = now8();
                    break;
                }
                __builtin_amdgcn_s_sleep(POLL_SLEEP);
            }
            STAMP(23);
            wait_vm<0>(); __builtin_amdgcn_s_waitcnt(0x0F70);
            for (int n = 0; n < 8; ++n) {
                if (n == 7 && lane == 0) __hip_atomic_store(ctl + 193, (tnext + OPS_OUT * TAU) | 1u, RLX, SC_WG);
#pragma unroll
                for (int i = 0; i < OPS_OUT; i += 2) {
                    wait_vm<WIN - 2>();
                    if (i == WIN - 2 && lane == 0) __hip_atomic_store(ctl + 192, (unsigned)n, RLX, SC_WG);
                    pace(tnext, 2u * TAU);
                    dma<true>(exs + (size_t)n * 16384 + i * 1024, lo, lds0 + (unsigned)n * 16384u + i * 1024);
                }
            }
            wait_vm<0>();
            STAMP(19);
        } else {
            while ((int)lds_get(ctl + 192) <= wid) __builtin_amdgcn_s_sleep(2);
        }
        asm volatile("" ::: "memory");
        {
            const unsigned hdr = lds_get(ctl + wid);
            const LAS unsigned char* src = lds + (unsigned)wid * 16384u + lo;
            if (PACK_OUT) {
                unsigned z[56];
#pragma unroll
                for (int i = 0; i < 14; ++i) { const u32x4 a = *(const LAS u32x4*)(src + i * 1024); z[4 * i] = a[0]; z[4 * i + 1] = a[1]; z[4 * i + 2] = a[2]; z[4 * i + 3] = a[3]; }
                const unsigned bb = (hdr & 255u) << 23;
#pragma unroll
                for (int g = 0; g < 8; ++g) {
                    if (!(hdr & (256u << g))) {
                        unsigned cc = 0;
#pragma unroll
                        for (int i = 0; i < 7; ++i) { cc |= (z[7 * g + i] & 0x78000000u) >> (27 - 4 * i); y[8 * g + i] = (z[7 * g + i] & 0x87ffffffu) + bb; }
                        y[8 * g + 7] = ((cc & 0x07ffffffu) | ((cc << 4) & 0x80000000u)) + bb;
                    } else {
#pragma unroll
                        for (int i = 0; i < 7; ++i) y[8 * g + i] = z[7 * g + i];
                        y[8 * g + 7] = ld4(exs + (size_t)wid * 16384 + 14336 + g * 256, lane * 4u);
                    }
                }
            } else {
#pragma unroll
                for (int i = 0; i < 16; ++i) { const u32x4 a = *(const LAS u32x4*)(src + i * 1024); y[4 * i] = a[0]; y[4 * i + 1] = a[1]; y[4 * i + 2] = a[2]; y[4 * i + 3] = a[3]; }
            }
            const char* ob = (const char*)(p.out + (size_t)((peer & 1) * 256 + wm * 64) * GN + (peer >> 1) * 4096 + cb * 128 + wn * 64);
            const bool dead = hdr >> 31;
#pragma unroll
            for (int i = 0; i < 16; ++i) {
                u32x4 v = {y[4 * i], y[4 * i + 1], y[4 * i + 2], y[4 * i + 3]};
                if (dead) v = (u32x4){0x7fc00000u, 0x7fc00000u, 0x7fc00000u, 0x7fc00000u};
                st16(ob + (size_t)((i >> 2) * 16 + (i & 3)) * (GN * 4), oo, v);
            }
        }
        if (wid == 0 && (idx & 3) != 3) {
            unsigned th;
            while ((th = lds_get(ctl + 193)) == 0u) __builtin_amdgcn_s_sleep(4);
            if (lane == 0) __hip_atomic_store((GAS u64*)(myws + WS_MBOX + (size_t)(blockIdx.x + 8) * 128), ((u64)call << 32) | th, RLX, SC_AGENT);
        }
    }
    if (tid == 0) __hip_atomic_store(ctr, call, RLX, SC_AGENT);
}

extern "C" void kernel_launch(int rank, int nranks, ncclComm_t comm, hipStream_t stream,
                              void* const* d_in, const int* in_sizes, int n_in,
                              void* d_out, int out_size, const int* in_dtypes, int out_dtype,
                              void* const* d_out_all, void* const* d_ws_all, size_t ws_bytes) {
    (void)comm; (void)in_dtypes; (void)out_dtype; (void)d_out_all;
    if (nranks != NR || (unsigned)rank >= (unsigned)NR || n_in != 2 || ws_bytes < WS_NEED ||
        in_sizes[0] != 4096 * KSH || in_sizes[1] != KSH * GN || out_size != 512 * GN) {
        fprintf(stderr, "kernel_launch: unexpected problem shape\n");
        return;
    }
    Params p;
    p.x = (const float*)d_in[0];
    p.w = (const float*)d_in[1];
    p.out = (float*)d_out;
    for (int i = 0; i < NR; ++i) p.ws[i] = (char*)d_ws_all[i];
    p.rank = rank;
    p.pad = 0;
    int n = 0, np = 0;
    bool px[8] = {};
    for (int key = 0; key < 16 + 3 * SKEW2; ++key) {
        for (int pass = 0; pass < 2; ++pass) {
            if ((pass == 0) == (key != 0)) {
                for (int e = 32; e < 44; ++e)
                    if (2 * ((e - 32) * 8 / 12) == key) {
                        p.order[n++] = (unsigned char)e;
                        for (int c = (e - 32) * 8 / 12; c <= ((e - 32) * 8 + 7) / 12; ++c) if (!px[c]) { px[c] = true; p.prod[np++] = (unsigned char)(32 + c); }
                    }
            } else {
                for (int b = 0; b < 4; ++b)
                    for (int c = 0; c < 8; ++c)
                        if (2 * c + b * SKEW2 == key) { p.order[n++] = (unsigned char)(c * 4 + b); p.prod[np++] = (unsigned char)(c * 4 + b); }
            }
        }
    }
    for (int e = 44; e < 48; ++e) p.order[n++] = (unsigned char)e;
    if (n != 48 || np != 40) { fprintf(stderr, "kernel_launch: bad schedule\n"); return; }
    hipLaunchKernelGGL(fused_gemm_rs, dim3(NBLK), dim3(NTHR), 0, stream, p);
    HIP_CHECK(hipGetLastError());
}
```

```cpp
#include <hip/hip_runtime.h>
#include <rccl/rccl.h>
#include <cstdio>

#define HIP_CHECK(expr) do { hipError_t _e = (expr); if (_e != hipSuccess) { \
    fprintf(stderr, "HIP error %s at %s:%d\n", hipGetErrorString(_e), __FILE__, __LINE__); \
    return; } } while (0)

typedef float f32x4 __attribute__((ext_vector_type(4)));
typedef unsigned u32x4 __attribute__((ext_vector_type(4)));
typedef unsigned u32x2 __attribute__((ext_vector_type(2)));
typedef unsigned short u16x2 __attribute__((ext_vector_type(2)));
typedef _Float16 h16x8 __attribute__((ext_vector_type(8)));
typedef unsigned long long u64;
#define LAS __attribute__((address_space(3)))
#define GAS __attribute__((address_space(1)))
#define RLX __ATOMIC_RELAXED
#define SC_WG __HIP_MEMORY_SCOPE_WORKGROUP
#define SC_AGENT __HIP_MEMORY_SCOPE_AGENT
#define SC_SYS __HIP_MEMORY_SCOPE_SYSTEM

#define PACK_OUT 1
#define TAU 109
#define CREDIT 30
#define WIN 10
#define PROD_GAP 50
#define PUSH_GAP 16
#define RAW_GROUPS 1
#define POLL_SLEEP 8
#define NFWD 3
#define XPOS 6
#define SKEW2 2

constexpr int NR = 8;
constexpr int GN = 16384;
constexpr int KSH = 512;
constexpr int NBLK = 256;
constexpr int NTHR = 512;
constexpr int NSTEP = 64;
constexpr unsigned A_STEP = 32768, B_STEP = 16384, STAGE = A_STEP + B_STEP;
constexpr int NU_OUT = 48 - NFWD, NU_PAIR = 48 + 6 * NFWD;
constexpr int RING = 8;
constexpr int QB = 3648;
constexpr int OPS_OUT = PACK_OUT ? 14 : 16;
constexpr unsigned UNIT_T = 57 * TAU / 4;
constexpr unsigned SLOT = 18432, SIDE = 16384;
constexpr size_t WS_CTR = 0;
constexpr size_t WS_FLAG = 32 << 10;
constexpr size_t WS_RDY = 48 << 10;
constexpr size_t WS_ORDY = 96 << 10;
constexpr size_t WS_MBOX = 128 << 10;
constexpr size_t WS_TMO = 192 << 10;
constexpr size_t WS_DIAG = 256 << 10;
constexpr size_t WS_STG = 1 << 20;
constexpr size_t STG_A = 8ull * NSTEP * A_STEP;
constexpr size_t STG_B = 32ull * NSTEP * B_STEP;
constexpr size_t WS_EXI = WS_STG + STG_A + STG_B;
constexpr size_t EXI_BYTES = 1536ull * SLOT;
constexpr size_t WS_EXO = WS_EXI + 2 * EXI_BYTES;
constexpr size_t WS_NEED = WS_EXO + (size_t)NBLK * 8 * 16384;
constexpr u64 DEADLINE_TICKS = 50000000ull;

struct Params {
    const float* x;
    const float* w;
    float* out;
    char* myws;
    char* ws[NR];
    int rank;
    int pad;
    unsigned prod[40];
    unsigned char order[2][128];
};

#define WAIT_L0() do { asm volatile("s_waitcnt lgkmcnt(0)" ::: "memory"); __builtin_amdgcn_s_waitcnt(0xC07F); } while (0)
#define BAR() do { asm volatile("" ::: "memory"); __builtin_amdgcn_s_barrier(); asm volatile("" ::: "memory"); } while (0)
#define SB() __builtin_amdgcn_sched_barrier(0)
#define UNI(x) __builtin_amdgcn_readfirstlane(x)
template <int N> __device__ __forceinline__ void wait_vm() { asm volatile("s_waitcnt vmcnt(%0)" ::"i"(N) : "memory"); }

__device__ __forceinline__ void st16(const void* base, unsigned voff, u32x4 v) {
    asm volatile("s_nop 4\n\tglobal_store_dwordx4 %0, %1, %2 sc0 sc1\n\ts_nop 1" ::"v"(voff), "v"(v), "s"(base) : "memory");
}
__device__ __forceinline__ void st8(const void* base, unsigned voff, u32x2 v) {
    asm volatile("s_nop 4\n\tglobal_store_dwordx2 %0, %1, %2 sc0 sc1" ::"v"(voff), "v"(v), "s"(base) : "memory");
}
__device__ __forceinline__ void st4(const void* base, unsigned voff, unsigned v) {
    asm volatile("s_nop 4\n\tglobal_store_dword %0, %1, %2 sc0 sc1" ::"v"(voff), "v"(v), "s"(base) : "memory");
}
__device__ __forceinline__ unsigned ld4(const void* base, unsigned voff) {
    unsigned r;
    asm volatile("s_nop 4\n\tglobal_load_dword %0, %1, %2 sc0 sc1\n\ts_waitcnt vmcnt(0)" : "=&v"(r) : "v"(voff), "s"(base) : "memory");
    return r;
}
__device__ __forceinline__ u32x2 ld8(const void* base, unsigned voff) {
    u32x2 r;
    asm volatile("s_nop 4\n\tglobal_load_dwordx2 %0, %1, %2 sc0 sc1\n\ts_waitcnt vmcnt(0)" : "=&v"(r) : "v"(voff), "s"(base) : "memory");
    return r;
}
template <bool TWO> __device__ __forceinline__ void dma(const void* base, unsigned voff, unsigned lds_dst) {
    unsigned keep;
    if (TWO) asm volatile("s_mov_b32 %0, m0\n\ts_mov_b32 m0, %3\n\ts_nop 4\n\tglobal_load_lds_dwordx4 %1, %2 sc0 sc1\n\tglobal_load_lds_dwordx4 %1, %2 offset:1024 sc0 sc1\n\ts_mov_b32 m0, %0"
                          : "=&s"(keep) : "v"(voff), "s"(base), "s"(lds_dst) : "memory");
    else asm volatile("s_mov_b32 %0, m0\n\ts_mov_b32 m0, %3\n\ts_nop 4\n\tglobal_load_lds_dwordx4 %1, %2 sc0 sc1\n\ts_mov_b32 m0, %0"
                      : "=&s"(keep) : "v"(voff), "s"(base), "s"(lds_dst) : "memory");
}
__device__ __forceinline__ unsigned now8() { return (unsigned)__builtin_amdgcn_s_memrealtime() << 3; }
__device__ __forceinline__ void pace(unsigned& t, unsigned period) {
    int late = (int)(now8() - t);
    while (late < 0) { __builtin_amdgcn_s_sleep(1); late = (int)(now8() - t); }
    t += period + (late > CREDIT * 8 ? (unsigned)(late - CREDIT * 8) : 0u);
}
__device__ __forceinline__ unsigned lds_get(const LAS unsigned* a) { return UNI(__hip_atomic_load(a, RLX, SC_WG)); }
__device__ __forceinline__ unsigned cvt2(float a, float b) {
    typedef _Float16 h2 __attribute__((ext_vector_type(2)));
    h2 r = {(_Float16)a, (_Float16)b};
    return __builtin_bit_cast(unsigned, r);
}
__device__ __forceinline__ int rb_owner(int d, int rb) { return rb >= 6 ? d : ((((d >> 1) + 1 + (rb >> 1)) & 3) << 1) | (rb & 1); }
__device__ __forceinline__ int rb_local(int d, int rb) { return rb >= 6 ? (rb - 6) * 256 : (d & 1) * 256; }
__device__ __forceinline__ int rb_of(int d, int o) { return ((((o >> 1) - (d >> 1) - 1) & 3) << 1) | (o & 1); }
__device__ __forceinline__ void unit_of(int e, int j, int& c, int& id) {
    if (e < 32) { c = e >> 2; id = j * 4 + (e & 3); }
    else if (e < 44) { const int u = (e - 32) * 8 + j; c = u / 12; id = 32 + u - c * 12; }
    else { const int u = (e & 1) * 8 + j; c = u >> 1; id = (e < 46 ? 46 : 44) + (u & 1); }
}
__device__ __forceinline__ int gid_of(int d, int id) {
    const int rb = (id - 32) >> 1;
    return id < 32 ? (d >> 1) * 32 + id : 128 + rb_owner(d, rb) * 4 + (rb_local(d, rb) >> 7) + (id & 1);
}
__device__ __forceinline__ size_t stage_off(int id, int ks) {
    return id < 32 ? STG_A + (size_t)(id * NSTEP + ks) * B_STEP : (size_t)(((id - 32) >> 1) * NSTEP + ks) * A_STEP + (id & 1) * 16384;
}
template <int BITS, int SH> __device__ __forceinline__ unsigned wave_max_exp(unsigned mag) {
    unsigned e = 0;
#pragma unroll
    for (int b = BITS - 1; b >= 0; --b) if (__any(mag >= ((e + (1u << b)) << SH))) e += 1u << b;
    return e;
}

__device__ __forceinline__ unsigned lane_prefix(unsigned cnt, unsigned& total) {
    unsigned pre = 0; total = 0;
#pragma unroll
    for (int b = 0; b < 4; ++b) {
        const u64 m = __ballot((cnt >> b) & 1u);
        pre += __builtin_amdgcn_mbcnt_hi((unsigned)(m >> 32), __builtin_amdgcn_mbcnt_lo((unsigned)m, 0u)) << b;
        total += (unsigned)__builtin_popcountll(m) << b;
    }
    return pre;
}
#define PK(x) __builtin_bit_cast(u16x2, (unsigned)(x))
#define DW(x) __builtin_bit_cast(unsigned, (u16x2)(x))

typedef float f32x2 __attribute__((ext_vector_type(2)));
__device__ __forceinline__ f32x2 gelu2(f32x2 y) {
    const float k1 = -2.0f * 0.7978845608028654f * 1.4426950408889634f, k3 = k1 * 0.044715f;
    const f32x2 a = __builtin_elementwise_fma(y * y, (f32x2){k3, k3}, (f32x2){k1, k1}) * y;
    const f32x2 d = (f32x2){__builtin_amdgcn_exp2f(a.x), __builtin_amdgcn_exp2f(a.y)} + 1.0f;
    return y * (f32x2){__builtin_amdgcn_rcpf(d.x), __builtin_amdgcn_rcpf(d.y)};
}
__device__ __forceinline__ float gelu1(float y) { return gelu2((f32x2){y, y}).x; }

#define STAMP(i) do {} while (0)
#define TIMED_OUT(t0) (UNI(__hip_atomic_load(tmo, RLX, SC_AGENT)) == call || __builtin_amdgcn_s_memrealtime() - (t0) > DEADLINE_TICKS)

__global__ __launch_bounds__(NTHR, 2) void fused_gemm_rs(Params p) {
    __shared__ __attribute__((aligned(16))) unsigned char lds_raw[3 * STAGE + 2048];
    LAS unsigned char* lds = (LAS unsigned char*)lds_raw;
    LAS unsigned* ctl = (LAS unsigned*)(lds + 3 * STAGE);
    const unsigned lds0 = (unsigned)(size_t)lds;
    const int tid = threadIdx.x, lane = tid & 63;
    const int wid = UNI(tid >> 6);
    const int fr = lane & 15, fq = lane >> 4;
    const unsigned lo = lane * 16u;
    const int xcd = blockIdx.x & 7, idx = blockIdx.x >> 3;
    const bool late = idx < 8;
    const int q = wid & 3;

    const int team = (wid >> 2) * 192 + (int)blockIdx.x - 64;
    f32x4 r[8];
    auto request = [&](int g, int u) {
        if (g < 32) {
            const GAS char* b = (const GAS char*)(p.w + (size_t)((g >> 2) * 64 + (q & 1) * 32) * GN + (u * 4 + (g & 3)) * 128 + (q >> 1) * 64) + (unsigned)(8 * fq * GN + 4 * fr) * 4u;
#pragma unroll
            for (int e = 0; e < 8; ++e) r[e] = *(const GAS f32x4*)(b + (size_t)e * (GN * 4));
        } else {
            const GAS char* b = (const GAS char*)(p.x + (size_t)(u * 128 + q * 32) * KSH + (g - 32) * 64) + (unsigned)(fr * KSH + 8 * fq) * 4u;
#pragma unroll
            for (int e = 0; e < 8; ++e) r[e] = *(const GAS f32x4*)(b + (e >> 2) * (16 * KSH * 4) + ((e >> 1) & 1) * 128 + (e & 1) * 16);
        }
    };
    if (!late && team < 64) request(team < 32 ? 0 : 32, team & 31);
    const unsigned t_entry = now8();

    const int rank = p.rank & 7;
    char* myws = p.myws;
    STAMP(0);
    GAS unsigned* ctr = (GAS unsigned*)(myws + WS_CTR + (size_t)blockIdx.x * 128);
    GAS unsigned* tmo = (GAS unsigned*)(myws + WS_TMO);
    const unsigned ctr_v = __hip_atomic_load(ctr, RLX, SC_AGENT);
    GAS unsigned* gflag = (GAS unsigned*)(myws + WS_FLAG);
    ctl[tid] = 0u;
    WAIT_L0();
    BAR();

    if (!late) {
        unsigned wlo = 0, whi = 0;
#pragma unroll
        for (int d = 0; d < NR; ++d) {
            unsigned a = (unsigned)(size_t)p.ws[d], b = (unsigned)((size_t)p.ws[d] >> 32);
            asm volatile("" : "+s"(a), "+s"(b));
            wlo = lane == d ? a : wlo;
            whi = lane == d ? b : whi;
        }
#pragma unroll 1
        for (int it = 0; it < 4; ++it) {
            const int t = it * 384 + team;
            if (t >= 1280) break;
            const int g = p.prod[t >> 5], u = t & 31;
            if (t >= 64) {
                const unsigned ts = t_entry + (unsigned)((t >> 5) - 1) * (PROD_GAP * 8);
                while ((int)(now8() - ts) < 0) __builtin_amdgcn_s_sleep(2);
                request(g, u);
            }
            const bool isw = g < 32;
            const int c = isw ? g >> 2 : g - 32;
            const int gid = isw ? u * 4 + (g & 3) : 128 + u;
            unsigned v[16];
            if (isw) {
#pragma unroll
                for (int i = 0; i < 16; ++i) v[i] = cvt2(r[(i & 3) * 2][i >> 2], r[(i & 3) * 2 + 1][i >> 2]);
            } else {
#pragma unroll
                for (int i = 0; i < 16; ++i) v[i] = cvt2(r[i >> 1][(i & 1) * 2], r[i >> 1][(i & 1) * 2 + 1]);
            }
            const unsigned call = UNI(ctr_v) + 1u;
            const size_t par = WS_EXI + (size_t)(call & 1u) * EXI_BYTES;
            const int o = u >> 2;
            const int myid = isw ? ((gid >> 5) == (rank >> 1) ? gid & 31 : -1)
                                 : (o == rank ? 44 + (u & 3) : ((o >> 1) != (rank >> 1) && ((u >> 1) & 1) == (rank & 1) ? 32 + rb_of(rank, o) * 2 + (u & 1) : -1));
            unsigned hdr = 0;
            {
                unsigned w[16];
#pragma unroll
                for (int i = 0; i < 16; ++i) w[i] = v[i];
                const char* home = myws + par + (size_t)(gid * 8 + c) * SLOT;
                unsigned pw = 0;
                hdr = 32u;
                if (t >= 32 * RAW_GROUPS) {
                    LAS unsigned* pool = (LAS unsigned*)(lds + wid * 128);
                    if (lane < 32) pool[lane] = 0u;
                    u16x2 mx = PK(w[0] & 0x7c007c00u);
#pragma unroll
                    for (int i = 1; i < 16; ++i) mx = __builtin_elementwise_max(mx, PK(w[i] & 0x7c007c00u));
                    const unsigned emax = wave_max_exp<5, 10>(mx.x > mx.y ? mx.x : mx.y);
                    const unsigned B = emax > 7u ? emax - 7u : 0u, bb = B * 0x04000400u;
                    unsigned str = 0, sh = 0, big = 0;
#pragma unroll
                    for (int i = 0; i < 16; ++i) {
                        const unsigned ef = w[i] & 0x7c007c00u;
                        const unsigned code = DW(__builtin_elementwise_sub_sat(PK(ef), PK(bb))), nib = DW(__builtin_elementwise_sub_sat(PK(bb), PK(ef)));
                        const unsigned step = DW(__builtin_elementwise_sub_sat(PK(0x00040004u), PK(code)));
                        big |= nib;
                        str |= ((nib >> 10) & 63u) << (sh & 31u); sh += step & 7u;
                        str |= (nib >> 26) << (sh & 31u); sh += step >> 16;
                        w[i] = (w[i] & 0x83ff83ffu) | code;
                    }
                    unsigned total;
                    const unsigned pos = lane_prefix(sh >> 2, total);
                    if (__all(sh <= 32u && !(big & 0x40004000u)) && total <= 128u) {
                        if (sh) {
                            const u64 sv = (u64)str << ((pos & 7u) * 4u);
                            __hip_atomic_fetch_or(pool + (pos >> 3), (unsigned)sv, RLX, SC_WG);
                            __hip_atomic_fetch_or(pool + (pos >> 3) + 1, (unsigned)(sv >> 32), RLX, SC_WG);
                        }
#pragma unroll
                        for (int k = 0; k < 2; ++k) {
                            const unsigned cc = (w[14 + k] & 0x1fff1fffu) | ((w[14 + k] >> 2) & 0x20002000u);
#pragma unroll
                            for (int i = 0; i < 7; ++i) w[7 * k + i] |= (cc << (13 - 2 * i)) & 0x60006000u;
                        }
                        pw = __hip_atomic_load(pool + (lane & 15), RLX, SC_WG);
                        hdr = B;
                    }
                }
                if (hdr == 32u) {
#pragma unroll
                    for (int i = 0; i < 14; ++i) w[i] = v[i];
                    st8(home + SIDE + q * 512, lane * 8u, (u32x2){v[14], v[15]});
                }
                const char* ex[2];
#pragma unroll
                for (int k = 0; k < 2; ++k) {
                    const int d = (u >> 3) * 2 + k;
                    ex[k] = t >= 32 ? (k ? nullptr : home) : (d == rank ? nullptr : p.ws[d] + par + (size_t)(1280 + rank * 8 + (u & 7)) * SLOT);
                }
#pragma unroll
                for (int i = 0; i < 4; ++i) {
                    if (i && t < 32) __builtin_amdgcn_s_sleep(PUSH_GAP);
#pragma unroll
                    for (int k = 0; k < 2; ++k) {
                        if (!ex[k]) continue;
                        if (i < 3) st16(ex[k] + q * QB + i * 1024, lo, (u32x4){w[4 * i], w[4 * i + 1], w[4 * i + 2], w[4 * i + 3]});
                        else {
                            st8(ex[k] + q * QB + 3072, lane * 8u, (u32x2){w[12], w[13]});
                            if (lane < 16 && hdr != 32u) st4(ex[k] + q * QB + 3584, lane * 4u, pw);
                        }
                    }
                }
            }
            if (myid >= 0) {
                const char* sg = myws + WS_STG + stage_off(myid, rank * 8 + c) + (isw ? (q >> 1) * 8 + (q & 1) : q * 4) * 1024;
#pragma unroll
                for (int i = 0; i < 4; ++i) st16(sg + i * (isw ? 2048 : 1024), lo, (u32x4){v[4 * i], v[4 * i + 1], v[4 * i + 2], v[4 * i + 3]});
            }
            wait_vm<0>();
            unsigned tw = (1u << 24) | (hdr << (6 * q));
            if (lane == 0) tw += __hip_atomic_fetch_add(ctl + 392 + (wid >> 2) * 4 + it, tw, RLX, SC_WG);
            tw = UNI(tw);
            if ((tw >> 24) == 4u) {
                const int d = lane & 7;
                int did = isw ? ((gid >> 5) == (d >> 1) ? gid & 31 : -1)
                              : (o == d ? 44 + (u & 3) : ((o >> 1) != (d >> 1) && ((u >> 1) & 1) == (d & 1) ? 32 + rb_of(d, o) * 2 + (u & 1) : -1));
                if (isw && (g & 3) == 3 && c < 2 * NFWD && (d >> 1) != (rank >> 1) && ((c ^ d) & 1)) did = -1;
                if (lane < 8 && did >= 0) {
                    if (d == rank) __hip_atomic_store(gflag + did * 64 + c * 8 + rank, call, RLX, SC_SYS);
                    else __hip_atomic_store((GAS u64*)((((size_t)whi << 32) | wlo) + WS_RDY) + (((call & 1u) * 8 + rank) * 48 + did) * 8 + c, ((u64)call << 32) | (tw & 0xffffffu), RLX, SC_SYS);
                }
            }
        }
        STAMP(9);
    } else if (idx < 7) {
        const int s = idx < 6 ? rb_owner(rank, idx) : rank ^ 1;
        const int nu = idx < 6 ? NU_OUT : NU_PAIR;
        unsigned desc[2];
#pragma unroll
        for (int k = 0; k < 2; ++k) {
            const int e = p.order[idx == 6][k * 64 + lane];
            int c, id, org = s, slot;
            if (e < 48) { unit_of(e, xcd, c, id); slot = gid_of(rank, id) * 8 + c; }
            else { const int kf = (e - 48) / 6; c = 2 * kf + (s & 1); id = xcd * 4 + 3; org = rb_owner(rank, e - 48 - kf * 6); slot = 1344 + (e - 48) * 8 + xcd; }
            desc[k] = (unsigned)(c | id << 3 | org << 9 | slot << 12);
        }
        auto desc_of = [&](int n) { return (unsigned)(n < 64 ? __builtin_amdgcn_readlane(desc[0], n) : __builtin_amdgcn_readlane(desc[1], n - 64)); };
        const unsigned call = UNI(ctr_v) + 1u;
        const size_t par = WS_EXI + (size_t)(call & 1u) * EXI_BYTES;
        if (wid < 4) {
            int pend = -1;
            auto staged = [&](int n) {
                wait_vm<0>();
                if (lane == 0 && __hip_atomic_fetch_add(ctl + 256 + n, 1u, RLX, SC_WG) == 3u) __hip_atomic_store(gflag + pend, call, RLX, SC_SYS);
                pend = -1;
            };
            for (int n = 0; n < nu; ++n) {
                const unsigned de = desc_of(n);
                const int c = de & 7, id = (de >> 3) & 63, org = (de >> 9) & 7;
                while ((int)lds_get(ctl + (n ? 384 : 386)) <= n) __builtin_amdgcn_s_sleep(1);
                asm volatile("" ::: "memory");
                const LAS unsigned char* src = lds + (unsigned)(n % RING) * 16384u + q * QB;
                unsigned w[16], h = 0;
#pragma unroll
                for (int i = 0; i < 3; ++i) { const u32x4 a = *(const LAS u32x4*)(src + i * 1024 + lo); w[4 * i] = a[0]; w[4 * i + 1] = a[1]; w[4 * i + 2] = a[2]; w[4 * i + 3] = a[3]; }
                { const u32x2 a = *(const LAS u32x2*)(src + 3072 + lane * 8); w[12] = a[0]; w[13] = a[1]; }
                h = (lds_get(ctl + n) >> (6 * q)) & 63u;
                if (h < 32u) {
#pragma unroll
                    for (int k = 0; k < 2; ++k) {
                        unsigned cc = 0;
#pragma unroll
                        for (int i = 0; i < 7; ++i) { cc |= (w[7 * k + i] & 0x60006000u) >> (13 - 2 * i); w[7 * k + i] &= 0x9fff9fffu; }
                        w[14 + k] = (cc & 0x1fff1fffu) | ((cc << 2) & 0x80008000u);
                    }
                    unsigned step[16], sum = 0;
#pragma unroll
                    for (int i = 0; i < 16; ++i) {
                        step[i] = DW(__builtin_elementwise_sub_sat(PK(0x00040004u), PK(w[i] & 0x1c001c00u)));
                        sum += step[i];
                    }
                    unsigned total;
                    const unsigned pos = lane_prefix(((sum & 0xffffu) + (sum >> 16)) >> 2, total);
                    const LAS unsigned* pool = (const LAS unsigned*)(src + 3584) + ((pos >> 3) & 15u);
                    unsigned str = (unsigned)((((u64)pool[1] << 32) | pool[0]) >> ((pos & 7u) * 4u));
                    WAIT_L0();
                    if (lane == 0) __hip_atomic_fetch_add(ctl + 128 + n, 1u, RLX, SC_WG);
                    if (pend >= 0) staged(n - 1);
                    const unsigned bb = h * 0x04000400u;
#pragma unroll
                    for (int i = 0; i < 16; ++i) {
                        const unsigned s1 = step[i] >> 16;
                        const unsigned n0 = __builtin_amdgcn_ubfe(str, 0u, step[i]); str >>= step[i] & 31u;
                        const unsigned n1 = __builtin_amdgcn_ubfe(str, 0u, s1); str >>= s1;
                        w[i] = w[i] + bb - (n0 << 10) - (n1 << 26);
                    }
                } else {
                    WAIT_L0();
                    if (lane == 0) __hip_atomic_fetch_add(ctl + 128 + n, 1u, RLX, SC_WG);
                    if (pend >= 0) staged(n - 1);
                    const u32x2 a = ld8(p.ws[org] + par + (size_t)(gid_of(rank, id) * 8 + c) * SLOT + SIDE + q * 512, lane * 8u);
                    w[14] = a[0]; w[15] = a[1];
                }
                const char* sg = myws + WS_STG + stage_off(id, org * 8 + c) + (id < 32 ? (q >> 1) * 8 + (q & 1) : q * 4) * 1024;
#pragma unroll
                for (int i = 0; i < 4; ++i) st16(sg + i * (id < 32 ? 2048 : 1024), lo, (u32x4){w[4 * i], w[4 * i + 1], w[4 * i + 2], w[4 * i + 3]});
                pend = id * 64 + c * 8 + org;
                if (n + 1 == nu || (int)lds_get(ctl + 384) <= n + 1) staged(n);
            }
        } else if (wid == 4) {
            const char* exs = p.ws[s] + par;
            unsigned tnext = t_entry + xcd * (TAU / 4);
            for (int n = 1; n < nu; ++n) {
                const char* src = exs + (size_t)(desc_of(n) >> 12) * SLOT;
                if (lds_get(ctl + n) == 0u || (n >= RING && lds_get(ctl + 128 + n - RING) < 5u)) {
                    wait_vm<0>();
                    if (lane == 0) __hip_atomic_store(ctl + 384, (unsigned)n, RLX, SC_WG);
                    while (lds_get(ctl + n) == 0u || (n >= RING && lds_get(ctl + 128 + n - RING) < 5u)) __builtin_amdgcn_s_sleep(1);
                }
                asm volatile("" ::: "memory");
                if (n == nu - 1 && lane == 0) __hip_atomic_store(ctl + 385, (tnext + UNIT_T) | 1u, RLX, SC_WG);
#pragma unroll
                for (int i = 0; i < 14; i += 2) {
                    wait_vm<WIN - 2>();
                    if (i == WIN - 2 && lane == 0) __hip_atomic_store(ctl + 384, (unsigned)n, RLX, SC_WG);
                    pace(tnext, i < 12 ? 2u * TAU : UNIT_T - 12u * TAU);
                    dma<true>(src + i * 1024, lo, lds0 + (unsigned)(n % RING) * 16384u + i * 1024);
                }
                if (lane < 16) dma<false>(src + 14336, lo, lds0 + (unsigned)(n % RING) * 16384u + 14336);
            }
            wait_vm<0>();
            if (lane == 0) __hip_atomic_store(ctl + 384, (unsigned)nu, RLX, SC_WG);
            STAMP(9);
        } else if (wid == 5) {
            const GAS u64* rp[2];
            bool pend[2];
#pragma unroll
            for (int k = 0; k < 2; ++k) {
                rp[k] = (const GAS u64*)(myws + WS_RDY) + (((call & 1u) * 8 + ((desc[k] >> 9) & 7)) * 48 + ((desc[k] >> 3) & 63)) * 8 + (desc[k] & 7);
                pend[k] = k * 64 + lane < nu;
            }
            const u64 t0 = __builtin_amdgcn_s_memrealtime();
            for (unsigned polls = 1;; ++polls) {
#pragma unroll
                for (int k = 0; k < 2; ++k)
                    if (pend[k] && (k == 0 || (polls & 7u) == 0u)) {
                        const u64 f = __hip_atomic_load(rp[k], RLX, SC_SYS);
                        if ((unsigned)(f >> 32) >= call) { __hip_atomic_store(ctl + k * 64 + lane, (unsigned)f | 0x80000000u, RLX, SC_WG); pend[k] = false; }
                    }
                if (!__any(pend[0] || pend[1])) break;
                if ((polls & 15u) == 0u && TIMED_OUT(t0)) {
                    if (lane == 0) __hip_atomic_store(tmo, call, RLX, SC_AGENT);
                    wait_vm<0>();
#pragma unroll
                    for (int k = 0; k < 2; ++k) if (pend[k]) __hip_atomic_store(ctl + k * 64 + lane, 0x80000000u, RLX, SC_WG);
                    break;
                }
                if (polls < 32u) __builtin_amdgcn_s_sleep(1); else __builtin_amdgcn_s_sleep(16);
            }
            if (idx < 6) {
                unsigned th;
                while ((th = lds_get(ctl + 385)) == 0u) __builtin_amdgcn_s_sleep(4);
                if (lane == 0) __hip_atomic_store((GAS u64*)(myws + WS_MBOX + (size_t)((8 + idx * 4) * 8 + xcd) * 128), ((u64)call << 32) | th, RLX, SC_AGENT);
            }
        } else if (wid == 6) {
            while (lds_get(ctl) == 0u) __builtin_amdgcn_s_sleep(2);
            const char* src = myws + par + (size_t)(1280 + s * 8 + xcd) * SLOT;
#pragma unroll
            for (int i = 0; i < 16; i += 2) dma<true>(src + i * 1024, lo, lds0 + i * 1024);
            wait_vm<0>();
            if (lane == 0) __hip_atomic_store(ctl + 386, 1u, RLX, SC_WG);
        } else {
            for (int n = 0; n < nu; ++n) {
                const unsigned de = desc_of(n);
                const int c = de & 7, id = (de >> 3) & 63;
                const bool relay = idx < 6 && id < 32 && (id & 3) == 3 && c < 2 * NFWD;
                if (relay) {
                    while ((int)lds_get(ctl + 384) <= n) __builtin_amdgcn_s_sleep(2);
                    asm volatile("" ::: "memory");
                    u32x4 v[15];
#pragma unroll
                    for (int i = 0; i < 15; ++i) v[i] = *(const LAS u32x4*)(lds + (unsigned)(n % RING) * 16384u + i * 1024 + lo);
                    const unsigned h = lds_get(ctl + n);
                    WAIT_L0();
                    if (lane == 0) __hip_atomic_fetch_add(ctl + 128 + n, 1u, RLX, SC_WG);
                    const char* dst = myws + par + (size_t)(1344 + ((c >> 1) * 6 + idx) * 8 + xcd) * SLOT;
#pragma unroll
                    for (int i = 0; i < 15; ++i) st16(dst + i * 1024, lo, v[i]);
                    wait_vm<0>();
                    if (lane == 0) __hip_atomic_store((GAS u64*)(p.ws[rank ^ 1] + WS_RDY) + (((call & 1u) * 8 + s) * 48 + id) * 8 + c, ((u64)call << 32) | (h & 0xffffffu), RLX, SC_SYS);
                } else if (lane == 0) __hip_atomic_fetch_add(ctl + 128 + n, 1u, RLX, SC_WG);
            }
        }
    }
    const unsigned call = UNI(ctr_v) + 1u;

    const int ntile = idx == 6 ? 0 : idx == 7 ? 2 : 1;
    const int wm = wid >> 1, wn = wid & 1;
#pragma unroll 1
    for (int rep = 0; rep < ntile; ++rep) {
    const int rb = late ? 6 + (idx >> 2) : (idx - 8) >> 2;
    const int cb = xcd * 4 + (idx == 7 ? rep : idx == 4 || idx == 5 ? idx - 2 : idx & 3);
    const char* stgA = myws + WS_STG + (size_t)(rb * NSTEP) * A_STEP + wid * 4096;
    const char* stgB = myws + WS_STG + STG_A + (size_t)(cb * NSTEP) * B_STEP + wid * 2048;
    u64 todo = ~0ull, avail = 0ull;
    auto pick = [&](int slot) {
        if (wid == 0) {
            if (avail == 0ull) {
                const u64 t0 = __builtin_amdgcn_s_memrealtime();
                for (unsigned polls = 1;; ++polls) {
                    const unsigned f0 = __hip_atomic_load(gflag + cb * 64 + lane, RLX, SC_SYS);
                    const unsigned f1 = __hip_atomic_load(gflag + (32 + rb * 2) * 64 + lane, RLX, SC_SYS);
                    const unsigned f2 = __hip_atomic_load(gflag + (33 + rb * 2) * 64 + lane, RLX, SC_SYS);
                    avail = __ballot(f0 >= call && f1 >= call && f2 >= call) & todo;
                    if (avail) break;
                    if ((polls & 15u) == 0u && TIMED_OUT(t0)) {
                        if (lane == 0) __hip_atomic_store(tmo, call, RLX, SC_AGENT);
                        avail = todo;
                        break;
                    }
                    __builtin_amdgcn_s_sleep(POLL_SLEEP);
                }
            }
            const int b = __builtin_ctzll(avail);
            avail &= avail - 1; todo &= ~(1ull << b);
            if (lane == 0) ctl[388 + slot] = (unsigned)((b & 7) * 8 + (b >> 3));
        }
    };
    auto g1 = [&](unsigned ks, unsigned buf, int i) {
        if (i < 4) __builtin_amdgcn_global_load_lds((const GAS unsigned*)(stgA + (size_t)ks * A_STEP + i * 1024 + lo), (LAS unsigned*)(lds + buf + wid * 4096 + i * 1024), 16, 0, 17);
        else __builtin_amdgcn_global_load_lds((const GAS unsigned*)(stgB + (size_t)ks * B_STEP + (i - 4) * 1024 + lo), (LAS unsigned*)(lds + buf + A_STEP + wid * 2048 + (i - 4) * 1024), 16, 0, 17);
    };
#define GETKS(slot) ((unsigned)UNI(ctl[388 + (slot)]) & 63u)

    f32x4 acc[4][4];
#pragma unroll
    for (int m = 0; m < 4; ++m)
#pragma unroll
        for (int n = 0; n < 4; ++n) acc[m][n] = (f32x4){0.f, 0.f, 0.f, 0.f};
    h16x8 a0[4], b0[4], a1[4], b1[4];
    const unsigned aoff = wm * 8192 + lo, boff = A_STEP + wn * 8192 + lo;
#define RDA(A, buf, kk, m) A[m] = *(const LAS h16x8*)(lds + (buf) + aoff + (m) * 2048 + (kk) * 1024)
#define RDB(B, buf, kk, n) B[n] = *(const LAS h16x8*)(lds + (buf) + boff + (n) * 2048 + (kk) * 1024)
#define ROW(A, B, m) _Pragma("unroll") for (int n = 0; n < 4; ++n) acc[m][n] = __builtin_amdgcn_mfma_f32_16x16x32_f16(A[m], B[n], acc[m][n], 0, 0, 0)

    STAMP(10);
    unsigned kq;
    pick(0); WAIT_L0(); BAR(); kq = GETKS(0);
#pragma unroll
    for (int i = 0; i < 6; ++i) g1(kq, 0u, i);
    pick(1); WAIT_L0(); BAR(); kq = GETKS(1);
#pragma unroll
    for (int i = 0; i < 6; ++i) g1(kq, STAGE, i);
    pick(2); WAIT_L0(); BAR(); kq = GETKS(2);
#pragma unroll
    for (int i = 0; i < 3; ++i) g1(kq, 2 * STAGE, i);
    wait_vm<9>(); BAR();
#pragma unroll
    for (int i = 0; i < 4; ++i) { RDB(b0, 0u, 0, i); RDA(a0, 0u, 0, i); }
    unsigned cur = 0, prv = 2 * STAGE;
    for (int t = 0; t < NSTEP; ++t) {
        const unsigned nxt = cur == 2 * STAGE ? 0u : cur + STAGE;
        const bool m2 = t + 2 < NSTEP, m3 = t + 3 < NSTEP;
        WAIT_L0(); SB();
        RDB(b1, cur, 1, 0); RDB(b1, cur, 1, 1); SB(); ROW(a0, b0, 0); SB(); if (m2) g1(kq, prv, 3); SB();
        RDB(b1, cur, 1, 2); RDB(b1, cur, 1, 3); SB(); ROW(a0, b0, 1); SB(); if (m2) g1(kq, prv, 4); SB();
        RDA(a1, cur, 1, 0); RDA(a1, cur, 1, 1); SB(); ROW(a0, b0, 2); SB(); if (m2) g1(kq, prv, 5); SB();
        RDA(a1, cur, 1, 2); RDA(a1, cur, 1, 3); SB(); ROW(a0, b0, 3); SB();
        if (m3) pick((t + 3) & 3);
        WAIT_L0();
        if (m2) wait_vm<6>(); else wait_vm<0>();
        BAR(); SB();
        const unsigned kv = ctl[388 + ((t + 3) & 3)];
        RDB(b0, nxt, 0, 0); RDB(b0, nxt, 0, 1); SB(); ROW(a1, b1, 0); SB();
        kq = (unsigned)UNI(kv) & 63u;
        if (m3) g1(kq, cur, 0); SB();
        RDB(b0, nxt, 0, 2); RDB(b0, nxt, 0, 3); SB(); ROW(a1, b1, 1); SB(); if (m3) g1(kq, cur, 1); SB();
        RDA(a0, nxt, 0, 0); RDA(a0, nxt, 0, 1); SB(); ROW(a1, b1, 2); SB(); if (m3) g1(kq, cur, 2); SB();
        RDA(a0, nxt, 0, 2); RDA(a0, nxt, 0, 3); SB(); ROW(a1, b1, 3); SB();
        prv = cur; cur = nxt;
    }

    const unsigned tmo_v = __hip_atomic_load(tmo, RLX, SC_AGENT);
    const unsigned oo = (unsigned)(fq * 4) * (GN * 4u) + fr * 16u;
    const u32x4 nan4 = {0x7fc00000u, 0x7fc00000u, 0x7fc00000u, 0x7fc00000u};
    unsigned y[64];
#pragma unroll
    for (int i = 0; i < 32; ++i) {
        const int m = i >> 3, n = (i >> 1) & 3, j = (i & 1) * 2;
        const f32x2 g = gelu2((f32x2){acc[m][n][j], acc[m][n][j + 1]});
        y[(m * 4 + j) * 4 + n] = __float_as_uint(g.x); y[(m * 4 + j + 1) * 4 + n] = __float_as_uint(g.y);
    }
    if (late) {
        const bool bad = UNI(tmo_v) == call;
        const char* ob = (const char*)(p.out + (size_t)(rb_local(rank, rb) + wm * 64) * GN + (rank >> 1) * 4096 + cb * 128 + wn * 64);
#pragma unroll
        for (int i = 0; i < 16; ++i) st16(ob + (size_t)((i >> 2) * 16 + (i & 3)) * (GN * 4), oo, bad ? nan4 : (u32x4){y[4 * i], y[4 * i + 1], y[4 * i + 2], y[4 * i + 3]});
    } else {
        const int peer = rb_owner(rank, rb);
        const int mirror = (8 + rb_of(peer, rank) * 4 + (idx & 3)) * 8 + xcd;
        char* const peerws = p.ws[peer];
        {
            const char* ex = myws + WS_EXO + (size_t)(blockIdx.x * 8 + wid) * 16384;
            unsigned hdr = 0;
            if (PACK_OUT) {
                float mx = 0.f;
#pragma unroll
                for (int i = 0; i < 64; i += 2) mx = __builtin_fmaxf(__builtin_fmaxf(mx, __builtin_fabsf(__builtin_bit_cast(float, y[i]))), __builtin_fabsf(__builtin_bit_cast(float, y[i + 1])));
                const unsigned emax = wave_max_exp<8, 23>(__builtin_bit_cast(unsigned, mx));
                const unsigned base = emax > 15u ? emax - 15u : 0u, bb = base << 23;
                hdr = base;
#pragma unroll
                for (int hf = 0; hf < 2; ++hf) {
                    unsigned z[28];
#pragma unroll
                    for (int g4 = 0; g4 < 4; ++g4) {
                        const int g = hf * 4 + g4;
                        unsigned dd[8], any = 0;
#pragma unroll
                        for (int i = 0; i < 8; ++i) { dd[i] = y[8 * g + i] - bb; any |= dd[i]; }
                        if (__all(!(any & 0x78000000u))) {
                            const unsigned cc = (dd[7] & 0x07ffffffu) | ((dd[7] >> 4) & 0x08000000u);
#pragma unroll
                            for (int i = 0; i < 7; ++i) z[7 * g4 + i] = dd[i] | ((cc << (27 - 4 * i)) & 0x78000000u);
                        } else {
#pragma unroll
                            for (int i = 0; i < 7; ++i) z[7 * g4 + i] = y[8 * g + i];
                            st4(ex + 14336 + g * 256, lane * 4u, y[8 * g + 7]);
                            hdr |= 256u << g;
                        }
                    }
#pragma unroll
                    for (int i = 0; i < 7; ++i) st16(ex + (hf * 7 + i) * 1024, lo, (u32x4){z[4 * i], z[4 * i + 1], z[4 * i + 2], z[4 * i + 3]});
                }
            } else {
#pragma unroll
                for (int i = 0; i < 16; ++i) st16(ex + i * 1024, lo, (u32x4){y[4 * i], y[4 * i + 1], y[4 * i + 2], y[4 * i + 3]});
            }
            wait_vm<0>();
            hdr |= UNI(tmo_v) == call ? 0xc0000000u : 0x40000000u;
            if (lane == 0) __hip_atomic_store((GAS u64*)(peerws + WS_ORDY + (size_t)mirror * 128) + wid, ((u64)call << 32) | hdr, RLX, SC_SYS);
        }
        WAIT_L0(); BAR();
        STAMP(22);
        const char* exs = peerws + WS_EXO + (size_t)mirror * (8 * 16384);
        constexpr int HALF = OPS_OUT / 2;
        unsigned raw[4];
        auto get_raw = [&](int blk, int hf, unsigned hdr) {
#pragma unroll
            for (int g4 = 0; g4 < 4; ++g4) {
                raw[g4] = 0u;
                if (PACK_OUT && (hdr & (256u << (hf * 4 + g4)))) raw[g4] = ld4(exs + (size_t)blk * 16384 + 14336 + (hf * 4 + g4) * 256, lane * 4u);
            }
        };
        auto unpack = [&](int blk, int hf, unsigned hdr) {
            const LAS unsigned char* src = lds + (unsigned)blk * 16384u + hf * (HALF * 1024) + lo;
            unsigned z[32], o[32];
#pragma unroll
            for (int i = 0; i < HALF; ++i) { const u32x4 a = *(const LAS u32x4*)(src + i * 1024); z[4 * i] = a[0]; z[4 * i + 1] = a[1]; z[4 * i + 2] = a[2]; z[4 * i + 3] = a[3]; }
            const unsigned bb = (hdr & 255u) << 23;
#pragma unroll
            for (int g4 = 0; g4 < 4; ++g4) {
                if (!PACK_OUT) {
#pragma unroll
                    for (int i = 0; i < 8; ++i) o[8 * g4 + i] = z[8 * g4 + i];
                } else if (!(hdr & (256u << (hf * 4 + g4)))) {
                    unsigned cc = 0;
#pragma unroll
                    for (int i = 0; i < 7; ++i) { cc |= (z[7 * g4 + i] & 0x78000000u) >> (27 - 4 * i); o[8 * g4 + i] = (z[7 * g4 + i] & 0x87ffffffu) + bb; }
                    o[8 * g4 + 7] = ((cc & 0x07ffffffu) | ((cc << 4) & 0x80000000u)) + bb;
                } else {
#pragma unroll
                    for (int i = 0; i < 7; ++i) o[8 * g4 + i] = z[7 * g4 + i];
                    o[8 * g4 + 7] = raw[g4];
                }
            }
            const char* ob = (const char*)(p.out + (size_t)((peer & 1) * 256 + (blk >> 1) * 64 + hf * 32) * GN + (peer >> 1) * 4096 + cb * 128 + (blk & 1) * 64);
#pragma unroll
            for (int i = 0; i < 8; ++i)
                st16(ob + (size_t)((i >> 2) * 16 + (i & 3)) * (GN * 4), oo, (hdr >> 31) ? nan4 : (u32x4){o[4 * i], o[4 * i + 1], o[4 * i + 2], o[4 * i + 3]});
        };
        if (wid == 7) {
            unsigned tnext;
            while ((tnext = lds_get(ctl + 8)) == 0u) __builtin_amdgcn_s_sleep(2);
            STAMP(23);
            wait_vm<0>(); __builtin_amdgcn_s_waitcnt(0x0F70);
            for (int n = 0; n < 8; ++n) {
                if (lds_get(ctl + n) == 0u) {
                    wait_vm<0>();
                    if (lane == 0) __hip_atomic_store(ctl + 384, (unsigned)n, RLX, SC_WG);
                    while (lds_get(ctl + n) == 0u) __builtin_amdgcn_s_sleep(1);
                }
                asm volatile("" ::: "memory");
                if (n == 7 && lane == 0) __hip_atomic_store(ctl + 385, (tnext + OPS_OUT * TAU) | 1u, RLX, SC_WG);
#pragma unroll
                for (int i = 0; i < OPS_OUT; i += 2) {
                    wait_vm<WIN - 2>();
                    if (i == WIN - 2 && lane == 0) __hip_atomic_store(ctl + 384, (unsigned)n, RLX, SC_WG);
                    pace(tnext, 2u * TAU);
                    dma<true>(exs + (size_t)n * 16384 + i * 1024, lo, lds0 + (unsigned)n * 16384u + i * 1024);
                }
            }
            wait_vm<HALF>();
            if (lane == 0) __hip_atomic_store(ctl + 384, 8u, RLX, SC_WG);
            wait_vm<0>();
            if (lane == 0) __hip_atomic_store(ctl + 384, 9u, RLX, SC_WG);
            STAMP(19);
        } else {
            if (wid == 6) {
                const GAS u64* fp = lane < 8 ? (const GAS u64*)(myws + WS_ORDY + (size_t)blockIdx.x * 128) + lane : (const GAS u64*)(myws + WS_MBOX + (size_t)blockIdx.x * 128);
                bool pend = lane < 9;
                const u64 t0 = __builtin_amdgcn_s_memrealtime();
                for (unsigned polls = 1;; ++polls) {
                    if (pend) {
                        const u64 f = __hip_atomic_load(fp, RLX, SC_SYS);
                        if ((unsigned)(f >> 32) >= call) { __hip_atomic_store(ctl + lane, (unsigned)f, RLX, SC_WG); pend = false; }
                    }
                    if (!__any(pend)) break;
                    if ((polls & 15u) == 0u && TIMED_OUT(t0)) {
                        if (lane == 0) __hip_atomic_store(tmo, call, RLX, SC_AGENT);
                        if (pend) __hip_atomic_store(ctl + lane, lane < 8 ? 0xc0000000u : now8() | 1u, RLX, SC_WG);
                        break;
                    }
                    __builtin_amdgcn_s_sleep(4);
                }
            }
            for (int job = 0; job < (wid == 1 || wid == 2 ? 3 : 2); ++job) {
                const int blk = job < 2 ? wid : 7, hf = job < 2 ? job : wid - 1, need = job < 2 ? wid : 6 + wid;
                unsigned hdr;
                while ((hdr = lds_get(ctl + blk)) == 0u) __builtin_amdgcn_s_sleep(4);
                get_raw(blk, hf, hdr);
                while ((int)lds_get(ctl + 384) <= need) __builtin_amdgcn_s_sleep(1);
                asm volatile("" ::: "memory");
                unpack(blk, hf, hdr);
            }
            if (wid == 0 && (idx & 3) != 3) {
                unsigned th;
                while ((th = lds_get(ctl + 385)) == 0u) __builtin_amdgcn_s_sleep(4);
                if (lane == 0) __hip_atomic_store((GAS u64*)(myws + WS_MBOX + (size_t)(blockIdx.x + 8) * 128), ((u64)call << 32) | th, RLX, SC_AGENT);
            }
        }
    }
    }
    if (ntile == 0) BAR();
    if (tid == 0) __hip_atomic_store(ctr, call, RLX, SC_AGENT);
}

extern "C" void kernel_launch(int rank, int nranks, ncclComm_t comm, hipStream_t stream,
                              void* const* d_in, const int* in_sizes, int n_in,
                              void* d_out, int out_size, const int* in_dtypes, int out_dtype,
                              void* const* d_out_all, void* const* d_ws_all, size_t ws_bytes) {
    (void)comm; (void)in_dtypes; (void)out_dtype; (void)d_out_all;
    if (nranks != NR || (unsigned)rank >= (unsigned)NR || n_in != 2 || ws_bytes < WS_NEED ||
        in_sizes[0] != 4096 * KSH || in_sizes[1] != KSH * GN || out_size != 512 * GN) {
        fprintf(stderr, "kernel_launch: unexpected problem shape\n");
        return;
    }
    Params p;
    p.x = (const float*)d_in[0];
    p.w = (const float*)d_in[1];
    p.out = (float*)d_out;
    p.myws = (char*)d_ws_all[rank];
    for (int i = 0; i < NR; ++i) p.ws[i] = (char*)d_ws_all[i];
    p.rank = rank;
    p.pad = 0;
    int n = 0, np = 0, nd = 0;
    unsigned char deferred[8];
    bool px[8] = {};
    for (int i = 0; i < 128; ++i) p.order[0][i] = p.order[1][i] = 0;
    auto need_x = [&](int c0, int c1) { for (int c = c0; c <= c1; ++c) if (!px[c]) { px[c] = true; p.prod[np++] = (unsigned)(32 + c); } };
    for (int key = 0; key < 16 + 3 * SKEW2; ++key) {
        for (int pass = 0; pass < 2; ++pass) {
            if ((pass == 0) == (key != 0)) {
                for (int e = 32; e < 44; ++e)
                    if (2 * ((e - 32) * 8 / 12) == key) { p.order[0][n++] = (unsigned char)e; need_x((e - 32) * 8 / 12, ((e - 32) * 8 + 7) / 12); }
                if (key == XPOS) { p.order[0][n++] = 44; need_x(0, 3); }
                if (key == XPOS + 2) { p.order[0][n++] = 45; need_x(4, 7); }
            } else {
                for (int b = 0; b < 4; ++b)
                    for (int c = 0; c < 8; ++c)
                        if (2 * c + b * SKEW2 == key) {
                            p.prod[np++] = (unsigned)(c * 4 + b);
                            if (b == 3 && c < 2 * NFWD && ((c ^ rank) & 1)) deferred[nd++] = (unsigned char)(c * 4 + b);
                            else p.order[0][n++] = (unsigned char)(c * 4 + b);
                        }
            }
        }
    }
    int m = 0;
    for (; m < n; ++m) p.order[1][m] = p.order[0][m];
    for (int i = 0; i < nd; ++i) p.order[1][m++] = deferred[i];
    for (int i = 0; i < 6 * NFWD; ++i) p.order[1][m++] = (unsigned char)(48 + i);
    for (int e = 46; e < 48; ++e) { p.order[0][n++] = (unsigned char)e; p.order[1][m++] = (unsigned char)e; }
    if (n != NU_OUT || m != NU_PAIR || np != 40 || p.prod[0] != 0 || p.prod[1] != 32) { fprintf(stderr, "kernel_launch: bad schedule\n"); return; }
    hipLaunchKernelGGL(fused_gemm_rs, dim3(NBLK), dim3(NTHR), 0, stream, p);
    HIP_CHECK(hipGetLastError());
}
```

```cpp
#include <hip/hip_runtime.h>
#include <rccl/rccl.h>
#include <cstdio>

#define HIP_CHECK(expr) do { hipError_t _e = (expr); if (_e != hipSuccess) { \
    fprintf(stderr, "HIP error %s at %s:%d\n", hipGetErrorString(_e), __FILE__, __LINE__); \
    return; } } while (0)

typedef float f32x4 __attribute__((ext_vector_type(4)));
typedef unsigned u32x4 __attribute__((ext_vector_type(4)));
typedef unsigned u32x2 __attribute__((ext_vector_type(2)));
typedef unsigned short u16x2 __attribute__((ext_vector_type(2)));
typedef _Float16 h16x8 __attribute__((ext_vector_type(8)));
typedef unsigned long long u64;
#define LAS __attribute__((address_space(3)))
#define GAS __attribute__((address_space(1)))
#define RLX __ATOMIC_RELAXED
#define SC_WG __HIP_MEMORY_SCOPE_WORKGROUP
#define SC_AGENT __HIP_MEMORY_SCOPE_AGENT
#define SC_SYS __HIP_MEMORY_SCOPE_SYSTEM

#define PACK_OUT 1
#define TAU 864
#define EQ_AT 5
#define CREDIT 30
#define WIN 10
#define PROD_GAP 40
#define NPUSH 1
#define PUSH_SLOT 112
#define PULL_DELAY 30
#define RSTRIDE 16
#define POLLW 12
#define GSC SC_AGENT
#define PROD_OFF -2
#define POLL_SLEEP 8
#define NFWD 3
#define XPOS 6
#define SKEW2 2

constexpr int NR = 8;
constexpr int GN = 16384;
constexpr int KSH = 512;
constexpr int NBLK = 256;
constexpr int NTHR = 512;
constexpr int NSTEP = 64;
constexpr unsigned A_STEP = 32768, B_STEP = 16384, STAGE = A_STEP + B_STEP;
constexpr int NU_OUT = 48 - NFWD, NU_PAIR = 48 + 6 * NFWD;
constexpr int RING = 8;
constexpr int QB = 3648;
constexpr int OPS_OUT = PACK_OUT ? 14 : 16;
constexpr unsigned UNIT_T = 57 * TAU / 4;
constexpr unsigned SLOT = 18432, SIDE = 16384;
constexpr size_t WS_CTR = 0;
constexpr size_t WS_FLAG = 32 << 10;
constexpr size_t WS_ORDY = 96 << 10;
constexpr size_t WS_MBOX = 128 << 10;
constexpr size_t WS_EQ = 160 << 10;
constexpr size_t WS_TMO = 192 << 10;
constexpr size_t WS_DIAG = 256 << 10;
constexpr size_t WS_STG = 1 << 20;
constexpr size_t STG_A = 8ull * NSTEP * A_STEP;
constexpr size_t STG_B = 32ull * NSTEP * B_STEP;
constexpr size_t WS_EXI = WS_STG + STG_A + STG_B;
constexpr size_t EXI_BYTES = 1600ull * SLOT;
constexpr size_t WS_EXO = WS_EXI + 2 * EXI_BYTES;
constexpr size_t WS_RDY = WS_EXO + (size_t)NBLK * 8 * 16384;
constexpr size_t WS_NEED = WS_RDY + 2 * 8 * 48 * 8 * 128;
constexpr u64 DEADLINE_TICKS = 50000000ull;

struct Params {
    const float* x;
    const float* w;
    float* out;
    char* myws;
    char* ws[NR];
    int rank;
    int pad;
    unsigned prod[40];
    unsigned char order[2][128];
};

#define WAIT_L0() do { asm volatile("s_waitcnt lgkmcnt(0)" ::: "memory"); __builtin_amdgcn_s_waitcnt(0xC07F); } while (0)
#define BAR() do { asm volatile("" ::: "memory"); __builtin_amdgcn_s_barrier(); asm volatile("" ::: "memory"); } while (0)
#define SB() __builtin_amdgcn_sched_barrier(0)
#define UNI(x) __builtin_amdgcn_readfirstlane(x)
template <int N> __device__ __forceinline__ void wait_vm() { asm volatile("s_waitcnt vmcnt(%0)" ::"i"(N) : "memory"); }

__device__ __forceinline__ void st16(const void* base, unsigned voff, u32x4 v) {
    asm volatile("s_nop 4\n\tglobal_store_dwordx4 %0, %1, %2 sc0 sc1\n\ts_nop 1" ::"v"(voff), "v"(v), "s"(base) : "memory");
}
__device__ __forceinline__ void st8(const void* base, unsigned voff, u32x2 v) {
    asm volatile("s_nop 4\n\tglobal_store_dwordx2 %0, %1, %2 sc0 sc1" ::"v"(voff), "v"(v), "s"(base) : "memory");
}
__device__ __forceinline__ void st4(const void* base, unsigned voff, unsigned v) {
    asm volatile("s_nop 4\n\tglobal_store_dword %0, %1, %2 sc0 sc1" ::"v"(voff), "v"(v), "s"(base) : "memory");
}
__device__ __forceinline__ unsigned ld4(const void* base, unsigned voff) {
    unsigned r;
    asm volatile("s_nop 4\n\tglobal_load_dword %0, %1, %2 sc0 sc1\n\ts_waitcnt vmcnt(0)" : "=&v"(r) : "v"(voff), "s"(base) : "memory");
    return r;
}
__device__ __forceinline__ u32x2 ld8(const void* base, unsigned voff) {
    u32x2 r;
    asm volatile("s_nop 4\n\tglobal_load_dwordx2 %0, %1, %2 sc0 sc1\n\ts_waitcnt vmcnt(0)" : "=&v"(r) : "v"(voff), "s"(base) : "memory");
    return r;
}
template <bool TWO> __device__ __forceinline__ void dma(const void* base, unsigned voff, unsigned lds_dst) {
    unsigned keep;
    if (TWO) asm volatile("s_mov_b32 %0, m0\n\ts_mov_b32 m0, %3\n\ts_nop 4\n\tglobal_load_lds_dwordx4 %1, %2 sc0 sc1\n\tglobal_load_lds_dwordx4 %1, %2 offset:1024 sc0 sc1\n\ts_mov_b32 m0, %0"
                          : "=&s"(keep) : "v"(voff), "s"(base), "s"(lds_dst) : "memory");
    else asm volatile("s_mov_b32 %0, m0\n\ts_mov_b32 m0, %3\n\ts_nop 4\n\tglobal_load_lds_dwordx4 %1, %2 sc0 sc1\n\ts_mov_b32 m0, %0"
                      : "=&s"(keep) : "v"(voff), "s"(base), "s"(lds_dst) : "memory");
}
__device__ __forceinline__ unsigned now8() { return (unsigned)__builtin_amdgcn_s_memrealtime() << 6; }
__device__ __forceinline__ void pace(unsigned& t, unsigned period) {
    int late = (int)(now8() - t);
    if (late < -(1 << 22)) { t += (unsigned)late; late = 0; }
    while (late < 0) { __builtin_amdgcn_s_sleep(1); late = (int)(now8() - t); }
    t += period + (late > CREDIT * 64 ? (unsigned)(late - CREDIT * 64) : 0u);
}
__device__ __forceinline__ unsigned lds_get(const LAS unsigned* a) { return UNI(__hip_atomic_load(a, RLX, SC_WG)); }
__device__ __forceinline__ unsigned cvt2(float a, float b) {
    typedef _Float16 h2 __attribute__((ext_vector_type(2)));
    h2 r = {(_Float16)a, (_Float16)b};
    return __builtin_bit_cast(unsigned, r);
}
__device__ __forceinline__ int rb_owner(int d, int rb) { return rb >= 6 ? d : ((((d >> 1) + 1 + (rb >> 1)) & 3) << 1) | (rb & 1); }
__device__ __forceinline__ int rb_local(int d, int rb) { return rb >= 6 ? (rb - 6) * 256 : (d & 1) * 256; }
__device__ __forceinline__ int rb_of(int d, int o) { return ((((o >> 1) - (d >> 1) - 1) & 3) << 1) | (o & 1); }
__device__ __forceinline__ void unit_of(int e, int j, int& c, int& id) {
    if (e < 32) { c = e >> 2; id = j * 4 + (e & 3); }
    else if (e < 44) { const int u = (e - 32) * 8 + j; c = u / 12; id = 32 + u - c * 12; }
    else { const int u = (e & 1) * 8 + j; c = u >> 1; id = (e < 46 ? 46 : 44) + (u & 1); }
}
__device__ __forceinline__ int gid_of(int d, int id) {
    const int rb = (id - 32) >> 1;
    return id < 32 ? (d >> 1) * 32 + id : 128 + rb_owner(d, rb) * 4 + (rb_local(d, rb) >> 7) + (id & 1);
}
__device__ __forceinline__ size_t stage_off(int id, int ks) {
    return id < 32 ? STG_A + (size_t)(id * NSTEP + ks) * B_STEP : (size_t)(((id - 32) >> 1) * NSTEP + ks) * A_STEP + (id & 1) * 16384;
}
template <int BITS, int SH> __device__ __forceinline__ unsigned wave_max_exp(unsigned mag) {
    unsigned e = 0;
#pragma unroll
    for (int b = BITS - 1; b >= 0; --b) if (__any(mag >= ((e + (1u << b)) << SH))) e += 1u << b;
    return e;
}

__device__ __forceinline__ unsigned lane_prefix(unsigned cnt, unsigned& total) {
    unsigned pre = 0; total = 0;
#pragma unroll
    for (int b = 0; b < 4; ++b) {
        const u64 m = __ballot((cnt >> b) & 1u);
        pre += __builtin_amdgcn_mbcnt_hi((unsigned)(m >> 32), __builtin_amdgcn_mbcnt_lo((unsigned)m, 0u)) << b;
        total += (unsigned)__builtin_popcountll(m) << b;
    }
    return pre;
}
#define PK(x) __builtin_bit_cast(u16x2, (unsigned)(x))
#define DW(x) __builtin_bit_cast(unsigned, (u16x2)(x))

typedef float f32x2 __attribute__((ext_vector_type(2)));
__device__ __forceinline__ f32x2 gelu2(f32x2 y) {
    const float k1 = -2.0f * 0.7978845608028654f * 1.4426950408889634f, k3 = k1 * 0.044715f;
    const f32x2 a = __builtin_elementwise_fma(y * y, (f32x2){k3, k3}, (f32x2){k1, k1}) * y;
    const f32x2 d = (f32x2){__builtin_amdgcn_exp2f(a.x), __builtin_amdgcn_exp2f(a.y)} + 1.0f;
    return y * (f32x2){__builtin_amdgcn_rcpf(d.x), __builtin_amdgcn_rcpf(d.y)};
}
__device__ __forceinline__ float gelu1(float y) { return gelu2((f32x2){y, y}).x; }

#define STAMP(i) do {} while (0)
#define TIMED_OUT(t0) (UNI(__hip_atomic_load(tmo, RLX, SC_AGENT)) == call || __builtin_amdgcn_s_memrealtime() - (t0) > DEADLINE_TICKS)

__global__ __launch_bounds__(NTHR, 2) void fused_gemm_rs(Params p) {
    __shared__ __attribute__((aligned(16))) unsigned char lds_raw[3 * STAGE + 2048];
    LAS unsigned char* lds = (LAS unsigned char*)lds_raw;
    LAS unsigned* ctl = (LAS unsigned*)(lds + 3 * STAGE);
    const unsigned lds0 = (unsigned)(size_t)lds;
    const int tid = threadIdx.x, lane = tid & 63;
    const int wid = UNI(tid >> 6);
    const int fr = lane & 15, fq = lane >> 4;
    const unsigned lo = lane * 16u;
    const int xcd = blockIdx.x & 7, idx = blockIdx.x >> 3;
    const bool late = idx < 8;
    const int q = wid & 3;

    const int team = (wid >> 2) * 192 + (int)blockIdx.x - 64;
    f32x4 r[8];
    auto request = [&](int g, int u) {
        if (g < 32) {
            const GAS char* b = (const GAS char*)(p.w + (size_t)((g >> 2) * 64 + (q & 1) * 32) * GN + (u * 4 + (g & 3)) * 128 + (q >> 1) * 64) + (unsigned)(8 * fq * GN + 4 * fr) * 4u;
#pragma unroll
            for (int e = 0; e < 8; ++e) r[e] = *(const GAS f32x4*)(b + (size_t)e * (GN * 4));
        } else {
            const GAS char* b = (const GAS char*)(p.x + (size_t)(u * 128 + q * 32) * KSH + (g - 32) * 64) + (unsigned)(fr * KSH + 8 * fq) * 4u;
#pragma unroll
            for (int e = 0; e < 8; ++e) r[e] = *(const GAS f32x4*)(b + (e >> 2) * (16 * KSH * 4) + ((e >> 1) & 1) * 128 + (e & 1) * 16);
        }
    };
    if (!late && team < 96) request(team >= 64 ? 32 : (team >> 5) * 4, team & 31);
    const unsigned t_entry = now8();

    const int rank = p.rank & 7;
    char* myws = p.myws;
    STAMP(0);
    GAS unsigned* ctr = (GAS unsigned*)(myws + WS_CTR + (size_t)blockIdx.x * 128);
    GAS unsigned* tmo = (GAS unsigned*)(myws + WS_TMO);
    const unsigned ctr_v = __hip_atomic_load(ctr, RLX, SC_AGENT);
    GAS unsigned* gflag = (GAS unsigned*)(myws + WS_FLAG);
    ctl[tid] = 0u;
    WAIT_L0();
    BAR();

    if (!late) {
        unsigned wlo = 0, whi = 0;
#pragma unroll
        for (int d = 0; d < NR; ++d) {
            unsigned a = (unsigned)(size_t)p.ws[d], b = (unsigned)((size_t)p.ws[d] >> 32);
            asm volatile("" : "+s"(a), "+s"(b));
            wlo = lane == d ? a : wlo;
            whi = lane == d ? b : whi;
        }
#pragma unroll 1
        for (int it = 0; it < 4; ++it) {
            const int t = it * 384 + team;
            if (t >= 1280) break;
            const int g = p.prod[t >> 5], u = t & 31;
            if (t >= 96) {
                const unsigned ts = t_entry + (unsigned)((t >> 5) - PROD_OFF) * (PROD_GAP * 64);
                while ((int)(now8() - ts) < 0) __builtin_amdgcn_s_sleep(2);
                request(g, u);
            }
            const bool isw = g < 32;
            const int c = isw ? g >> 2 : g - 32;
            const int gid = isw ? u * 4 + (g & 3) : 128 + u;
            unsigned v[16];
            if (isw) {
#pragma unroll
                for (int i = 0; i < 16; ++i) v[i] = cvt2(r[(i & 3) * 2][i >> 2], r[(i & 3) * 2 + 1][i >> 2]);
            } else {
#pragma unroll
                for (int i = 0; i < 16; ++i) v[i] = cvt2(r[i >> 1][(i & 1) * 2], r[i >> 1][(i & 1) * 2 + 1]);
            }
            asm volatile("" : "+v"(v[15]));
            const unsigned t_loaded = now8();
            const bool pushed = t < 32 * NPUSH;
            const unsigned call = UNI(ctr_v) + 1u;
            const size_t par = WS_EXI + (size_t)(call & 1u) * EXI_BYTES;
            const int o = u >> 2;
            const int myid = isw ? ((gid >> 5) == (rank >> 1) ? gid & 31 : -1)
                                 : (o == rank ? 44 + (u & 3) : ((o >> 1) != (rank >> 1) && ((u >> 1) & 1) == (rank & 1) ? 32 + rb_of(rank, o) * 2 + (u & 1) : -1));
            unsigned hdr = 0;
            {
                unsigned w[16];
#pragma unroll
                for (int i = 0; i < 16; ++i) w[i] = v[i];
                const char* home = myws + par + (size_t)(gid * 8 + c) * SLOT;
                unsigned pw = 0;
                hdr = 32u;
                if (!pushed) {
                    LAS unsigned* pool = (LAS unsigned*)(lds + wid * 128);
                    if (lane < 32) pool[lane] = 0u;
                    u16x2 mx = PK(w[0] & 0x7c007c00u);
#pragma unroll
                    for (int i = 1; i < 16; ++i) mx = __builtin_elementwise_max(mx, PK(w[i] & 0x7c007c00u));
                    const unsigned emax = wave_max_exp<5, 10>(mx.x > mx.y ? mx.x : mx.y);
                    const unsigned B = emax > 7u ? emax - 7u : 0u, bb = B * 0x04000400u;
                    unsigned str = 0, sh = 0, big = 0;
#pragma unroll
                    for (int i = 0; i < 16; ++i) {
                        const unsigned ef = w[i] & 0x7c007c00u;
                        const unsigned code = DW(__builtin_elementwise_sub_sat(PK(ef), PK(bb))), nib = DW(__builtin_elementwise_sub_sat(PK(bb), PK(ef)));
                        const unsigned step = DW(__builtin_elementwise_sub_sat(PK(0x00040004u), PK(code)));
                        big |= nib;
                        str |= ((nib >> 10) & 63u) << (sh & 31u); sh += step & 7u;
                        str |= (nib >> 26) << (sh & 31u); sh += step >> 16;
                        w[i] = (w[i] & 0x83ff83ffu) | code;
                    }
                    unsigned total;
                    const unsigned pos = lane_prefix(sh >> 2, total);
                    if (__all(sh <= 32u && !(big & 0x40004000u)) && total <= 128u) {
                        if (sh) {
                            const u64 sv = (u64)str << ((pos & 7u) * 4u);
                            __hip_atomic_fetch_or(pool + (pos >> 3), (unsigned)sv, RLX, SC_WG);
                            __hip_atomic_fetch_or(pool + (pos >> 3) + 1, (unsigned)(sv >> 32), RLX, SC_WG);
                        }
#pragma unroll
                        for (int k = 0; k < 2; ++k) {
                            const unsigned cc = (w[14 + k] & 0x1fff1fffu) | ((w[14 + k] >> 2) & 0x20002000u);
#pragma unroll
                            for (int i = 0; i < 7; ++i) w[7 * k + i] |= (cc << (13 - 2 * i)) & 0x60006000u;
                        }
                        pw = __hip_atomic_load(pool + (lane & 15), RLX, SC_WG);
                        hdr = B;
                    }
                }
                if (hdr == 32u) {
#pragma unroll
                    for (int i = 0; i < 14; ++i) w[i] = v[i];
                }
                if (pushed) {
                    unsigned tp = t_loaded + (unsigned)((t >> 5) * 128 + (u & 7) * 4 + q) * PUSH_SLOT;
#pragma unroll
                    for (int i = 0; i < 4; ++i) {
                        pace(tp, 32u * PUSH_SLOT);
#pragma unroll
                        for (int k = 0; k < 2; ++k) {
                            const int d = (u >> 3) * 2 + k;
                            if (d != rank) st16(p.ws[d] + par + (size_t)(1280 + ((t >> 5) * 8 + rank) * 8 + (u & 7)) * SLOT + q * 4096 + i * 1024, lo, (u32x4){v[4 * i], v[4 * i + 1], v[4 * i + 2], v[4 * i + 3]});
                        }
                    }
                } else {
#pragma unroll
                    for (int i = 0; i < 3; ++i) st16(home + q * QB + i * 1024, lo, (u32x4){w[4 * i], w[4 * i + 1], w[4 * i + 2], w[4 * i + 3]});
                    st8(home + q * QB + 3072, lane * 8u, (u32x2){w[12], w[13]});
                    if (hdr == 32u) st8(home + SIDE + q * 512, lane * 8u, (u32x2){v[14], v[15]});
                    else if (lane < 16) st4(home + q * QB + 3584, lane * 4u, pw);
                }
            }
            if (myid >= 0) {
                const char* sg = myws + WS_STG + stage_off(myid, rank * 8 + c) + (isw ? (q >> 1) * 8 + (q & 1) : q * 4) * 1024;
#pragma unroll
                for (int i = 0; i < 4; ++i) st16(sg + i * (isw ? 2048 : 1024), lo, (u32x4){v[4 * i], v[4 * i + 1], v[4 * i + 2], v[4 * i + 3]});
            }
            wait_vm<0>();
            unsigned tw = (1u << 24) | (hdr << (6 * q));
            if (lane == 0) tw += __hip_atomic_fetch_add(ctl + 392 + (wid >> 2) * 4 + it, tw, RLX, SC_WG);
            tw = UNI(tw);
            if ((tw >> 24) == 4u) {
                const int d = lane & 7;
                int did = isw ? ((gid >> 5) == (d >> 1) ? gid & 31 : -1)
                              : (o == d ? 44 + (u & 3) : ((o >> 1) != (d >> 1) && ((u >> 1) & 1) == (d & 1) ? 32 + rb_of(d, o) * 2 + (u & 1) : -1));
                if (isw && (g & 3) == 3 && c < 2 * NFWD && (d >> 1) != (rank >> 1) && ((c ^ d) & 1)) did = -1;
                if (lane < 8 && did >= 0) {
                    if (d == rank) __hip_atomic_store(gflag + did * 64 + c * 8 + rank, call, RLX, GSC);
                    else __hip_atomic_store((GAS u64*)((((size_t)whi << 32) | wlo) + WS_RDY) + ((((call & 1u) * 8 + rank) * 48 + did) * 8 + c) * RSTRIDE, ((u64)call << 32) | (tw & 0xffffffu), RLX, SC_SYS);
                }
            }
        }
        STAMP(9);
    } else if (idx < 7) {
        const int s = idx < 6 ? rb_owner(rank, idx) : rank ^ 1;
        const int nu = idx < 6 ? NU_OUT : NU_PAIR;
        unsigned desc[2];
#pragma unroll
        for (int k = 0; k < 2; ++k) {
            const int e = p.order[idx == 6][k * 64 + lane];
            int c, id, org = s, slot;
            if (e < 48) { unit_of(e, xcd, c, id); slot = gid_of(rank, id) * 8 + c; }
            else { const int kf = (e - 48) / 6; c = 2 * kf + (s & 1); id = xcd * 4 + 3; org = rb_owner(rank, e - 48 - kf * 6); slot = 1408 + (e - 48) * 8 + xcd; }
            desc[k] = (unsigned)(c | id << 3 | org << 9 | slot << 12);
        }
        auto desc_of = [&](int n) { return (unsigned)(n < 64 ? __builtin_amdgcn_readlane(desc[0], n) : __builtin_amdgcn_readlane(desc[1], n - 64)); };
        const unsigned call = UNI(ctr_v) + 1u;
        const size_t par = WS_EXI + (size_t)(call & 1u) * EXI_BYTES;
        if (wid < 4) {
            int pend = -1;
            auto staged = [&](int n) {
                wait_vm<0>();
                if (lane == 0 && __hip_atomic_fetch_add(ctl + 256 + n, 1u, RLX, SC_WG) == 3u) __hip_atomic_store(gflag + pend, call, RLX, GSC);
                pend = -1;
            };
            for (int n = 0; n < nu; ++n) {
                while ((int)lds_get(ctl + (n >= NPUSH ? 384 : 386)) <= n) __builtin_amdgcn_s_sleep(1);
                asm volatile("" ::: "memory");
                const int un = lds_get(ctl + 416 + n);
                const unsigned de = desc_of(un);
                const int c = de & 7, id = (de >> 3) & 63, org = (de >> 9) & 7;
                const LAS unsigned char* src = lds + (unsigned)(n % RING) * 16384u + q * (n < NPUSH ? 4096 : QB);
                unsigned w[16], h = 0;
#pragma unroll
                for (int i = 0; i < 3; ++i) { const u32x4 a = *(const LAS u32x4*)(src + i * 1024 + lo); w[4 * i] = a[0]; w[4 * i + 1] = a[1]; w[4 * i + 2] = a[2]; w[4 * i + 3] = a[3]; }
                { const u32x2 a = *(const LAS u32x2*)(src + 3072 + lane * 8); w[12] = a[0]; w[13] = a[1]; }
                h = (lds_get(ctl + un) >> (6 * q)) & 63u;
                if (h < 32u) {
#pragma unroll
                    for (int k = 0; k < 2; ++k) {
                        unsigned cc = 0;
#pragma unroll
                        for (int i = 0; i < 7; ++i) { cc |= (w[7 * k + i] & 0x60006000u) >> (13 - 2 * i); w[7 * k + i] &= 0x9fff9fffu; }
                        w[14 + k] = (cc & 0x1fff1fffu) | ((cc << 2) & 0x80008000u);
                    }
                    unsigned step[16], sum = 0;
#pragma unroll
                    for (int i = 0; i < 16; ++i) {
                        step[i] = DW(__builtin_elementwise_sub_sat(PK(0x00040004u), PK(w[i] & 0x1c001c00u)));
                        sum += step[i];
                    }
                    unsigned total;
                    const unsigned pos = lane_prefix(((sum & 0xffffu) + (sum >> 16)) >> 2, total);
                    const LAS unsigned* pool = (const LAS unsigned*)(src + 3584) + ((pos >> 3) & 15u);
                    unsigned str = (unsigned)((((u64)pool[1] << 32) | pool[0]) >> ((pos & 7u) * 4u));
                    WAIT_L0();
                    if (lane == 0) __hip_atomic_fetch_add(ctl + 128 + n, 1u, RLX, SC_WG);
                    if (pend >= 0) staged(n - 1);
                    const unsigned bb = h * 0x04000400u;
#pragma unroll
                    for (int i = 0; i < 16; ++i) {
                        const unsigned s1 = step[i] >> 16;
                        const unsigned n0 = __builtin_amdgcn_ubfe(str, 0u, step[i]); str >>= step[i] & 31u;
                        const unsigned n1 = __builtin_amdgcn_ubfe(str, 0u, s1); str >>= s1;
                        w[i] = w[i] + bb - (n0 << 10) - (n1 << 26);
                    }
                } else {
                    if (n < NPUSH) { const u32x4 a = *(const LAS u32x4*)(src + 3072 + lo); w[12] = a[0]; w[13] = a[1]; w[14] = a[2]; w[15] = a[3]; }
                    WAIT_L0();
                    if (lane == 0) __hip_atomic_fetch_add(ctl + 128 + n, 1u, RLX, SC_WG);
                    if (pend >= 0) staged(n - 1);
                    if (n >= NPUSH) {
                        const u32x2 a = ld8(p.ws[org] + par + (size_t)(gid_of(rank, id) * 8 + c) * SLOT + SIDE + q * 512, lane * 8u);
                        w[14] = a[0]; w[15] = a[1];
                    }
                }
                const char* sg = myws + WS_STG + stage_off(id, org * 8 + c) + (id < 32 ? (q >> 1) * 8 + (q & 1) : q * 4) * 1024;
#pragma unroll
                for (int i = 0; i < 4; ++i) st16(sg + i * (id < 32 ? 2048 : 1024), lo, (u32x4){w[4 * i], w[4 * i + 1], w[4 * i + 2], w[4 * i + 3]});
                pend = id * 64 + c * 8 + org;
                if (n + 1 == nu || (int)lds_get(ctl + (n + 1 >= NPUSH ? 384 : 386)) <= n + 1) staged(n);
            }
        } else if (wid == 4) {
            const char* exs = p.ws[s] + par;
            unsigned tnext = t_entry + xcd * (TAU / 4);
            u64 todo = ~0ull << NPUSH;
            __builtin_amdgcn_s_setprio(3);
            const char* src = nullptr;
            unsigned trim = 0;
            auto choose = [&](int n) {
                const unsigned rdy = __hip_atomic_load(ctl + (n >= 64 ? n : lane), RLX, SC_WG);
                const unsigned fre = __hip_atomic_load(ctl + (n >= RING ? 128 + n - RING : 400), RLX, SC_WG), tr = __hip_atomic_load(ctl + 400, RLX, SC_WG);
                const u64 rd = __ballot(rdy != 0u) & (n >= 64 ? 1ull : todo);
                if (!rd || (n >= RING && UNI(fre) < 5u)) return;
                const int un = n >= 64 ? n : __builtin_ctzll(rd);
                todo &= ~(1ull << (un & 63));
                if (lane == 0) __hip_atomic_store(ctl + 416 + n, (unsigned)un, RLX, SC_WG);
                trim = n >= EQ_AT + 4 && n < EQ_AT + 36 ? UNI(tr) : 0u;
                src = exs + (size_t)(desc_of(un) >> 12) * SLOT;
            };
            if (NPUSH > 1 && PULL_DELAY >= 0) {
                while (lds_get(ctl) == 0u) __builtin_amdgcn_s_sleep(1);
                if (lds_get(ctl + 1) == 0u) tnext = now8() + PULL_DELAY * 64;
            }
            for (int n = NPUSH; n < nu; ++n) {
                if (!src) {
                    choose(n);
                    if (!src) {
                        wait_vm<0>();
                        if (lane == 0) __hip_atomic_store(ctl + 384, (unsigned)n, RLX, SC_WG);
                        for (choose(n); !src; choose(n)) __builtin_amdgcn_s_sleep(1);
                    }
                }
                const char* cur = src;
                const unsigned dst = lds0 + (unsigned)(n % RING) * 16384u, tr = trim;
                src = nullptr;
                asm volatile("" ::: "memory");
#pragma unroll
                for (int i = 0; i < 14; i += 2) {
                    wait_vm<WIN - 2>();
                    if (i == WIN - 2 && lane == 0) __hip_atomic_store(ctl + 384, (unsigned)n, RLX, SC_WG);
                    pace(tnext, (i < 12 ? 2u * TAU : UNIT_T - 12u * TAU) - tr);
                    dma<true>(cur + i * 1024, lo, dst + i * 1024);
                    if (i == 0 && lane == 0) {
                        if (n == EQ_AT) __hip_atomic_store(ctl + 387, tnext | 1u, RLX, SC_WG);
                        if (n == nu - 1) __hip_atomic_store(ctl + 385, (tnext + UNIT_T - 2u * TAU) | 1u, RLX, SC_WG);
                    }
                    if (i == 10 && n + 1 < nu) choose(n + 1);
                }
                if (lane < 16) dma<false>(cur + 14336, lo, dst + 14336);
            }
            __builtin_amdgcn_s_setprio(0);
            wait_vm<0>();
            if (lane == 0) __hip_atomic_store(ctl + 384, (unsigned)nu, RLX, SC_WG);
            STAMP(9);
        } else if (wid == 5) {
            const GAS u64* rp[2];
            bool pend[2];
#pragma unroll
            for (int k = 0; k < 2; ++k) {
                rp[k] = (const GAS u64*)(myws + WS_RDY) + ((((call & 1u) * 8 + ((desc[k] >> 9) & 7)) * 48 + ((desc[k] >> 3) & 63)) * 8 + (desc[k] & 7)) * RSTRIDE;
                pend[k] = k * 64 + lane < nu;
            }
            const u64 t0 = __builtin_amdgcn_s_memrealtime();
            for (unsigned polls = 1;; ++polls) {
                const u64 pm = __ballot(pend[0]);
                const bool front = __builtin_amdgcn_mbcnt_hi((unsigned)(pm >> 32), __builtin_amdgcn_mbcnt_lo((unsigned)pm, 0u)) < POLLW;
#pragma unroll
                for (int k = 0; k < 2; ++k)
                    if (pend[k] && ((k == 0 && front) || (polls & 7u) == 0u)) {
                        const u64 f = __hip_atomic_load(rp[k], RLX, SC_SYS);
                        if ((unsigned)(f >> 32) >= call) { __hip_atomic_store(ctl + k * 64 + lane, (unsigned)f | 0x80000000u, RLX, SC_WG); pend[k] = false; }
                    }
                if (!__any(pend[0] || pend[1])) break;
                if ((polls & 15u) == 0u && TIMED_OUT(t0)) {
                    if (lane == 0) __hip_atomic_store(tmo, call, RLX, SC_AGENT);
                    wait_vm<0>();
#pragma unroll
                    for (int k = 0; k < 2; ++k) if (pend[k]) __hip_atomic_store(ctl + k * 64 + lane, 0x80000000u, RLX, SC_WG);
                    break;
                }
                if (polls < 32u) __builtin_amdgcn_s_sleep(1); else __builtin_amdgcn_s_sleep(16);
            }
            if (idx < 6) {
                unsigned th;
                while ((th = lds_get(ctl + 385)) == 0u) __builtin_amdgcn_s_sleep(4);
                if (lane == 0) __hip_atomic_store((GAS u64*)(myws + WS_MBOX + (size_t)((8 + idx * 4) * 8 + xcd) * 128), ((u64)call << 32) | th, RLX, SC_AGENT);
            }
        } else if (wid == 6) {
            for (int n = 0; n < NPUSH; ++n) {
                while (lds_get(ctl + n) == 0u) __builtin_amdgcn_s_sleep(2);
                const char* src = myws + par + (size_t)(1280 + (n * 8 + s) * 8 + xcd) * SLOT;
#pragma unroll
                for (int i = 0; i < 16; i += 2) dma<true>(src + i * 1024, lo, lds0 + n * 16384 + i * 1024);
                if (lane == 0) __hip_atomic_store(ctl + 416 + n, (unsigned)n, RLX, SC_WG);
                wait_vm<0>();
                if (lane == 0) __hip_atomic_store(ctl + 386, (unsigned)n + 1u, RLX, SC_WG);
            }
            if (EQ_AT) {
                GAS u64* eq = (GAS u64*)(myws + WS_EQ) + idx * 16;
                unsigned mine;
                while ((mine = lds_get(ctl + 387)) == 0u) __builtin_amdgcn_s_sleep(4);
                if (lane == 0) __hip_atomic_store(eq + xcd, ((u64)call << 32) | mine, RLX, SC_AGENT);
                while ((int)lds_get(ctl + 384) < EQ_AT + 3) {
                    const u64 f = __hip_atomic_load(eq + (lane & 7), RLX, SC_AGENT);
                    if (__all((unsigned)(f >> 32) == call)) {
                        int dlt = (int)(mine - (unsigned)f);
                        dlt += __shfl_xor(dlt, 1); dlt += __shfl_xor(dlt, 2); dlt += __shfl_xor(dlt, 4);
                        dlt /= 8 * 32 * 7;
                        dlt = dlt > TAU / 8 ? TAU / 8 : dlt < -(TAU / 8) ? -(TAU / 8) : dlt;
                        if (lane == 0) __hip_atomic_store(ctl + 400, (unsigned)dlt, RLX, SC_WG);
                        break;
                    }
                    __builtin_amdgcn_s_sleep(8);
                }
            }
        } else {
            for (int n = 0; n < nu; ++n) {
                int un = 0;
                if (n) while ((un = lds_get(ctl + 416 + n)) == 0) __builtin_amdgcn_s_sleep(4);
                const unsigned de = desc_of(un);
                const int c = de & 7, id = (de >> 3) & 63;
                const bool relay = idx < 6 && id < 32 && (id & 3) == 3 && c < 2 * NFWD;
                if (relay) {
                    while ((int)lds_get(ctl + 384) <= n) __builtin_amdgcn_s_sleep(2);
                    asm volatile("" ::: "memory");
                    u32x4 v[15];
#pragma unroll
                    for (int i = 0; i < 15; ++i) v[i] = *(const LAS u32x4*)(lds + (unsigned)(n % RING) * 16384u + i * 1024 + lo);
                    const unsigned h = lds_get(ctl + un);
                    WAIT_L0();
                    if (lane == 0) __hip_atomic_fetch_add(ctl + 128 + n, 1u, RLX, SC_WG);
                    const char* dst = myws + par + (size_t)(1408 + ((c >> 1) * 6 + idx) * 8 + xcd) * SLOT;
#pragma unroll
                    for (int i = 0; i < 15; ++i) st16(dst + i * 1024, lo, v[i]);
                    wait_vm<0>();
                    if (lane == 0) __hip_atomic_store((GAS u64*)(p.ws[rank ^ 1] + WS_RDY) + ((((call & 1u) * 8 + s) * 48 + id) * 8 + c) * RSTRIDE, ((u64)call << 32) | (h & 0xffffffu), RLX, SC_SYS);
                } else if (lane == 0) __hip_atomic_fetch_add(ctl + 128 + n, 1u, RLX, SC_WG);
            }
        }
    }
    const unsigned call = UNI(ctr_v) + 1u;

    const int ntile = idx == 7 ? 2 : 1;
    const int wm = wid >> 1, wn = wid & 1;
#pragma unroll 1
    for (int rep = 0; rep < ntile; ++rep) {
    const int rb = late ? 6 + (idx >> 2) : (idx - 8) >> 2;
    const int cb = xcd * 4 + (idx == 7 ? rep : idx == 6 ? 0 : idx == 4 || idx == 5 ? idx - 2 : idx & 3);
    const bool part1 = idx == 7 && rep == 0, part2 = idx == 6;
    const int nstep = part1 ? 56 : part2 ? 8 : NSTEP;
    const char* stgA = myws + WS_STG + (size_t)(rb * NSTEP) * A_STEP + wid * 4096;
    const char* stgB = myws + WS_STG + STG_A + (size_t)(cb * NSTEP) * B_STEP + wid * 2048;
    const u64 pbits = 0x0101010101010101ull << (rank ^ 1);
    u64 todo = part1 ? ~pbits : part2 ? pbits : ~0ull, avail = 0ull;
    auto pick = [&](int slot) {
        if (wid == 0) {
            if (avail == 0ull) {
                const u64 t0 = __builtin_amdgcn_s_memrealtime();
                for (unsigned polls = 1;; ++polls) {
                    const unsigned f0 = __hip_atomic_load(gflag + cb * 64 + lane, RLX, GSC);
                    const unsigned f1 = __hip_atomic_load(gflag + (32 + rb * 2) * 64 + lane, RLX, GSC);
                    const unsigned f2 = __hip_atomic_load(gflag + (33 + rb * 2) * 64 + lane, RLX, GSC);
                    avail = __ballot(f0 >= call && f1 >= call && f2 >= call) & todo;
                    if (avail) break;
                    if ((polls & 15u) == 0u && TIMED_OUT(t0)) {
                        if (lane == 0) __hip_atomic_store(tmo, call, RLX, SC_AGENT);
                        avail = todo;
                        break;
                    }
                    __builtin_amdgcn_s_sleep(POLL_SLEEP);
                }
            }
            const int b = __builtin_ctzll(avail);
            avail &= avail - 1; todo &= ~(1ull << b);
            if (lane == 0) ctl[388 + slot] = (unsigned)((b & 7) * 8 + (b >> 3));
        }
    };
    auto g1 = [&](unsigned ks, unsigned buf, int i) {
        if (i < 4) __builtin_amdgcn_global_load_lds((const GAS unsigned*)(stgA + (size_t)ks * A_STEP + i * 1024 + lo), (LAS unsigned*)(lds + buf + wid * 4096 + i * 1024), 16, 0, 17);
        else __builtin_amdgcn_global_load_lds((const GAS unsigned*)(stgB + (size_t)ks * B_STEP + (i - 4) * 1024 + lo), (LAS unsigned*)(lds + buf + A_STEP + wid * 2048 + (i - 4) * 1024), 16, 0, 17);
    };
#define GETKS(slot) ((unsigned)UNI(ctl[388 + (slot)]) & 63u)

    f32x4 acc[4][4];
#pragma unroll
    for (int m = 0; m < 4; ++m)
#pragma unroll
        for (int n = 0; n < 4; ++n) acc[m][n] = (f32x4){0.f, 0.f, 0.f, 0.f};
    h16x8 a0[4], b0[4], a1[4], b1[4];
    const unsigned aoff = wm * 8192 + lo, boff = A_STEP + wn * 8192 + lo;
#define RDA(A, buf, kk, m) A[m] = *(const LAS h16x8*)(lds + (buf) + aoff + (m) * 2048 + (kk) * 1024)
#define RDB(B, buf, kk, n) B[n] = *(const LAS h16x8*)(lds + (buf) + boff + (n) * 2048 + (kk) * 1024)
#define ROW(A, B, m) _Pragma("unroll") for (int n = 0; n < 4; ++n) acc[m][n] = __builtin_amdgcn_mfma_f32_16x16x32_f16(A[m], B[n], acc[m][n], 0, 0, 0)

    STAMP(10);
    unsigned kq;
    pick(0); WAIT_L0(); BAR(); kq = GETKS(0);
#pragma unroll
    for (int i = 0; i < 6; ++i) g1(kq, 0u, i);
    pick(1); WAIT_L0(); BAR(); kq = GETKS(1);
#pragma unroll
    for (int i = 0; i < 6; ++i) g1(kq, STAGE, i);
    pick(2); WAIT_L0(); BAR(); kq = GETKS(2);
#pragma unroll
    for (int i = 0; i < 3; ++i) g1(kq, 2 * STAGE, i);
    wait_vm<9>(); BAR();
#pragma unroll
    for (int i = 0; i < 4; ++i) { RDB(b0, 0u, 0, i); RDA(a0, 0u, 0, i); }
    unsigned cur = 0, prv = 2 * STAGE;
    for (int t = 0; t < nstep; ++t) {
        const unsigned nxt = cur == 2 * STAGE ? 0u : cur + STAGE;
        const bool m2 = t + 2 < nstep, m3 = t + 3 < nstep;
        WAIT_L0(); SB();
        RDB(b1, cur, 1, 0); RDB(b1, cur, 1, 1); SB(); ROW(a0, b0, 0); SB(); if (m2) g1(kq, prv, 3); SB();
        RDB(b1, cur, 1, 2); RDB(b1, cur, 1, 3); SB(); ROW(a0, b0, 1); SB(); if (m2) g1(kq, prv, 4); SB();
        RDA(a1, cur, 1, 0); RDA(a1, cur, 1, 1); SB(); ROW(a0, b0, 2); SB(); if (m2) g1(kq, prv, 5); SB();
        RDA(a1, cur, 1, 2); RDA(a1, cur, 1, 3); SB(); ROW(a0, b0, 3); SB();
        if (m3) pick((t + 3) & 3);
        WAIT_L0();
        if (m2) wait_vm<6>(); else wait_vm<0>();
        BAR(); SB();
        const unsigned kv = ctl[388 + ((t + 3) & 3)];
        RDB(b0, nxt, 0, 0); RDB(b0, nxt, 0, 1); SB(); ROW(a1, b1, 0); SB();
        kq = (unsigned)UNI(kv) & 63u;
        if (m3) g1(kq, cur, 0); SB();
        RDB(b0, nxt, 0, 2); RDB(b0, nxt, 0, 3); SB(); ROW(a1, b1, 1); SB(); if (m3) g1(kq, cur, 1); SB();
        RDA(a0, nxt, 0, 0); RDA(a0, nxt, 0, 1); SB(); ROW(a1, b1, 2); SB(); if (m3) g1(kq, cur, 2); SB();
        RDA(a0, nxt, 0, 2); RDA(a0, nxt, 0, 3); SB(); ROW(a1, b1, 3); SB();
        prv = cur; cur = nxt;
    }

    if (part1 || part2) {
        const char* park = myws + WS_EXO + (size_t)(xcd * 8 + wid) * 16384;
        GAS u64* pf = (GAS u64*)(myws + WS_ORDY + (size_t)xcd * 128) + wid;
        if (part1) {
#pragma unroll
            for (int i = 0; i < 16; ++i) st16(park + i * 1024, lo, __builtin_bit_cast(u32x4, acc[i >> 2][i & 3]));
            wait_vm<0>();
            if (lane == 0) __hip_atomic_store(pf, (u64)call << 32, RLX, SC_SYS);
            continue;
        }
        const u64 t0 = __builtin_amdgcn_s_memrealtime();
        while ((unsigned)UNI((unsigned)(__hip_atomic_load(pf, RLX, SC_SYS) >> 32)) < call) {
            if (TIMED_OUT(t0)) { if (lane == 0) __hip_atomic_store(tmo, call, RLX, SC_AGENT); wait_vm<0>(); break; }
            __builtin_amdgcn_s_sleep(4);
        }
#pragma unroll
        for (int m = 0; m < 4; ++m) {
            f32x4 e0, e1, e2, e3;
            asm volatile("s_nop 4\n\tglobal_load_dwordx4 %0, %4, %5 sc0 sc1\n\tglobal_load_dwordx4 %1, %4, %5 offset:1024 sc0 sc1\n\t"
                         "global_load_dwordx4 %2, %4, %5 offset:2048 sc0 sc1\n\tglobal_load_dwordx4 %3, %4, %5 offset:3072 sc0 sc1\n\ts_waitcnt vmcnt(0)"
                         : "=&v"(e0), "=&v"(e1), "=&v"(e2), "=&v"(e3) : "v"(lo), "s"(park + m * 4096) : "memory");
            acc[m][0] += e0; acc[m][1] += e1; acc[m][2] += e2; acc[m][3] += e3;
        }
    }
    const unsigned tmo_v = __hip_atomic_load(tmo, RLX, SC_AGENT);
    const unsigned oo = (unsigned)(fq * 4) * (GN * 4u) + fr * 16u;
    const u32x4 nan4 = {0x7fc00000u, 0x7fc00000u, 0x7fc00000u, 0x7fc00000u};
    unsigned y[64];
#pragma unroll
    for (int i = 0; i < 32; ++i) {
        const int m = i >> 3, n = (i >> 1) & 3, j = (i & 1) * 2;
        const f32x2 g = gelu2((f32x2){acc[m][n][j], acc[m][n][j + 1]});
        y[(m * 4 + j) * 4 + n] = __float_as_uint(g.x); y[(m * 4 + j + 1) * 4 + n] = __float_as_uint(g.y);
    }
    if (late) {
        const bool bad = UNI(tmo_v) == call;
        const char* ob = (const char*)(p.out + (size_t)(rb_local(rank, rb) + wm * 64) * GN + (rank >> 1) * 4096 + cb * 128 + wn * 64);
#pragma unroll
        for (int i = 0; i < 16; ++i) st16(ob + (size_t)((i >> 2) * 16 + (i & 3)) * (GN * 4), oo, bad ? nan4 : (u32x4){y[4 * i], y[4 * i + 1], y[4 * i + 2], y[4 * i + 3]});
    } else {
        const int peer = rb_owner(rank, rb);
        const int mirror = (8 + rb_of(peer, rank) * 4 + (idx & 3)) * 8 + xcd;
        char* const peerws = p.ws[peer];
        {
            const char* ex = myws + WS_EXO + (size_t)(blockIdx.x * 8 + wid) * 16384;
            unsigned hdr = 0;
            if (PACK_OUT) {
                float mx = 0.f;
#pragma unroll
                for (int i = 0; i < 64; i += 2) mx = __builtin_fmaxf(__builtin_fmaxf(mx, __builtin_fabsf(__builtin_bit_cast(float, y[i]))), __builtin_fabsf(__builtin_bit_cast(float, y[i + 1])));
                const unsigned emax = wave_max_exp<8, 23>(__builtin_bit_cast(unsigned, mx));
                const unsigned base = emax > 15u ? emax - 15u : 0u, bb = base << 23;
                hdr = base;
#pragma unroll
                for (int hf = 0; hf < 2; ++hf) {
                    unsigned z[28];
#pragma unroll
                    for (int g4 = 0; g4 < 4; ++g4) {
                        const int g = hf * 4 + g4;
                        unsigned dd[8], any = 0;
#pragma unroll
                        for (int i = 0; i < 8; ++i) { dd[i] = y[8 * g + i] - bb; any |= dd[i]; }
                        if (__all(!(any & 0x78000000u))) {
                            const unsigned cc = (dd[7] & 0x07ffffffu) | ((dd[7] >> 4) & 0x08000000u);
#pragma unroll
                            for (int i = 0; i < 7; ++i) z[7 * g4 + i] = dd[i] | ((cc << (27 - 4 * i)) & 0x78000000u);
                        } else {
#pragma unroll
                            for (int i = 0; i < 7; ++i) z[7 * g4 + i] = y[8 * g + i];
                            st4(ex + 14336 + g * 256, lane * 4u, y[8 * g + 7]);
                            hdr |= 256u << g;
                        }
                    }
#pragma unroll
                    for (int i = 0; i < 7; ++i) st16(ex + (hf * 7 + i) * 1024, lo, (u32x4){z[4 * i], z[4 * i + 1], z[4 * i + 2], z[4 * i + 3]});
                }
            } else {
#pragma unroll
                for (int i = 0; i < 16; ++i) st16(ex + i * 1024, lo, (u32x4){y[4 * i], y[4 * i + 1], y[4 * i + 2], y[4 * i + 3]});
            }
            wait_vm<0>();
            hdr |= UNI(tmo_v) == call ? 0xc0000000u : 0x40000000u;
            if (lane == 0) __hip_atomic_store((GAS u64*)(peerws + WS_ORDY + (size_t)mirror * 128) + wid, ((u64)call << 32) | hdr, RLX, SC_SYS);
        }
        WAIT_L0(); BAR();
        STAMP(22);
        const char* exs = peerws + WS_EXO + (size_t)mirror * (8 * 16384);
        constexpr int HALF = OPS_OUT / 2;
        unsigned raw[4];
        auto get_raw = [&](int blk, int hf, unsigned hdr) {
#pragma unroll
            for (int g4 = 0; g4 < 4; ++g4) {
                raw[g4] = 0u;
                if (PACK_OUT && (hdr & (256u << (hf * 4 + g4)))) raw[g4] = ld4(exs + (size_t)blk * 16384 + 14336 + (hf * 4 + g4) * 256, lane * 4u);
            }
        };
        auto unpack = [&](int blk, int hf, unsigned hdr) {
            const LAS unsigned char* src = lds + (unsigned)blk * 16384u + hf * (HALF * 1024) + lo;
            unsigned z[32], o[32];
#pragma unroll
            for (int i = 0; i < HALF; ++i) { const u32x4 a = *(const LAS u32x4*)(src + i * 1024); z[4 * i] = a[0]; z[4 * i + 1] = a[1]; z[4 * i + 2] = a[2]; z[4 * i + 3] = a[3]; }
            const unsigned bb = (hdr & 255u) << 23;
#pragma unroll
            for (int g4 = 0; g4 < 4; ++g4) {
                if (!PACK_OUT) {
#pragma unroll
                    for (int i = 0; i < 8; ++i) o[8 * g4 + i] = z[8 * g4 + i];
                } else if (!(hdr & (256u << (hf * 4 + g4)))) {
                    unsigned cc = 0;
#pragma unroll
                    for (int i = 0; i < 7; ++i) { cc |= (z[7 * g4 + i] & 0x78000000u) >> (27 - 4 * i); o[8 * g4 + i] = (z[7 * g4 + i] & 0x87ffffffu) + bb; }
                    o[8 * g4 + 7] = ((cc & 0x07ffffffu) | ((cc << 4) & 0x80000000u)) + bb;
                } else {
#pragma unroll
                    for (int i = 0; i < 7; ++i) o[8 * g4 + i] = z[7 * g4 + i];
                    o[8 * g4 + 7] = raw[g4];
                }
            }
            const char* ob = (const char*)(p.out + (size_t)((peer & 1) * 256 + (blk >> 1) * 64 + hf * 32) * GN + (peer >> 1) * 4096 + cb * 128 + (blk & 1) * 64);
#pragma unroll
            for (int i = 0; i < 8; ++i)
                st16(ob + (size_t)((i >> 2) * 16 + (i & 3)) * (GN * 4), oo, (hdr >> 31) ? nan4 : (u32x4){o[4 * i], o[4 * i + 1], o[4 * i + 2], o[4 * i + 3]});
        };
        if (wid == 7) {
            unsigned tnext;
            while ((tnext = lds_get(ctl + 8)) == 0u) __builtin_amdgcn_s_sleep(2);
            STAMP(23);
            __builtin_amdgcn_s_setprio(3);
            wait_vm<0>(); __builtin_amdgcn_s_waitcnt(0x0F70);
            for (int n = 0; n < 8; ++n) {
                if (lds_get(ctl + n) == 0u) {
                    wait_vm<0>();
                    if (lane == 0) __hip_atomic_store(ctl + 384, (unsigned)n, RLX, SC_WG);
                    while (lds_get(ctl + n) == 0u) __builtin_amdgcn_s_sleep(1);
                }
                asm volatile("" ::: "memory");
                if (n == 7 && lane == 0) __hip_atomic_store(ctl + 385, (tnext + OPS_OUT * TAU) | 1u, RLX, SC_WG);
#pragma unroll
                for (int i = 0; i < OPS_OUT; i += 2) {
                    wait_vm<WIN - 2>();
                    if (i == WIN - 2 && lane == 0) __hip_atomic_store(ctl + 384, (unsigned)n, RLX, SC_WG);
                    pace(tnext, 2u * TAU);
                    dma<true>(exs + (size_t)n * 16384 + i * 1024, lo, lds0 + (unsigned)n * 16384u + i * 1024);
                }
            }
            wait_vm<HALF>();
            if (lane == 0) __hip_atomic_store(ctl + 384, 8u, RLX, SC_WG);
            wait_vm<0>();
            if (lane == 0) __hip_atomic_store(ctl + 384, 9u, RLX, SC_WG);
            STAMP(19);
        } else {
            if (wid == 6) {
                const GAS u64* fp = lane < 8 ? (const GAS u64*)(myws + WS_ORDY + (size_t)blockIdx.x * 128) + lane : (const GAS u64*)(myws + WS_MBOX + (size_t)blockIdx.x * 128);
                bool pend = lane < 9;
                const u64 t0 = __builtin_amdgcn_s_memrealtime();
                for (unsigned polls = 1;; ++polls) {
                    if (pend) {
                        const u64 f = __hip_atomic_load(fp, RLX, SC_SYS);
                        if ((unsigned)(f >> 32) >= call) { __hip_atomic_store(ctl + lane, (unsigned)f, RLX, SC_WG); pend = false; }
                    }
                    if (!__any(pend)) break;
                    if ((polls & 15u) == 0u && TIMED_OUT(t0)) {
                        if (lane == 0) __hip_atomic_store(tmo, call, RLX, SC_AGENT);
                        if (pend) __hip_atomic_store(ctl + lane, lane < 8 ? 0xc0000000u : now8() | 1u, RLX, SC_WG);
                        break;
                    }
                    __builtin_amdgcn_s_sleep(4);
                }
            }
            for (int job = 0; job < (wid == 1 || wid == 2 ? 3 : 2); ++job) {
                const int blk = job < 2 ? wid : 7, hf = job < 2 ? job : wid - 1, need = job < 2 ? wid : 6 + wid;
                unsigned hdr;
                while ((hdr = lds_get(ctl + blk)) == 0u) __builtin_amdgcn_s_sleep(4);
                get_raw(blk, hf, hdr);
                while ((int)lds_get(ctl + 384) <= need) __builtin_amdgcn_s_sleep(1);
                asm volatile("" ::: "memory");
                unpack(blk, hf, hdr);
            }
            if (wid == 0 && (idx & 3) != 3) {
                unsigned th;
                while ((th = lds_get(ctl + 385)) == 0u) __builtin_amdgcn_s_sleep(4);
                if (lane == 0) __hip_atomic_store((GAS u64*)(myws + WS_MBOX + (size_t)(blockIdx.x + 8) * 128), ((u64)call << 32) | th, RLX, SC_AGENT);
            }
        }
    }
    }
    if (tid == 0) __hip_atomic_store(ctr, call, RLX, SC_AGENT);
}

extern "C" void kernel_launch(int rank, int nranks, ncclComm_t comm, hipStream_t stream,
                              void* const* d_in, const int* in_sizes, int n_in,
                              void* d_out, int out_size, const int* in_dtypes, int out_dtype,
                              void* const* d_out_all, void* const* d_ws_all, size_t ws_bytes) {
    (void)comm; (void)in_dtypes; (void)out_dtype; (void)d_out_all;
    if (nranks != NR || (unsigned)rank >= (unsigned)NR || n_in != 2 || ws_bytes < WS_NEED ||
        in_sizes[0] != 4096 * KSH || in_sizes[1] != KSH * GN || out_size != 512 * GN) {
        fprintf(stderr, "kernel_launch: unexpected problem shape\n");
        return;
    }
    Params p;
    p.x = (const float*)d_in[0];
    p.w = (const float*)d_in[1];
    p.out = (float*)d_out;
    p.myws = (char*)d_ws_all[rank];
    for (int i = 0; i < NR; ++i) p.ws[i] = (char*)d_ws_all[i];
    p.rank = rank;
    p.pad = 0;
    int n = 0, np = 0, nd = 0;
    unsigned char deferred[8];
    bool px[8] = {};
    for (int i = 0; i < 128; ++i) p.order[0][i] = p.order[1][i] = 0;
    auto need_x = [&](int c0, int c1) { for (int c = c0; c <= c1; ++c) if (!px[c]) { px[c] = true; p.prod[np++] = (unsigned)(32 + c); } };
    for (int key = 0; key < 16 + 3 * SKEW2; ++key) {
        for (int pass = 0; pass < 2; ++pass) {
            if ((pass == 0) == (key != 0)) {
                for (int e = 32; e < 44; ++e)
                    if (2 * ((e - 32) * 8 / 12) == key) { p.order[0][n++] = (unsigned char)e; need_x((e - 32) * 8 / 12, ((e - 32) * 8 + 7) / 12); }
                if (key == XPOS) { p.order[0][n++] = 44; need_x(0, 3); }
                if (key == XPOS + 2) { p.order[0][n++] = 45; need_x(4, 7); }
            } else {
                for (int b = 0; b < 4; ++b)
                    for (int c = 0; c < 8; ++c)
                        if ((b == 0 && c == 1 ? 0 : 2 * c + b * SKEW2) == key) {
                            p.prod[np++] = (unsigned)(c * 4 + b);
                            if (b == 3 && c < 2 * NFWD && ((c ^ rank) & 1)) deferred[nd++] = (unsigned char)(c * 4 + b);
                            else p.order[0][n++] = (unsigned char)(c * 4 + b);
                        }
            }
        }
    }
    int m = 0;
    for (int i = 0; i < n; ++i) if (p.order[0][i] < 44) p.order[1][m++] = p.order[0][i];
    for (int i = 0; i < nd; ++i) p.order[1][m++] = deferred[i];
    for (int i = 0; i < 6 * NFWD; ++i) p.order[1][m++] = (unsigned char)(48 + i);
    for (int e = 44; e < 48; ++e) { if (e >= 46) p.order[0][n++] = (unsigned char)e; p.order[1][m++] = (unsigned char)e; }
    if (n != NU_OUT || m != NU_PAIR || np != 40 || p.prod[0] != 0 || p.prod[1] != 4 || p.prod[2] != 32) { fprintf(stderr, "kernel_launch: bad schedule\n"); return; }
    hipLaunchKernelGGL(fused_gemm_rs, dim3(NBLK), dim3(NTHR), 0, stream, p);
    HIP_CHECK(hipGetLastError());
}
```

```cpp
#include <hip/hip_runtime.h>
#include <rccl/rccl.h>
#include <cstdio>

#define HIP_CHECK(expr) do { hipError_t _e = (expr); if (_e != hipSuccess) { \
    fprintf(stderr, "HIP error %s at %s:%d\n", hipGetErrorString(_e), __FILE__, __LINE__); \
    return; } } while (0)

typedef float f32x4 __attribute__((ext_vector_type(4)));
typedef unsigned u32x4 __attribute__((ext_vector_type(4)));
typedef unsigned u32x3 __attribute__((ext_vector_type(3)));
typedef unsigned short u16x2 __attribute__((ext_vector_type(2)));
typedef _Float16 h16x8 __attribute__((ext_vector_type(8)));
typedef unsigned long long u64;
#define LAS __attribute__((address_space(3)))
#define GAS __attribute__((address_space(1)))
#define RLX __ATOMIC_RELAXED
#define SC_WG __HIP_MEMORY_SCOPE_WORKGROUP
#define SC_AGENT __HIP_MEMORY_SCOPE_AGENT
#define SC_SYS __HIP_MEMORY_SCOPE_SYSTEM

#define PACK_OUT 1
#define TAU 864
#define TAU_OUT TAU
#define WIN_OUT WIN
#define EQ_AT 5
#define CREDIT 30
#define WIN 10
#define PROD_GAP 40
#define PUSH_SLOT 112
#define RSTRIDE 16
#define POLLW 12
#define PROD_OFF -2
#define POLL_SLEEP 8
#define NFWD 3
#define XPOS 6
#define SKEW2 2

constexpr int NR = 8;
constexpr int GN = 16384;
constexpr int KSH = 512;
constexpr int NBLK = 256;
constexpr int NTHR = 512;
constexpr int NSTEP = 64;
constexpr unsigned A_STEP = 32768, B_STEP = 16384, STAGE = A_STEP + B_STEP;
constexpr int NU_OUT = 48 - NFWD, NU_PAIR = 48 + 6 * NFWD;
constexpr int RING = 8;
constexpr int PW = 13;
constexpr int QB = 3328 + 16 * PW;
constexpr int OPS_OUT = PACK_OUT ? 14 : 16;
constexpr unsigned UNIT_T = QB * TAU / 256;
constexpr unsigned SLOT = 18432, SIDE = 14336;
constexpr size_t WS_CTR = 0;
constexpr size_t WS_FLAG = 32 << 10;
constexpr size_t WS_ORDY = 96 << 10;
constexpr size_t WS_MBOX = 128 << 10;
constexpr size_t WS_EQ = 160 << 10;
constexpr size_t WS_TMO = 192 << 10;
constexpr size_t WS_DIAG = 256 << 10;
constexpr size_t WS_STG = 1 << 20;
constexpr size_t STG_A = 8ull * NSTEP * A_STEP;
constexpr size_t STG_B = 32ull * NSTEP * B_STEP;
constexpr size_t WS_EXI = WS_STG + STG_A + STG_B;
constexpr size_t EXI_BYTES = 1600ull * SLOT;
constexpr size_t WS_EXO = WS_EXI + 2 * EXI_BYTES;
constexpr size_t WS_RDY = WS_EXO + (size_t)NBLK * 8 * 16384;
constexpr size_t WS_NEED = WS_RDY + 2 * 8 * 48 * 8 * 128;
constexpr u64 DEADLINE_TICKS = 50000000ull;

struct Params {
    const float* x;
    const float* w;
    float* out;
    char* myws;
    char* ws[NR];
    int rank;
    int pad;
    unsigned prod[40];
    unsigned char order[2][128];
};

#define WAIT_L0() do { asm volatile("s_waitcnt lgkmcnt(0)" ::: "memory"); __builtin_amdgcn_s_waitcnt(0xC07F); } while (0)
#define BAR() do { asm volatile("" ::: "memory"); __builtin_amdgcn_s_barrier(); asm volatile("" ::: "memory"); } while (0)
#define SB() __builtin_amdgcn_sched_barrier(0)
#define UNI(x) __builtin_amdgcn_readfirstlane(x)
template <int N> __device__ __forceinline__ void wait_vm() { asm volatile("s_waitcnt vmcnt(%0)" ::"i"(N) : "memory"); }

__device__ __forceinline__ void st16(const void* base, unsigned voff, u32x4 v) {
    asm volatile("s_nop 4\n\tglobal_store_dwordx4 %0, %1, %2 sc0 sc1\n\ts_nop 1" ::"v"(voff), "v"(v), "s"(base) : "memory");
}
__device__ __forceinline__ void st12(const void* base, unsigned voff, u32x3 v) {
    asm volatile("s_nop 4\n\tglobal_store_dwordx3 %0, %1, %2 sc0 sc1\n\ts_nop 1" ::"v"(voff), "v"(v), "s"(base) : "memory");
}
__device__ __forceinline__ void st4(const void* base, unsigned voff, unsigned v) {
    asm volatile("s_nop 4\n\tglobal_store_dword %0, %1, %2 sc0 sc1" ::"v"(voff), "v"(v), "s"(base) : "memory");
}
__device__ __forceinline__ unsigned ld4(const void* base, unsigned voff) {
    unsigned r;
    asm volatile("s_nop 4\n\tglobal_load_dword %0, %1, %2 sc0 sc1\n\ts_waitcnt vmcnt(0)" : "=&v"(r) : "v"(voff), "s"(base) : "memory");
    return r;
}
__device__ __forceinline__ u32x3 ld12(const void* base, unsigned voff) {
    u32x3 r;
    asm volatile("s_nop 4\n\tglobal_load_dwordx3 %0, %1, %2 sc0 sc1\n\ts_waitcnt vmcnt(0)" : "=&v"(r) : "v"(voff), "s"(base) : "memory");
    return r;
}
__device__ __forceinline__ void ld64(const void* base, unsigned voff, u32x4& a, u32x4& b, u32x4& c, u32x4& d) {
    asm volatile("s_nop 4\n\tglobal_load_dwordx4 %0, %4, %5 sc0 sc1\n\tglobal_load_dwordx4 %1, %4, %5 offset:1024 sc0 sc1\n\t"
                 "global_load_dwordx4 %2, %4, %5 offset:2048 sc0 sc1\n\tglobal_load_dwordx4 %3, %4, %5 offset:3072 sc0 sc1\n\ts_waitcnt vmcnt(0)"
                 : "=&v"(a), "=&v"(b), "=&v"(c), "=&v"(d) : "v"(voff), "s"(base) : "memory");
}
template <bool TWO> __device__ __forceinline__ void dma(const void* base, unsigned voff, unsigned lds_dst) {
    unsigned keep;
    if (TWO) asm volatile("s_mov_b32 %0, m0\n\ts_mov_b32 m0, %3\n\ts_nop 4\n\tglobal_load_lds_dwordx4 %1, %2 sc0 sc1\n\tglobal_load_lds_dwordx4 %1, %2 offset:1024 sc0 sc1\n\ts_mov_b32 m0, %0"
                          : "=&s"(keep) : "v"(voff), "s"(base), "s"(lds_dst) : "memory");
    else asm volatile("s_mov_b32 %0, m0\n\ts_mov_b32 m0, %3\n\ts_nop 4\n\tglobal_load_lds_dwordx4 %1, %2 sc0 sc1\n\ts_mov_b32 m0, %0"
                      : "=&s"(keep) : "v"(voff), "s"(base), "s"(lds_dst) : "memory");
}
__device__ __forceinline__ unsigned now8() { return (unsigned)__builtin_amdgcn_s_memrealtime() << 6; }
__device__ __forceinline__ void pace(unsigned& t, unsigned period) {
    int late = (int)(now8() - t);
    if (late < -(1 << 22)) { t += (unsigned)late; late = 0; }
    while (late < 0) { __builtin_amdgcn_s_sleep(1); late = (int)(now8() - t); }
    t += period + (late > CREDIT * 64 ? (unsigned)(late - CREDIT * 64) : 0u);
}
__device__ __forceinline__ unsigned lds_get(const LAS unsigned* a) { return UNI(__hip_atomic_load(a, RLX, SC_WG)); }
__device__ __forceinline__ unsigned cvt2(float a, float b) {
    typedef _Float16 h2 __attribute__((ext_vector_type(2)));
    h2 r = {(_Float16)a, (_Float16)b};
    return __builtin_bit_cast(unsigned, r);
}
__device__ __forceinline__ int rb_owner(int d, int rb) { return rb >= 6 ? d : ((((d >> 1) + 1 + (rb >> 1)) & 3) << 1) | (rb & 1); }
__device__ __forceinline__ int rb_local(int d, int rb) { return rb >= 6 ? (rb - 6) * 256 : (d & 1) * 256; }
__device__ __forceinline__ int rb_of(int d, int o) { return ((((o >> 1) - (d >> 1) - 1) & 3) << 1) | (o & 1); }
__device__ __forceinline__ void unit_of(int e, int j, int& c, int& id) {
    if (e < 32) { c = e >> 2; id = j * 4 + (e & 3); }
    else if (e < 44) { const int u = (e - 32) * 8 + j; c = u / 12; id = 32 + u - c * 12; }
    else { const int u = (e & 1) * 8 + j; c = u >> 1; id = (e < 46 ? 46 : 44) + (u & 1); }
}
__device__ __forceinline__ int gid_of(int d, int id) {
    const int rb = (id - 32) >> 1;
    return id < 32 ? (d >> 1) * 32 + id : 128 + rb_owner(d, rb) * 4 + (rb_local(d, rb) >> 7) + (id & 1);
}
__device__ __forceinline__ size_t stage_off(int id, int ks) {
    return id < 32 ? STG_A + (size_t)(id * NSTEP + ks) * B_STEP : (size_t)(((id - 32) >> 1) * NSTEP + ks) * A_STEP + (id & 1) * 16384;
}
template <int BITS, int SH> __device__ __forceinline__ unsigned wave_max_exp(unsigned mag) {
    unsigned e = 0;
#pragma unroll
    for (int b = BITS - 1; b >= 0; --b) if (__any(mag >= ((e + (1u << b)) << SH))) e += 1u << b;
    return e;
}

__device__ __forceinline__ unsigned row_scan(unsigned x) {
    x += __builtin_amdgcn_update_dpp(0, x, 0x111, 0xf, 0xf, true);
    x += __builtin_amdgcn_update_dpp(0, x, 0x112, 0xf, 0xf, true);
    x += __builtin_amdgcn_update_dpp(0, x, 0x114, 0xf, 0xf, true);
    x += __builtin_amdgcn_update_dpp(0, x, 0x118, 0xf, 0xf, true);
    return x;
}
__device__ __forceinline__ unsigned ffbl(unsigned x) { unsigned r; asm("v_ffbl_b32 %0, %1" : "=v"(r) : "v"(x)); return r; }
#define PK(x) __builtin_bit_cast(u16x2, (unsigned)(x))
#define DW(x) __builtin_bit_cast(unsigned, (u16x2)(x))
#define SUBS(a, b) DW(__builtin_elementwise_sub_sat(PK(a), PK(b)))
#define MIN2(a, b) DW(__builtin_elementwise_min(PK(a), PK(b)))

typedef float f32x2 __attribute__((ext_vector_type(2)));
__device__ __forceinline__ f32x2 gelu2(f32x2 y) {
    const float k1 = -2.0f * 0.7978845608028654f * 1.4426950408889634f, k3 = k1 * 0.044715f;
    const f32x2 a = __builtin_elementwise_fma(y * y, (f32x2){k3, k3}, (f32x2){k1, k1}) * y;
    const f32x2 d = (f32x2){__builtin_amdgcn_exp2f(a.x), __builtin_amdgcn_exp2f(a.y)} + 1.0f;
    return y * (f32x2){__builtin_amdgcn_rcpf(d.x), __builtin_amdgcn_rcpf(d.y)};
}
__device__ __forceinline__ float gelu1(float y) { return gelu2((f32x2){y, y}).x; }

#define STAMP(i) do {} while (0)
#define TIMED_OUT(t0) (UNI(__hip_atomic_load(tmo, RLX, SC_AGENT)) == call || __builtin_amdgcn_s_memrealtime() - (t0) > DEADLINE_TICKS)

__global__ __launch_bounds__(NTHR, 2) void fused_gemm_rs(Params p) {
    __shared__ __attribute__((aligned(16))) unsigned char lds_raw[3 * STAGE + 2048];
    LAS unsigned char* lds = (LAS unsigned char*)lds_raw;
    LAS unsigned* ctl = (LAS unsigned*)(lds + 3 * STAGE);
    const unsigned lds0 = (unsigned)(size_t)lds;
    const int tid = threadIdx.x, lane = tid & 63;
    const int wid = UNI(tid >> 6);
    const int fr = lane & 15, fq = lane >> 4;
    const unsigned lo = lane * 16u;
    const int xcd = blockIdx.x & 7, idx = blockIdx.x >> 3;
    const bool late = idx < 8;
    const int q = wid & 3;

    const int team = (wid >> 2) * 192 + (int)blockIdx.x - 64;
    f32x4 r[8];
    auto request = [&](int g, int u) {
        if (g < 32) {
            const GAS char* b = (const GAS char*)(p.w + (size_t)((g >> 2) * 64 + (q & 1) * 32) * GN + (u * 4 + (g & 3)) * 128 + (q >> 1) * 64) + (unsigned)(8 * fq * GN + 4 * fr) * 4u;
#pragma unroll
            for (int e = 0; e < 8; ++e) r[e] = *(const GAS f32x4*)(b + (size_t)e * (GN * 4));
        } else {
            const GAS char* b = (const GAS char*)(p.x + (size_t)(u * 128 + q * 32) * KSH + (g - 32) * 64) + (unsigned)(fr * KSH + 8 * fq) * 4u;
#pragma unroll
            for (int e = 0; e < 8; ++e) r[e] = *(const GAS f32x4*)(b + (e >> 2) * (16 * KSH * 4) + ((e >> 1) & 1) * 128 + (e & 1) * 16);
        }
    };
    if (!late && team < 96) request(team >= 64 ? 32 : (team >> 5) * 4, team & 31);
    const unsigned t_entry = now8();

    const int rank = p.rank & 7;
    char* myws = p.myws;
    STAMP(0);
    GAS unsigned* ctr = (GAS unsigned*)(myws + WS_CTR + (size_t)blockIdx.x * 128);
    GAS unsigned* tmo = (GAS unsigned*)(myws + WS_TMO);
    const unsigned ctr_v = __hip_atomic_load(ctr, RLX, SC_AGENT);
    GAS unsigned* gflag = (GAS unsigned*)(myws + WS_FLAG);
    ctl[tid] = 0u;
    WAIT_L0();
    BAR();

    if (!late) {
        unsigned wlo = 0, whi = 0;
#pragma unroll
        for (int d = 0; d < NR; ++d) {
            unsigned a = (unsigned)(size_t)p.ws[d], b = (unsigned)((size_t)p.ws[d] >> 32);
            asm volatile("" : "+s"(a), "+s"(b));
            wlo = lane == d ? a : wlo;
            whi = lane == d ? b : whi;
        }
#pragma unroll 1
        for (int it = 0; it < 4; ++it) {
            const int t = it * 384 + team;
            if (t >= 1280) break;
            const int g = p.prod[t >> 5], u = t & 31;
            if (t >= 96) {
                const unsigned ts = t_entry + (unsigned)((t >> 5) - PROD_OFF) * (PROD_GAP * 64);
                while ((int)(now8() - ts) < 0) __builtin_amdgcn_s_sleep(2);
                request(g, u);
            }
            const bool isw = g < 32;
            const int c = isw ? g >> 2 : g - 32;
            const int gid = isw ? u * 4 + (g & 3) : 128 + u;
            unsigned v[16];
            if (isw) {
#pragma unroll
                for (int i = 0; i < 16; ++i) v[i] = cvt2(r[(i & 3) * 2][i >> 2], r[(i & 3) * 2 + 1][i >> 2]);
            } else {
#pragma unroll
                for (int i = 0; i < 16; ++i) v[i] = cvt2(r[i >> 1][(i & 1) * 2], r[i >> 1][(i & 1) * 2 + 1]);
            }
            asm volatile("" : "+v"(v[15]));
            const unsigned t_loaded = now8();
            const bool pushed = t < 32;
            const unsigned call = UNI(ctr_v) + 1u;
            const size_t par = WS_EXI + (size_t)(call & 1u) * EXI_BYTES;
            const int o = u >> 2;
            const int myid = isw ? ((gid >> 5) == (rank >> 1) ? gid & 31 : -1)
                                 : (o == rank ? 44 + (u & 3) : ((o >> 1) != (rank >> 1) && ((u >> 1) & 1) == (rank & 1) ? 32 + rb_of(rank, o) * 2 + (u & 1) : -1));
            unsigned hdr = 0;
            {
                unsigned w[16];
#pragma unroll
                for (int i = 0; i < 16; ++i) w[i] = v[i];
                const char* home = myws + par + (size_t)(gid * 8 + c) * SLOT;
                unsigned pw = 0;
                hdr = 32u;
                if (!pushed) {
                    LAS unsigned* pool = (LAS unsigned*)(lds + wid * 256);
                    pool[lane] = 0u;
                    unsigned ef[16];
#pragma unroll
                    for (int i = 0; i < 16; ++i) ef[i] = (w[i] >> 10) & 0x001f001fu;
                    u16x2 mx = PK(ef[0]);
#pragma unroll
                    for (int i = 1; i < 16; ++i) mx = __builtin_elementwise_max(mx, PK(ef[i]));
                    const int emax = (int)wave_max_exp<5, 0>(mx.x > mx.y ? mx.x : mx.y);
                    int up = 0, mid = 0;
#pragma unroll
                    for (int i = 0; i < 6; ++i) {
                        const int e = (int)(ef[i] >> 16);
                        up += __builtin_popcountll(__ballot(e == emax)) - __builtin_popcountll(__ballot(e == emax - 3));
                        mid += __builtin_popcountll(__ballot(e == emax - 1)) - __builtin_popcountll(__ballot(e == emax - 4));
                    }
                    const int bs = emax - (up > 0 ? 3 : mid > 0 ? 4 : 5);
                    const unsigned B = bs > 0 ? (unsigned)bs : 0u, b2 = B * 0x00010001u;
                    const unsigned lut = 0x0300u | ((B + 2u + (B ? 1u : 0u)) << 16);
                    u64 str = 0;
                    unsigned sh = 0;
#pragma unroll
                    for (int i = 0; i < 16; ++i) {
                        const unsigned k1 = SUBS(b2 + 0x00010001u, ef[i]), d = SUBS(ef[i], b2);
                        const unsigned len = k1 + MIN2(SUBS(k1, 0x00020002u), 0x00010001u) + __builtin_amdgcn_perm(0u, lut, SUBS(d, 0x00030003u) | 0x0c000c00u);
                        sh += len & 0xffffu; str |= 1ull << (sh & 63u);
                        sh += len >> 16; str |= 1ull << (sh & 63u);
                        w[i] = (w[i] & 0x83ff83ffu) | ((MIN2(d, 0x00040004u) << 10) & 0x0c000c00u);
                    }
                    const unsigned end = row_scan(sh), pos = end - sh;
                    if (__all(sh <= 63u && end <= 32u * PW)) {
                        str >>= 1;
                        const u64 s0 = str << (pos & 31u);
                        LAS unsigned* pp = pool + fq * PW + (pos >> 5);
                        __hip_atomic_fetch_or(pp, (unsigned)s0, RLX, SC_WG);
                        __hip_atomic_fetch_or(pp + 1, (unsigned)(s0 >> 32), RLX, SC_WG);
                        __hip_atomic_fetch_or(pp + 2, (unsigned)(((str >> 32) << (pos & 31u)) >> 32), RLX, SC_WG);
#pragma unroll
                        for (int k = 0; k < 3; ++k) {
#pragma unroll
                            for (int i = 0; i < 4; ++i) w[4 * k + i] |= (w[13 + k] << (12 - 3 * i)) & 0x70007000u;
                            w[12] |= (w[13 + k] >> (1 + k)) & (0x40004000u >> k);
                        }
                        pw = __hip_atomic_load(pool + lane, RLX, SC_WG);
                        hdr = B;
                    }
                }
                if (hdr == 32u) {
#pragma unroll
                    for (int i = 0; i < 13; ++i) w[i] = v[i];
                }
                if (pushed) {
                    unsigned tp = t_loaded + (unsigned)((u & 7) * 4 + q) * PUSH_SLOT;
#pragma unroll
                    for (int i = 0; i < 4; ++i) {
                        pace(tp, 32u * PUSH_SLOT);
#pragma unroll
                        for (int k = 0; k < 2; ++k) {
                            const int d = (u >> 3) * 2 + k;
                            if (d != rank) st16(p.ws[d] + par + (size_t)(1280 + rank * 8 + (u & 7)) * SLOT + q * 4096 + i * 1024, lo, (u32x4){v[4 * i], v[4 * i + 1], v[4 * i + 2], v[4 * i + 3]});
                        }
                    }
                } else {
#pragma unroll
                    for (int i = 0; i < 3; ++i) st16(home + q * QB + i * 1024, lo, (u32x4){w[4 * i], w[4 * i + 1], w[4 * i + 2], w[4 * i + 3]});
                    st4(home + q * QB + 3072, lane * 4u, w[12]);
                    if (hdr == 32u) st12(home + SIDE + q * 768, lane * 12u, (u32x3){v[13], v[14], v[15]});
                    else if (lane < 4 * PW) st4(home + q * QB + 3328, lane * 4u, pw);
                }
            }
            if (myid >= 0) {
                const char* sg = myws + WS_STG + stage_off(myid, rank * 8 + c) + (isw ? (q >> 1) * 8 + (q & 1) : q * 4) * 1024;
#pragma unroll
                for (int i = 0; i < 4; ++i) st16(sg + i * (isw ? 2048 : 1024), lo, (u32x4){v[4 * i], v[4 * i + 1], v[4 * i + 2], v[4 * i + 3]});
            }
            wait_vm<0>();
            unsigned tw = (1u << 24) | (hdr << (6 * q));
            if (lane == 0) tw += __hip_atomic_fetch_add(ctl + 392 + (wid >> 2) * 4 + it, tw, RLX, SC_WG);
            tw = UNI(tw);
            if ((tw >> 24) == 4u) {
                const int d = lane & 7;
                int did = isw ? ((gid >> 5) == (d >> 1) ? gid & 31 : -1)
                              : (o == d ? 44 + (u & 3) : ((o >> 1) != (d >> 1) && ((u >> 1) & 1) == (d & 1) ? 32 + rb_of(d, o) * 2 + (u & 1) : -1));
                if (isw && (g & 3) == 3 && c < 2 * NFWD && (d >> 1) != (rank >> 1) && ((c ^ d) & 1)) did = -1;
                if (lane < 8 && did >= 0) {
                    if (d == rank) __hip_atomic_store(gflag + did * 64 + c * 8 + rank, call, RLX, SC_AGENT);
                    else __hip_atomic_store((GAS u64*)((((size_t)whi << 32) | wlo) + WS_RDY) + ((((call & 1u) * 8 + rank) * 48 + did) * 8 + c) * RSTRIDE, ((u64)call << 32) | (tw & 0xffffffu), RLX, SC_SYS);
                }
            }
        }
        STAMP(9);
    } else if (idx < 7) {
        const int s = idx < 6 ? rb_owner(rank, idx) : rank ^ 1;
        const int nu = idx < 6 ? NU_OUT : NU_PAIR;
        unsigned desc[2];
#pragma unroll
        for (int k = 0; k < 2; ++k) {
            const int e = p.order[idx == 6][k * 64 + lane];
            int c, id, org = s, slot;
            if (e < 48) { unit_of(e, xcd, c, id); slot = gid_of(rank, id) * 8 + c; }
            else { const int kf = (e - 48) / 6; c = 2 * kf + (s & 1); id = xcd * 4 + 3; org = rb_owner(rank, e - 48 - kf * 6); slot = 1408 + (e - 48) * 8 + xcd; }
            desc[k] = (unsigned)(c | id << 3 | org << 9 | slot << 12);
        }
        auto desc_of = [&](int n) { return (unsigned)(n < 64 ? __builtin_amdgcn_readlane(desc[0], n) : __builtin_amdgcn_readlane(desc[1], n - 64)); };
        const unsigned call = UNI(ctr_v) + 1u;
        const size_t par = WS_EXI + (size_t)(call & 1u) * EXI_BYTES;
        if (wid < 6) {
            int pend = -1, pend_n = 0;
            unsigned pend_h = 0;
            auto staged = [&]() {
                wait_vm<0>();
                if (lane == 0 && __hip_atomic_fetch_add(ctl + 256 + pend_n, 1u, RLX, SC_WG) == 3u) {
                    __hip_atomic_store(gflag + pend, call, RLX, SC_AGENT);
                    if (pend_h) __hip_atomic_store((GAS u64*)(p.ws[rank ^ 1] + WS_RDY) + ((((call & 1u) * 8 + s) * 48 + (pend >> 6)) * 8 + ((pend >> 3) & 7)) * RSTRIDE,
                                                   ((u64)call << 32) | (pend_h & 0xffffffu), RLX, SC_SYS);
                }
                pend = -1;
            };
            auto need = [&](int job) { return (job >> 2) * 14 - 14 + (((job & 3) + 1) * QB + 1023) / 1024; };
            auto ticket = [&]() { unsigned tk = 0; if (lane == 0) tk = __hip_atomic_fetch_add(ctl + 401, 1u, RLX, SC_WG); return (int)UNI(tk); };
            for (int job = ticket(); job < 4 * nu;) {
                const int n = job >> 2, qq = job & 3;
                unsigned w[16];
                int un = 0;
                unsigned relay_h = 0;
                if (n == 0) {
                    while (lds_get(ctl) == 0u) __builtin_amdgcn_s_sleep(1);
                    u32x4 a0, a1, a2, a3;
                    ld64(myws + par + (size_t)(1280 + s * 8 + xcd) * SLOT + qq * 4096, lo, a0, a1, a2, a3);
#pragma unroll
                    for (int i = 0; i < 4; ++i) { w[i] = a0[i]; w[4 + i] = a1[i]; w[8 + i] = a2[i]; w[12 + i] = a3[i]; }
                } else {
                    while ((int)lds_get(ctl + 384) < need(job)) __builtin_amdgcn_s_sleep(1);
                    asm volatile("" ::: "memory");
                    un = lds_get(ctl + 416 + n);
                }
                const unsigned de = desc_of(un);
                const int c = de & 7, id = (de >> 3) & 63, org = (de >> 9) & 7;
                if (n) {
                    const LAS unsigned char* src = lds + (unsigned)(n % RING) * 16384u + qq * QB;
#pragma unroll
                    for (int i = 0; i < 3; ++i) { const u32x4 a = *(const LAS u32x4*)(src + i * 1024 + lo); w[4 * i] = a[0]; w[4 * i + 1] = a[1]; w[4 * i + 2] = a[2]; w[4 * i + 3] = a[3]; }
                    w[12] = *(const LAS unsigned*)(src + 3072 + lane * 4);
                    const LAS unsigned* pool = (const LAS unsigned*)(src + 3328);
                    const unsigned hu = lds_get(ctl + un), h = (hu >> (6 * qq)) & 63u;
                    if (idx < 6 && id < 32 && (id & 3) == 3 && c < 2 * NFWD) {
                        const unsigned pw = pool[lane];
                        if (pend >= 0) staged();
                        const char* dst = myws + par + (size_t)(1408 + ((c >> 1) * 6 + idx) * 8 + xcd) * SLOT + qq * QB;
#pragma unroll
                        for (int i = 0; i < 3; ++i) st16(dst + i * 1024, lo, (u32x4){w[4 * i], w[4 * i + 1], w[4 * i + 2], w[4 * i + 3]});
                        st4(dst + 3072, lane * 4u, w[12]);
                        if (lane < 4 * PW) st4(dst + 3328, lane * 4u, pw);
                        relay_h = hu;
                    }
                    if (h < 32u) {
                        unsigned pd[PW];
#pragma unroll
                        for (int i = 0; i < PW; ++i) pd[i] = pool[fq * PW + i];
#pragma unroll
                        for (int k = 0; k < 3; ++k) {
                            unsigned cc = (w[12] << (1 + k)) & 0x80008000u;
#pragma unroll
                            for (int i = 0; i < 4; ++i) cc |= (w[4 * k + i] >> (12 - 3 * i)) & (0x00070007u << (3 * i));
                            w[13 + k] = cc;
                        }
                        unsigned ne[16], sum = 0;
#pragma unroll
                        for (int i = 0; i < 16; ++i) { ne[i] = MIN2(w[i] & 0x0c000c00u, 0x00010001u); sum += ne[i]; }
                        sum = 32u - (sum & 0xffffu) - (sum >> 16);
                        const unsigned skip = row_scan(sum) - sum;
                        unsigned cum = 0, at = 0, below = 0;
#pragma unroll
                        for (int i = 0; i < PW - 1; ++i) {
                            cum += __builtin_popcount(pd[i]);
                            const bool lt = cum < skip;
                            at += lt; below = lt ? cum : below;
                        }
                        const unsigned d0 = pool[fq * PW + at], d1 = pool[fq * PW + at + 1], d2 = pool[fq * PW + at + 2];
                        WAIT_L0();
                        if (lane == 0) __hip_atomic_fetch_add(ctl + 128 + n, 1u, RLX, SC_WG);
                        if (pend >= 0) staged();
                        unsigned rem = skip - below, pos = 0;
#pragma unroll
                        for (int sft = 16; sft; sft >>= 1) {
                            const unsigned t = __builtin_popcount(__builtin_amdgcn_ubfe(d0, pos, sft));
                            const bool lt = t < rem;
                            rem -= lt ? t : 0u; pos += lt ? sft : 0u;
                        }
                        pos = skip ? pos + 1u : 0u;
                        u64 str = (unsigned)((((u64)d1 << 32) | d0) >> pos) | ((((u64)d2 << 32) | d1) >> pos << 32);
                        const unsigned b2 = h * 0x00010001u;
#pragma unroll
                        for (int i = 0; i < 16; ++i) {
                            const unsigned l0 = ffbl(((unsigned)str << 1) | (ne[i] & 1u)); str >>= l0 & 63u;
                            const unsigned l1 = ffbl(((unsigned)str << 1) | (ne[i] >> 16)); str >>= l1 & 63u;
                            const unsigned len = l0 | (l1 << 16);
                            const unsigned adj = __builtin_amdgcn_perm(0x02u, 0x07010100u, MIN2(len, 0x00040004u) | 0x0c000c00u);
                            const unsigned e = MIN2(DW(PK(b2 + adj) - PK(len)), b2 + 0x00050005u);
                            w[i] = (w[i] & 0x8fff8fffu) + (e << 10);
                        }
                    } else {
                        WAIT_L0();
                        if (lane == 0) __hip_atomic_fetch_add(ctl + 128 + n, 1u, RLX, SC_WG);
                        if (pend >= 0) staged();
                        const u32x3 a = ld12(p.ws[org] + par + (size_t)(gid_of(rank, id) * 8 + c) * SLOT + SIDE + qq * 768, lane * 12u);
                        w[13] = a[0]; w[14] = a[1]; w[15] = a[2];
                    }
                }
                const char* sg = myws + WS_STG + stage_off(id, org * 8 + c) + (id < 32 ? (qq >> 1) * 8 + (qq & 1) : qq * 4) * 1024;
#pragma unroll
                for (int i = 0; i < 4; ++i) st16(sg + i * (id < 32 ? 2048 : 1024), lo, (u32x4){w[4 * i], w[4 * i + 1], w[4 * i + 2], w[4 * i + 3]});
                pend = id * 64 + c * 8 + org; pend_n = n; pend_h = relay_h;
                job = ticket();
                if (job >= 4 * nu || job < 4 || (int)lds_get(ctl + 384) < need(job)) staged();
            }
        } else if (wid == 6) {
            const char* exs = p.ws[s] + par;
            unsigned tnext = t_entry + xcd * (TAU / 4);
            u64 todo = ~1ull;
            __builtin_amdgcn_s_setprio(3);
            const char* src = nullptr;
            unsigned trim = 0;
            auto choose = [&](int n) {
                const unsigned rdy = __hip_atomic_load(ctl + (n >= 64 ? n : lane), RLX, SC_WG);
                const unsigned fre = __hip_atomic_load(ctl + (n > RING ? 128 + n - RING : 400), RLX, SC_WG), tr = __hip_atomic_load(ctl + 400, RLX, SC_WG);
                const u64 rd = __ballot(rdy != 0u) & (n >= 64 ? 1ull : todo);
                if (!rd || (n > RING && UNI(fre) < 4u)) return;
                const int un = n >= 64 ? n : __builtin_ctzll(rd);
                todo &= ~(1ull << (un & 63));
                if (lane == 0) __hip_atomic_store(ctl + 416 + n, (unsigned)un, RLX, SC_WG);
                trim = n >= EQ_AT + 4 && n < EQ_AT + 36 ? UNI(tr) : 0u;
                src = exs + (size_t)(desc_of(un) >> 12) * SLOT;
            };
            for (int n = 1; n < nu; ++n) {
                bool drained = false;
                if (!src) {
                    choose(n);
                    if (!src) {
                        wait_vm<0>();
                        if (lane == 0) __hip_atomic_store(ctl + 384, (unsigned)(n - 1) * 14u, RLX, SC_WG);
                        drained = true;
                        for (choose(n); !src; choose(n)) __builtin_amdgcn_s_sleep(1);
                    }
                }
                const char* cur = src;
                const unsigned dst = lds0 + (unsigned)(n % RING) * 16384u, tr = trim;
                src = nullptr;
                asm volatile("" ::: "memory");
#pragma unroll
                for (int i = 0; i < 14; i += 2) {
                    wait_vm<WIN - 2>();
                    if ((i > WIN - 2 || (!drained && n > 1)) && lane == 0) __hip_atomic_store(ctl + 384, (unsigned)((n - 1) * 14 + i - (WIN - 2)), RLX, SC_WG);
                    pace(tnext, (i < 12 ? 2u * TAU : UNIT_T - 12u * TAU) - tr);
                    if (i < 12) dma<true>(cur + i * 1024, lo, dst + i * 1024);
                    else {
                        dma<false>(cur + 12288, lo, dst + 12288);
                        if (lane < (4 * QB - 13312) / 16) dma<false>(cur + 13312, lo, dst + 13312);
                    }
                    if (i == 0 && lane == 0) {
                        if (n == EQ_AT) __hip_atomic_store(ctl + 387, tnext | 1u, RLX, SC_WG);
                        if (n == nu - 1) __hip_atomic_store(ctl + 385, (tnext + UNIT_T - 2u * TAU) | 1u, RLX, SC_WG);
                    }
                    if (i == 10 && n + 1 < nu) choose(n + 1);
                }
            }
            __builtin_amdgcn_s_setprio(0);
            wait_vm<0>();
            if (lane == 0) __hip_atomic_store(ctl + 384, (unsigned)(nu - 1) * 14u, RLX, SC_WG);
            STAMP(9);
        } else {
            const GAS u64* rp[2];
            bool pend[2];
#pragma unroll
            for (int k = 0; k < 2; ++k) {
                rp[k] = (const GAS u64*)(myws + WS_RDY) + ((((call & 1u) * 8 + ((desc[k] >> 9) & 7)) * 48 + ((desc[k] >> 3) & 63)) * 8 + (desc[k] & 7)) * RSTRIDE;
                pend[k] = k * 64 + lane < nu;
            }
            GAS u64* eq = (GAS u64*)(myws + WS_EQ) + idx * 16;
            unsigned mine = 0;
            bool eq_open = EQ_AT != 0;
            const u64 t0 = __builtin_amdgcn_s_memrealtime();
            for (unsigned polls = 1;; ++polls) {
                const u64 pm = __ballot(pend[0]);
                const bool front = __builtin_amdgcn_mbcnt_hi((unsigned)(pm >> 32), __builtin_amdgcn_mbcnt_lo((unsigned)pm, 0u)) < POLLW;
#pragma unroll
                for (int k = 0; k < 2; ++k)
                    if (pend[k] && ((k == 0 && front) || (polls & 7u) == 0u)) {
                        const u64 f = __hip_atomic_load(rp[k], RLX, SC_SYS);
                        if ((unsigned)(f >> 32) >= call) { __hip_atomic_store(ctl + k * 64 + lane, (unsigned)f | 0x80000000u, RLX, SC_WG); pend[k] = false; }
                    }
                if (eq_open) {
                    if (!mine) {
                        mine = lds_get(ctl + 387);
                        if (mine && lane == 0) __hip_atomic_store(eq + xcd, ((u64)call << 32) | mine, RLX, SC_AGENT);
                    } else {
                        const u64 f = __hip_atomic_load(eq + (lane & 7), RLX, SC_AGENT);
                        if (__all((unsigned)(f >> 32) == call)) {
                            int dlt = (int)(mine - (unsigned)f);
                            dlt += __shfl_xor(dlt, 1); dlt += __shfl_xor(dlt, 2); dlt += __shfl_xor(dlt, 4);
                            dlt /= 8 * 32 * 7;
                            dlt = dlt > TAU / 8 ? TAU / 8 : dlt < -(TAU / 8) ? -(TAU / 8) : dlt;
                            if (lane == 0) __hip_atomic_store(ctl + 400, (unsigned)dlt, RLX, SC_WG);
                            eq_open = false;
                        } else if ((int)lds_get(ctl + 384) >= (EQ_AT + 2) * 14) eq_open = false;
                    }
                }
                const bool waiting = __any(pend[0] || pend[1]);
                if (!waiting && !eq_open) break;
                if (waiting && (polls & 15u) == 0u && TIMED_OUT(t0)) {
                    if (lane == 0) __hip_atomic_store(tmo, call, RLX, SC_AGENT);
                    wait_vm<0>();
#pragma unroll
                    for (int k = 0; k < 2; ++k) if (pend[k]) __hip_atomic_store(ctl + k * 64 + lane, 0x80000000u, RLX, SC_WG);
                    break;
                }
                if (polls < 32u) __builtin_amdgcn_s_sleep(1); else __builtin_amdgcn_s_sleep(16);
            }
            if (idx < 6) {
                unsigned th;
                while ((th = lds_get(ctl + 385)) == 0u) __builtin_amdgcn_s_sleep(4);
                if (lane == 0) __hip_atomic_store((GAS u64*)(myws + WS_MBOX + (size_t)((8 + idx * 4) * 8 + xcd) * 128), ((u64)call << 32) | th, RLX, SC_AGENT);
            }
        }
    }
    const unsigned call = UNI(ctr_v) + 1u;

    const int ntile = idx == 7 ? 2 : 1;
    const int wm = wid >> 1, wn = wid & 1;
#pragma unroll 1
    for (int rep = 0; rep < ntile; ++rep) {
    const int rb = late ? 6 + (idx >> 2) : (idx - 8) >> 2;
    const int cb = xcd * 4 + (idx == 7 ? rep : idx == 6 ? 0 : idx == 4 || idx == 5 ? idx - 2 : idx & 3);
    const bool part1 = idx == 7 && rep == 0, part2 = idx == 6;
    const int nstep = part1 ? 56 : part2 ? 8 : NSTEP;
    const char* stgA = myws + WS_STG + (size_t)(rb * NSTEP) * A_STEP + wid * 4096;
    const char* stgB = myws + WS_STG + STG_A + (size_t)(cb * NSTEP) * B_STEP + wid * 2048;
    const u64 pbits = 0x0101010101010101ull << (rank ^ 1);
    u64 todo = part1 ? ~pbits : part2 ? pbits : ~0ull, avail = 0ull;
    int slot = 0;
    auto poll = [&]() {
        if (wid == 0) {
            const u64 t0 = __builtin_amdgcn_s_memrealtime();
            u64 got;
            for (unsigned polls = 1;; ++polls) {
                const unsigned f0 = __hip_atomic_load(gflag + cb * 64 + lane, RLX, SC_AGENT);
                const unsigned f1 = __hip_atomic_load(gflag + (32 + rb * 2) * 64 + lane, RLX, SC_AGENT);
                const unsigned f2 = __hip_atomic_load(gflag + (33 + rb * 2) * 64 + lane, RLX, SC_AGENT);
                got = __ballot(f0 >= call && f1 >= call && f2 >= call) & todo;
                if (got) break;
                if ((polls & 15u) == 0u && TIMED_OUT(t0)) {
                    if (lane == 0) __hip_atomic_store(tmo, call, RLX, SC_AGENT);
                    got = todo;
                    break;
                }
                __builtin_amdgcn_s_sleep(POLL_SLEEP);
            }
            if (lane < 2) ctl[388 + slot * 2 + lane] = (unsigned)(got >> (32 * lane));
        }
    };
    auto news = [&]() { avail = ((u64)(unsigned)UNI(ctl[389 + slot * 2]) << 32) | (unsigned)UNI(ctl[388 + slot * 2]); slot ^= 1; };
    auto take = [&]() {
        const unsigned b = (unsigned)__builtin_ctzll(avail) & 63u;
        avail &= avail - 1; todo &= ~(1ull << b);
        return (b & 7u) * 8u + (b >> 3);
    };
    auto g1 = [&](unsigned ks, unsigned buf, int i) {
        if (i < 4) __builtin_amdgcn_global_load_lds((const GAS unsigned*)(stgA + (size_t)ks * A_STEP + i * 1024 + lo), (LAS unsigned*)(lds + buf + wid * 4096 + i * 1024), 16, 0, 17);
        else __builtin_amdgcn_global_load_lds((const GAS unsigned*)(stgB + (size_t)ks * B_STEP + (i - 4) * 1024 + lo), (LAS unsigned*)(lds + buf + A_STEP + wid * 2048 + (i - 4) * 1024), 16, 0, 17);
    };

    f32x4 acc[4][4];
#pragma unroll
    for (int m = 0; m < 4; ++m)
#pragma unroll
        for (int n = 0; n < 4; ++n) acc[m][n] = (f32x4){0.f, 0.f, 0.f, 0.f};
    h16x8 a0[4], b0[4], a1[4], b1[4];
    const unsigned aoff = wm * 8192 + lo, boff = A_STEP + wn * 8192 + lo;
#define RDA(A, buf, kk, m) A[m] = *(const LAS h16x8*)(lds + (buf) + aoff + (m) * 2048 + (kk) * 1024)
#define RDB(B, buf, kk, n) B[n] = *(const LAS h16x8*)(lds + (buf) + boff + (n) * 2048 + (kk) * 1024)
#define ROW(A, B, m) _Pragma("unroll") for (int n = 0; n < 4; ++n) acc[m][n] = __builtin_amdgcn_mfma_f32_16x16x32_f16(A[m], B[n], acc[m][n], 0, 0, 0)

    STAMP(10);
    unsigned kq;
#pragma unroll
    for (int k = 0; k < 3; ++k) {
        if (!avail) { poll(); WAIT_L0(); BAR(); news(); }
        kq = take();
#pragma unroll
        for (int i = 0; i < (k < 2 ? 6 : 3); ++i) g1(kq, k * STAGE, i);
    }
    wait_vm<9>(); BAR();
#pragma unroll
    for (int i = 0; i < 4; ++i) { RDB(b0, 0u, 0, i); RDA(a0, 0u, 0, i); }
    unsigned cur = 0, prv = 2 * STAGE;
    for (int t = 0; t < nstep; ++t) {
        const unsigned nxt = cur == 2 * STAGE ? 0u : cur + STAGE;
        const bool m2 = t + 2 < nstep, m3 = t + 3 < nstep;
        const bool dry = m3 && !avail;
        WAIT_L0(); SB();
        RDB(b1, cur, 1, 0); RDB(b1, cur, 1, 1); SB(); ROW(a0, b0, 0); SB(); if (m2) g1(kq, prv, 3); SB();
        RDB(b1, cur, 1, 2); RDB(b1, cur, 1, 3); SB(); ROW(a0, b0, 1); SB(); if (m2) g1(kq, prv, 4); SB();
        RDA(a1, cur, 1, 0); RDA(a1, cur, 1, 1); SB(); ROW(a0, b0, 2); SB(); if (m2) g1(kq, prv, 5); SB();
        RDA(a1, cur, 1, 2); RDA(a1, cur, 1, 3); SB(); ROW(a0, b0, 3); SB();
        if (dry) poll();
        WAIT_L0();
        if (m2) wait_vm<6>(); else wait_vm<0>();
        BAR(); SB();
        RDB(b0, nxt, 0, 0); RDB(b0, nxt, 0, 1); SB(); ROW(a1, b1, 0); SB();
        if (dry) news();
        if (m3) { kq = take(); g1(kq, cur, 0); } SB();
        RDB(b0, nxt, 0, 2); RDB(b0, nxt, 0, 3); SB(); ROW(a1, b1, 1); SB(); if (m3) g1(kq, cur, 1); SB();
        RDA(a0, nxt, 0, 0); RDA(a0, nxt, 0, 1); SB(); ROW(a1, b1, 2); SB(); if (m3) g1(kq, cur, 2); SB();
        RDA(a0, nxt, 0, 2); RDA(a0, nxt, 0, 3); SB(); ROW(a1, b1, 3); SB();
        prv = cur; cur = nxt;
    }

    if (part1 || part2) {
        const char* park = myws + WS_EXO + (size_t)(xcd * 8 + wid) * 16384;
        GAS u64* pf = (GAS u64*)(myws + WS_ORDY + (size_t)xcd * 128) + wid;
        if (part1) {
#pragma unroll
            for (int i = 0; i < 16; ++i) st16(park + i * 1024, lo, __builtin_bit_cast(u32x4, acc[i >> 2][i & 3]));
            wait_vm<0>();
            if (lane == 0) __hip_atomic_store(pf, (u64)call << 32, RLX, SC_SYS);
            continue;
        }
        const u64 t0 = __builtin_amdgcn_s_memrealtime();
        while ((unsigned)UNI((unsigned)(__hip_atomic_load(pf, RLX, SC_SYS) >> 32)) < call) {
            if (TIMED_OUT(t0)) { if (lane == 0) __hip_atomic_store(tmo, call, RLX, SC_AGENT); wait_vm<0>(); break; }
            __builtin_amdgcn_s_sleep(4);
        }
#pragma unroll
        for (int m = 0; m < 4; ++m) {
            u32x4 e0, e1, e2, e3;
            ld64(park + m * 4096, lo, e0, e1, e2, e3);
            acc[m][0] += __builtin_bit_cast(f32x4, e0); acc[m][1] += __builtin_bit_cast(f32x4, e1);
            acc[m][2] += __builtin_bit_cast(f32x4, e2); acc[m][3] += __builtin_bit_cast(f32x4, e3);
        }
    }
    const unsigned tmo_v = __hip_atomic_load(tmo, RLX, SC_AGENT);
    const unsigned oo = (unsigned)(fq * 4) * (GN * 4u) + fr * 16u;
    const u32x4 nan4 = {0x7fc00000u, 0x7fc00000u, 0x7fc00000u, 0x7fc00000u};
    unsigned y[64];
#pragma unroll
    for (int i = 0; i < 32; ++i) {
        const int m = i >> 3, n = (i >> 1) & 3, j = (i & 1) * 2;
        const f32x2 g = gelu2((f32x2){acc[m][n][j], acc[m][n][j + 1]});
        y[(m * 4 + j) * 4 + n] = __float_as_uint(g.x); y[(m * 4 + j + 1) * 4 + n] = __float_as_uint(g.y);
    }
    if (late) {
        const bool bad = UNI(tmo_v) == call;
        const char* ob = (const char*)(p.out + (size_t)(rb_local(rank, rb) + wm * 64) * GN + (rank >> 1) * 4096 + cb * 128 + wn * 64);
#pragma unroll
        for (int i = 0; i < 16; ++i) st16(ob + (size_t)((i >> 2) * 16 + (i & 3)) * (GN * 4), oo, bad ? nan4 : (u32x4){y[4 * i], y[4 * i + 1], y[4 * i + 2], y[4 * i + 3]});
    } else {
        const int peer = rb_owner(rank, rb);
        const int mirror = (8 + rb_of(peer, rank) * 4 + (idx & 3)) * 8 + xcd;
        char* const peerws = p.ws[peer];
        {
            const char* ex = myws + WS_EXO + (size_t)(blockIdx.x * 8 + wid) * 16384;
            unsigned hdr = 0;
            if (PACK_OUT) {
                float mx = 0.f;
#pragma unroll
                for (int i = 0; i < 64; i += 2) mx = __builtin_fmaxf(__builtin_fmaxf(mx, __builtin_fabsf(__builtin_bit_cast(float, y[i]))), __builtin_fabsf(__builtin_bit_cast(float, y[i + 1])));
                const unsigned emax = wave_max_exp<8, 23>(__builtin_bit_cast(unsigned, mx));
                const unsigned base = emax > 15u ? emax - 15u : 0u, bb = base << 23;
                hdr = base;
#pragma unroll
                for (int hf = 0; hf < 2; ++hf) {
                    unsigned z[28];
#pragma unroll
                    for (int g4 = 0; g4 < 4; ++g4) {
                        const int g = hf * 4 + g4;
                        unsigned dd[8], any = 0;
#pragma unroll
                        for (int i = 0; i < 8; ++i) { dd[i] = y[8 * g + i] - bb; any |= dd[i]; }
                        if (__all(!(any & 0x78000000u))) {
                            const unsigned cc = (dd[7] & 0x07ffffffu) | ((dd[7] >> 4) & 0x08000000u);
#pragma unroll
                            for (int i = 0; i < 7; ++i) z[7 * g4 + i] = dd[i] | ((cc << (27 - 4 * i)) & 0x78000000u);
                        } else {
#pragma unroll
                            for (int i = 0; i < 7; ++i) z[7 * g4 + i] = y[8 * g + i];
                            st4(ex + 14336 + g * 256, lane * 4u, y[8 * g + 7]);
                            hdr |= 256u << g;
                        }
                    }
#pragma unroll
                    for (int i = 0; i < 7; ++i) st16(ex + (hf * 7 + i) * 1024, lo, (u32x4){z[4 * i], z[4 * i + 1], z[4 * i + 2], z[4 * i + 3]});
                }
            } else {
#pragma unroll
                for (int i = 0; i < 16; ++i) st16(ex + i * 1024, lo, (u32x4){y[4 * i], y[4 * i + 1], y[4 * i + 2], y[4 * i + 3]});
            }
            wait_vm<0>();
            hdr |= UNI(tmo_v) == call ? 0xc0000000u : 0x40000000u;
            if (lane == 0) __hip_atomic_store((GAS u64*)(peerws + WS_ORDY + (size_t)mirror * 128) + wid, ((u64)call << 32) | hdr, RLX, SC_SYS);
        }
        WAIT_L0(); BAR();
        STAMP(22);
        const char* exs = peerws + WS_EXO + (size_t)mirror * (8 * 16384);
        constexpr int HALF = OPS_OUT / 2;
        unsigned raw[4];
        auto get_raw = [&](int blk, int hf, unsigned hdr) {
#pragma unroll
            for (int g4 = 0; g4 < 4; ++g4) {
                raw[g4] = 0u;
                if (PACK_OUT && (hdr & (256u << (hf * 4 + g4)))) raw[g4] = ld4(exs + (size_t)blk * 16384 + 14336 + (hf * 4 + g4) * 256, lane * 4u);
            }
        };
        auto unpack = [&](int blk, int hf, unsigned hdr) {
            const LAS unsigned char* src = lds + (unsigned)blk * 16384u + hf * (HALF * 1024) + lo;
            unsigned z[32], o[32];
#pragma unroll
            for (int i = 0; i < HALF; ++i) { const u32x4 a = *(const LAS u32x4*)(src + i * 1024); z[4 * i] = a[0]; z[4 * i + 1] = a[1]; z[4 * i + 2] = a[2]; z[4 * i + 3] = a[3]; }
            const unsigned bb = (hdr & 255u) << 23;
#pragma unroll
            for (int g4 = 0; g4 < 4; ++g4) {
                if (!PACK_OUT) {
#pragma unroll
                    for (int i = 0; i < 8; ++i) o[8 * g4 + i] = z[8 * g4 + i];
                } else if (!(hdr & (256u << (hf * 4 + g4)))) {
                    unsigned cc = 0;
#pragma unroll
                    for (int i = 0; i < 7; ++i) { cc |= (z[7 * g4 + i] & 0x78000000u) >> (27 - 4 * i); o[8 * g4 + i] = (z[7 * g4 + i] & 0x87ffffffu) + bb; }
                    o[8 * g4 + 7] = ((cc & 0x07ffffffu) | ((cc << 4) & 0x80000000u)) + bb;
                } else {
#pragma unroll
                    for (int i = 0; i < 7; ++i) o[8 * g4 + i] = z[7 * g4 + i];
                    o[8 * g4 + 7] = raw[g4];
                }
            }
            const char* ob = (const char*)(p.out + (size_t)((peer & 1) * 256 + (blk >> 1) * 64 + hf * 32) * GN + (peer >> 1) * 4096 + cb * 128 + (blk & 1) * 64);
#pragma unroll
            for (int i = 0; i < 8; ++i)
                st16(ob + (size_t)((i >> 2) * 16 + (i & 3)) * (GN * 4), oo, (hdr >> 31) ? nan4 : (u32x4){o[4 * i], o[4 * i + 1], o[4 * i + 2], o[4 * i + 3]});
        };
        if (wid == 7) {
            unsigned tnext;
            while ((tnext = lds_get(ctl + 8)) == 0u) __builtin_amdgcn_s_sleep(2);
            STAMP(23);
            __builtin_amdgcn_s_setprio(3);
            wait_vm<0>(); __builtin_amdgcn_s_waitcnt(0x0F70);
            for (int n = 0; n < 8; ++n) {
                if (lds_get(ctl + n) == 0u) {
                    wait_vm<0>();
                    if (lane == 0) __hip_atomic_store(ctl + 384, (unsigned)n, RLX, SC_WG);
                    while (lds_get(ctl + n) == 0u) __builtin_amdgcn_s_sleep(1);
                }
                asm volatile("" ::: "memory");
                if (n == 7 && lane == 0) __hip_atomic_store(ctl + 385, (tnext + OPS_OUT * TAU_OUT) | 1u, RLX, SC_WG);
#pragma unroll
                for (int i = 0; i < OPS_OUT; i += 2) {
                    wait_vm<WIN_OUT - 2>();
                    if (i == WIN_OUT - 2 && lane == 0) __hip_atomic_store(ctl + 384, (unsigned)n, RLX, SC_WG);
                    pace(tnext, 2u * TAU_OUT);
                    dma<true>(exs + (size_t)n * 16384 + i * 1024, lo, lds0 + (unsigned)n * 16384u + i * 1024);
                }
            }
            wait_vm<HALF>();
            if (lane == 0) __hip_atomic_store(ctl + 384, 8u, RLX, SC_WG);
            wait_vm<0>();
            if (lane == 0) __hip_atomic_store(ctl + 384, 9u, RLX, SC_WG);
            STAMP(19);
        } else {
            if (wid == 6) {
                const GAS u64* fp = lane < 8 ? (const GAS u64*)(myws + WS_ORDY + (size_t)blockIdx.x * 128) + lane : (const GAS u64*)(myws + WS_MBOX + (size_t)blockIdx.x * 128);
                bool pend = lane < 9;
                const u64 t0 = __builtin_amdgcn_s_memrealtime();
                for (unsigned polls = 1;; ++polls) {
                    if (pend) {
                        const u64 f = __hip_atomic_load(fp, RLX, SC_SYS);
                        if ((unsigned)(f >> 32) >= call) { __hip_atomic_store(ctl + lane, (unsigned)f, RLX, SC_WG); pend = false; }
                    }
                    if (!__any(pend)) break;
                    if ((polls & 15u) == 0u && TIMED_OUT(t0)) {
                        if (lane == 0) __hip_atomic_store(tmo, call, RLX, SC_AGENT);
                        if (pend) __hip_atomic_store(ctl + lane, lane < 8 ? 0xc0000000u : now8() | 1u, RLX, SC_WG);
                        break;
                    }
                    __builtin_amdgcn_s_sleep(4);
                }
            }
            for (int job = 0; job < (wid == 1 || wid == 2 ? 3 : 2); ++job) {
                const int blk = job < 2 ? wid : 7, hf = job < 2 ? job : wid - 1, need = job < 2 ? wid : 6 + wid;
                unsigned hdr;
                while ((hdr = lds_get(ctl + blk)) == 0u) __builtin_amdgcn_s_sleep(4);
                get_raw(blk, hf, hdr);
                while ((int)lds_get(ctl + 384) <= need) __builtin_amdgcn_s_sleep(1);
                asm volatile("" ::: "memory");
                unpack(blk, hf, hdr);
            }
            if (wid == 0 && (idx & 3) != 3) {
                unsigned th;
                while ((th = lds_get(ctl + 385)) == 0u) __builtin_amdgcn_s_sleep(4);
                if (lane == 0) __hip_atomic_store((GAS u64*)(myws + WS_MBOX + (size_t)(blockIdx.x + 8) * 128), ((u64)call << 32) | th, RLX, SC_AGENT);
            }
        }
    }
    }
    if (tid == 0) __hip_atomic_store(ctr, call, RLX, SC_AGENT);
}

extern "C" void kernel_launch(int rank, int nranks, ncclComm_t comm, hipStream_t stream,
                              void* const* d_in, const int* in_sizes, int n_in,
                              void* d_out, int out_size, const int* in_dtypes, int out_dtype,
                              void* const* d_out_all, void* const* d_ws_all, size_t ws_bytes) {
    (void)comm; (void)in_dtypes; (void)out_dtype; (void)d_out_all;
    if (nranks != NR || (unsigned)rank >= (unsigned)NR || n_in != 2 || ws_bytes < WS_NEED ||
        in_sizes[0] != 4096 * KSH || in_sizes[1] != KSH * GN || out_size != 512 * GN) {
        fprintf(stderr, "kernel_launch: unexpected problem shape\n");
        return;
    }
    Params p;
    p.x = (const float*)d_in[0];
    p.w = (const float*)d_in[1];
    p.out = (float*)d_out;
    p.myws = (char*)d_ws_all[rank];
    for (int i = 0; i < NR; ++i) p.ws[i] = (char*)d_ws_all[i];
    p.rank = rank;
    p.pad = 0;
    int n = 0, np = 0, nd = 0;
    unsigned char deferred[8];
    bool px[8] = {};
    for (int i = 0; i < 128; ++i) p.order[0][i] = p.order[1][i] = 0;
    auto need_x = [&](int c0, int c1) { for (int c = c0; c <= c1; ++c) if (!px[c]) { px[c] = true; p.prod[np++] = (unsigned)(32 + c); } };
    for (int key = 0; key < 16 + 3 * SKEW2; ++key) {
        for (int pass = 0; pass < 2; ++pass) {
            if ((pass == 0) == (key != 0)) {
                for (int e = 32; e < 44; ++e)
                    if (2 * ((e - 32) * 8 / 12) == key) { p.order[0][n++] = (unsigned char)e; need_x((e - 32) * 8 / 12, ((e - 32) * 8 + 7) / 12); }
                if (key == XPOS) { p.order[0][n++] = 44; need_x(0, 3); }
                if (key == XPOS + 2) { p.order[0][n++] = 45; need_x(4, 7); }
            } else {
                for (int b = 0; b < 4; ++b)
                    for (int c = 0; c < 8; ++c)
                        if ((b == 0 && c == 1 ? 0 : 2 * c + b * SKEW2) == key) {
                            p.prod[np++] = (unsigned)(c * 4 + b);
                            if (b == 3 && c < 2 * NFWD && ((c ^ rank) & 1)) deferred[nd++] = (unsigned char)(c * 4 + b);
                            else p.order[0][n++] = (unsigned char)(c * 4 + b);
                        }
            }
        }
    }
    int m = 0;
    for (int i = 0; i < n; ++i) if (p.order[0][i] < 44) p.order[1][m++] = p.order[0][i];
    for (int i = 0; i < nd; ++i) p.order[1][m++] = deferred[i];
    for (int i = 0; i < 6 * NFWD; ++i) p.order[1][m++] = (unsigned char)(48 + i);
    for (int e = 44; e < 48; ++e) { if (e >= 46) p.order[0][n++] = (unsigned char)e; p.order[1][m++] = (unsigned char)e; }
    if (n != NU_OUT || m != NU_PAIR || np != 40 || p.prod[0] != 0 || p.prod[1] != 4 || p.prod[2] != 32) { fprintf(stderr, "kernel_launch: bad schedule\n"); return; }
    hipLaunchKernelGGL(fused_gemm_rs, dim3(NBLK), dim3(NTHR), 0, stream, p);
    HIP_CHECK(hipGetLastError());
}
```

```cpp
#include <hip/hip_runtime.h>
#include <rccl/rccl.h>
#include <cstdio>

#define HIP_CHECK(expr) do { hipError_t _e = (expr); if (_e != hipSuccess) { \
    fprintf(stderr, "HIP error %s at %s:%d\n", hipGetErrorString(_e), __FILE__, __LINE__); \
    return; } } while (0)

typedef float f32x4 __attribute__((ext_vector_type(4)));
typedef unsigned u32x4 __attribute__((ext_vector_type(4)));
typedef unsigned u32x3 __attribute__((ext_vector_type(3)));
typedef unsigned u32x2 __attribute__((ext_vector_type(2)));
typedef unsigned short u16x2 __attribute__((ext_vector_type(2)));
typedef _Float16 h16x8 __attribute__((ext_vector_type(8)));
typedef unsigned long long u64;
#define LAS __attribute__((address_space(3)))
#define GAS __attribute__((address_space(1)))
#define RLX __ATOMIC_RELAXED
#define SC_WG __HIP_MEMORY_SCOPE_WORKGROUP
#define SC_AGENT __HIP_MEMORY_SCOPE_AGENT
#define SC_SYS __HIP_MEMORY_SCOPE_SYSTEM

#define NPLAIN 3
#define TAU 864
#define TAU_OUT TAU
#define WIN_OUT WIN
#define EQ_AT 5
#define CREDIT 30
#define WIN 10
#define PROD_GAP 40
#define PUSH_SLOT 112
#define RSTRIDE 16
#define POLLW 12
#define PROD_OFF -2
#define B0_ITS 2
#define POLL_SLEEP 8
#define NFWD 3
#define XPOS 6
#define LPOS 18
#define SKEW2 2

constexpr int NR = 8;
constexpr int GN = 16384;
constexpr int KSH = 512;
constexpr int NBLK = 256;
constexpr int NTHR = 512;
constexpr int NSTEP = 64;
constexpr unsigned A_STEP = 32768, B_STEP = 16384, STAGE = A_STEP + B_STEP;
constexpr int NU_OUT = 48 - NFWD, NU_PAIR = 48 + 6 * NFWD;
constexpr int RING = 8;
constexpr int PW = 13;
constexpr int QB = 3328 + 16 * PW;
constexpr int PB = 6656 + 16 * PW;
constexpr unsigned UNIT_T = QB * TAU / 256;
constexpr unsigned SLOT = 18432, SIDE = 14336;
constexpr size_t WS_CTR = 0;
constexpr size_t WS_FLAG = 32 << 10;
constexpr size_t WS_ORDY = 96 << 10;
constexpr size_t WS_MBOX = 128 << 10;
constexpr size_t WS_EQ = 160 << 10;
constexpr size_t WS_TMO = 192 << 10;
constexpr size_t WS_DIAG = 256 << 10;
constexpr size_t WS_STG = 1 << 20;
constexpr size_t STG_A = 8ull * NSTEP * A_STEP;
constexpr size_t STG_B = 32ull * NSTEP * B_STEP;
constexpr size_t WS_EXI = WS_STG + STG_A + STG_B;
constexpr size_t EXI_BYTES = 1600ull * SLOT;
constexpr size_t WS_EXO = WS_EXI + 2 * EXI_BYTES;
constexpr size_t WS_RDY = WS_EXO + (size_t)NBLK * 8 * 16384;
constexpr size_t WS_NEED = WS_RDY + 2 * 8 * 48 * 8 * 128;
constexpr u64 DEADLINE_TICKS = 50000000ull;

struct Params {
    const float* x;
    const float* w;
    float* out;
    char* myws;
    char* ws[NR];
    int rank;
    int pad;
    unsigned prod[40];
    unsigned char order[2][128];
};

#define WAIT_L0() do { asm volatile("s_waitcnt lgkmcnt(0)" ::: "memory"); __builtin_amdgcn_s_waitcnt(0xC07F); } while (0)
#define BAR() do { asm volatile("" ::: "memory"); __builtin_amdgcn_s_barrier(); asm volatile("" ::: "memory"); } while (0)
#define SB() __builtin_amdgcn_sched_barrier(0)
#define UNI(x) __builtin_amdgcn_readfirstlane(x)
template <int N> __device__ __forceinline__ void wait_vm() { asm volatile("s_waitcnt vmcnt(%0)" ::"i"(N) : "memory"); }

__device__ __forceinline__ void st16(const void* base, unsigned voff, u32x4 v) {
    asm volatile("s_nop 4\n\tglobal_store_dwordx4 %0, %1, %2 sc0 sc1\n\ts_nop 1" ::"v"(voff), "v"(v), "s"(base) : "memory");
}
__device__ __forceinline__ void st12(const void* base, unsigned voff, u32x3 v) {
    asm volatile("s_nop 4\n\tglobal_store_dwordx3 %0, %1, %2 sc0 sc1\n\ts_nop 1" ::"v"(voff), "v"(v), "s"(base) : "memory");
}
__device__ __forceinline__ void st8(const void* base, unsigned voff, u32x2 v) {
    asm volatile("s_nop 4\n\tglobal_store_dwordx2 %0, %1, %2 sc0 sc1" ::"v"(voff), "v"(v), "s"(base) : "memory");
}
__device__ __forceinline__ void st4(const void* base, unsigned voff, unsigned v) {
    asm volatile("s_nop 4\n\tglobal_store_dword %0, %1, %2 sc0 sc1" ::"v"(voff), "v"(v), "s"(base) : "memory");
}
__device__ __forceinline__ u32x3 ld12(const void* base, unsigned voff) {
    u32x3 r;
    asm volatile("s_nop 4\n\tglobal_load_dwordx3 %0, %1, %2 sc0 sc1\n\ts_waitcnt vmcnt(0)" : "=&v"(r) : "v"(voff), "s"(base) : "memory");
    return r;
}
__device__ __forceinline__ void ld64(const void* base, unsigned voff, u32x4& a, u32x4& b, u32x4& c, u32x4& d) {
    asm volatile("s_nop 4\n\tglobal_load_dwordx4 %0, %4, %5 sc0 sc1\n\tglobal_load_dwordx4 %1, %4, %5 offset:1024 sc0 sc1\n\t"
                 "global_load_dwordx4 %2, %4, %5 offset:2048 sc0 sc1\n\tglobal_load_dwordx4 %3, %4, %5 offset:3072 sc0 sc1\n\ts_waitcnt vmcnt(0)"
                 : "=&v"(a), "=&v"(b), "=&v"(c), "=&v"(d) : "v"(voff), "s"(base) : "memory");
}
template <bool TWO> __device__ __forceinline__ void dma(const void* base, unsigned voff, unsigned lds_dst) {
    unsigned keep;
    if (TWO) asm volatile("s_mov_b32 %0, m0\n\ts_mov_b32 m0, %3\n\ts_nop 4\n\tglobal_load_lds_dwordx4 %1, %2 sc0 sc1\n\tglobal_load_lds_dwordx4 %1, %2 offset:1024 sc0 sc1\n\ts_mov_b32 m0, %0"
                          : "=&s"(keep) : "v"(voff), "s"(base), "s"(lds_dst) : "memory");
    else asm volatile("s_mov_b32 %0, m0\n\ts_mov_b32 m0, %3\n\ts_nop 4\n\tglobal_load_lds_dwordx4 %1, %2 sc0 sc1\n\ts_mov_b32 m0, %0"
                      : "=&s"(keep) : "v"(voff), "s"(base), "s"(lds_dst) : "memory");
}
__device__ __forceinline__ unsigned now8() { return (unsigned)__builtin_amdgcn_s_memrealtime() << 6; }
__device__ __forceinline__ void pace(unsigned& t, unsigned period) {
    int late = (int)(now8() - t);
    if (late < -(1 << 22)) { t += (unsigned)late; late = 0; }
    while (late < 0) { __builtin_amdgcn_s_sleep(1); late = (int)(now8() - t); }
    t += period + (late > CREDIT * 64 ? (unsigned)(late - CREDIT * 64) : 0u);
}
__device__ __forceinline__ unsigned lds_get(const LAS unsigned* a) { return UNI(__hip_atomic_load(a, RLX, SC_WG)); }
__device__ __forceinline__ unsigned cvt2(float a, float b) {
    typedef _Float16 h2 __attribute__((ext_vector_type(2)));
    h2 r = {(_Float16)a, (_Float16)b};
    return __builtin_bit_cast(unsigned, r);
}
__device__ __forceinline__ int rb_owner(int d, int rb) { return rb >= 6 ? d : ((((d >> 1) + 1 + (rb >> 1)) & 3) << 1) | (rb & 1); }
__device__ __forceinline__ int rb_local(int d, int rb) { return rb >= 6 ? (rb - 6) * 256 : (d & 1) * 256; }
__device__ __forceinline__ int rb_of(int d, int o) { return ((((o >> 1) - (d >> 1) - 1) & 3) << 1) | (o & 1); }
__device__ __forceinline__ void unit_of(int e, int j, int& c, int& id) {
    if (e < 32) { c = e >> 2; id = j * 4 + (e & 3); }
    else if (e < 44) { const int u = (e - 32) * 8 + j; c = u / 12; id = 32 + u - c * 12; }
    else { const int u = (e & 1) * 8 + j; c = u >> 1; id = (e < 46 ? 46 : 44) + (u & 1); }
}
__device__ __forceinline__ int gid_of(int d, int id) {
    const int rb = (id - 32) >> 1;
    return id < 32 ? (d >> 1) * 32 + id : 128 + rb_owner(d, rb) * 4 + (rb_local(d, rb) >> 7) + (id & 1);
}
__device__ __forceinline__ size_t stage_off(int id, int ks) {
    return id < 32 ? STG_A + (size_t)(id * NSTEP + ks) * B_STEP : (size_t)(((id - 32) >> 1) * NSTEP + ks) * A_STEP + (id & 1) * 16384;
}
template <int BITS, int SH> __device__ __forceinline__ unsigned wave_max_exp(unsigned mag) {
    unsigned e = 0;
#pragma unroll
    for (int b = BITS - 1; b >= 0; --b) if (__any(mag >= ((e + (1u << b)) << SH))) e += 1u << b;
    return e;
}

__device__ __forceinline__ unsigned row_scan(unsigned x) {
    x += __builtin_amdgcn_update_dpp(0, x, 0x111, 0xf, 0xf, true);
    x += __builtin_amdgcn_update_dpp(0, x, 0x112, 0xf, 0xf, true);
    x += __builtin_amdgcn_update_dpp(0, x, 0x114, 0xf, 0xf, true);
    x += __builtin_amdgcn_update_dpp(0, x, 0x118, 0xf, 0xf, true);
    return x;
}
__device__ __forceinline__ unsigned ffbl(unsigned x) { unsigned r; asm("v_ffbl_b32 %0, %1" : "=v"(r) : "v"(x)); return r; }
#define PK(x) __builtin_bit_cast(u16x2, (unsigned)(x))
#define DW(x) __builtin_bit_cast(unsigned, (u16x2)(x))
#define SUBS(a, b) DW(__builtin_elementwise_sub_sat(PK(a), PK(b)))
#define MIN2(a, b) DW(__builtin_elementwise_min(PK(a), PK(b)))

template <int BITS> __device__ __forceinline__ int code_base(const unsigned (&ef)[16]) {
    u16x2 mx = PK(ef[0]);
#pragma unroll
    for (int i = 1; i < 16; ++i) mx = __builtin_elementwise_max(mx, PK(ef[i]));
    const int emax = (int)wave_max_exp<BITS, 0>(mx.x > mx.y ? mx.x : mx.y);
    int up = 0, mid = 0;
#pragma unroll
    for (int i = 0; i < 6; ++i) {
        const int e = (int)(ef[i] >> 16);
        up += __builtin_popcountll(__ballot(e == emax)) - __builtin_popcountll(__ballot(e == emax - 3));
        mid += __builtin_popcountll(__ballot(e == emax - 1)) - __builtin_popcountll(__ballot(e == emax - 4));
    }
    const int bs = emax - (up > 0 ? 3 : mid > 0 ? 4 : 5);
    return bs > 0 ? bs : 0;
}
__device__ __forceinline__ bool code_put(unsigned (&ef)[16], unsigned B, LAS unsigned* pool, int row) {
    const unsigned b2 = B * 0x00010001u, lut = 0x0300u | ((B + 2u + (B ? 1u : 0u)) << 16);
    u64 str = 0;
    unsigned sh = 0;
#pragma unroll
    for (int i = 0; i < 16; ++i) {
        const unsigned k1 = SUBS(b2 + 0x00010001u, ef[i]), d = SUBS(ef[i], b2);
        const unsigned len = k1 + MIN2(SUBS(k1, 0x00020002u), 0x00010001u) + __builtin_amdgcn_perm(0u, lut, SUBS(d, 0x00030003u) | 0x0c000c00u);
        sh += len & 0xffffu; str |= 1ull << (sh & 63u);
        sh += len >> 16; str |= 1ull << (sh & 63u);
        ef[i] = MIN2(d, 0x00040004u) & 0x00030003u;
    }
    const unsigned end = row_scan(sh), pos = end - sh;
    if (!__all(sh <= 63u && end <= 32u * PW)) return false;
    str >>= 1;
    const u64 s0 = str << (pos & 31u);
    LAS unsigned* pp = pool + row * PW + (pos >> 5);
    __hip_atomic_fetch_or(pp, (unsigned)s0, RLX, SC_WG);
    __hip_atomic_fetch_or(pp + 1, (unsigned)(s0 >> 32), RLX, SC_WG);
    __hip_atomic_fetch_or(pp + 2, (unsigned)(((str >> 32) << (pos & 31u)) >> 32), RLX, SC_WG);
    return true;
}
template <int SH> __device__ __forceinline__ u64 code_find(const unsigned (&w)[16], unsigned (&ne)[16], const LAS unsigned* pool, int row) {
    unsigned pd[PW - 1], sum = 0;
#pragma unroll
    for (int i = 0; i < PW - 1; ++i) pd[i] = pool[row * PW + i];
#pragma unroll
    for (int i = 0; i < 16; ++i) { ne[i] = MIN2(w[i] & (0x00030003u << SH), 0x00010001u); sum += ne[i]; }
    sum = 32u - (sum & 0xffffu) - (sum >> 16);
    const unsigned skip = row_scan(sum) - sum;
    unsigned cum = 0, at = 0, below = 0;
#pragma unroll
    for (int i = 0; i < PW - 1; ++i) {
        cum += __builtin_popcount(pd[i]);
        const bool lt = cum < skip;
        at += lt; below = lt ? cum : below;
    }
    const unsigned d0 = pool[row * PW + at], d1 = pool[row * PW + at + 1], d2 = pool[row * PW + at + 2];
    unsigned rem = skip - below, pos = 0;
#pragma unroll
    for (int sft = 16; sft; sft >>= 1) {
        const unsigned t = __builtin_popcount(__builtin_amdgcn_ubfe(d0, pos, sft));
        const bool lt = t < rem;
        rem -= lt ? t : 0u; pos += lt ? sft : 0u;
    }
    pos = skip ? pos + 1u : 0u;
    return (unsigned)((((u64)d1 << 32) | d0) >> pos) | ((((u64)d2 << 32) | d1) >> pos << 32);
}
__device__ __forceinline__ unsigned code_get(u64& str, unsigned ne, unsigned b2) {
    const unsigned l0 = ffbl(((unsigned)str << 1) | (ne & 1u)); str >>= l0 & 63u;
    const unsigned l1 = ffbl(((unsigned)str << 1) | (ne >> 16)); str >>= l1 & 63u;
    const unsigned len = l0 | (l1 << 16);
    const unsigned adj = __builtin_amdgcn_perm(0x02u, 0x07010100u, MIN2(len, 0x00040004u) | 0x0c000c00u);
    return MIN2(DW(PK(b2 + adj) - PK(len)), b2 + 0x00050005u);
}

typedef float f32x2 __attribute__((ext_vector_type(2)));
__device__ __forceinline__ f32x2 gelu2(f32x2 y) {
    const float k1 = -2.0f * 0.7978845608028654f * 1.4426950408889634f, k3 = k1 * 0.044715f;
    const f32x2 a = __builtin_elementwise_fma(y * y, (f32x2){k3, k3}, (f32x2){k1, k1}) * y;
    const f32x2 d = (f32x2){__builtin_amdgcn_exp2f(a.x), __builtin_amdgcn_exp2f(a.y)} + 1.0f;
    return y * (f32x2){__builtin_amdgcn_rcpf(d.x), __builtin_amdgcn_rcpf(d.y)};
}
__device__ __forceinline__ float gelu1(float y) { return gelu2((f32x2){y, y}).x; }

#define STAMP(i) do {} while (0)
#define TIMED_OUT(t0) (UNI(__hip_atomic_load(tmo, RLX, SC_AGENT)) == call || __builtin_amdgcn_s_memrealtime() - (t0) > DEADLINE_TICKS)

__global__ __launch_bounds__(NTHR, 2) void fused_gemm_rs(Params p) {
    __shared__ __attribute__((aligned(16))) unsigned char lds_raw[3 * STAGE + 2048];
    LAS unsigned char* lds = (LAS unsigned char*)lds_raw;
    LAS unsigned* ctl = (LAS unsigned*)(lds + 3 * STAGE);
    const unsigned lds0 = (unsigned)(size_t)lds;
    const int tid = threadIdx.x, lane = tid & 63;
    const int wid = UNI(tid >> 6);
    const int fr = lane & 15, fq = lane >> 4;
    const unsigned lo = lane * 16u;
    const int xcd = blockIdx.x & 7, idx = blockIdx.x >> 3;
    const bool late = idx < 8;
    const int q = wid & 3;

    const int team = (wid >> 2) * 192 + (int)blockIdx.x - 64;
    f32x4 r[8];
    auto request = [&](int g, int u) {
        if (g < 32) {
            const GAS char* b = (const GAS char*)(p.w + (size_t)((g >> 2) * 64 + (q & 1) * 32) * GN + (u * 4 + (g & 3)) * 128 + (q >> 1) * 64) + (unsigned)(8 * fq * GN + 4 * fr) * 4u;
#pragma unroll
            for (int e = 0; e < 8; ++e) r[e] = *(const GAS f32x4*)(b + (size_t)e * (GN * 4));
        } else {
            const GAS char* b = (const GAS char*)(p.x + (size_t)(u * 128 + q * 32) * KSH + (g - 32) * 64) + (unsigned)(fr * KSH + 8 * fq) * 4u;
#pragma unroll
            for (int e = 0; e < 8; ++e) r[e] = *(const GAS f32x4*)(b + (e >> 2) * (16 * KSH * 4) + ((e >> 1) & 1) * 128 + (e & 1) * 16);
        }
    };
    if (!late && team < 96) request(team >= 64 ? 32 : (team >> 5) * 4, team & 31);
    const unsigned t_entry = now8();

    const int rank = p.rank & 7;
    char* myws = p.myws;
    STAMP(0);
    GAS unsigned* ctr = (GAS unsigned*)(myws + WS_CTR + (size_t)blockIdx.x * 128);
    GAS unsigned* tmo = (GAS unsigned*)(myws + WS_TMO);
    const unsigned ctr_v = __hip_atomic_load(ctr, RLX, SC_AGENT);
    GAS unsigned* gflag = (GAS unsigned*)(myws + WS_FLAG);
    ctl[tid] = 0u;
    WAIT_L0();
    BAR();

    if (!late) {
        unsigned wlo = 0, whi = 0;
#pragma unroll
        for (int d = 0; d < NR; ++d) {
            unsigned a = (unsigned)(size_t)p.ws[d], b = (unsigned)((size_t)p.ws[d] >> 32);
            asm volatile("" : "+s"(a), "+s"(b));
            wlo = lane == d ? a : wlo;
            whi = lane == d ? b : whi;
        }
#pragma unroll 1
        for (int it = 0; it < 5; ++it) {
            const int t = it < B0_ITS ? it * 384 + team : B0_ITS * 384 + (it - B0_ITS) * 288 + (wid >> 2) * 144 + (((idx - 8) >> 2) * 3 + (idx & 3) - 1) * 8 + xcd;
            if (t >= 1280 || (it >= B0_ITS && (idx & 3) == 0)) break;
            const int g = p.prod[t >> 5], u = t & 31;
            if (t >= 96) {
                const unsigned ts = t_entry + (unsigned)((t >> 5) - PROD_OFF) * (PROD_GAP * 64);
                while ((int)(now8() - ts) < 0) __builtin_amdgcn_s_sleep(2);
                request(g, u);
            }
            const bool isw = g < 32;
            const int c = isw ? g >> 2 : g - 32;
            const int gid = isw ? u * 4 + (g & 3) : 128 + u;
            unsigned v[16];
            if (isw) {
#pragma unroll
                for (int i = 0; i < 16; ++i) v[i] = cvt2(r[(i & 3) * 2][i >> 2], r[(i & 3) * 2 + 1][i >> 2]);
            } else {
#pragma unroll
                for (int i = 0; i < 16; ++i) v[i] = cvt2(r[i >> 1][(i & 1) * 2], r[i >> 1][(i & 1) * 2 + 1]);
            }
            asm volatile("" : "+v"(v[15]));
            const unsigned t_loaded = now8();
            const bool pushed = t < 32;
            const unsigned call = UNI(ctr_v) + 1u;
            const size_t par = WS_EXI + (size_t)(call & 1u) * EXI_BYTES;
            const int o = u >> 2;
            const int myid = isw ? ((gid >> 5) == (rank >> 1) ? gid & 31 : -1)
                                 : (o == rank ? 44 + (u & 3) : ((o >> 1) != (rank >> 1) && ((u >> 1) & 1) == (rank & 1) ? 32 + rb_of(rank, o) * 2 + (u & 1) : -1));
            unsigned hdr = 0;
            {
                unsigned w[16];
#pragma unroll
                for (int i = 0; i < 16; ++i) w[i] = v[i];
                const char* home = myws + par + (size_t)(gid * 8 + c) * SLOT;
                unsigned pw = 0;
                hdr = 32u;
                if (!pushed) {
                    LAS unsigned* pool = (LAS unsigned*)(lds + wid * 256);
                    pool[lane] = 0u;
                    unsigned ef[16];
#pragma unroll
                    for (int i = 0; i < 16; ++i) ef[i] = (w[i] >> 10) & 0x001f001fu;
                    const unsigned B = (unsigned)code_base<5>(ef);
                    if (code_put(ef, B, pool, fq)) {
#pragma unroll
                        for (int i = 0; i < 16; ++i) w[i] = (w[i] & 0x83ff83ffu) | (ef[i] << 10);
#pragma unroll
                        for (int k = 0; k < 3; ++k) {
#pragma unroll
                            for (int i = 0; i < 4; ++i) w[4 * k + i] |= (w[13 + k] << (12 - 3 * i)) & 0x70007000u;
                            w[12] |= (w[13 + k] >> (1 + k)) & (0x40004000u >> k);
                        }
                        pw = __hip_atomic_load(pool + lane, RLX, SC_WG);
                        hdr = B;
                    }
                }
                if (pushed) {
                    unsigned tp = t_loaded + (unsigned)((u & 7) * 4 + q) * PUSH_SLOT;
#pragma unroll
                    for (int i = 0; i < 4; ++i) {
                        pace(tp, 32u * PUSH_SLOT);
#pragma unroll
                        for (int k = 0; k < 2; ++k) {
                            const int d = (u >> 3) * 2 + k;
                            if (d != rank) st16(p.ws[d] + par + (size_t)(1280 + rank * 8 + (u & 7)) * SLOT + q * 4096 + i * 1024, lo, (u32x4){v[4 * i], v[4 * i + 1], v[4 * i + 2], v[4 * i + 3]});
                        }
                    }
                } else {
#pragma unroll
                    for (int i = 0; i < 3; ++i) st16(home + q * QB + i * 1024, lo, (u32x4){w[4 * i], w[4 * i + 1], w[4 * i + 2], w[4 * i + 3]});
                    st4(home + q * QB + 3072, lane * 4u, w[12]);
                    if (hdr == 32u) st12(home + SIDE + q * 768, lane * 12u, (u32x3){v[13], v[14], v[15]});
                    else if (lane < 4 * PW) st4(home + q * QB + 3328, lane * 4u, pw);
                }
            }
            if (myid >= 0) {
                const char* sg = myws + WS_STG + stage_off(myid, rank * 8 + c) + (isw ? (q >> 1) * 8 + (q & 1) : q * 4) * 1024;
#pragma unroll
                for (int i = 0; i < 4; ++i) st16(sg + i * (isw ? 2048 : 1024), lo, (u32x4){v[4 * i], v[4 * i + 1], v[4 * i + 2], v[4 * i + 3]});
            }
            wait_vm<0>();
            unsigned tw = (1u << 24) | (hdr << (6 * q));
            if (lane == 0) tw += __hip_atomic_fetch_add(ctl + 402 + (wid >> 2) * 5 + it, tw, RLX, SC_WG);
            tw = UNI(tw);
            if ((tw >> 24) == 4u) {
                const int d = lane & 7;
                int did = isw ? ((gid >> 5) == (d >> 1) ? gid & 31 : -1)
                              : (o == d ? 44 + (u & 3) : ((o >> 1) != (d >> 1) && ((u >> 1) & 1) == (d & 1) ? 32 + rb_of(d, o) * 2 + (u & 1) : -1));
                if (isw && (g & 3) == 3 && c < 2 * NFWD && (d >> 1) != (rank >> 1) && ((c ^ d) & 1)) did = -1;
                if (lane < 8 && did >= 0) {
                    if (d == rank) __hip_atomic_store(gflag + did * 64 + c * 8 + rank, call, RLX, SC_AGENT);
                    else __hip_atomic_store((GAS u64*)((((size_t)whi << 32) | wlo) + WS_RDY) + ((((call & 1u) * 8 + rank) * 48 + did) * 8 + c) * RSTRIDE, ((u64)call << 32) | (tw & 0xffffffu), RLX, SC_SYS);
                }
            }
        }
        STAMP(9);
    } else if (idx < 7) {
        const int s = idx < 6 ? rb_owner(rank, idx) : rank ^ 1;
        const int nu = idx < 6 ? NU_OUT : NU_PAIR;
        unsigned desc[2];
#pragma unroll
        for (int k = 0; k < 2; ++k) {
            const int e = p.order[idx == 6][k * 64 + lane];
            int c, id, org = s, slot;
            if (e < 48) { unit_of(e, xcd, c, id); slot = gid_of(rank, id) * 8 + c; }
            else { const int kf = (e - 48) / 6; c = 2 * kf + (s & 1); id = xcd * 4 + 3; org = rb_owner(rank, e - 48 - kf * 6); slot = 1408 + (e - 48) * 8 + xcd; }
            desc[k] = (unsigned)(c | id << 3 | org << 9 | slot << 12);
        }
        auto desc_of = [&](int n) { return (unsigned)(n < 64 ? __builtin_amdgcn_readlane(desc[0], n) : __builtin_amdgcn_readlane(desc[1], n - 64)); };
        const unsigned call = UNI(ctr_v) + 1u;
        const size_t par = WS_EXI + (size_t)(call & 1u) * EXI_BYTES;
        if (wid < 6) {
            int pend = -1, pend_n = 0;
            unsigned pend_h = 0;
            auto staged = [&]() {
                wait_vm<0>();
                if (lane == 0 && __hip_atomic_fetch_add(ctl + 256 + pend_n, 1u, RLX, SC_WG) == 3u) {
                    __hip_atomic_store(gflag + pend, call, RLX, SC_AGENT);
                    if (pend_h) __hip_atomic_store((GAS u64*)(p.ws[rank ^ 1] + WS_RDY) + ((((call & 1u) * 8 + s) * 48 + (pend >> 6)) * 8 + ((pend >> 3) & 7)) * RSTRIDE,
                                                   ((u64)call << 32) | (pend_h & 0xffffffu), RLX, SC_SYS);
                }
                pend = -1;
            };
            auto need = [&](int job) { return (job >> 2) * 14 - 14 + (((job & 3) + 1) * QB + 1023) / 1024; };
            auto ticket = [&]() { unsigned tk = 0; if (lane == 0) tk = __hip_atomic_fetch_add(ctl + 401, 1u, RLX, SC_WG); return (int)UNI(tk); };
            for (int job = ticket(); job < 4 * nu;) {
                const int n = job >> 2, qq = job & 3;
                unsigned w[16];
                int un = 0;
                unsigned relay_h = 0;
                if (n == 0) {
                    while (lds_get(ctl) == 0u) __builtin_amdgcn_s_sleep(1);
                    u32x4 a0, a1, a2, a3;
                    ld64(myws + par + (size_t)(1280 + s * 8 + xcd) * SLOT + qq * 4096, lo, a0, a1, a2, a3);
#pragma unroll
                    for (int i = 0; i < 4; ++i) { w[i] = a0[i]; w[4 + i] = a1[i]; w[8 + i] = a2[i]; w[12 + i] = a3[i]; }
                } else {
                    while ((int)lds_get(ctl + 384) < need(job)) __builtin_amdgcn_s_sleep(1);
                    asm volatile("" ::: "memory");
                    un = lds_get(ctl + 416 + n);
                }
                const unsigned de = desc_of(un);
                const int c = de & 7, id = (de >> 3) & 63, org = (de >> 9) & 7;
                if (n) {
                    const LAS unsigned char* src = lds + (unsigned)(n % RING) * 16384u + qq * QB;
#pragma unroll
                    for (int i = 0; i < 3; ++i) { const u32x4 a = *(const LAS u32x4*)(src + i * 1024 + lo); w[4 * i] = a[0]; w[4 * i + 1] = a[1]; w[4 * i + 2] = a[2]; w[4 * i + 3] = a[3]; }
                    w[12] = *(const LAS unsigned*)(src + 3072 + lane * 4);
                    const LAS unsigned* pool = (const LAS unsigned*)(src + 3328);
                    const unsigned hu = lds_get(ctl + un), h = (hu >> (6 * qq)) & 63u;
                    if (idx < 6 && id < 32 && (id & 3) == 3 && c < 2 * NFWD) {
                        const unsigned pw = pool[lane];
                        if (pend >= 0) staged();
                        const char* dst = myws + par + (size_t)(1408 + ((c >> 1) * 6 + idx) * 8 + xcd) * SLOT + qq * QB;
#pragma unroll
                        for (int i = 0; i < 3; ++i) st16(dst + i * 1024, lo, (u32x4){w[4 * i], w[4 * i + 1], w[4 * i + 2], w[4 * i + 3]});
                        st4(dst + 3072, lane * 4u, w[12]);
                        if (lane < 4 * PW) st4(dst + 3328, lane * 4u, pw);
                        relay_h = hu;
                    }
                    if (h < 32u) {
#pragma unroll
                        for (int k = 0; k < 3; ++k) {
                            unsigned cc = (w[12] << (1 + k)) & 0x80008000u;
#pragma unroll
                            for (int i = 0; i < 4; ++i) cc |= (w[4 * k + i] >> (12 - 3 * i)) & (0x00070007u << (3 * i));
                            w[13 + k] = cc;
                        }
                        unsigned ne[16];
                        u64 str = code_find<10>(w, ne, pool, fq);
                        WAIT_L0();
                        if (lane == 0) __hip_atomic_fetch_add(ctl + 128 + n, 1u, RLX, SC_WG);
                        if (pend >= 0) staged();
#pragma unroll
                        for (int i = 0; i < 16; ++i) w[i] = (w[i] & 0x8fff8fffu) + (code_get(str, ne[i], h * 0x00010001u) << 10);
                    } else {
                        WAIT_L0();
                        if (lane == 0) __hip_atomic_fetch_add(ctl + 128 + n, 1u, RLX, SC_WG);
                        if (pend >= 0) staged();
                        const u32x3 a = ld12(p.ws[org] + par + (size_t)(gid_of(rank, id) * 8 + c) * SLOT + SIDE + qq * 768, lane * 12u);
                        w[13] = a[0]; w[14] = a[1]; w[15] = a[2];
                    }
                }
                const char* sg = myws + WS_STG + stage_off(id, org * 8 + c) + (id < 32 ? (qq >> 1) * 8 + (qq & 1) : qq * 4) * 1024;
#pragma unroll
                for (int i = 0; i < 4; ++i) st16(sg + i * (id < 32 ? 2048 : 1024), lo, (u32x4){w[4 * i], w[4 * i + 1], w[4 * i + 2], w[4 * i + 3]});
                pend = id * 64 + c * 8 + org; pend_n = n; pend_h = relay_h;
                job = ticket();
                if (job >= 4 * nu || job < 4 || (int)lds_get(ctl + 384) < need(job)) staged();
            }
        } else if (wid == 6) {
            const char* exs = p.ws[s] + par;
            unsigned tnext = t_entry + xcd * (TAU / 4);
            u64 todo = ~1ull;
            __builtin_amdgcn_s_setprio(3);
            const char* src = nullptr;
            unsigned trim = 0;
            auto choose = [&](int n) {
                const unsigned rdy = __hip_atomic_load(ctl + (n >= 64 ? n : lane), RLX, SC_WG);
                const unsigned fre = __hip_atomic_load(ctl + (n > RING ? 128 + n - RING : 400), RLX, SC_WG), tr = __hip_atomic_load(ctl + 400, RLX, SC_WG);
                const u64 rd = __ballot(rdy != 0u) & (n >= 64 ? 1ull : todo);
                if (!rd || (n > RING && UNI(fre) < 4u)) return;
                const int un = n >= 64 ? n : __builtin_ctzll(rd);
                todo &= ~(1ull << (un & 63));
                if (lane == 0) __hip_atomic_store(ctl + 416 + n, (unsigned)un, RLX, SC_WG);
                trim = n >= EQ_AT + 4 && n < EQ_AT + 36 ? UNI(tr) : 0u;
                src = exs + (size_t)(desc_of(un) >> 12) * SLOT;
            };
            for (int n = 1; n < nu; ++n) {
                bool drained = false;
                if (!src) {
                    choose(n);
                    if (!src) {
                        wait_vm<0>();
                        if (lane == 0) __hip_atomic_store(ctl + 384, (unsigned)(n - 1) * 14u, RLX, SC_WG);
                        drained = true;
                        for (choose(n); !src; choose(n)) __builtin_amdgcn_s_sleep(1);
                    }
                }
                const char* cur = src;
                const unsigned dst = lds0 + (unsigned)(n % RING) * 16384u, tr = trim;
                src = nullptr;
                asm volatile("" ::: "memory");
#pragma unroll
                for (int i = 0; i < 14; i += 2) {
                    wait_vm<WIN - 2>();
                    if ((i > WIN - 2 || (!drained && n > 1)) && lane == 0) __hip_atomic_store(ctl + 384, (unsigned)((n - 1) * 14 + i - (WIN - 2)), RLX, SC_WG);
                    pace(tnext, (i < 12 ? 2u * TAU : UNIT_T - 12u * TAU) - tr);
                    if (i < 12) dma<true>(cur + i * 1024, lo, dst + i * 1024);
                    else {
                        dma<false>(cur + 12288, lo, dst + 12288);
                        if (lane < (4 * QB - 13312) / 16) dma<false>(cur + 13312, lo, dst + 13312);
                    }
                    if (i == 0 && lane == 0) {
                        if (n == EQ_AT) __hip_atomic_store(ctl + 387, tnext | 1u, RLX, SC_WG);
                        if (n == nu - 1) __hip_atomic_store(ctl + 385, (tnext + UNIT_T - 2u * TAU) | 1u, RLX, SC_WG);
                    }
                    if (i == 10 && n + 1 < nu) choose(n + 1);
                }
            }
            __builtin_amdgcn_s_setprio(0);
            wait_vm<0>();
            if (lane == 0) __hip_atomic_store(ctl + 384, (unsigned)(nu - 1) * 14u, RLX, SC_WG);
            STAMP(9);
        } else {
            const GAS u64* rp[2];
            bool pend[2];
#pragma unroll
            for (int k = 0; k < 2; ++k) {
                rp[k] = (const GAS u64*)(myws + WS_RDY) + ((((call & 1u) * 8 + ((desc[k] >> 9) & 7)) * 48 + ((desc[k] >> 3) & 63)) * 8 + (desc[k] & 7)) * RSTRIDE;
                pend[k] = k * 64 + lane < nu;
            }
            GAS u64* eq = (GAS u64*)(myws + WS_EQ) + idx * 16;
            unsigned mine = 0;
            bool eq_open = EQ_AT != 0;
            const u64 t0 = __builtin_amdgcn_s_memrealtime();
            for (unsigned polls = 1;; ++polls) {
                const u64 pm = __ballot(pend[0]);
                const bool front = __builtin_amdgcn_mbcnt_hi((unsigned)(pm >> 32), __builtin_amdgcn_mbcnt_lo((unsigned)pm, 0u)) < POLLW;
#pragma unroll
                for (int k = 0; k < 2; ++k)
                    if (pend[k] && ((k == 0 && front) || (polls & 7u) == 0u)) {
                        const u64 f = __hip_atomic_load(rp[k], RLX, SC_SYS);
                        if ((unsigned)(f >> 32) >= call) { __hip_atomic_store(ctl + k * 64 + lane, (unsigned)f | 0x80000000u, RLX, SC_WG); pend[k] = false; }
                    }
                if (eq_open) {
                    if (!mine) {
                        mine = lds_get(ctl + 387);
                        if (mine && lane == 0) __hip_atomic_store(eq + xcd, ((u64)call << 32) | mine, RLX, SC_AGENT);
                    } else {
                        const u64 f = __hip_atomic_load(eq + (lane & 7), RLX, SC_AGENT);
                        if (__all((unsigned)(f >> 32) == call)) {
                            int dlt = (int)(mine - (unsigned)f);
                            dlt += __shfl_xor(dlt, 1); dlt += __shfl_xor(dlt, 2); dlt += __shfl_xor(dlt, 4);
                            dlt /= 8 * 32 * 7;
                            dlt = dlt > TAU / 8 ? TAU / 8 : dlt < -(TAU / 8) ? -(TAU / 8) : dlt;
                            if (lane == 0) __hip_atomic_store(ctl + 400, (unsigned)dlt, RLX, SC_WG);
                            eq_open = false;
                        } else if ((int)lds_get(ctl + 384) >= (EQ_AT + 2) * 14) eq_open = false;
                    }
                }
                const bool waiting = __any(pend[0] || pend[1]);
                if (!waiting && !eq_open) break;
                if (waiting && (polls & 15u) == 0u && TIMED_OUT(t0)) {
                    if (lane == 0) __hip_atomic_store(tmo, call, RLX, SC_AGENT);
                    wait_vm<0>();
#pragma unroll
                    for (int k = 0; k < 2; ++k) if (pend[k]) __hip_atomic_store(ctl + k * 64 + lane, 0x80000000u, RLX, SC_WG);
                    break;
                }
                if (polls < 32u) __builtin_amdgcn_s_sleep(1); else __builtin_amdgcn_s_sleep(16);
            }
            if (idx < 6) {
                unsigned th;
                while ((th = lds_get(ctl + 385)) == 0u) __builtin_amdgcn_s_sleep(4);
                if (lane == 0) __hip_atomic_store((GAS u64*)(myws + WS_MBOX + (size_t)((8 + idx * 4) * 8 + xcd) * 128), ((u64)call << 32) | th, RLX, SC_AGENT);
            }
        }
    }
    const unsigned call = UNI(ctr_v) + 1u;

    const int ntile = idx == 7 ? 2 : 1;
    const int wm = wid >> 1, wn = wid & 1;
#pragma unroll 1
    for (int rep = 0; rep < ntile; ++rep) {
    const int rb = late ? 6 + (idx >> 2) : (idx - 8) >> 2;
    const int cb = xcd * 4 + (idx == 7 ? rep : idx == 6 ? 0 : idx == 4 || idx == 5 ? idx - 2 : idx & 3);
    const bool part1 = idx == 7 && rep == 0, part2 = idx == 6;
    const int nstep = part1 ? 56 : part2 ? 8 : NSTEP;
    const char* stgA = myws + WS_STG + (size_t)(rb * NSTEP) * A_STEP + wid * 4096;
    const char* stgB = myws + WS_STG + STG_A + (size_t)(cb * NSTEP) * B_STEP + wid * 2048;
    const u64 pbits = 0x0101010101010101ull << (rank ^ 1);
    u64 todo = part1 ? ~pbits : part2 ? pbits : ~0ull, avail = 0ull;
    int slot = 0;
    auto poll = [&]() {
        if (wid == 0) {
            const u64 t0 = __builtin_amdgcn_s_memrealtime();
            u64 got;
            for (unsigned polls = 1;; ++polls) {
                const unsigned f0 = __hip_atomic_load(gflag + cb * 64 + lane, RLX, SC_AGENT);
                const unsigned f1 = __hip_atomic_load(gflag + (32 + rb * 2) * 64 + lane, RLX, SC_AGENT);
                const unsigned f2 = __hip_atomic_load(gflag + (33 + rb * 2) * 64 + lane, RLX, SC_AGENT);
                got = __ballot(f0 >= call && f1 >= call && f2 >= call) & todo;
                if (got) break;
                if ((polls & 15u) == 0u && TIMED_OUT(t0)) {
                    if (lane == 0) __hip_atomic_store(tmo, call, RLX, SC_AGENT);
                    got = todo;
                    break;
                }
                __builtin_amdgcn_s_sleep(POLL_SLEEP);
            }
            if (lane < 2) ctl[388 + slot * 2 + lane] = (unsigned)(got >> (32 * lane));
        }
    };
    auto news = [&]() { avail = ((u64)(unsigned)UNI(ctl[389 + slot * 2]) << 32) | (unsigned)UNI(ctl[388 + slot * 2]); slot ^= 1; };
    auto take = [&]() {
        const unsigned b = (unsigned)__builtin_ctzll(avail) & 63u;
        avail &= avail - 1; todo &= ~(1ull << b);
        return (b & 7u) * 8u + (b >> 3);
    };
    auto g1 = [&](unsigned ks, unsigned buf, int i) {
        if (i < 4) __builtin_amdgcn_global_load_lds((const GAS unsigned*)(stgA + (size_t)ks * A_STEP + i * 1024 + lo), (LAS unsigned*)(lds + buf + wid * 4096 + i * 1024), 16, 0, 17);
        else __builtin_amdgcn_global_load_lds((const GAS unsigned*)(stgB + (size_t)ks * B_STEP + (i - 4) * 1024 + lo), (LAS unsigned*)(lds + buf + A_STEP + wid * 2048 + (i - 4) * 1024), 16, 0, 17);
    };

    f32x4 acc[4][4];
#pragma unroll
    for (int m = 0; m < 4; ++m)
#pragma unroll
        for (int n = 0; n < 4; ++n) acc[m][n] = (f32x4){0.f, 0.f, 0.f, 0.f};
    h16x8 a0[4], b0[4], a1[4], b1[4];
    const unsigned aoff = wm * 8192 + lo, boff = A_STEP + wn * 8192 + lo;
#define RDA(A, buf, kk, m) A[m] = *(const LAS h16x8*)(lds + (buf) + aoff + (m) * 2048 + (kk) * 1024)
#define RDB(B, buf, kk, n) B[n] = *(const LAS h16x8*)(lds + (buf) + boff + (n) * 2048 + (kk) * 1024)
#define ROW(A, B, m) _Pragma("unroll") for (int n = 0; n < 4; ++n) acc[m][n] = __builtin_amdgcn_mfma_f32_16x16x32_f16(A[m], B[n], acc[m][n], 0, 0, 0)

    STAMP(10);
    unsigned kq;
#pragma unroll
    for (int k = 0; k < 3; ++k) {
        if (!avail) { poll(); WAIT_L0(); BAR(); news(); }
        kq = take();
#pragma unroll
        for (int i = 0; i < (k < 2 ? 6 : 3); ++i) g1(kq, k * STAGE, i);
    }
    wait_vm<9>(); BAR();
#pragma unroll
    for (int i = 0; i < 4; ++i) { RDB(b0, 0u, 0, i); RDA(a0, 0u, 0, i); }
    unsigned cur = 0, prv = 2 * STAGE;
    for (int t = 0; t < nstep; ++t) {
        const unsigned nxt = cur == 2 * STAGE ? 0u : cur + STAGE;
        const bool m2 = t + 2 < nstep, m3 = t + 3 < nstep;
        const bool dry = m3 && !avail;
        WAIT_L0(); SB();
        RDB(b1, cur, 1, 0); RDB(b1, cur, 1, 1); SB(); ROW(a0, b0, 0); SB(); if (m2) g1(kq, prv, 3); SB();
        RDB(b1, cur, 1, 2); RDB(b1, cur, 1, 3); SB(); ROW(a0, b0, 1); SB(); if (m2) g1(kq, prv, 4); SB();
        RDA(a1, cur, 1, 0); RDA(a1, cur, 1, 1); SB(); ROW(a0, b0, 2); SB(); if (m2) g1(kq, prv, 5); SB();
        RDA(a1, cur, 1, 2); RDA(a1, cur, 1, 3); SB(); ROW(a0, b0, 3); SB();
        if (dry) poll();
        WAIT_L0();
        if (m2) wait_vm<6>(); else wait_vm<0>();
        BAR(); SB();
        RDB(b0, nxt, 0, 0); RDB(b0, nxt, 0, 1); SB(); ROW(a1, b1, 0); SB();
        if (dry) news();
        if (m3) { kq = take(); g1(kq, cur, 0); } SB();
        RDB(b0, nxt, 0, 2); RDB(b0, nxt, 0, 3); SB(); ROW(a1, b1, 1); SB(); if (m3) g1(kq, cur, 1); SB();
        RDA(a0, nxt, 0, 0); RDA(a0, nxt, 0, 1); SB(); ROW(a1, b1, 2); SB(); if (m3) g1(kq, cur, 2); SB();
        RDA(a0, nxt, 0, 2); RDA(a0, nxt, 0, 3); SB(); ROW(a1, b1, 3); SB();
        prv = cur; cur = nxt;
    }

    if (part1 || part2) {
        const char* park = myws + WS_EXO + (size_t)(xcd * 8 + wid) * 16384;
        GAS u64* pf = (GAS u64*)(myws + WS_ORDY + (size_t)xcd * 128) + wid;
        if (part1) {
#pragma unroll
            for (int i = 0; i < 16; ++i) st16(park + i * 1024, lo, __builtin_bit_cast(u32x4, acc[i >> 2][i & 3]));
            wait_vm<0>();
            if (lane == 0) __hip_atomic_store(pf, (u64)call << 32, RLX, SC_SYS);
            continue;
        }
        const u64 t0 = __builtin_amdgcn_s_memrealtime();
        while ((unsigned)UNI((unsigned)(__hip_atomic_load(pf, RLX, SC_SYS) >> 32)) < call) {
            if (TIMED_OUT(t0)) { if (lane == 0) __hip_atomic_store(tmo, call, RLX, SC_AGENT); wait_vm<0>(); break; }
            __builtin_amdgcn_s_sleep(4);
        }
#pragma unroll
        for (int m = 0; m < 4; ++m) {
            u32x4 e0, e1, e2, e3;
            ld64(park + m * 4096, lo, e0, e1, e2, e3);
            acc[m][0] += __builtin_bit_cast(f32x4, e0); acc[m][1] += __builtin_bit_cast(f32x4, e1);
            acc[m][2] += __builtin_bit_cast(f32x4, e2); acc[m][3] += __builtin_bit_cast(f32x4, e3);
        }
    }
    const unsigned tmo_v = __hip_atomic_load(tmo, RLX, SC_AGENT);
    const unsigned oo = (unsigned)(fq * 4) * (GN * 4u) + fr * 16u;
    const u32x4 nan4 = {0x7fc00000u, 0x7fc00000u, 0x7fc00000u, 0x7fc00000u};
    auto finish = [&](unsigned (&o)[32], int i, float a, float b) {
        const f32x2 g = gelu2((f32x2){a, b});
        const int at = ((i >> 3) * 4 + (i & 1) * 2) * 4 + ((i >> 1) & 3);
        o[at] = __float_as_uint(g.x); o[at + 4] = __float_as_uint(g.y);
    };
#define ACC(hf, i, k) acc[2 * (hf) + ((i) >> 3)][((i) >> 1) & 3][((i) & 1) * 2 + (k)]
    if (late) {
        const bool bad = UNI(tmo_v) == call;
        const char* ob = (const char*)(p.out + (size_t)(rb_local(rank, rb) + wm * 64) * GN + (rank >> 1) * 4096 + cb * 128 + wn * 64);
#pragma unroll
        for (int hf = 0; hf < 2; ++hf) {
            unsigned o[32];
#pragma unroll
            for (int i = 0; i < 16; ++i) finish(o, i, ACC(hf, i, 0), ACC(hf, i, 1));
#pragma unroll
            for (int i = 0; i < 8; ++i) st16(ob + (size_t)(hf * 32 + (i >> 2) * 16 + (i & 3)) * (GN * 4), oo, bad ? nan4 : (u32x4){o[4 * i], o[4 * i + 1], o[4 * i + 2], o[4 * i + 3]});
        }
    } else {
        const int peer = rb_owner(rank, rb);
        const int mirror = (8 + rb_of(peer, rank) * 4 + (idx & 3)) * 8 + xcd;
        char* const peerws = p.ws[peer];
        WAIT_L0(); BAR();
        if (wid < 4) __builtin_amdgcn_s_setprio(2);
#pragma unroll
        for (int hf = 0; hf < 2; ++hf) {
            const int k = (wid >> 2) * 8 + hf * 4 + q;
            const char* ex = myws + WS_EXO + (size_t)(blockIdx.x * 16 + k) * 8192;
            unsigned hdr = 1u << 13;
            if ((idx & 3) != 3 || k < 16 - NPLAIN) {
                LAS unsigned* pool = (LAS unsigned*)(lds + 131072 + wid * 256);
                pool[lane] = 0u;
                unsigned hi[16], ef[16], under = 0;
#pragma unroll
                for (int i = 0; i < 16; ++i) {
                    hi[i] = __builtin_amdgcn_perm(__float_as_uint(ACC(hf, i, 1)), __float_as_uint(ACC(hf, i, 0)), 0x07060302u);
                    ef[i] = (hi[i] >> 7) & 0x00ff00ffu;
                }
                const unsigned bs = (unsigned)code_base<8>(ef), e0 = bs > 20u ? bs - 20u : 0u;
#pragma unroll
                for (int i = 0; i < 16; ++i) { under |= SUBS(e0 * 0x00010001u, ef[i]); ef[i] = SUBS(ef[i], e0 * 0x00010001u); }
                if (__all(under == 0u) && code_put(ef, bs - e0, pool, fq)) {
#pragma unroll
                    for (int i = 0; i < 16; ++i) hi[i] = (hi[i] & 0x807f807fu) | (ef[i] << 7);
#pragma unroll
                    for (int c = 0; c < 6; ++c) {
#pragma unroll
                        for (int i = 0; i < 3; ++i) { const int fld = c * 3 + i, at = 9 + 3 * (fld & 1); hi[fld >> 1] |= (hi[10 + c] << (at - 3 * i)) & (0x00070007u << at); }
                        hi[9] |= (hi[10 + c] >> (1 + c)) & (0x40004000u >> c);
                    }
                    const unsigned pw = __hip_atomic_load(pool + lane, RLX, SC_WG);
#pragma unroll
                    for (int i = 0; i < 4; ++i) {
                        u32x4 v;
#pragma unroll
                        for (int e = 0; e < 4; ++e) v[e] = __builtin_amdgcn_perm(__float_as_uint(ACC(hf, 4 * i + e, 1)), __float_as_uint(ACC(hf, 4 * i + e, 0)), 0x05040100u);
                        st16(ex + i * 1024, lo, v);
                    }
                    st16(ex + 4096, lo, (u32x4){hi[0], hi[1], hi[2], hi[3]});
                    st16(ex + 5120, lo, (u32x4){hi[4], hi[5], hi[6], hi[7]});
                    st8(ex + 6144, lane * 8u, (u32x2){hi[8], hi[9]});
                    if (lane < 4 * PW) st4(ex + 6656, lane * 4u, pw);
                    hdr = e0 | ((bs - e0) << 8);
                }
            }
            if (hdr >> 13) {
                unsigned o[32];
#pragma unroll
                for (int i = 0; i < 16; ++i) finish(o, i, ACC(hf, i, 0), ACC(hf, i, 1));
#pragma unroll
                for (int i = 0; i < 8; ++i) st16(ex + i * 1024, lo, (u32x4){o[4 * i], o[4 * i + 1], o[4 * i + 2], o[4 * i + 3]});
            }
            wait_vm<0>();
            hdr |= UNI(tmo_v) == call ? 0xc0000000u : 0x40000000u;
            if (lane == 0) __hip_atomic_store((GAS u64*)(peerws + WS_ORDY + (size_t)mirror * 128) + k, ((u64)call << 32) | hdr, RLX, SC_SYS);
        }
        __builtin_amdgcn_s_setprio(0);
        STAMP(22);
#pragma unroll 1
        for (int sk = 0; (idx & 3) + sk <= 3 && (idx & 3) != 3; sk += 3) {
        if (sk) {
            BAR();
            if (tid < 32 || tid == 384 || tid == 401 || (tid >= 416 && tid < 432)) ctl[tid] = 0u;
            WAIT_L0(); BAR();
        }
        const int me = blockIdx.x + 8 * sk;
        const char* exs = peerws + WS_EXO + (size_t)(mirror + 8 * sk) * (16 * 8192);
        if (wid == 0) {
            unsigned tnext;
            while ((tnext = lds_get(ctl + 16)) == 0u) __builtin_amdgcn_s_sleep(2);
            STAMP(23);
            __builtin_amdgcn_s_setprio(3);
            wait_vm<0>(); __builtin_amdgcn_s_waitcnt(0x0F70);
            int issued = 0, told = 0;
            unsigned hn = lds_get(ctl);
            for (int n = 0; n < 16; ++n) {
                if (hn == 0u && (hn = lds_get(ctl + n)) == 0u) {
                    wait_vm<0>();
                    told = issued;
                    if (lane == 0) __hip_atomic_store(ctl + 384, (unsigned)told, RLX, SC_WG);
                    while ((hn = lds_get(ctl + n)) == 0u) __builtin_amdgcn_s_sleep(1);
                }
                asm volatile("" ::: "memory");
                const bool plain = (hn >> 13) & 1u;
                const char* cur = exs + (size_t)n * 8192;
                const unsigned dst = lds0 + (unsigned)n * 8192u;
#pragma unroll
                for (int i = 0; i < 8; i += 2) {
                    wait_vm<WIN_OUT - 2>();
                    if (issued - (WIN_OUT - 2) > told) { told = issued - (WIN_OUT - 2); if (lane == 0) __hip_atomic_store(ctl + 384, (unsigned)told, RLX, SC_WG); }
                    pace(tnext, i < 6 || plain ? 2u * TAU_OUT : PB * TAU_OUT / 1024 - 6u * TAU_OUT);
                    if (i < 6 || plain) { dma<true>(cur + i * 1024, lo, dst + i * 1024); issued += 2; }
                    else { if (lane < (PB - 6144) / 16) dma<false>(cur + 6144, lo, dst + 6144); issued += 1; }
                    if (i == 0) {
                        if (lane == 0) __hip_atomic_store(ctl + 416 + n, (unsigned)issued + (plain ? 6u : 5u), RLX, SC_WG);
                        if (n == 14 && !sk && lane == 0)
                            __hip_atomic_store((GAS u64*)(myws + WS_MBOX + (size_t)(blockIdx.x + 8) * 128), ((u64)call << 32) | ((tnext + 2u * (PB * TAU_OUT / 1024) - 2u * TAU_OUT) | 1u), RLX, SC_AGENT);
                    }
                    if (i == 2) hn = n < 15 ? lds_get(ctl + n + 1) : 0u;
                }
            }
            wait_vm<7>();
            if (issued - 7 > told && lane == 0) __hip_atomic_store(ctl + 384, (unsigned)issued - 7u, RLX, SC_WG);
            wait_vm<0>();
            if (lane == 0) __hip_atomic_store(ctl + 384, (unsigned)issued, RLX, SC_WG);
            __builtin_amdgcn_s_setprio(0);
            STAMP(19);
        } else {
            if (wid == 1) {
                const GAS u64* fp = lane < 16 ? (const GAS u64*)(myws + WS_ORDY + (size_t)me * 128) + lane : (const GAS u64*)(myws + WS_MBOX + (size_t)me * 128);
                bool pend = lane < 17;
                const u64 t0 = __builtin_amdgcn_s_memrealtime();
                for (unsigned polls = 1;; ++polls) {
                    if (pend) {
                        const u64 f = __hip_atomic_load(fp, RLX, SC_SYS);
                        if ((unsigned)(f >> 32) >= call) { __hip_atomic_store(ctl + lane, (unsigned)f, RLX, SC_WG); pend = false; }
                    }
                    if (!__any(pend)) break;
                    if ((polls & 15u) == 0u && TIMED_OUT(t0)) {
                        if (lane == 0) __hip_atomic_store(tmo, call, RLX, SC_AGENT);
                        if (pend) __hip_atomic_store(ctl + lane, lane < 16 ? 0xc0002000u : now8() | 1u, RLX, SC_WG);
                        break;
                    }
                    __builtin_amdgcn_s_sleep(4);
                }
            }
            for (;;) {
                unsigned tk = 0;
                if (lane == 0) tk = __hip_atomic_fetch_add(ctl + 401, 1u, RLX, SC_WG);
                const int k = UNI(tk);
                if (k >= 16) break;
                unsigned need;
                while ((need = lds_get(ctl + 416 + k)) == 0u) __builtin_amdgcn_s_sleep(4);
                const unsigned hdr = lds_get(ctl + k);
                while (lds_get(ctl + 384) < need) __builtin_amdgcn_s_sleep(1);
                asm volatile("" ::: "memory");
                const LAS unsigned char* src = lds + (unsigned)k * 8192u;
                unsigned o[32];
                if ((hdr >> 13) & 1u) {
#pragma unroll
                    for (int i = 0; i < 8; ++i) { const u32x4 a = *(const LAS u32x4*)(src + i * 1024 + lo); o[4 * i] = a[0]; o[4 * i + 1] = a[1]; o[4 * i + 2] = a[2]; o[4 * i + 3] = a[3]; }
                } else {
                    unsigned lw[16], hi[16], ne[16];
#pragma unroll
                    for (int i = 0; i < 2; ++i) { const u32x4 a = *(const LAS u32x4*)(src + 4096 + i * 1024 + lo); hi[4 * i] = a[0]; hi[4 * i + 1] = a[1]; hi[4 * i + 2] = a[2]; hi[4 * i + 3] = a[3]; }
                    { const u32x2 a = *(const LAS u32x2*)(src + 6144 + lane * 8); hi[8] = a[0]; hi[9] = a[1]; }
#pragma unroll
                    for (int i = 0; i < 4; ++i) { const u32x4 a = *(const LAS u32x4*)(src + i * 1024 + lo); lw[4 * i] = a[0]; lw[4 * i + 1] = a[1]; lw[4 * i + 2] = a[2]; lw[4 * i + 3] = a[3]; }
#pragma unroll
                    for (int c = 0; c < 6; ++c) {
                        unsigned cc = (hi[9] << (1 + c)) & 0x80008000u;
#pragma unroll
                        for (int i = 0; i < 3; ++i) { const int fld = c * 3 + i, at = 9 + 3 * (fld & 1); cc |= (hi[fld >> 1] >> (at - 3 * i)) & (0x00070007u << (3 * i)); }
                        hi[10 + c] = cc;
                    }
                    u64 str = code_find<7>(hi, ne, (const LAS unsigned*)(src + 6656), fq);
                    const unsigned e0 = (hdr & 255u) * 0x00800080u, b2 = ((hdr >> 8) & 31u) * 0x00010001u;
#pragma unroll
                    for (int i = 0; i < 16; ++i) {
                        const unsigned h2 = (hi[i] & 0x81ff81ffu) + e0 + (code_get(str, ne[i], b2) << 7);
                        finish(o, i, __uint_as_float(__builtin_amdgcn_perm(h2, lw[i], 0x05040100u)), __uint_as_float(__builtin_amdgcn_perm(h2, lw[i], 0x07060302u)));
                    }
                }
                const int blk = (k >> 3) * 4 + (k & 3);
                const char* ob = (const char*)(p.out + (size_t)((peer & 1) * 256 + (blk >> 1) * 64 + ((k >> 2) & 1) * 32) * GN + (peer >> 1) * 4096 + (cb + sk) * 128 + (blk & 1) * 64);
#pragma unroll
                for (int i = 0; i < 8; ++i)
                    st16(ob + (size_t)((i >> 2) * 16 + (i & 3)) * (GN * 4), oo, (hdr >> 31) ? nan4 : (u32x4){o[4 * i], o[4 * i + 1], o[4 * i + 2], o[4 * i + 3]});
            }
        }
        }
    }
    }
    if (tid == 0) __hip_atomic_store(ctr, call, RLX, SC_AGENT);
}

extern "C" void kernel_launch(int rank, int nranks, ncclComm_t comm, hipStream_t stream,
                              void* const* d_in, const int* in_sizes, int n_in,
                              void* d_out, int out_size, const int* in_dtypes, int out_dtype,
                              void* const* d_out_all, void* const* d_ws_all, size_t ws_bytes) {
    (void)comm; (void)in_dtypes; (void)out_dtype; (void)d_out_all;
    if (nranks != NR || (unsigned)rank >= (unsigned)NR || n_in != 2 || ws_bytes < WS_NEED ||
        in_sizes[0] != 4096 * KSH || in_sizes[1] != KSH * GN || out_size != 512 * GN) {
        fprintf(stderr, "kernel_launch: unexpected problem shape\n");
        return;
    }
    Params p;
    p.x = (const float*)d_in[0];
    p.w = (const float*)d_in[1];
    p.out = (float*)d_out;
    p.myws = (char*)d_ws_all[rank];
    for (int i = 0; i < NR; ++i) p.ws[i] = (char*)d_ws_all[i];
    p.rank = rank;
    p.pad = 0;
    int n = 0, np = 0, nd = 0;
    unsigned char deferred[8];
    bool px[8] = {};
    for (int i = 0; i < 128; ++i) p.order[0][i] = p.order[1][i] = 0;
    auto need_x = [&](int c0, int c1) { for (int c = c0; c <= c1; ++c) if (!px[c]) { px[c] = true; p.prod[np++] = (unsigned)(32 + c); } };
    for (int key = 0; key < 16 + 3 * SKEW2; ++key) {
        for (int pass = 0; pass < 2; ++pass) {
            if ((pass == 0) == (key != 0)) {
                for (int e = 32; e < 44; ++e)
                    if (2 * ((e - 32) * 8 / 12) == key) { p.order[0][n++] = (unsigned char)e; need_x((e - 32) * 8 / 12, ((e - 32) * 8 + 7) / 12); }
                if (key == XPOS) { p.order[0][n++] = 44; need_x(0, 3); }
                if (key == XPOS + 2) { p.order[0][n++] = 45; need_x(4, 7); }
                if (key == LPOS) { p.order[0][n++] = 46; p.order[0][n++] = 47; }
            } else {
                for (int b = 0; b < 4; ++b)
                    for (int c = 0; c < 8; ++c)
                        if ((b == 0 && c == 1 ? 0 : 2 * c + b * SKEW2) == key) {
                            p.prod[np++] = (unsigned)(c * 4 + b);
                            if (b == 3 && c < 2 * NFWD && ((c ^ rank) & 1)) deferred[nd++] = (unsigned char)(c * 4 + b);
                            else p.order[0][n++] = (unsigned char)(c * 4 + b);
                        }
            }
        }
    }
    int m = 0;
    for (int i = 0; i < n; ++i) if (p.order[0][i] < 44) p.order[1][m++] = p.order[0][i];
    for (int i = 0; i < nd; ++i) p.order[1][m++] = deferred[i];
    for (int i = 0; i < 6 * NFWD; ++i) p.order[1][m++] = (unsigned char)(48 + i);
    for (int e = 44; e < 48; ++e) p.order[1][m++] = (unsigned char)e;
    if (n != NU_OUT || m != NU_PAIR || np != 40 || p.prod[0] != 0 || p.prod[1] != 4 || p.prod[2] != 32) { fprintf(stderr, "kernel_launch: bad schedule\n"); return; }
    hipLaunchKernelGGL(fused_gemm_rs, dim3(NBLK), dim3(NTHR), 0, stream, p);
    HIP_CHECK(hipGetLastError());
}
```

```cpp
#include <hip/hip_runtime.h>
#include <rccl/rccl.h>
#include <cstdio>

#define HIP_CHECK(expr) do { hipError_t _e = (expr); if (_e != hipSuccess) { \
    fprintf(stderr, "HIP error %s at %s:%d\n", hipGetErrorString(_e), __FILE__, __LINE__); \
    return; } } while (0)

typedef float f32x4 __attribute__((ext_vector_type(4)));
typedef unsigned u32x4 __attribute__((ext_vector_type(4)));
typedef unsigned u32x3 __attribute__((ext_vector_type(3)));
typedef unsigned u32x2 __attribute__((ext_vector_type(2)));
typedef unsigned short u16x2 __attribute__((ext_vector_type(2)));
typedef _Float16 h16x8 __attribute__((ext_vector_type(8)));
typedef unsigned long long u64;
#define LAS __attribute__((address_space(3)))
#define GAS __attribute__((address_space(1)))
#define RLX __ATOMIC_RELAXED
#define SC_WG __HIP_MEMORY_SCOPE_WORKGROUP
#define SC_AGENT __HIP_MEMORY_SCOPE_AGENT
#define SC_SYS __HIP_MEMORY_SCOPE_SYSTEM

#define NPLAIN 3
#define TAU 864
#define TAU_OUT TAU
#define WIN_OUT WIN
#define EQ_AT 5
#define CREDIT 36
#define WIN 10
#define PROD_GAP 40
#define PUSH_SLOT 112
#define RSTRIDE 16
#define POLLW 12
#define PROD_OFF -2
#define B0_ITS 2
#define ROT 64
#define EQ2 1
#define POLL_SLEEP 8
#define NFWD 3
#define XPOS 6
#define LPOS 18
#define SKEW2 2

constexpr int NR = 8;
constexpr int GN = 16384;
constexpr int KSH = 512;
constexpr int NBLK = 256;
constexpr int NTHR = 512;
constexpr int NSTEP = 64;
constexpr unsigned A_STEP = 32768, B_STEP = 16384, STAGE = A_STEP + B_STEP;
constexpr int NU_OUT = 48 - NFWD, NU_PAIR = 48 + 6 * NFWD;
constexpr int RING = 8;
constexpr int PW = 14;
constexpr int POOL_IN = 48, POOL_OUT = 60;
constexpr int QB = 3328 + 4 * POOL_IN;
constexpr int PB = 6656 + 4 * POOL_OUT;
constexpr unsigned UNIT_T = (4 * QB + 127) / 128 * TAU / 8, PIECE_T = (PB + 127) / 128 * TAU_OUT / 8;
constexpr unsigned SLOT = 18432, SIDE = 14336;
constexpr size_t WS_CTR = 0;
constexpr size_t WS_FLAG = 32 << 10;
constexpr size_t WS_ORDY = 96 << 10;
constexpr size_t WS_MBOX = 128 << 10;
constexpr size_t WS_EQ = 160 << 10;
constexpr size_t WS_TMO = 192 << 10;
constexpr size_t WS_DIAG = 256 << 10;
constexpr size_t WS_STG = 1 << 20;
constexpr size_t STG_A = 8ull * NSTEP * A_STEP;
constexpr size_t STG_B = 32ull * NSTEP * B_STEP;
constexpr size_t WS_EXI = WS_STG + STG_A + STG_B;
constexpr size_t EXI_BYTES = 1600ull * SLOT;
constexpr size_t WS_EXO = WS_EXI + 2 * EXI_BYTES;
constexpr size_t WS_RDY = WS_EXO + (size_t)NBLK * 8 * 16384;
constexpr size_t WS_NEED = WS_RDY + 2 * 8 * 48 * 8 * 128;
constexpr u64 DEADLINE_TICKS = 50000000ull;

struct Params {
    const float* x;
    const float* w;
    float* out;
    char* myws;
    char* ws[NR];
    int rank;
    int pad;
    unsigned prod[40];
    unsigned char order[2][128];
};

#define WAIT_L0() do { asm volatile("s_waitcnt lgkmcnt(0)" ::: "memory"); __builtin_amdgcn_s_waitcnt(0xC07F); } while (0)
#define BAR() do { asm volatile("" ::: "memory"); __builtin_amdgcn_s_barrier(); asm volatile("" ::: "memory"); } while (0)
#define SB() __builtin_amdgcn_sched_barrier(0)
#define UNI(x) __builtin_amdgcn_readfirstlane(x)
template <int N> __device__ __forceinline__ void wait_vm() { asm volatile("s_waitcnt vmcnt(%0)" ::"i"(N) : "memory"); }

__device__ __forceinline__ void st16(const void* base, unsigned voff, u32x4 v) {
    asm volatile("s_nop 4\n\tglobal_store_dwordx4 %0, %1, %2 sc0 sc1\n\ts_nop 1" ::"v"(voff), "v"(v), "s"(base) : "memory");
}
__device__ __forceinline__ void st12(const void* base, unsigned voff, u32x3 v) {
    asm volatile("s_nop 4\n\tglobal_store_dwordx3 %0, %1, %2 sc0 sc1\n\ts_nop 1" ::"v"(voff), "v"(v), "s"(base) : "memory");
}
__device__ __forceinline__ void st8(const void* base, unsigned voff, u32x2 v) {
    asm volatile("s_nop 4\n\tglobal_store_dwordx2 %0, %1, %2 sc0 sc1" ::"v"(voff), "v"(v), "s"(base) : "memory");
}
__device__ __forceinline__ void st4(const void* base, unsigned voff, unsigned v) {
    asm volatile("s_nop 4\n\tglobal_store_dword %0, %1, %2 sc0 sc1" ::"v"(voff), "v"(v), "s"(base) : "memory");
}
__device__ __forceinline__ u32x3 ld12(const void* base, unsigned voff) {
    u32x3 r;
    asm volatile("s_nop 4\n\tglobal_load_dwordx3 %0, %1, %2 sc0 sc1\n\ts_waitcnt vmcnt(0)" : "=&v"(r) : "v"(voff), "s"(base) : "memory");
    return r;
}
__device__ __forceinline__ void ld64(const void* base, unsigned voff, u32x4& a, u32x4& b, u32x4& c, u32x4& d) {
    asm volatile("s_nop 4\n\tglobal_load_dwordx4 %0, %4, %5 sc0 sc1\n\tglobal_load_dwordx4 %1, %4, %5 offset:1024 sc0 sc1\n\t"
                 "global_load_dwordx4 %2, %4, %5 offset:2048 sc0 sc1\n\tglobal_load_dwordx4 %3, %4, %5 offset:3072 sc0 sc1\n\ts_waitcnt vmcnt(0)"
                 : "=&v"(a), "=&v"(b), "=&v"(c), "=&v"(d) : "v"(voff), "s"(base) : "memory");
}
template <bool TWO> __device__ __forceinline__ void dma(const void* base, unsigned voff, unsigned lds_dst) {
    unsigned keep;
    if (TWO) asm volatile("s_mov_b32 %0, m0\n\ts_mov_b32 m0, %3\n\ts_nop 4\n\tglobal_load_lds_dwordx4 %1, %2 sc0 sc1\n\tglobal_load_lds_dwordx4 %1, %2 offset:1024 sc0 sc1\n\ts_mov_b32 m0, %0"
                          : "=&s"(keep) : "v"(voff), "s"(base), "s"(lds_dst) : "memory");
    else asm volatile("s_mov_b32 %0, m0\n\ts_mov_b32 m0, %3\n\ts_nop 4\n\tglobal_load_lds_dwordx4 %1, %2 sc0 sc1\n\ts_mov_b32 m0, %0"
                      : "=&s"(keep) : "v"(voff), "s"(base), "s"(lds_dst) : "memory");
}
__device__ __forceinline__ unsigned now8() { return (unsigned)__builtin_amdgcn_s_memrealtime() << 6; }
__device__ __forceinline__ void pace(unsigned& t, unsigned period) {
    int late = (int)(now8() - t);
    if (late < -(1 << 22)) { t += (unsigned)late; late = 0; }
    while (late < 0) { __builtin_amdgcn_s_sleep(1); late = (int)(now8() - t); }
    t += period + (late > CREDIT * 64 ? (unsigned)(late - CREDIT * 64) : 0u);
}
__device__ __forceinline__ unsigned lds_get(const LAS unsigned* a) { return UNI(__hip_atomic_load(a, RLX, SC_WG)); }
__device__ __forceinline__ unsigned cvt2(float a, float b) {
    typedef _Float16 h2 __attribute__((ext_vector_type(2)));
    h2 r = {(_Float16)a, (_Float16)b};
    return __builtin_bit_cast(unsigned, r);
}
__device__ __forceinline__ int rb_owner(int d, int rb) { return rb >= 6 ? d : ((((d >> 1) + 1 + (rb >> 1)) & 3) << 1) | (rb & 1); }
__device__ __forceinline__ int rb_local(int d, int rb) { return rb >= 6 ? (rb - 6) * 256 : (d & 1) * 256; }
__device__ __forceinline__ int rb_of(int d, int o) { return ((((o >> 1) - (d >> 1) - 1) & 3) << 1) | (o & 1); }
__device__ __forceinline__ void unit_of(int e, int j, int& c, int& id) {
    if (e < 32) { c = e >> 2; id = j * 4 + (e & 3); }
    else if (e < 44) { const int u = (e - 32) * 8 + j; c = u / 12; id = 32 + u - c * 12; }
    else { const int u = (e & 1) * 8 + j; c = u >> 1; id = (e < 46 ? 46 : 44) + (u & 1); }
}
__device__ __forceinline__ int gid_of(int d, int id) {
    const int rb = (id - 32) >> 1;
    return id < 32 ? (d >> 1) * 32 + id : 128 + rb_owner(d, rb) * 4 + (rb_local(d, rb) >> 7) + (id & 1);
}
__device__ __forceinline__ size_t stage_off(int id, int ks) {
    return id < 32 ? STG_A + (size_t)(id * NSTEP + ks) * B_STEP : (size_t)(((id - 32) >> 1) * NSTEP + ks) * A_STEP + (id & 1) * 16384;
}
template <int BITS, int SH> __device__ __forceinline__ unsigned wave_max_exp(unsigned mag) {
    unsigned e = 0;
#pragma unroll
    for (int b = BITS - 1; b >= 0; --b) if (__any(mag >= ((e + (1u << b)) << SH))) e += 1u << b;
    return e;
}

__device__ __forceinline__ unsigned row_scan(unsigned x) {
    x += __builtin_amdgcn_update_dpp(0, x, 0x111, 0xf, 0xf, true);
    x += __builtin_amdgcn_update_dpp(0, x, 0x112, 0xf, 0xf, true);
    x += __builtin_amdgcn_update_dpp(0, x, 0x114, 0xf, 0xf, true);
    x += __builtin_amdgcn_update_dpp(0, x, 0x118, 0xf, 0xf, true);
    return x;
}
__device__ __forceinline__ unsigned ffbl(unsigned x) { unsigned r; asm("v_ffbl_b32 %0, %1" : "=v"(r) : "v"(x)); return r; }
#define PK(x) __builtin_bit_cast(u16x2, (unsigned)(x))
#define DW(x) __builtin_bit_cast(unsigned, (u16x2)(x))
#define SUBS(a, b) DW(__builtin_elementwise_sub_sat(PK(a), PK(b)))
#define MIN2(a, b) DW(__builtin_elementwise_min(PK(a), PK(b)))

template <int BITS> __device__ __forceinline__ int code_base(const unsigned (&ef)[16]) {
    u16x2 mx = PK(ef[0]);
#pragma unroll
    for (int i = 1; i < 16; ++i) mx = __builtin_elementwise_max(mx, PK(ef[i]));
    const int emax = (int)wave_max_exp<BITS, 0>(mx.x > mx.y ? mx.x : mx.y);
    int up = 0, mid = 0;
#pragma unroll
    for (int i = 0; i < 6; ++i) {
        const int e = (int)(ef[i] >> 16);
        up += __builtin_popcountll(__ballot(e == emax)) - __builtin_popcountll(__ballot(e == emax - 3));
        mid += __builtin_popcountll(__ballot(e == emax - 1)) - __builtin_popcountll(__ballot(e == emax - 4));
    }
    const int bs = emax - (up > 0 ? 3 : mid > 0 ? 4 : 5);
    return bs > 0 ? bs : 0;
}
template <int POOL> __device__ __forceinline__ bool code_put(unsigned (&ef)[16], unsigned B, LAS unsigned* pool, int row) {
    const unsigned b2 = B * 0x00010001u, lut = 0x0300u | ((B + 2u + (B ? 1u : 0u)) << 16);
    u64 str = 0;
    unsigned sh = 0;
#pragma unroll
    for (int i = 0; i < 16; ++i) {
        const unsigned k1 = SUBS(b2 + 0x00010001u, ef[i]), d = SUBS(ef[i], b2);
        const unsigned len = k1 + MIN2(SUBS(k1, 0x00020002u), 0x00010001u) + __builtin_amdgcn_perm(0u, lut, SUBS(d, 0x00030003u) | 0x0c000c00u);
        sh += len & 0xffffu; str |= 1ull << (sh & 63u);
        sh += len >> 16; str |= 1ull << (sh & 63u);
        ef[i] = MIN2(d, 0x00040004u) & 0x00030003u;
    }
    const unsigned end = row_scan(sh);
    const unsigned r0 = __builtin_amdgcn_readlane(end, 15), r1 = __builtin_amdgcn_readlane(end, 31), r2 = __builtin_amdgcn_readlane(end, 47);
    if (!__all(sh <= 63u && end <= 32u * (PW - 1)) || r0 + r1 + r2 + __builtin_amdgcn_readlane(end, 63) > 32u * (POOL - 1)) return false;
    const unsigned pos = 32u + end - sh + (row > 0 ? r0 : 0u) + (row > 1 ? r1 : 0u) + (row > 2 ? r2 : 0u);
    str >>= 1;
    const u64 s0 = str << (pos & 31u);
    LAS unsigned* pp = pool + (pos >> 5);
    __hip_atomic_store(pool, r0 | r1 << 10 | r2 << 20, RLX, SC_WG);
    __hip_atomic_fetch_or(pp, (unsigned)s0, RLX, SC_WG);
    __hip_atomic_fetch_or(pp + 1, (unsigned)(s0 >> 32), RLX, SC_WG);
    __hip_atomic_fetch_or(pp + 2, (unsigned)(((str >> 32) << (pos & 31u)) >> 32), RLX, SC_WG);
    return true;
}
template <int SH> __device__ __forceinline__ u64 code_find(const unsigned (&w)[16], unsigned (&ne)[16], const LAS unsigned* pool, int row) {
    const unsigned rl = pool[0];
    const unsigned start = 32u + (row > 0 ? rl & 1023u : 0u) + (row > 1 ? (rl >> 10) & 1023u : 0u) + (row > 2 ? (rl >> 20) & 1023u : 0u);
    pool += start >> 5;
    unsigned pd[PW - 1], sum = 0;
#pragma unroll
    for (int i = 0; i < PW - 1; ++i) pd[i] = pool[i];
#pragma unroll
    for (int i = 0; i < 16; ++i) { ne[i] = MIN2(w[i] & (0x00030003u << SH), 0x00010001u); sum += ne[i]; }
    sum = 32u - (sum & 0xffffu) - (sum >> 16);
    const unsigned skip = row_scan(sum) - sum + __builtin_popcount(__builtin_amdgcn_ubfe(pd[0], 0u, start));
    unsigned cum = 0, at = 0, below = 0;
#pragma unroll
    for (int i = 0; i < PW - 1; ++i) {
        cum += __builtin_popcount(pd[i]);
        const bool lt = cum < skip;
        at += lt; below = lt ? cum : below;
    }
    const unsigned d0 = pool[at], d1 = pool[at + 1], d2 = pool[at + 2];
    unsigned rem = skip - below, pos = 0;
#pragma unroll
    for (int sft = 16; sft; sft >>= 1) {
        const unsigned t = __builtin_popcount(__builtin_amdgcn_ubfe(d0, pos, sft));
        const bool lt = t < rem;
        rem -= lt ? t : 0u; pos += lt ? sft : 0u;
    }
    pos = skip ? pos + 1u : 0u;
    return (unsigned)((((u64)d1 << 32) | d0) >> pos) | ((((u64)d2 << 32) | d1) >> pos << 32);
}
__device__ __forceinline__ unsigned code_get(u64& str, unsigned ne, unsigned b2) {
    const unsigned l0 = ffbl(((unsigned)str << 1) | (ne & 1u)); str >>= l0 & 63u;
    const unsigned l1 = ffbl(((unsigned)str << 1) | (ne >> 16)); str >>= l1 & 63u;
    const unsigned len = l0 | (l1 << 16);
    const unsigned adj = __builtin_amdgcn_perm(0x02u, 0x07010100u, MIN2(len, 0x00040004u) | 0x0c000c00u);
    return MIN2(DW(PK(b2 + adj) - PK(len)), b2 + 0x00050005u);
}

typedef float f32x2 __attribute__((ext_vector_type(2)));
__device__ __forceinline__ f32x2 gelu2(f32x2 y) {
    const float k1 = -2.0f * 0.7978845608028654f * 1.4426950408889634f, k3 = k1 * 0.044715f;
    const f32x2 a = __builtin_elementwise_fma(y * y, (f32x2){k3, k3}, (f32x2){k1, k1}) * y;
    const f32x2 d = (f32x2){__builtin_amdgcn_exp2f(a.x), __builtin_amdgcn_exp2f(a.y)} + 1.0f;
    return y * (f32x2){__builtin_amdgcn_rcpf(d.x), __builtin_amdgcn_rcpf(d.y)};
}
__device__ __forceinline__ float gelu1(float y) { return gelu2((f32x2){y, y}).x; }

#define STAMP(i) do {} while (0)
#define TIMED_OUT(t0) (UNI(__hip_atomic_load(tmo, RLX, SC_AGENT)) == call || __builtin_amdgcn_s_memrealtime() - (t0) > DEADLINE_TICKS)

__global__ __launch_bounds__(NTHR, 2) void fused_gemm_rs(Params p) {
    __shared__ __attribute__((aligned(16))) unsigned char lds_raw[3 * STAGE + 2048];
    LAS unsigned char* lds = (LAS unsigned char*)lds_raw;
    LAS unsigned* ctl = (LAS unsigned*)(lds + 3 * STAGE);
    const unsigned lds0 = (unsigned)(size_t)lds;
    const int tid = threadIdx.x, lane = tid & 63;
    const int wid = UNI(tid >> 6);
    const int fr = lane & 15, fq = lane >> 4;
    const unsigned lo = lane * 16u;
    const unsigned bid = (blockIdx.x + ROT) & 255u;
    const int xcd = bid & 7, idx = bid >> 3;
    const bool late = idx < 8;
    const int q = wid & 3;

    const int team = (wid >> 2) * 192 + (int)bid - 64;
    f32x4 r[8];
    auto request = [&](int g, int u) {
        if (g < 32) {
            const GAS char* b = (const GAS char*)(p.w + (size_t)((g >> 2) * 64 + (q & 1) * 32) * GN + (u * 4 + (g & 3)) * 128 + (q >> 1) * 64) + (unsigned)(8 * fq * GN + 4 * fr) * 4u;
#pragma unroll
            for (int e = 0; e < 8; ++e) r[e] = *(const GAS f32x4*)(b + (size_t)e * (GN * 4));
        } else {
            const GAS char* b = (const GAS char*)(p.x + (size_t)(u * 128 + q * 32) * KSH + (g - 32) * 64) + (unsigned)(fr * KSH + 8 * fq) * 4u;
#pragma unroll
            for (int e = 0; e < 8; ++e) r[e] = *(const GAS f32x4*)(b + (e >> 2) * (16 * KSH * 4) + ((e >> 1) & 1) * 128 + (e & 1) * 16);
        }
    };
    if (!late && team < 96) request(team >= 64 ? 32 : (team >> 5) * 4, team & 31);
    const unsigned t_entry = now8();

    const int rank = p.rank & 7;
    char* myws = p.myws;
    STAMP(0);
    GAS unsigned* ctr = (GAS unsigned*)(myws + WS_CTR + (size_t)bid * 128);
    GAS unsigned* tmo = (GAS unsigned*)(myws + WS_TMO);
    const unsigned ctr_v = __hip_atomic_load(ctr, RLX, SC_AGENT);
    GAS unsigned* gflag = (GAS unsigned*)(myws + WS_FLAG);
    ctl[tid] = 0u;
    WAIT_L0();
    BAR();

    if (!late) {
        unsigned wlo = 0, whi = 0;
#pragma unroll
        for (int d = 0; d < NR; ++d) {
            unsigned a = (unsigned)(size_t)p.ws[d], b = (unsigned)((size_t)p.ws[d] >> 32);
            asm volatile("" : "+s"(a), "+s"(b));
            wlo = lane == d ? a : wlo;
            whi = lane == d ? b : whi;
        }
#pragma unroll 1
        for (int it = 0; it < 5; ++it) {
            const int t = it < B0_ITS ? it * 384 + team : B0_ITS * 384 + (it - B0_ITS) * 288 + (wid >> 2) * 144 + (((idx - 8) >> 2) * 3 + (idx & 3) - 1) * 8 + xcd;
            if (t >= 1280 || (it >= B0_ITS && (idx & 3) == 0)) break;
            const int g = p.prod[t >> 5], u = t & 31;
            if (t >= 96) {
                const unsigned ts = t_entry + (unsigned)((t >> 5) - PROD_OFF) * (PROD_GAP * 64);
                while ((int)(now8() - ts) < 0) __builtin_amdgcn_s_sleep(2);
                request(g, u);
            }
            const bool isw = g < 32;
            const int c = isw ? g >> 2 : g - 32;
            const int gid = isw ? u * 4 + (g & 3) : 128 + u;
            unsigned v[16];
            if (isw) {
#pragma unroll
                for (int i = 0; i < 16; ++i) v[i] = cvt2(r[(i & 3) * 2][i >> 2], r[(i & 3) * 2 + 1][i >> 2]);
            } else {
#pragma unroll
                for (int i = 0; i < 16; ++i) v[i] = cvt2(r[i >> 1][(i & 1) * 2], r[i >> 1][(i & 1) * 2 + 1]);
            }
            asm volatile("" : "+v"(v[15]));
            const unsigned t_loaded = now8();
            const bool pushed = t < 32;
            const unsigned call = UNI(ctr_v) + 1u;
            const size_t par = WS_EXI + (size_t)(call & 1u) * EXI_BYTES;
            const int o = u >> 2;
            const int myid = isw ? ((gid >> 5) == (rank >> 1) ? gid & 31 : -1)
                                 : (o == rank ? 44 + (u & 3) : ((o >> 1) != (rank >> 1) && ((u >> 1) & 1) == (rank & 1) ? 32 + rb_of(rank, o) * 2 + (u & 1) : -1));
            unsigned hdr = 0;
            {
                unsigned w[16];
#pragma unroll
                for (int i = 0; i < 16; ++i) w[i] = v[i];
                const char* home = myws + par + (size_t)(gid * 8 + c) * SLOT;
                unsigned pw = 0;
                hdr = 32u;
                if (!pushed) {
                    LAS unsigned* pool = (LAS unsigned*)(lds + wid * 256);
                    pool[lane] = 0u;
                    unsigned ef[16];
#pragma unroll
                    for (int i = 0; i < 16; ++i) ef[i] = (w[i] >> 10) & 0x001f001fu;
                    const unsigned B = (unsigned)code_base<5>(ef);
                    if (code_put<POOL_IN>(ef, B, pool, fq)) {
#pragma unroll
                        for (int i = 0; i < 16; ++i) w[i] = (w[i] & 0x83ff83ffu) | (ef[i] << 10);
#pragma unroll
                        for (int k = 0; k < 3; ++k) {
#pragma unroll
                            for (int i = 0; i < 4; ++i) w[4 * k + i] |= (w[13 + k] << (12 - 3 * i)) & 0x70007000u;
                            w[12] |= (w[13 + k] >> (1 + k)) & (0x40004000u >> k);
                        }
                        pw = __hip_atomic_load(pool + lane, RLX, SC_WG);
                        hdr = B;
                    }
                }
                if (pushed) {
                    unsigned tp = t_loaded + (unsigned)((u & 7) * 4 + q) * PUSH_SLOT;
#pragma unroll
                    for (int i = 0; i < 4; ++i) {
                        pace(tp, 32u * PUSH_SLOT);
#pragma unroll
                        for (int k = 0; k < 2; ++k) {
                            const int d = (u >> 3) * 2 + k;
                            if (d != rank) st16(p.ws[d] + par + (size_t)(1280 + rank * 8 + (u & 7)) * SLOT + q * 4096 + i * 1024, lo, (u32x4){v[4 * i], v[4 * i + 1], v[4 * i + 2], v[4 * i + 3]});
                        }
                    }
                } else {
#pragma unroll
                    for (int i = 0; i < 3; ++i) st16(home + q * QB + i * 1024, lo, (u32x4){w[4 * i], w[4 * i + 1], w[4 * i + 2], w[4 * i + 3]});
                    st4(home + q * QB + 3072, lane * 4u, w[12]);
                    if (hdr == 32u) st12(home + SIDE + q * 768, lane * 12u, (u32x3){v[13], v[14], v[15]});
                    else if (lane < POOL_IN) st4(home + q * QB + 3328, lane * 4u, pw);
                }
            }
            if (myid >= 0) {
                const char* sg = myws + WS_STG + stage_off(myid, rank * 8 + c) + (isw ? (q >> 1) * 8 + (q & 1) : q * 4) * 1024;
#pragma unroll
                for (int i = 0; i < 4; ++i) st16(sg + i * (isw ? 2048 : 1024), lo, (u32x4){v[4 * i], v[4 * i + 1], v[4 * i + 2], v[4 * i + 3]});
            }
            wait_vm<0>();
            unsigned tw = (1u << 24) | (hdr << (6 * q));
            if (lane == 0) tw += __hip_atomic_fetch_add(ctl + 402 + (wid >> 2) * 5 + it, tw, RLX, SC_WG);
            tw = UNI(tw);
            if ((tw >> 24) == 4u) {
                const int d = lane & 7;
                int did = isw ? ((gid >> 5) == (d >> 1) ? gid & 31 : -1)
                              : (o == d ? 44 + (u & 3) : ((o >> 1) != (d >> 1) && ((u >> 1) & 1) == (d & 1) ? 32 + rb_of(d, o) * 2 + (u & 1) : -1));
                if (isw && (g & 3) == 3 && c < 2 * NFWD && (d >> 1) != (rank >> 1) && ((c ^ d) & 1)) did = -1;
                if (lane < 8 && did >= 0) {
                    if (d == rank) __hip_atomic_store(gflag + did * 64 + c * 8 + rank, call, RLX, SC_AGENT);
                    else __hip_atomic_store((GAS u64*)((((size_t)whi << 32) | wlo) + WS_RDY) + ((((call & 1u) * 8 + rank) * 48 + did) * 8 + c) * RSTRIDE, ((u64)call << 32) | (tw & 0xffffffu), RLX, SC_SYS);
                }
            }
        }
        STAMP(9);
    } else if (idx < 7) {
        const int s = idx < 6 ? rb_owner(rank, idx) : rank ^ 1;
        const int nu = idx < 6 ? NU_OUT : NU_PAIR;
        unsigned desc[2];
#pragma unroll
        for (int k = 0; k < 2; ++k) {
            const int e = p.order[idx == 6][k * 64 + lane];
            int c, id, org = s, slot;
            if (e < 48) { unit_of(e, xcd, c, id); slot = gid_of(rank, id) * 8 + c; }
            else { const int kf = (e - 48) / 6; c = 2 * kf + (s & 1); id = xcd * 4 + 3; org = rb_owner(rank, e - 48 - kf * 6); slot = 1408 + (e - 48) * 8 + xcd; }
            desc[k] = (unsigned)(c | id << 3 | org << 9 | slot << 12);
        }
        auto desc_of = [&](int n) { return (unsigned)(n < 64 ? __builtin_amdgcn_readlane(desc[0], n) : __builtin_amdgcn_readlane(desc[1], n - 64)); };
        const unsigned call = UNI(ctr_v) + 1u;
        const size_t par = WS_EXI + (size_t)(call & 1u) * EXI_BYTES;
        if (wid < 6) {
            int pend = -1, pend_n = 0;
            unsigned pend_h = 0;
            auto staged = [&]() {
                wait_vm<0>();
                if (lane == 0 && __hip_atomic_fetch_add(ctl + 256 + pend_n, 1u, RLX, SC_WG) == 3u) {
                    __hip_atomic_store(gflag + pend, call, RLX, SC_AGENT);
                    if (pend_h) __hip_atomic_store((GAS u64*)(p.ws[rank ^ 1] + WS_RDY) + ((((call & 1u) * 8 + s) * 48 + (pend >> 6)) * 8 + ((pend >> 3) & 7)) * RSTRIDE,
                                                   ((u64)call << 32) | (pend_h & 0xffffffu), RLX, SC_SYS);
                }
                pend = -1;
            };
            auto need = [&](int job) { return (job >> 2) * 14 - 14 + (((job & 3) + 1) * QB + 1023) / 1024; };
            auto ticket = [&]() { unsigned tk = 0; if (lane == 0) tk = __hip_atomic_fetch_add(ctl + 401, 1u, RLX, SC_WG); return (int)UNI(tk); };
            for (int job = ticket(); job < 4 * nu;) {
                const int n = job >> 2, qq = job & 3;
                unsigned w[16];
                int un = 0;
                unsigned relay_h = 0;
                if (n == 0) {
                    while (lds_get(ctl) == 0u) __builtin_amdgcn_s_sleep(1);
                    u32x4 a0, a1, a2, a3;
                    ld64(myws + par + (size_t)(1280 + s * 8 + xcd) * SLOT + qq * 4096, lo, a0, a1, a2, a3);
#pragma unroll
                    for (int i = 0; i < 4; ++i) { w[i] = a0[i]; w[4 + i] = a1[i]; w[8 + i] = a2[i]; w[12 + i] = a3[i]; }
                } else {
                    while ((int)lds_get(ctl + 384) < need(job)) __builtin_amdgcn_s_sleep(1);
                    asm volatile("" ::: "memory");
                    un = lds_get(ctl + 416 + n);
                }
                const unsigned de = desc_of(un);
                const int c = de & 7, id = (de >> 3) & 63, org = (de >> 9) & 7;
                if (n) {
                    const LAS unsigned char* src = lds + (unsigned)(n % RING) * 16384u + qq * QB;
#pragma unroll
                    for (int i = 0; i < 3; ++i) { const u32x4 a = *(const LAS u32x4*)(src + i * 1024 + lo); w[4 * i] = a[0]; w[4 * i + 1] = a[1]; w[4 * i + 2] = a[2]; w[4 * i + 3] = a[3]; }
                    w[12] = *(const LAS unsigned*)(src + 3072 + lane * 4);
                    const LAS unsigned* pool = (const LAS unsigned*)(src + 3328);
                    const unsigned hu = lds_get(ctl + un), h = (hu >> (6 * qq)) & 63u;
                    if (idx < 6 && id < 32 && (id & 3) == 3 && c < 2 * NFWD) {
                        const unsigned pw = pool[lane];
                        if (pend >= 0) staged();
                        const char* dst = myws + par + (size_t)(1408 + ((c >> 1) * 6 + idx) * 8 + xcd) * SLOT + qq * QB;
#pragma unroll
                        for (int i = 0; i < 3; ++i) st16(dst + i * 1024, lo, (u32x4){w[4 * i], w[4 * i + 1], w[4 * i + 2], w[4 * i + 3]});
                        st4(dst + 3072, lane * 4u, w[12]);
                        if (lane < POOL_IN) st4(dst + 3328, lane * 4u, pw);
                        relay_h = hu;
                    }
                    if (h < 32u) {
#pragma unroll
                        for (int k = 0; k < 3; ++k) {
                            unsigned cc = (w[12] << (1 + k)) & 0x80008000u;
#pragma unroll
                            for (int i = 0; i < 4; ++i) cc |= (w[4 * k + i] >> (12 - 3 * i)) & (0x00070007u << (3 * i));
                            w[13 + k] = cc;
                        }
                        unsigned ne[16];
                        u64 str = code_find<10>(w, ne, pool, fq);
                        WAIT_L0();
                        if (lane == 0) __hip_atomic_fetch_add(ctl + 128 + n, 1u, RLX, SC_WG);
                        if (pend >= 0) staged();
#pragma unroll
                        for (int i = 0; i < 16; ++i) w[i] = (w[i] & 0x8fff8fffu) + (code_get(str, ne[i], h * 0x00010001u) << 10);
                    } else {
                        WAIT_L0();
                        if (lane == 0) __hip_atomic_fetch_add(ctl + 128 + n, 1u, RLX, SC_WG);
                        if (pend >= 0) staged();
                        const u32x3 a = ld12(p.ws[org] + par + (size_t)(gid_of(rank, id) * 8 + c) * SLOT + SIDE + qq * 768, lane * 12u);
                        w[13] = a[0]; w[14] = a[1]; w[15] = a[2];
                    }
                }
                const char* sg = myws + WS_STG + stage_off(id, org * 8 + c) + (id < 32 ? (qq >> 1) * 8 + (qq & 1) : qq * 4) * 1024;
#pragma unroll
                for (int i = 0; i < 4; ++i) st16(sg + i * (id < 32 ? 2048 : 1024), lo, (u32x4){w[4 * i], w[4 * i + 1], w[4 * i + 2], w[4 * i + 3]});
                pend = id * 64 + c * 8 + org; pend_n = n; pend_h = relay_h;
                job = ticket();
                if (job >= 4 * nu || job < 4 || (int)lds_get(ctl + 384) < need(job)) staged();
            }
        } else if (wid == 6) {
            const char* exs = p.ws[s] + par;
            unsigned tnext = t_entry + xcd * (TAU / 4);
            u64 todo = ~1ull;
            __builtin_amdgcn_s_setprio(3);
            const char* src = nullptr;
            unsigned trim = 0;
            auto choose = [&](int n) {
                const unsigned rdy = __hip_atomic_load(ctl + (n >= 64 ? n : lane), RLX, SC_WG);
                const unsigned fre = __hip_atomic_load(ctl + (n > RING ? 128 + n - RING : 400), RLX, SC_WG), tr = __hip_atomic_load(ctl + 400, RLX, SC_WG);
                const u64 rd = __ballot(rdy != 0u) & (n >= 64 ? 1ull : todo);
                if (!rd || (n > RING && UNI(fre) < 4u)) return;
                const int un = n >= 64 ? n : __builtin_ctzll(rd);
                todo &= ~(1ull << (un & 63));
                if (lane == 0) __hip_atomic_store(ctl + 416 + n, (unsigned)un, RLX, SC_WG);
                trim = n >= EQ_AT + 4 && n < EQ_AT + 36 ? UNI(tr) : 0u;
                src = exs + (size_t)(desc_of(un) >> 12) * SLOT;
            };
            for (int n = 1; n < nu; ++n) {
                bool drained = false;
                if (!src) {
                    choose(n);
                    if (!src) {
                        wait_vm<0>();
                        if (lane == 0) __hip_atomic_store(ctl + 384, (unsigned)(n - 1) * 14u, RLX, SC_WG);
                        drained = true;
                        for (choose(n); !src; choose(n)) __builtin_amdgcn_s_sleep(1);
                    }
                }
                const char* cur = src;
                const unsigned dst = lds0 + (unsigned)(n % RING) * 16384u, tr = trim;
                src = nullptr;
                asm volatile("" ::: "memory");
#pragma unroll
                for (int i = 0; i < 14; i += 2) {
                    wait_vm<WIN - 2>();
                    if ((i > WIN - 2 || (!drained && n > 1)) && lane == 0) __hip_atomic_store(ctl + 384, (unsigned)((n - 1) * 14 + i - (WIN - 2)), RLX, SC_WG);
                    pace(tnext, (i < 12 ? 2u * TAU : UNIT_T - 12u * TAU) - tr);
                    if (i < 12) dma<true>(cur + i * 1024, lo, dst + i * 1024);
                    else {
                        dma<false>(cur + 12288, lo, dst + 12288);
                        if (lane < (4 * QB - 13312) / 16) dma<false>(cur + 13312, lo, dst + 13312);
                    }
                    if (i == 0 && lane == 0) {
                        if (n == EQ_AT) __hip_atomic_store(ctl + 387, tnext | 1u, RLX, SC_WG);
                        if (n == nu - 3) __hip_atomic_store(ctl + 386, tnext | 1u, RLX, SC_WG);
                        if (n == nu - 1) __hip_atomic_store(ctl + 385, (tnext + UNIT_T - 2u * TAU) | 1u, RLX, SC_WG);
                    }
                    if (i == 10 && n + 1 < nu) choose(n + 1);
                }
            }
            __builtin_amdgcn_s_setprio(0);
            wait_vm<0>();
            if (lane == 0) __hip_atomic_store(ctl + 384, (unsigned)(nu - 1) * 14u, RLX, SC_WG);
            STAMP(9);
        } else {
            const GAS u64* rp[2];
            bool pend[2];
#pragma unroll
            for (int k = 0; k < 2; ++k) {
                rp[k] = (const GAS u64*)(myws + WS_RDY) + ((((call & 1u) * 8 + ((desc[k] >> 9) & 7)) * 48 + ((desc[k] >> 3) & 63)) * 8 + (desc[k] & 7)) * RSTRIDE;
                pend[k] = k * 64 + lane < nu;
            }
            GAS u64* eq = (GAS u64*)(myws + WS_EQ) + idx * 16;
            unsigned mine = 0;
            bool eq_open = EQ_AT != 0;
            const u64 t0 = __builtin_amdgcn_s_memrealtime();
            for (unsigned polls = 1;; ++polls) {
                const u64 pm = __ballot(pend[0]);
                const bool front = __builtin_amdgcn_mbcnt_hi((unsigned)(pm >> 32), __builtin_amdgcn_mbcnt_lo((unsigned)pm, 0u)) < POLLW;
#pragma unroll
                for (int k = 0; k < 2; ++k)
                    if (pend[k] && ((k == 0 && front) || (polls & 7u) == 0u)) {
                        const u64 f = __hip_atomic_load(rp[k], RLX, SC_SYS);
                        if ((unsigned)(f >> 32) >= call) { __hip_atomic_store(ctl + k * 64 + lane, (unsigned)f | 0x80000000u, RLX, SC_WG); pend[k] = false; }
                    }
                if (eq_open) {
                    if (!mine) {
                        mine = lds_get(ctl + 387);
                        if (mine && lane == 0) __hip_atomic_store(eq + xcd, ((u64)call << 32) | mine, RLX, SC_AGENT);
                    } else {
                        const u64 f = __hip_atomic_load(eq + (lane & 7), RLX, SC_AGENT);
                        if (__all((unsigned)(f >> 32) == call)) {
                            int dlt = (int)(mine - (unsigned)f);
                            dlt += __shfl_xor(dlt, 1); dlt += __shfl_xor(dlt, 2); dlt += __shfl_xor(dlt, 4);
                            dlt /= 8 * 32 * 7;
                            dlt = dlt > TAU / 8 ? TAU / 8 : dlt < -(TAU / 8) ? -(TAU / 8) : dlt;
                            if (lane == 0) __hip_atomic_store(ctl + 400, (unsigned)dlt, RLX, SC_WG);
                            eq_open = false;
                        } else if ((int)lds_get(ctl + 384) >= (EQ_AT + 2) * 14) eq_open = false;
                    }
                }
                const bool waiting = __any(pend[0] || pend[1]);
                if (!waiting && !eq_open) break;
                if (waiting && (polls & 15u) == 0u && TIMED_OUT(t0)) {
                    if (lane == 0) __hip_atomic_store(tmo, call, RLX, SC_AGENT);
                    wait_vm<0>();
#pragma unroll
                    for (int k = 0; k < 2; ++k) if (pend[k]) __hip_atomic_store(ctl + k * 64 + lane, 0x80000000u, RLX, SC_WG);
                    break;
                }
                if (polls < 32u) __builtin_amdgcn_s_sleep(1); else __builtin_amdgcn_s_sleep(16);
            }
            if (idx < 6) {
                while ((mine = lds_get(ctl + 386)) == 0u) __builtin_amdgcn_s_sleep(8);
                if (lane == 0) __hip_atomic_store(eq + 8 + xcd, ((u64)call << 32) | mine, RLX, SC_AGENT);
                unsigned th;
                int dlt = 0;
                bool open = EQ2;
                while ((th = lds_get(ctl + 385)) == 0u) {
                    if (open) {
                        const u64 f = __hip_atomic_load(eq + 8 + (lane & 7), RLX, SC_AGENT);
                        if (__all((unsigned)(f >> 32) == call)) {
                            dlt = (int)(mine - (unsigned)f);
                            dlt += __shfl_xor(dlt, 1); dlt += __shfl_xor(dlt, 2); dlt += __shfl_xor(dlt, 4);
                            dlt /= 8 * 256;
                            dlt = dlt > 100 ? 100 : dlt < -100 ? -100 : dlt;
                            open = false;
                        }
                    }
                    __builtin_amdgcn_s_sleep(4);
                }
                if (lane == 0) __hip_atomic_store((GAS u64*)(myws + WS_MBOX + (size_t)((8 + idx * 4) * 8 + xcd) * 128), ((u64)call << 32) | (th & ~255u) | (unsigned)(dlt + 128), RLX, SC_AGENT);
            }
        }
    }
    const unsigned call = UNI(ctr_v) + 1u;

    const int ntile = idx == 7 ? 2 : 1;
    const int wm = wid >> 1, wn = wid & 1;
#pragma unroll 1
    for (int rep = 0; rep < ntile; ++rep) {
    const int rb = late ? 6 + (idx >> 2) : (idx - 8) >> 2;
    const int cb = xcd * 4 + (idx == 7 ? rep : idx == 6 ? 0 : idx == 4 || idx == 5 ? idx - 2 : idx & 3);
    const bool part1 = idx == 7 && rep == 0, part2 = idx == 6;
    const int nstep = part1 ? 56 : part2 ? 8 : NSTEP;
    const char* stgA = myws + WS_STG + (size_t)(rb * NSTEP) * A_STEP + wid * 4096;
    const char* stgB = myws + WS_STG + STG_A + (size_t)(cb * NSTEP) * B_STEP + wid * 2048;
    const u64 pbits = 0x0101010101010101ull << (rank ^ 1);
    u64 todo = part1 ? ~pbits : part2 ? pbits : ~0ull, avail = 0ull;
    int slot = 0;
    auto poll = [&]() {
        if (wid == 0) {
            const u64 t0 = __builtin_amdgcn_s_memrealtime();
            u64 got;
            for (unsigned polls = 1;; ++polls) {
                const unsigned f0 = __hip_atomic_load(gflag + cb * 64 + lane, RLX, SC_AGENT);
                const unsigned f1 = __hip_atomic_load(gflag + (32 + rb * 2) * 64 + lane, RLX, SC_AGENT);
                const unsigned f2 = __hip_atomic_load(gflag + (33 + rb * 2) * 64 + lane, RLX, SC_AGENT);
                got = __ballot(f0 >= call && f1 >= call && f2 >= call) & todo;
                if (got) break;
                if ((polls & 15u) == 0u && TIMED_OUT(t0)) {
                    if (lane == 0) __hip_atomic_store(tmo, call, RLX, SC_AGENT);
                    got = todo;
                    break;
                }
                __builtin_amdgcn_s_sleep(POLL_SLEEP);
            }
            if (lane < 2) ctl[388 + slot * 2 + lane] = (unsigned)(got >> (32 * lane));
        }
    };
    auto news = [&]() { avail = ((u64)(unsigned)UNI(ctl[389 + slot * 2]) << 32) | (unsigned)UNI(ctl[388 + slot * 2]); slot ^= 1; };
    auto take = [&]() {
        const unsigned b = (unsigned)__builtin_ctzll(avail) & 63u;
        avail &= avail - 1; todo &= ~(1ull << b);
        return (b & 7u) * 8u + (b >> 3);
    };
    auto g1 = [&](unsigned ks, unsigned buf, int i) {
        if (i < 4) __builtin_amdgcn_global_load_lds((const GAS unsigned*)(stgA + (size_t)ks * A_STEP + i * 1024 + lo), (LAS unsigned*)(lds + buf + wid * 4096 + i * 1024), 16, 0, 17);
        else __builtin_amdgcn_global_load_lds((const GAS unsigned*)(stgB + (size_t)ks * B_STEP + (i - 4) * 1024 + lo), (LAS unsigned*)(lds + buf + A_STEP + wid * 2048 + (i - 4) * 1024), 16, 0, 17);
    };

    f32x4 acc[4][4];
#pragma unroll
    for (int m = 0; m < 4; ++m)
#pragma unroll
        for (int n = 0; n < 4; ++n) acc[m][n] = (f32x4){0.f, 0.f, 0.f, 0.f};
    h16x8 a0[4], b0[4], a1[4], b1[4];
    const unsigned aoff = wm * 8192 + lo, boff = A_STEP + wn * 8192 + lo;
#define RDA(A, buf, kk, m) A[m] = *(const LAS h16x8*)(lds + (buf) + aoff + (m) * 2048 + (kk) * 1024)
#define RDB(B, buf, kk, n) B[n] = *(const LAS h16x8*)(lds + (buf) + boff + (n) * 2048 + (kk) * 1024)
#define ROW(A, B, m) _Pragma("unroll") for (int n = 0; n < 4; ++n) acc[m][n] = __builtin_amdgcn_mfma_f32_16x16x32_f16(A[m], B[n], acc[m][n], 0, 0, 0)

    STAMP(10);
    unsigned kq;
#pragma unroll
    for (int k = 0; k < 3; ++k) {
        if (!avail) { poll(); WAIT_L0(); BAR(); news(); }
        kq = take();
#pragma unroll
        for (int i = 0; i < (k < 2 ? 6 : 3); ++i) g1(kq, k * STAGE, i);
    }
    wait_vm<9>(); BAR();
#pragma unroll
    for (int i = 0; i < 4; ++i) { RDB(b0, 0u, 0, i); RDA(a0, 0u, 0, i); }
    unsigned cur = 0, prv = 2 * STAGE;
    for (int t = 0; t < nstep; ++t) {
        const unsigned nxt = cur == 2 * STAGE ? 0u : cur + STAGE;
        const bool m2 = t + 2 < nstep, m3 = t + 3 < nstep;
        const bool dry = m3 && !avail;
        WAIT_L0(); SB();
        RDB(b1, cur, 1, 0); RDB(b1, cur, 1, 1); SB(); ROW(a0, b0, 0); SB(); if (m2) g1(kq, prv, 3); SB();
        RDB(b1, cur, 1, 2); RDB(b1, cur, 1, 3); SB(); ROW(a0, b0, 1); SB(); if (m2) g1(kq, prv, 4); SB();
        RDA(a1, cur, 1, 0); RDA(a1, cur, 1, 1); SB(); ROW(a0, b0, 2); SB(); if (m2) g1(kq, prv, 5); SB();
        RDA(a1, cur, 1, 2); RDA(a1, cur, 1, 3); SB(); ROW(a0, b0, 3); SB();
        if (dry) poll();
        WAIT_L0();
        if (m2) wait_vm<6>(); else wait_vm<0>();
        BAR(); SB();
        RDB(b0, nxt, 0, 0); RDB(b0, nxt, 0, 1); SB(); ROW(a1, b1, 0); SB();
        if (dry) news();
        if (m3) { kq = take(); g1(kq, cur, 0); } SB();
        RDB(b0, nxt, 0, 2); RDB(b0, nxt, 0, 3); SB(); ROW(a1, b1, 1); SB(); if (m3) g1(kq, cur, 1); SB();
        RDA(a0, nxt, 0, 0); RDA(a0, nxt, 0, 1); SB(); ROW(a1, b1, 2); SB(); if (m3) g1(kq, cur, 2); SB();
        RDA(a0, nxt, 0, 2); RDA(a0, nxt, 0, 3); SB(); ROW(a1, b1, 3); SB();
        prv = cur; cur = nxt;
    }

    if (part1 || part2) {
        const char* park = myws + WS_EXO + (size_t)(xcd * 8 + wid) * 16384;
        GAS u64* pf = (GAS u64*)(myws + WS_ORDY + (size_t)xcd * 128) + wid;
        if (part1) {
#pragma unroll
            for (int i = 0; i < 16; ++i) st16(park + i * 1024, lo, __builtin_bit_cast(u32x4, acc[i >> 2][i & 3]));
            wait_vm<0>();
            if (lane == 0) __hip_atomic_store(pf, (u64)call << 32, RLX, SC_SYS);
            continue;
        }
        const u64 t0 = __builtin_amdgcn_s_memrealtime();
        while ((unsigned)UNI((unsigned)(__hip_atomic_load(pf, RLX, SC_SYS) >> 32)) < call) {
            if (TIMED_OUT(t0)) { if (lane == 0) __hip_atomic_store(tmo, call, RLX, SC_AGENT); wait_vm<0>(); break; }
            __builtin_amdgcn_s_sleep(4);
        }
#pragma unroll
        for (int m = 0; m < 4; ++m) {
            u32x4 e0, e1, e2, e3;
            ld64(park + m * 4096, lo, e0, e1, e2, e3);
            acc[m][0] += __builtin_bit_cast(f32x4, e0); acc[m][1] += __builtin_bit_cast(f32x4, e1);
            acc[m][2] += __builtin_bit_cast(f32x4, e2); acc[m][3] += __builtin_bit_cast(f32x4, e3);
        }
    }
    const unsigned tmo_v = __hip_atomic_load(tmo, RLX, SC_AGENT);
    const unsigned oo = (unsigned)(fq * 4) * (GN * 4u) + fr * 16u;
    const u32x4 nan4 = {0x7fc00000u, 0x7fc00000u, 0x7fc00000u, 0x7fc00000u};
    auto finish = [&](unsigned (&o)[32], int i, float a, float b) {
        const f32x2 g = gelu2((f32x2){a, b});
        const int at = ((i >> 3) * 4 + (i & 1) * 2) * 4 + ((i >> 1) & 3);
        o[at] = __float_as_uint(g.x); o[at + 4] = __float_as_uint(g.y);
    };
#define ACC(hf, i, k) acc[2 * (hf) + ((i) >> 3)][((i) >> 1) & 3][((i) & 1) * 2 + (k)]
    if (late) {
        const bool bad = UNI(tmo_v) == call;
        const char* ob = (const char*)(p.out + (size_t)(rb_local(rank, rb) + wm * 64) * GN + (rank >> 1) * 4096 + cb * 128 + wn * 64);
#pragma unroll
        for (int hf = 0; hf < 2; ++hf) {
            unsigned o[32];
#pragma unroll
            for (int i = 0; i < 16; ++i) finish(o, i, ACC(hf, i, 0), ACC(hf, i, 1));
#pragma unroll
            for (int i = 0; i < 8; ++i) st16(ob + (size_t)(hf * 32 + (i >> 2) * 16 + (i & 3)) * (GN * 4), oo, bad ? nan4 : (u32x4){o[4 * i], o[4 * i + 1], o[4 * i + 2], o[4 * i + 3]});
        }
    } else {
        const int peer = rb_owner(rank, rb);
        const int mirror = (8 + rb_of(peer, rank) * 4 + (idx & 3)) * 8 + xcd;
        char* const peerws = p.ws[peer];
        WAIT_L0(); BAR();
        if (wid < 4) __builtin_amdgcn_s_setprio(2);
#pragma unroll
        for (int hf = 0; hf < 2; ++hf) {
            const int k = (wid >> 2) * 8 + hf * 4 + q;
            const char* ex = myws + WS_EXO + (size_t)(bid * 16 + k) * 8192;
            unsigned hdr = 1u << 13;
            if ((idx & 3) != 3 || k < 16 - NPLAIN) {
                LAS unsigned* pool = (LAS unsigned*)(lds + 131072 + wid * 256);
                pool[lane] = 0u;
                unsigned hi[16], ef[16], under = 0;
#pragma unroll
                for (int i = 0; i < 16; ++i) {
                    hi[i] = __builtin_amdgcn_perm(__float_as_uint(ACC(hf, i, 1)), __float_as_uint(ACC(hf, i, 0)), 0x07060302u);
                    ef[i] = (hi[i] >> 7) & 0x00ff00ffu;
                }
                const unsigned bs = (unsigned)code_base<8>(ef), e0 = bs > 20u ? bs - 20u : 0u;
#pragma unroll
                for (int i = 0; i < 16; ++i) { under |= SUBS(e0 * 0x00010001u, ef[i]); ef[i] = SUBS(ef[i], e0 * 0x00010001u); }
                if (__all(under == 0u) && code_put<POOL_OUT>(ef, bs - e0, pool, fq)) {
#pragma unroll
                    for (int i = 0; i < 16; ++i) hi[i] = (hi[i] & 0x807f807fu) | (ef[i] << 7);
#pragma unroll
                    for (int c = 0; c < 6; ++c) {
#pragma unroll
                        for (int i = 0; i < 3; ++i) { const int fld = c * 3 + i, at = 9 + 3 * (fld & 1); hi[fld >> 1] |= (hi[10 + c] << (at - 3 * i)) & (0x00070007u << at); }
                        hi[9] |= (hi[10 + c] >> (1 + c)) & (0x40004000u >> c);
                    }
                    const unsigned pw = __hip_atomic_load(pool + lane, RLX, SC_WG);
#pragma unroll
                    for (int i = 0; i < 4; ++i) {
                        u32x4 v;
#pragma unroll
                        for (int e = 0; e < 4; ++e) v[e] = __builtin_amdgcn_perm(__float_as_uint(ACC(hf, 4 * i + e, 1)), __float_as_uint(ACC(hf, 4 * i + e, 0)), 0x05040100u);
                        st16(ex + i * 1024, lo, v);
                    }
                    st16(ex + 4096, lo, (u32x4){hi[0], hi[1], hi[2], hi[3]});
                    st16(ex + 5120, lo, (u32x4){hi[4], hi[5], hi[6], hi[7]});
                    st8(ex + 6144, lane * 8u, (u32x2){hi[8], hi[9]});
                    if (lane < POOL_OUT) st4(ex + 6656, lane * 4u, pw);
                    hdr = e0 | ((bs - e0) << 8);
                }
            }
            if (hdr >> 13) {
                unsigned o[32];
#pragma unroll
                for (int i = 0; i < 16; ++i) finish(o, i, ACC(hf, i, 0), ACC(hf, i, 1));
#pragma unroll
                for (int i = 0; i < 8; ++i) st16(ex + i * 1024, lo, (u32x4){o[4 * i], o[4 * i + 1], o[4 * i + 2], o[4 * i + 3]});
            }
            wait_vm<0>();
            hdr |= UNI(tmo_v) == call ? 0xc0000000u : 0x40000000u;
            if (lane == 0) __hip_atomic_store((GAS u64*)(peerws + WS_ORDY + (size_t)mirror * 128) + k, ((u64)call << 32) | hdr, RLX, SC_SYS);
        }
        __builtin_amdgcn_s_setprio(0);
        STAMP(22);
#pragma unroll 1
        for (int sk = 0; (idx & 3) + sk <= 3 && (idx & 3) != 3; sk += 3) {
        if (sk) {
            BAR();
            if (tid < 32 || tid == 384 || tid == 401 || (tid >= 416 && tid < 432)) ctl[tid] = 0u;
            WAIT_L0(); BAR();
        }
        const int me = bid + 8 * sk;
        const char* exs = peerws + WS_EXO + (size_t)(mirror + 8 * sk) * (16 * 8192);
        if (wid == 0) {
            unsigned tnext;
            while ((tnext = lds_get(ctl + 16)) == 0u) __builtin_amdgcn_s_sleep(2);
            STAMP(23);
            const unsigned trb = tnext & 255u, tr = trb - 128u;
            tnext = (tnext & ~255u) | 128u;
            __builtin_amdgcn_s_setprio(3);
            wait_vm<0>(); __builtin_amdgcn_s_waitcnt(0x0F70);
            int issued = 0, told = 0;
            unsigned hn = lds_get(ctl);
            for (int n = 0; n < 16; ++n) {
                if (hn == 0u && (hn = lds_get(ctl + n)) == 0u) {
                    wait_vm<0>();
                    told = issued;
                    if (lane == 0) __hip_atomic_store(ctl + 384, (unsigned)told, RLX, SC_WG);
                    while ((hn = lds_get(ctl + n)) == 0u) __builtin_amdgcn_s_sleep(1);
                }
                asm volatile("" ::: "memory");
                const bool plain = (hn >> 13) & 1u;
                const char* cur = exs + (size_t)n * 8192;
                const unsigned dst = lds0 + (unsigned)n * 8192u;
#pragma unroll
                for (int i = 0; i < 8; i += 2) {
                    wait_vm<WIN_OUT - 2>();
                    if (issued - (WIN_OUT - 2) > told) { told = issued - (WIN_OUT - 2); if (lane == 0) __hip_atomic_store(ctl + 384, (unsigned)told, RLX, SC_WG); }
                    pace(tnext, (i < 6 || plain ? 2u * TAU_OUT : PIECE_T - 6u * TAU_OUT) - tr);
                    if (i < 6 || plain) { dma<true>(cur + i * 1024, lo, dst + i * 1024); issued += 2; }
                    else { if (lane < (PB - 6144) / 16) dma<false>(cur + 6144, lo, dst + 6144); issued += 1; }
                    if (i == 0) {
                        if (lane == 0) __hip_atomic_store(ctl + 416 + n, (unsigned)issued + (plain ? 6u : 5u), RLX, SC_WG);
                        if (n == 14 && !sk && lane == 0)
                            __hip_atomic_store((GAS u64*)(myws + WS_MBOX + (size_t)(bid + 8) * 128), ((u64)call << 32) | ((tnext + 2u * (PIECE_T) - 2u * TAU_OUT - 7u * tr) & ~255u) | trb, RLX, SC_AGENT);
                    }
                    if (i == 2) hn = n < 15 ? lds_get(ctl + n + 1) : 0u;
                }
            }
            wait_vm<7>();
            if (issued - 7 > told && lane == 0) __hip_atomic_store(ctl + 384, (unsigned)issued - 7u, RLX, SC_WG);
            wait_vm<0>();
            if (lane == 0) __hip_atomic_store(ctl + 384, (unsigned)issued, RLX, SC_WG);
            __builtin_amdgcn_s_setprio(0);
            STAMP(19);
        } else {
            if (wid == 1) {
                const GAS u64* fp = lane < 16 ? (const GAS u64*)(myws + WS_ORDY + (size_t)me * 128) + lane : (const GAS u64*)(myws + WS_MBOX + (size_t)me * 128);
                bool pend = lane < 17;
                const u64 t0 = __builtin_amdgcn_s_memrealtime();
                for (unsigned polls = 1;; ++polls) {
                    if (pend) {
                        const u64 f = __hip_atomic_load(fp, RLX, SC_SYS);
                        if ((unsigned)(f >> 32) >= call) { __hip_atomic_store(ctl + lane, (unsigned)f, RLX, SC_WG); pend = false; }
                    }
                    if (!__any(pend)) break;
                    if ((polls & 15u) == 0u && TIMED_OUT(t0)) {
                        if (lane == 0) __hip_atomic_store(tmo, call, RLX, SC_AGENT);
                        if (pend) __hip_atomic_store(ctl + lane, lane < 16 ? 0xc0002000u : (now8() & ~255u) | 128u, RLX, SC_WG);
                        break;
                    }
                    __builtin_amdgcn_s_sleep(4);
                }
            }
            for (;;) {
                unsigned tk = 0;
                if (lane == 0) tk = __hip_atomic_fetch_add(ctl + 401, 1u, RLX, SC_WG);
                const int k = UNI(tk);
                if (k >= 16) break;
                unsigned need;
                while ((need = lds_get(ctl + 416 + k)) == 0u) __builtin_amdgcn_s_sleep(4);
                const unsigned hdr = lds_get(ctl + k);
                while (lds_get(ctl + 384) < need) __builtin_amdgcn_s_sleep(1);
                asm volatile("" ::: "memory");
                const LAS unsigned char* src = lds + (unsigned)k * 8192u;
                unsigned o[32];
                if ((hdr >> 13) & 1u) {
#pragma unroll
                    for (int i = 0; i < 8; ++i) { const u32x4 a = *(const LAS u32x4*)(src + i * 1024 + lo); o[4 * i] = a[0]; o[4 * i + 1] = a[1]; o[4 * i + 2] = a[2]; o[4 * i + 3] = a[3]; }
                } else {
                    unsigned lw[16], hi[16], ne[16];
#pragma unroll
                    for (int i = 0; i < 2; ++i) { const u32x4 a = *(const LAS u32x4*)(src + 4096 + i * 1024 + lo); hi[4 * i] = a[0]; hi[4 * i + 1] = a[1]; hi[4 * i + 2] = a[2]; hi[4 * i + 3] = a[3]; }
                    { const u32x2 a = *(const LAS u32x2*)(src + 6144 + lane * 8); hi[8] = a[0]; hi[9] = a[1]; }
#pragma unroll
                    for (int i = 0; i < 4; ++i) { const u32x4 a = *(const LAS u32x4*)(src + i * 1024 + lo); lw[4 * i] = a[0]; lw[4 * i + 1] = a[1]; lw[4 * i + 2] = a[2]; lw[4 * i + 3] = a[3]; }
#pragma unroll
                    for (int c = 0; c < 6; ++c) {
                        unsigned cc = (hi[9] << (1 + c)) & 0x80008000u;
#pragma unroll
                        for (int i = 0; i < 3; ++i) { const int fld = c * 3 + i, at = 9 + 3 * (fld & 1); cc |= (hi[fld >> 1] >> (at - 3 * i)) & (0x00070007u << (3 * i)); }
                        hi[10 + c] = cc;
                    }
                    u64 str = code_find<7>(hi, ne, (const LAS unsigned*)(src + 6656), fq);
                    const unsigned e0 = (hdr & 255u) * 0x00800080u, b2 = ((hdr >> 8) & 31u) * 0x00010001u;
#pragma unroll
                    for (int i = 0; i < 16; ++i) {
                        const unsigned h2 = (hi[i] & 0x81ff81ffu) + e0 + (code_get(str, ne[i], b2) << 7);
                        finish(o, i, __uint_as_float(__builtin_amdgcn_perm(h2, lw[i], 0x05040100u)), __uint_as_float(__builtin_amdgcn_perm(h2, lw[i], 0x07060302u)));
                    }
                }
                const int blk = (k >> 3) * 4 + (k & 3);
                const char* ob = (const char*)(p.out + (size_t)((peer & 1) * 256 + (blk >> 1) * 64 + ((k >> 2) & 1) * 32) * GN + (peer >> 1) * 4096 + (cb + sk) * 128 + (blk & 1) * 64);
#pragma unroll
                for (int i = 0; i < 8; ++i)
                    st16(ob + (size_t)((i >> 2) * 16 + (i & 3)) * (GN * 4), oo, (hdr >> 31) ? nan4 : (u32x4){o[4 * i], o[4 * i + 1], o[4 * i + 2], o[4 * i + 3]});
            }
        }
        }
    }
    }
    if (tid == 0) __hip_atomic_store(ctr, call, RLX, SC_AGENT);
}

extern "C" void kernel_launch(int rank, int nranks, ncclComm_t comm, hipStream_t stream,
                              void* const* d_in, const int* in_sizes, int n_in,
                              void* d_out, int out_size, const int* in_dtypes, int out_dtype,
                              void* const* d_out_all, void* const* d_ws_all, size_t ws_bytes) {
    (void)comm; (void)in_dtypes; (void)out_dtype; (void)d_out_all;
    if (nranks != NR || (unsigned)rank >= (unsigned)NR || n_in != 2 || ws_bytes < WS_NEED ||
        in_sizes[0] != 4096 * KSH || in_sizes[1] != KSH * GN || out_size != 512 * GN) {
        fprintf(stderr, "kernel_launch: unexpected problem shape\n");
        return;
    }
    Params p;
    p.x = (const float*)d_in[0];
    p.w = (const float*)d_in[1];
    p.out = (float*)d_out;
    p.myws = (char*)d_ws_all[rank];
    for (int i = 0; i < NR; ++i) p.ws[i] = (char*)d_ws_all[i];
    p.rank = rank;
    p.pad = 0;
    int n = 0, np = 0, nd = 0;
    unsigned char deferred[8];
    bool px[8] = {};
    for (int i = 0; i < 128; ++i) p.order[0][i] = p.order[1][i] = 0;
    auto need_x = [&](int c0, int c1) { for (int c = c0; c <= c1; ++c) if (!px[c]) { px[c] = true; p.prod[np++] = (unsigned)(32 + c); } };
    for (int key = 0; key < 16 + 3 * SKEW2; ++key) {
        for (int pass = 0; pass < 2; ++pass) {
            if ((pass == 0) == (key != 0)) {
                for (int e = 32; e < 44; ++e)
                    if (2 * ((e - 32) * 8 / 12) == key) { p.order[0][n++] = (unsigned char)e; need_x((e - 32) * 8 / 12, ((e - 32) * 8 + 7) / 12); }
                if (key == XPOS) { p.order[0][n++] = 44; need_x(0, 3); }
                if (key == XPOS + 2) { p.order[0][n++] = 45; need_x(4, 7); }
                if (key == LPOS) { p.order[0][n++] = 46; p.order[0][n++] = 47; }
            } else {
                for (int b = 0; b < 4; ++b)
                    for (int c = 0; c < 8; ++c)
                        if ((b == 0 && c == 1 ? 0 : 2 * c + b * SKEW2) == key) {
                            p.prod[np++] = (unsigned)(c * 4 + b);
                            if (b == 3 && c < 2 * NFWD && ((c ^ rank) & 1)) deferred[nd++] = (unsigned char)(c * 4 + b);
                            else p.order[0][n++] = (unsigned char)(c * 4 + b);
                        }
            }
        }
    }
    int m = 0;
    for (int i = 0; i < n; ++i) if (p.order[0][i] < 44) p.order[1][m++] = p.order[0][i];
    for (int i = 0; i < nd; ++i) p.order[1][m++] = deferred[i];
    for (int i = 0; i < 6 * NFWD; ++i) p.order[1][m++] = (unsigned char)(48 + i);
    for (int e = 44; e < 48; ++e) p.order[1][m++] = (unsigned char)e;
    if (n != NU_OUT || m != NU_PAIR || np != 40 || p.prod[0] != 0 || p.prod[1] != 4 || p.prod[2] != 32) { fprintf(stderr, "kernel_launch: bad schedule\n"); return; }
    hipLaunchKernelGGL(fused_gemm_rs, dim3(NBLK), dim3(NTHR), 0, stream, p);
    HIP_CHECK(hipGetLastError());
}
```

```cpp
#include <hip/hip_runtime.h>
#include <rccl/rccl.h>
#include <cstdio>

#define HIP_CHECK(expr) do { hipError_t _e = (expr); if (_e != hipSuccess) { \
    fprintf(stderr, "HIP error %s at %s:%d\n", hipGetErrorString(_e), __FILE__, __LINE__); \
    return; } } while (0)

typedef float f32x4 __attribute__((ext_vector_type(4)));
typedef unsigned u32x4 __attribute__((ext_vector_type(4)));
typedef unsigned u32x3 __attribute__((ext_vector_type(3)));
typedef unsigned u32x2 __attribute__((ext_vector_type(2)));
typedef unsigned short u16x2 __attribute__((ext_vector_type(2)));
typedef _Float16 h16x8 __attribute__((ext_vector_type(8)));
typedef unsigned long long u64;
#define LAS __attribute__((address_space(3)))
#define GAS __attribute__((address_space(1)))
#define RLX __ATOMIC_RELAXED
#define SC_WG __HIP_MEMORY_SCOPE_WORKGROUP
#define SC_AGENT __HIP_MEMORY_SCOPE_AGENT
#define SC_SYS __HIP_MEMORY_SCOPE_SYSTEM

constexpr int TAU = 864;
constexpr int WIN = 10;
constexpr int CREDIT = 36;
constexpr int EQ_AT = 5;
constexpr int PUSH_SLOT = 112;
constexpr int PROD_GAP = 40;
constexpr int NFWD = 3;
constexpr int NPLAIN = 3;
constexpr int SKEW2 = 2;
constexpr int XPOS = 6, LPOS = 18;
constexpr int RSTRIDE = 16;
constexpr int POLLW = 12;

constexpr int NR = 8;
constexpr int GN = 16384;
constexpr int KSH = 512;
constexpr int NBLK = 256;
constexpr int NTHR = 512;
constexpr int NSTEP = 64;
constexpr unsigned A_STEP = 32768, B_STEP = 16384, STAGE = A_STEP + B_STEP;
constexpr int NU_OUT = 48 - NFWD, NU_PAIR = 48 + 6 * NFWD;
constexpr int RING = 10;
constexpr int PW = 14;
constexpr int POOL_IN = 48, POOL_OUT = 60;
constexpr int QB = 3328 + 4 * POOL_IN;
constexpr int PB = 6656 + 4 * POOL_OUT;
constexpr unsigned UNIT_T = (4 * QB + 127) / 128 * TAU / 8, PIECE_T = (PB + 127) / 128 * TAU / 8;
constexpr unsigned SLOT = 18432, SIDE = 14336;
constexpr size_t WS_CTR = 0;
constexpr size_t WS_FLAG = 32 << 10;
constexpr size_t WS_ORDY = 96 << 10;
constexpr size_t WS_MBOX = 128 << 10;
constexpr size_t WS_EQ = 160 << 10;
constexpr size_t WS_TMO = 192 << 10;
constexpr size_t WS_STG = 1 << 20;
constexpr size_t STG_A = 8ull * NSTEP * A_STEP;
constexpr size_t STG_B = 32ull * NSTEP * B_STEP;
constexpr size_t WS_EXI = WS_STG + STG_A + STG_B;
constexpr size_t EXI_BYTES = 1600ull * SLOT;
constexpr size_t WS_EXO = WS_EXI + 2 * EXI_BYTES;
constexpr size_t WS_RDY = WS_EXO + (size_t)NBLK * 8 * 16384;
constexpr size_t WS_NEED = WS_RDY + 2 * 8 * 48 * 8 * 128;
constexpr u64 DEADLINE_TICKS = 50000000ull;

struct Params {
    const float* x;
    const float* w;
    float* out;
    char* myws;
    char* ws[NR];
    int rank;
    int pad;
    unsigned prod[40];
    unsigned char order[2][128];
};

#define WAIT_L0() do { asm volatile("s_waitcnt lgkmcnt(0)" ::: "memory"); __builtin_amdgcn_s_waitcnt(0xC07F); } while (0)
#define BAR() do { asm volatile("" ::: "memory"); __builtin_amdgcn_s_barrier(); asm volatile("" ::: "memory"); } while (0)
#define SB() __builtin_amdgcn_sched_barrier(0)
#define UNI(x) __builtin_amdgcn_readfirstlane(x)
template <int N> __device__ __forceinline__ void wait_vm() { asm volatile("s_waitcnt vmcnt(%0)" ::"i"(N) : "memory"); }

__device__ __forceinline__ void st16(const void* base, unsigned voff, u32x4 v) {
    asm volatile("s_nop 4\n\tglobal_store_dwordx4 %0, %1, %2 sc0 sc1\n\ts_nop 1" ::"v"(voff), "v"(v), "s"(base) : "memory");
}
__device__ __forceinline__ void st12(const void* base, unsigned voff, u32x3 v) {
    asm volatile("s_nop 4\n\tglobal_store_dwordx3 %0, %1, %2 sc0 sc1\n\ts_nop 1" ::"v"(voff), "v"(v), "s"(base) : "memory");
}
__device__ __forceinline__ void st8(const void* base, unsigned voff, u32x2 v) {
    asm volatile("s_nop 4\n\tglobal_store_dwordx2 %0, %1, %2 sc0 sc1" ::"v"(voff), "v"(v), "s"(base) : "memory");
}
__device__ __forceinline__ void st4(const void* base, unsigned voff, unsigned v) {
    asm volatile("s_nop 4\n\tglobal_store_dword %0, %1, %2 sc0 sc1" ::"v"(voff), "v"(v), "s"(base) : "memory");
}
__device__ __forceinline__ u32x3 ld12(const void* base, unsigned voff) {
    u32x3 r;
    asm volatile("s_nop 4\n\tglobal_load_dwordx3 %0, %1, %2 sc0 sc1\n\ts_waitcnt vmcnt(0)" : "=&v"(r) : "v"(voff), "s"(base) : "memory");
    return r;
}
__device__ __forceinline__ void ld64(const void* base, unsigned voff, u32x4& a, u32x4& b, u32x4& c, u32x4& d) {
    asm volatile("s_nop 4\n\tglobal_load_dwordx4 %0, %4, %5 sc0 sc1\n\tglobal_load_dwordx4 %1, %4, %5 offset:1024 sc0 sc1\n\t"
                 "global_load_dwordx4 %2, %4, %5 offset:2048 sc0 sc1\n\tglobal_load_dwordx4 %3, %4, %5 offset:3072 sc0 sc1\n\ts_waitcnt vmcnt(0)"
                 : "=&v"(a), "=&v"(b), "=&v"(c), "=&v"(d) : "v"(voff), "s"(base) : "memory");
}
template <bool TWO> __device__ __forceinline__ void dma(const void* base, unsigned voff, unsigned lds_dst) {
    unsigned keep;
    if (TWO) asm volatile("s_mov_b32 %0, m0\n\ts_mov_b32 m0, %3\n\ts_nop 4\n\tglobal_load_lds_dwordx4 %1, %2 sc0 sc1\n\tglobal_load_lds_dwordx4 %1, %2 offset:1024 sc0 sc1\n\ts_mov_b32 m0, %0"
                          : "=&s"(keep) : "v"(voff), "s"(base), "s"(lds_dst) : "memory");
    else asm volatile("s_mov_b32 %0, m0\n\ts_mov_b32 m0, %3\n\ts_nop 4\n\tglobal_load_lds_dwordx4 %1, %2 sc0 sc1\n\ts_mov_b32 m0, %0"
                      : "=&s"(keep) : "v"(voff), "s"(base), "s"(lds_dst) : "memory");
}
__device__ __forceinline__ unsigned now64() { return (unsigned)__builtin_amdgcn_s_memrealtime() << 6; }
__device__ __forceinline__ void pace(unsigned& t, unsigned period) {
    int late = (int)(now64() - t);
    if (late < -(1 << 22)) { t += (unsigned)late; late = 0; }
    while (late < 0) { __builtin_amdgcn_s_sleep(1); late = (int)(now64() - t); }
    t += period + (late > CREDIT * 64 ? (unsigned)(late - CREDIT * 64) : 0u);
}
__device__ __forceinline__ unsigned lds_get(const LAS unsigned* a) { return UNI(__hip_atomic_load(a, RLX, SC_WG)); }
__device__ __forceinline__ unsigned cvt2(float a, float b) {
    typedef _Float16 h2 __attribute__((ext_vector_type(2)));
    h2 r = {(_Float16)a, (_Float16)b};
    return __builtin_bit_cast(unsigned, r);
}
__device__ __forceinline__ int rb_owner(int d, int rb) { return rb >= 6 ? d : ((((d >> 1) + 1 + (rb >> 1)) & 3) << 1) | (rb & 1); }
__device__ __forceinline__ int rb_local(int d, int rb) { return rb >= 6 ? (rb - 6) * 256 : (d & 1) * 256; }
__device__ __forceinline__ int rb_of(int d, int o) { return ((((o >> 1) - (d >> 1) - 1) & 3) << 1) | (o & 1); }
__device__ __forceinline__ void unit_of(int e, int j, int& c, int& id) {
    if (e < 32) { c = e >> 2; id = j * 4 + (e & 3); }
    else if (e < 44) { const int u = (e - 32) * 8 + j; c = u / 12; id = 32 + u - c * 12; }
    else { const int u = (e & 1) * 8 + j; c = u >> 1; id = (e < 46 ? 46 : 44) + (u & 1); }
}
__device__ __forceinline__ int gid_of(int d, int id) {
    const int rb = (id - 32) >> 1;
    return id < 32 ? (d >> 1) * 32 + id : 128 + rb_owner(d, rb) * 4 + (rb_local(d, rb) >> 7) + (id & 1);
}
__device__ __forceinline__ size_t stage_off(int id, int ks) {
    return id < 32 ? STG_A + (size_t)(id * NSTEP + ks) * B_STEP : (size_t)(((id - 32) >> 1) * NSTEP + ks) * A_STEP + (id & 1) * 16384;
}
template <int BITS, int SH> __device__ __forceinline__ unsigned wave_max_exp(unsigned mag) {
    unsigned e = 0;
#pragma unroll
    for (int b = BITS - 1; b >= 0; --b) if (__any(mag >= ((e + (1u << b)) << SH))) e += 1u << b;
    return e;
}

__device__ __forceinline__ unsigned row_scan(unsigned x) {
    x += __builtin_amdgcn_update_dpp(0, x, 0x111, 0xf, 0xf, true);
    x += __builtin_amdgcn_update_dpp(0, x, 0x112, 0xf, 0xf, true);
    x += __builtin_amdgcn_update_dpp(0, x, 0x114, 0xf, 0xf, true);
    x += __builtin_amdgcn_update_dpp(0, x, 0x118, 0xf, 0xf, true);
    return x;
}
__device__ __forceinline__ unsigned ffbl(unsigned x) { unsigned r; asm("v_ffbl_b32 %0, %1" : "=v"(r) : "v"(x)); return r; }
#define PK(x) __builtin_bit_cast(u16x2, (unsigned)(x))
#define DW(x) __builtin_bit_cast(unsigned, (u16x2)(x))
#define SUBS(a, b) DW(__builtin_elementwise_sub_sat(PK(a), PK(b)))
#define MIN2(a, b) DW(__builtin_elementwise_min(PK(a), PK(b)))

template <int BITS> __device__ __forceinline__ int code_base(const unsigned (&ef)[16]) {
    u16x2 mx = PK(ef[0]);
#pragma unroll
    for (int i = 1; i < 16; ++i) mx = __builtin_elementwise_max(mx, PK(ef[i]));
    const int emax = (int)wave_max_exp<BITS, 0>(mx.x > mx.y ? mx.x : mx.y);
    int up = 0, mid = 0;
#pragma unroll
    for (int i = 0; i < 6; ++i) {
        const int e = (int)(ef[i] >> 16);
        up += __builtin_popcountll(__ballot(e == emax)) - __builtin_popcountll(__ballot(e == emax - 3));
        mid += __builtin_popcountll(__ballot(e == emax - 1)) - __builtin_popcountll(__ballot(e == emax - 4));
    }
    const int bs = emax - (up > 0 ? 3 : mid > 0 ? 4 : 5);
    return bs > 0 ? bs : 0;
}
template <int POOL> __device__ __forceinline__ bool code_put(unsigned (&ef)[16], unsigned B, LAS unsigned* pool, int row) {
    const unsigned b2 = B * 0x00010001u, lut = 0x0300u | ((B + 2u + (B ? 1u : 0u)) << 16);
    u64 str = 0;
    unsigned sh = 0;
#pragma unroll
    for (int i = 0; i < 16; ++i) {
        const unsigned k1 = SUBS(b2 + 0x00010001u, ef[i]), d = SUBS(ef[i], b2);
        const unsigned len = k1 + MIN2(SUBS(k1, 0x00020002u), 0x00010001u) + __builtin_amdgcn_perm(0u, lut, SUBS(d, 0x00030003u) | 0x0c000c00u);
        sh += len & 0xffffu; str |= 1ull << (sh & 63u);
        sh += len >> 16; str |= 1ull << (sh & 63u);
        ef[i] = MIN2(d, 0x00040004u) & 0x00030003u;
    }
    const unsigned end = row_scan(sh);
    const unsigned r0 = __builtin_amdgcn_readlane(end, 15), r1 = __builtin_amdgcn_readlane(end, 31), r2 = __builtin_amdgcn_readlane(end, 47);
    if (!__all(sh <= 63u && end <= 32u * (PW - 1)) || r0 + r1 + r2 + __builtin_amdgcn_readlane(end, 63) > 32u * (POOL - 1)) return false;
    const unsigned pos = 32u + end - sh + (row > 0 ? r0 : 0u) + (row > 1 ? r1 : 0u) + (row > 2 ? r2 : 0u);
    str >>= 1;
    const u64 s0 = str << (pos & 31u);
    LAS unsigned* pp = pool + (pos >> 5);
    __hip_atomic_store(pool, r0 | r1 << 10 | r2 << 20, RLX, SC_WG);
    __hip_atomic_fetch_or(pp, (unsigned)s0, RLX, SC_WG);
    __hip_atomic_fetch_or(pp + 1, (unsigned)(s0 >> 32), RLX, SC_WG);
    __hip_atomic_fetch_or(pp + 2, (unsigned)(((str >> 32) << (pos & 31u)) >> 32), RLX, SC_WG);
    return true;
}
template <int SH> __device__ __forceinline__ u64 code_find(const unsigned (&w)[16], unsigned (&ne)[16], const LAS unsigned* pool, int row) {
    const unsigned rl = pool[0];
    const unsigned start = 32u + (row > 0 ? rl & 1023u : 0u) + (row > 1 ? (rl >> 10) & 1023u : 0u) + (row > 2 ? (rl >> 20) & 1023u : 0u);
    pool += start >> 5;
    unsigned pd[PW - 1], sum = 0;
#pragma unroll
    for (int i = 0; i < PW - 1; ++i) pd[i] = pool[i];
#pragma unroll
    for (int i = 0; i < 16; ++i) { ne[i] = MIN2(w[i] & (0x00030003u << SH), 0x00010001u); sum += ne[i]; }
    sum = 32u - (sum & 0xffffu) - (sum >> 16);
    const unsigned skip = row_scan(sum) - sum + __builtin_popcount(__builtin_amdgcn_ubfe(pd[0], 0u, start));
    unsigned cum = 0, at = 0, below = 0;
#pragma unroll
    for (int i = 0; i < PW - 1; ++i) {
        cum += __builtin_popcount(pd[i]);
        const bool lt = cum < skip;
        at += lt; below = lt ? cum : below;
    }
    const unsigned d0 = pool[at], d1 = pool[at + 1], d2 = pool[at + 2];
    unsigned rem = skip - below, pos = 0;
#pragma unroll
    for (int sft = 16; sft; sft >>= 1) {
        const unsigned t = __builtin_popcount(__builtin_amdgcn_ubfe(d0, pos, sft));
        const bool lt = t < rem;
        rem -= lt ? t : 0u; pos += lt ? sft : 0u;
    }
    pos = skip ? pos + 1u : 0u;
    return (unsigned)((((u64)d1 << 32) | d0) >> pos) | ((((u64)d2 << 32) | d1) >> pos << 32);
}
__device__ __forceinline__ unsigned code_get(u64& str, unsigned ne, unsigned b2) {
    const unsigned l0 = ffbl(((unsigned)str << 1) | (ne & 1u)); str >>= l0 & 63u;
    const unsigned l1 = ffbl(((unsigned)str << 1) | (ne >> 16)); str >>= l1 & 63u;
    const unsigned len = l0 | (l1 << 16);
    const unsigned adj = __builtin_amdgcn_perm(0x02u, 0x07010100u, MIN2(len, 0x00040004u) | 0x0c000c00u);
    return MIN2(DW(PK(b2 + adj) - PK(len)), b2 + 0x00050005u);
}

typedef float f32x2 __attribute__((ext_vector_type(2)));
__device__ __forceinline__ f32x2 gelu2(f32x2 y) {
    const float k1 = -2.0f * 0.7978845608028654f * 1.4426950408889634f, k3 = k1 * 0.044715f;
    const f32x2 a = __builtin_elementwise_fma(y * y, (f32x2){k3, k3}, (f32x2){k1, k1}) * y;
    const f32x2 d = (f32x2){__builtin_amdgcn_exp2f(a.x), __builtin_amdgcn_exp2f(a.y)} + 1.0f;
    return y * (f32x2){__builtin_amdgcn_rcpf(d.x), __builtin_amdgcn_rcpf(d.y)};
}
__device__ __forceinline__ float gelu1(float y) { return gelu2((f32x2){y, y}).x; }

#define TIMED_OUT(t0) (UNI(__hip_atomic_load(tmo, RLX, SC_AGENT)) == call || __builtin_amdgcn_s_memrealtime() - (t0) > DEADLINE_TICKS)

__global__ __launch_bounds__(NTHR, 2) void fused_gemm_rs(Params p) {
    __shared__ __attribute__((aligned(16))) unsigned char lds_raw[3 * STAGE + 2048];
    LAS unsigned char* lds = (LAS unsigned char*)lds_raw;
    LAS unsigned* ctl = (LAS unsigned*)(lds + 3 * STAGE);
    const unsigned lds0 = (unsigned)(size_t)lds;
    const int tid = threadIdx.x, lane = tid & 63;
    const int wid = UNI(tid >> 6);
    const int fr = lane & 15, fq = lane >> 4;
    const unsigned lo = lane * 16u;
    const unsigned bid = (blockIdx.x + 64u) & 255u;
    const int xcd = bid & 7, idx = bid >> 3;
    const bool late = idx < 8;
    const int q = wid & 3;

    const int team = (wid >> 2) * 192 + (int)bid - 64;
    f32x4 r[8];
    auto request = [&](int g, int u) {
        if (g < 32) {
            const GAS char* b = (const GAS char*)(p.w + (size_t)((g >> 2) * 64 + (q & 1) * 32) * GN + (u * 4 + (g & 3)) * 128 + (q >> 1) * 64) + (unsigned)(8 * fq * GN + 4 * fr) * 4u;
#pragma unroll
            for (int e = 0; e < 8; ++e) r[e] = *(const GAS f32x4*)(b + (size_t)e * (GN * 4));
        } else {
            const GAS char* b = (const GAS char*)(p.x + (size_t)(u * 128 + q * 32) * KSH + (g - 32) * 64) + (unsigned)(fr * KSH + 8 * fq) * 4u;
#pragma unroll
            for (int e = 0; e < 8; ++e) r[e] = *(const GAS f32x4*)(b + (e >> 2) * (16 * KSH * 4) + ((e >> 1) & 1) * 128 + (e & 1) * 16);
        }
    };
    if (!late && team < 96) request(team >= 64 ? 32 : (team >> 5) * 4, team & 31);
    const unsigned t_entry = now64();

    const int rank = p.rank & 7;
    char* myws = p.myws;
    GAS unsigned* ctr = (GAS unsigned*)(myws + WS_CTR + (size_t)bid * 128);
    GAS unsigned* tmo = (GAS unsigned*)(myws + WS_TMO);
    const unsigned ctr_v = __hip_atomic_load(ctr, RLX, SC_AGENT);
    GAS unsigned* gflag = (GAS unsigned*)(myws + WS_FLAG);
    ctl[tid] = 0u;
    WAIT_L0();
    BAR();

    if (!late) {
        unsigned wlo = 0, whi = 0;
#pragma unroll
        for (int d = 0; d < NR; ++d) {
            unsigned a = (unsigned)(size_t)p.ws[d], b = (unsigned)((size_t)p.ws[d] >> 32);
            asm volatile("" : "+s"(a), "+s"(b));
            wlo = lane == d ? a : wlo;
            whi = lane == d ? b : whi;
        }
#pragma unroll 1
        for (int it = 0; it < 5; ++it) {
            const int t = it < 2 ? it * 384 + team : 2 * 384 + (it - 2) * 288 + (wid >> 2) * 144 + (((idx - 8) >> 2) * 3 + (idx & 3) - 1) * 8 + xcd;
            if (t >= 1280 || (it >= 2 && (idx & 3) == 0)) break;
            const int g = p.prod[t >> 5], u = t & 31;
            if (t >= 96) {
                const unsigned ts = t_entry + (unsigned)((t >> 5) + 2) * (PROD_GAP * 64);
                while ((int)(now64() - ts) < 0) __builtin_amdgcn_s_sleep(2);
                request(g, u);
            }
            const bool isw = g < 32;
            const int c = isw ? g >> 2 : g - 32;
            const int gid = isw ? u * 4 + (g & 3) : 128 + u;
            unsigned v[16];
            if (isw) {
#pragma unroll
                for (int i = 0; i < 16; ++i) v[i] = cvt2(r[(i & 3) * 2][i >> 2], r[(i & 3) * 2 + 1][i >> 2]);
            } else {
#pragma unroll
                for (int i = 0; i < 16; ++i) v[i] = cvt2(r[i >> 1][(i & 1) * 2], r[i >> 1][(i & 1) * 2 + 1]);
            }
            asm volatile("" : "+v"(v[15]));
            const unsigned t_loaded = now64();
            const bool pushed = t < 32;
            const unsigned call = UNI(ctr_v) + 1u;
            const size_t par = WS_EXI + (size_t)(call & 1u) * EXI_BYTES;
            const int o = u >> 2;
            const int myid = isw ? ((gid >> 5) == (rank >> 1) ? gid & 31 : -1)
                                 : (o == rank ? 44 + (u & 3) : ((o >> 1) != (rank >> 1) && ((u >> 1) & 1) == (rank & 1) ? 32 + rb_of(rank, o) * 2 + (u & 1) : -1));
            unsigned hdr = 0;
            {
                unsigned w[16];
#pragma unroll
                for (int i = 0; i < 16; ++i) w[i] = v[i];
                const char* home = myws + par + (size_t)(gid * 8 + c) * SLOT;
                unsigned pw = 0;
                hdr = 32u;
                if (!pushed) {
                    LAS unsigned* pool = (LAS unsigned*)(lds + wid * 256);
                    pool[lane] = 0u;
                    unsigned ef[16];
#pragma unroll
                    for (int i = 0; i < 16; ++i) ef[i] = (w[i] >> 10) & 0x001f001fu;
                    const unsigned B = (unsigned)code_base<5>(ef);
                    if (code_put<POOL_IN>(ef, B, pool, fq)) {
#pragma unroll
                        for (int i = 0; i < 16; ++i) w[i] = (w[i] & 0x83ff83ffu) | (ef[i] << 10);
#pragma unroll
                        for (int k = 0; k < 3; ++k) {
#pragma unroll
                            for (int i = 0; i < 4; ++i) w[4 * k + i] |= (w[13 + k] << (12 - 3 * i)) & 0x70007000u;
                            w[12] |= (w[13 + k] >> (1 + k)) & (0x40004000u >> k);
                        }
                        pw = __hip_atomic_load(pool + lane, RLX, SC_WG);
                        hdr = B;
                    }
                }
                if (pushed) {
                    unsigned tp = t_loaded + (unsigned)((u & 7) * 4 + q) * PUSH_SLOT;
#pragma unroll
                    for (int i = 0; i < 4; ++i) {
                        pace(tp, 32u * PUSH_SLOT);
#pragma unroll
                        for (int k = 0; k < 2; ++k) {
                            const int d = (u >> 3) * 2 + k;
                            if (d != rank) st16(p.ws[d] + par + (size_t)(1280 + rank * 8 + (u & 7)) * SLOT + q * 4096 + i * 1024, lo, (u32x4){v[4 * i], v[4 * i + 1], v[4 * i + 2], v[4 * i + 3]});
                        }
                    }
                } else {
#pragma unroll
                    for (int i = 0; i < 3; ++i) st16(home + q * QB + i * 1024, lo, (u32x4){w[4 * i], w[4 * i + 1], w[4 * i + 2], w[4 * i + 3]});
                    st4(home + q * QB + 3072, lane * 4u, w[12]);
                    if (hdr == 32u) st12(home + SIDE + q * 768, lane * 12u, (u32x3){v[13], v[14], v[15]});
                    else if (lane < POOL_IN) st4(home + q * QB + 3328, lane * 4u, pw);
                }
            }
            if (myid >= 0) {
                const char* sg = myws + WS_STG + stage_off(myid, rank * 8 + c) + (isw ? (q >> 1) * 8 + (q & 1) : q * 4) * 1024;
#pragma unroll
                for (int i = 0; i < 4; ++i) st16(sg + i * (isw ? 2048 : 1024), lo, (u32x4){v[4 * i], v[4 * i + 1], v[4 * i + 2], v[4 * i + 3]});
            }
            wait_vm<0>();
            unsigned tw = (1u << 24) | (hdr << (6 * q));
            if (lane == 0) tw += __hip_atomic_fetch_add(ctl + 402 + (wid >> 2) * 5 + it, tw, RLX, SC_WG);
            tw = UNI(tw);
            if ((tw >> 24) == 4u) {
                const int d = lane & 7;
                int did = isw ? ((gid >> 5) == (d >> 1) ? gid & 31 : -1)
                              : (o == d ? 44 + (u & 3) : ((o >> 1) != (d >> 1) && ((u >> 1) & 1) == (d & 1) ? 32 + rb_of(d, o) * 2 + (u & 1) : -1));
                if (isw && (g & 3) == 3 && c < 2 * NFWD && (d >> 1) != (rank >> 1) && ((c ^ d) & 1)) did = -1;
                if (lane < 8 && did >= 0) {
                    if (d == rank) __hip_atomic_store(gflag + did * 64 + c * 8 + rank, call, RLX, SC_AGENT);
                    else __hip_atomic_store((GAS u64*)((((size_t)whi << 32) | wlo) + WS_RDY) + ((((call & 1u) * 8 + rank) * 48 + did) * 8 + c) * RSTRIDE, ((u64)call << 32) | (tw & 0xffffffu), RLX, SC_SYS);
                }
            }
        }
    } else if (idx < 7) {
        const int s = idx < 6 ? rb_owner(rank, idx) : rank ^ 1;
        const int nu = idx < 6 ? NU_OUT : NU_PAIR;
        unsigned desc[2];
#pragma unroll
        for (int k = 0; k < 2; ++k) {
            const int e = p.order[idx == 6][k * 64 + lane];
            int c, id, org = s, slot;
            if (e < 48) { unit_of(e, xcd, c, id); slot = gid_of(rank, id) * 8 + c; }
            else { const int kf = (e - 48) / 6; c = 2 * kf + (s & 1); id = xcd * 4 + 3; org = rb_owner(rank, e - 48 - kf * 6); slot = 1408 + (e - 48) * 8 + xcd; }
            desc[k] = (unsigned)(c | id << 3 | org << 9 | slot << 12);
        }
        auto desc_of = [&](int n) { return (unsigned)(n < 64 ? __builtin_amdgcn_readlane(desc[0], n) : __builtin_amdgcn_readlane(desc[1], n - 64)); };
        const unsigned call = UNI(ctr_v) + 1u;
        const size_t par = WS_EXI + (size_t)(call & 1u) * EXI_BYTES;
        GAS u64* eq = (GAS u64*)(myws + WS_EQ) + idx * 16;
        if (wid == 6) {
            const char* exs = p.ws[s] + par;
            unsigned tnext = t_entry + xcd * (TAU / 4);
            u64 todo = ~1ull;
            __builtin_amdgcn_s_setprio(3);
            const char* src = nullptr;
            unsigned trim = 0;
            auto choose = [&](int n) {
                const unsigned rdy = __hip_atomic_load(ctl + (n >= 64 ? n : lane), RLX, SC_WG);
                const unsigned fre = __hip_atomic_load(ctl + (n > RING ? 128 + n - RING : 400), RLX, SC_WG), tr = __hip_atomic_load(ctl + 400, RLX, SC_WG);
                const u64 rd = __ballot(rdy != 0u) & (n >= 64 ? 1ull : todo);
                if (!rd || (n > RING && UNI(fre) < 4u)) return;
                const int un = n >= 64 ? n : __builtin_ctzll(rd);
                todo &= ~(1ull << (un & 63));
                if (lane == 0) __hip_atomic_store(ctl + 416 + n, (unsigned)un, RLX, SC_WG);
                trim = n >= EQ_AT + 4 && n < EQ_AT + 36 ? UNI(tr) : 0u;
                src = exs + (size_t)(desc_of(un) >> 12) * SLOT;
            };
            for (int n = 1; n < nu; ++n) {
                bool drained = false;
                if (!src) {
                    choose(n);
                    if (!src) {
                        wait_vm<0>();
                        if (lane == 0) __hip_atomic_store(ctl + 384, (unsigned)(n - 1) * 14u, RLX, SC_WG);
                        drained = true;
                        for (choose(n); !src; choose(n)) __builtin_amdgcn_s_sleep(1);
                    }
                }
                const char* cur = src;
                const unsigned dst = lds0 + (unsigned)(n % RING) * 14336u, tr = trim;
                src = nullptr;
                asm volatile("" ::: "memory");
#pragma unroll
                for (int i = 0; i < 14; i += 2) {
                    wait_vm<WIN - 2>();
                    if ((i > WIN - 2 || (!drained && n > 1)) && lane == 0) __hip_atomic_store(ctl + 384, (unsigned)((n - 1) * 14 + i - (WIN - 2)), RLX, SC_WG);
                    pace(tnext, (i < 12 ? 2u * TAU : UNIT_T - 12u * TAU) - tr);
                    if (i < 12) dma<true>(cur + i * 1024, lo, dst + i * 1024);
                    else {
                        dma<false>(cur + 12288, lo, dst + 12288);
                        if (lane < (4 * QB - 13312) / 16) dma<false>(cur + 13312, lo, dst + 13312);
                    }
                    if (i == 0 && lane == 0) {
                        if (n == EQ_AT) __hip_atomic_store(ctl + 387, tnext | 1u, RLX, SC_WG);
                        if (n == nu - 3) __hip_atomic_store(ctl + 386, tnext | 1u, RLX, SC_WG);
                        if (n == nu - 1) __hip_atomic_store(ctl + 385, (tnext + UNIT_T - 2u * TAU) | 1u, RLX, SC_WG);
                    }
                    if (i == 10 && n + 1 < nu) choose(n + 1);
                }
            }
            __builtin_amdgcn_s_setprio(0);
            wait_vm<0>();
            if (lane == 0) __hip_atomic_store(ctl + 384, (unsigned)(nu - 1) * 14u, RLX, SC_WG);
        } else if (wid == 7) {
            const GAS u64* rp[2];
            bool pend[2];
#pragma unroll
            for (int k = 0; k < 2; ++k) {
                rp[k] = (const GAS u64*)(myws + WS_RDY) + ((((call & 1u) * 8 + ((desc[k] >> 9) & 7)) * 48 + ((desc[k] >> 3) & 63)) * 8 + (desc[k] & 7)) * RSTRIDE;
                pend[k] = k * 64 + lane < nu;
            }
            unsigned mine = 0;
            bool eq_open = true;
            const u64 t0 = __builtin_amdgcn_s_memrealtime();
            for (unsigned polls = 1;; ++polls) {
                const u64 pm = __ballot(pend[0]);
                const bool front = __builtin_amdgcn_mbcnt_hi((unsigned)(pm >> 32), __builtin_amdgcn_mbcnt_lo((unsigned)pm, 0u)) < POLLW;
#pragma unroll
                for (int k = 0; k < 2; ++k)
                    if (pend[k] && ((k == 0 && front) || (polls & 7u) == 0u)) {
                        const u64 f = __hip_atomic_load(rp[k], RLX, SC_SYS);
                        if ((unsigned)(f >> 32) >= call) { __hip_atomic_store(ctl + k * 64 + lane, (unsigned)f | 0x80000000u, RLX, SC_WG); pend[k] = false; }
                    }
                if (eq_open) {
                    if (!mine) {
                        mine = lds_get(ctl + 387);
                        if (mine && lane == 0) __hip_atomic_store(eq + xcd, ((u64)call << 32) | mine, RLX, SC_AGENT);
                    } else {
                        const u64 f = __hip_atomic_load(eq + (lane & 7), RLX, SC_AGENT);
                        if (__all((unsigned)(f >> 32) == call)) {
                            int dlt = (int)(mine - (unsigned)f);
                            dlt += __shfl_xor(dlt, 1); dlt += __shfl_xor(dlt, 2); dlt += __shfl_xor(dlt, 4);
                            dlt /= 8 * 32 * 7;
                            dlt = dlt > TAU / 8 ? TAU / 8 : dlt < -(TAU / 8) ? -(TAU / 8) : dlt;
                            if (lane == 0) __hip_atomic_store(ctl + 400, (unsigned)dlt, RLX, SC_WG);
                            eq_open = false;
                        } else if ((int)lds_get(ctl + 384) >= (EQ_AT + 2) * 14) eq_open = false;
                    }
                }
                const bool waiting = __any(pend[0] || pend[1]);
                if (!waiting && !eq_open) break;
                if (waiting && (polls & 15u) == 0u && TIMED_OUT(t0)) {
                    if (lane == 0) __hip_atomic_store(tmo, call, RLX, SC_AGENT);
                    wait_vm<0>();
#pragma unroll
                    for (int k = 0; k < 2; ++k) if (pend[k]) __hip_atomic_store(ctl + k * 64 + lane, 0x80000000u, RLX, SC_WG);
                    break;
                }
                if (polls < 32u) __builtin_amdgcn_s_sleep(1); else __builtin_amdgcn_s_sleep(16);
            }
            if (idx < 6) {
                while ((mine = lds_get(ctl + 386)) == 0u) __builtin_amdgcn_s_sleep(8);
                if (lane == 0) __hip_atomic_store(eq + 8 + xcd, ((u64)call << 32) | mine, RLX, SC_AGENT);
                unsigned th;
                int dlt = 0;
                bool open = true;
                while ((th = lds_get(ctl + 385)) == 0u) {
                    if (open) {
                        const u64 f = __hip_atomic_load(eq + 8 + (lane & 7), RLX, SC_AGENT);
                        if (__all((unsigned)(f >> 32) == call)) {
                            dlt = (int)(mine - (unsigned)f);
                            dlt += __shfl_xor(dlt, 1); dlt += __shfl_xor(dlt, 2); dlt += __shfl_xor(dlt, 4);
                            dlt /= 8 * 256;
                            dlt = dlt > 100 ? 100 : dlt < -100 ? -100 : dlt;
                            open = false;
                        }
                    }
                    __builtin_amdgcn_s_sleep(4);
                }
                if (lane == 0) __hip_atomic_store((GAS u64*)(myws + WS_MBOX + (size_t)((8 + idx * 4) * 8 + xcd) * 128), ((u64)call << 32) | (th & ~255u) | (unsigned)(dlt + 128), RLX, SC_AGENT);
            }
        } else {
            int pend = -1, pend_n = 0;
            unsigned pend_h = 0;
            auto staged = [&]() {
                wait_vm<0>();
                if (lane == 0 && __hip_atomic_fetch_add(ctl + 256 + pend_n, 1u, RLX, SC_WG) == 3u) {
                    __hip_atomic_store(gflag + pend, call, RLX, SC_AGENT);
                    if (pend_h) __hip_atomic_store((GAS u64*)(p.ws[rank ^ 1] + WS_RDY) + ((((call & 1u) * 8 + s) * 48 + (pend >> 6)) * 8 + ((pend >> 3) & 7)) * RSTRIDE,
                                                   ((u64)call << 32) | (pend_h & 0xffffffu), RLX, SC_SYS);
                }
                pend = -1;
            };
            auto need = [&](int job) { return (job >> 2) * 14 - 14 + (((job & 3) + 1) * QB + 1023) / 1024; };
            auto ticket = [&]() { unsigned tk = 0; if (lane == 0) tk = __hip_atomic_fetch_add(ctl + 401, 1u, RLX, SC_WG); return (int)UNI(tk); };
            for (int job = ticket(); job < 4 * nu;) {
                const int n = job >> 2, qq = job & 3;
                unsigned w[16];
                int un = 0;
                unsigned relay_h = 0;
                if (n == 0) {
                    while (lds_get(ctl) == 0u) __builtin_amdgcn_s_sleep(1);
                    u32x4 a0, a1, a2, a3;
                    ld64(myws + par + (size_t)(1280 + s * 8 + xcd) * SLOT + qq * 4096, lo, a0, a1, a2, a3);
#pragma unroll
                    for (int i = 0; i < 4; ++i) { w[i] = a0[i]; w[4 + i] = a1[i]; w[8 + i] = a2[i]; w[12 + i] = a3[i]; }
                } else {
                    while ((int)lds_get(ctl + 384) < need(job)) __builtin_amdgcn_s_sleep(1);
                    asm volatile("" ::: "memory");
                    un = lds_get(ctl + 416 + n);
                }
                const unsigned de = desc_of(un);
                const int c = de & 7, id = (de >> 3) & 63, org = (de >> 9) & 7;
                if (n) {
                    const LAS unsigned char* src = lds + (unsigned)(n % RING) * 14336u + qq * QB;
#pragma unroll
                    for (int i = 0; i < 3; ++i) { const u32x4 a = *(const LAS u32x4*)(src + i * 1024 + lo); w[4 * i] = a[0]; w[4 * i + 1] = a[1]; w[4 * i + 2] = a[2]; w[4 * i + 3] = a[3]; }
                    w[12] = *(const LAS unsigned*)(src + 3072 + lane * 4);
                    const LAS unsigned* pool = (const LAS unsigned*)(src + 3328);
                    const unsigned hu = lds_get(ctl + un), h = (hu >> (6 * qq)) & 63u;
                    if (idx < 6 && id < 32 && (id & 3) == 3 && c < 2 * NFWD) {
                        const unsigned pw = pool[lane];
                        if (pend >= 0) staged();
                        const char* dst = myws + par + (size_t)(1408 + ((c >> 1) * 6 + idx) * 8 + xcd) * SLOT + qq * QB;
#pragma unroll
                        for (int i = 0; i < 3; ++i) st16(dst + i * 1024, lo, (u32x4){w[4 * i], w[4 * i + 1], w[4 * i + 2], w[4 * i + 3]});
                        st4(dst + 3072, lane * 4u, w[12]);
                        if (lane < POOL_IN) st4(dst + 3328, lane * 4u, pw);
                        relay_h = hu;
                    }
                    if (h < 32u) {
#pragma unroll
                        for (int k = 0; k < 3; ++k) {
                            unsigned cc = (w[12] << (1 + k)) & 0x80008000u;
#pragma unroll
                            for (int i = 0; i < 4; ++i) cc |= (w[4 * k + i] >> (12 - 3 * i)) & (0x00070007u << (3 * i));
                            w[13 + k] = cc;
                        }
                        unsigned ne[16];
                        u64 str = code_find<10>(w, ne, pool, fq);
                        WAIT_L0();
                        if (lane == 0) __hip_atomic_fetch_add(ctl + 128 + n, 1u, RLX, SC_WG);
                        if (pend >= 0) staged();
#pragma unroll
                        for (int i = 0; i < 16; ++i) w[i] = (w[i] & 0x8fff8fffu) + (code_get(str, ne[i], h * 0x00010001u) << 10);
                    } else {
                        WAIT_L0();
                        if (lane == 0) __hip_atomic_fetch_add(ctl + 128 + n, 1u, RLX, SC_WG);
                        if (pend >= 0) staged();
                        const u32x3 a = ld12(p.ws[org] + par + (size_t)(gid_of(rank, id) * 8 + c) * SLOT + SIDE + qq * 768, lane * 12u);
                        w[13] = a[0]; w[14] = a[1]; w[15] = a[2];
                    }
                }
                const char* sg = myws + WS_STG + stage_off(id, org * 8 + c) + (id < 32 ? (qq >> 1) * 8 + (qq & 1) : qq * 4) * 1024;
#pragma unroll
                for (int i = 0; i < 4; ++i) st16(sg + i * (id < 32 ? 2048 : 1024), lo, (u32x4){w[4 * i], w[4 * i + 1], w[4 * i + 2], w[4 * i + 3]});
                pend = id * 64 + c * 8 + org; pend_n = n; pend_h = relay_h;
                job = ticket();
                if (job >= 4 * nu || job < 4 || (int)lds_get(ctl + 384) < need(job)) staged();
            }
        }
    }
    const unsigned call = UNI(ctr_v) + 1u;

    const int ntile = idx == 7 ? 2 : 1;
    const int wm = wid >> 1, wn = wid & 1;
#pragma unroll 1
    for (int rep = 0; rep < ntile; ++rep) {
    const int rb = late ? 6 + (idx >> 2) : (idx - 8) >> 2;
    const int cb = xcd * 4 + (idx == 7 ? rep : idx == 6 ? 0 : idx == 4 || idx == 5 ? idx - 2 : idx & 3);
    const bool part1 = idx == 7 && rep == 0, part2 = idx == 6;
    const int nstep = part1 ? 56 : part2 ? 8 : NSTEP;
    const char* stgA = myws + WS_STG + (size_t)(rb * NSTEP) * A_STEP + wid * 4096;
    const char* stgB = myws + WS_STG + STG_A + (size_t)(cb * NSTEP) * B_STEP + wid * 2048;
    const u64 pbits = 0x0101010101010101ull << (rank ^ 1);
    u64 todo = part1 ? ~pbits : part2 ? pbits : ~0ull, avail = 0ull;
    int slot = 0;
    auto poll = [&]() {
        if (wid == 0) {
            const u64 t0 = __builtin_amdgcn_s_memrealtime();
            u64 got;
            for (unsigned polls = 1;; ++polls) {
                const unsigned f0 = __hip_atomic_load(gflag + cb * 64 + lane, RLX, SC_AGENT);
                const unsigned f1 = __hip_atomic_load(gflag + (32 + rb * 2) * 64 + lane, RLX, SC_AGENT);
                const unsigned f2 = __hip_atomic_load(gflag + (33 + rb * 2) * 64 + lane, RLX, SC_AGENT);
                got = __ballot(f0 >= call && f1 >= call && f2 >= call) & todo;
                if (got) break;
                if ((polls & 15u) == 0u && TIMED_OUT(t0)) {
                    if (lane == 0) __hip_atomic_store(tmo, call, RLX, SC_AGENT);
                    got = todo;
                    break;
                }
                __builtin_amdgcn_s_sleep(8);
            }
            if (lane < 2) ctl[388 + slot * 2 + lane] = (unsigned)(got >> (32 * lane));
        }
    };
    auto news = [&]() { avail = ((u64)(unsigned)UNI(ctl[389 + slot * 2]) << 32) | (unsigned)UNI(ctl[388 + slot * 2]); slot ^= 1; };
    auto take = [&]() {
        const unsigned b = (unsigned)__builtin_ctzll(avail) & 63u;
        avail &= avail - 1; todo &= ~(1ull << b);
        return (b & 7u) * 8u + (b >> 3);
    };
    auto g1 = [&](unsigned ks, unsigned buf, int i) {
        if (i < 4) __builtin_amdgcn_global_load_lds((const GAS unsigned*)(stgA + (size_t)ks * A_STEP + i * 1024 + lo), (LAS unsigned*)(lds + buf + wid * 4096 + i * 1024), 16, 0, 17);
        else __builtin_amdgcn_global_load_lds((const GAS unsigned*)(stgB + (size_t)ks * B_STEP + (i - 4) * 1024 + lo), (LAS unsigned*)(lds + buf + A_STEP + wid * 2048 + (i - 4) * 1024), 16, 0, 17);
    };

    f32x4 acc[4][4];
#pragma unroll
    for (int m = 0; m < 4; ++m)
#pragma unroll
        for (int n = 0; n < 4; ++n) acc[m][n] = (f32x4){0.f, 0.f, 0.f, 0.f};
    h16x8 a0[4], b0[4], a1[4], b1[4];
    const unsigned aoff = wm * 8192 + lo, boff = A_STEP + wn * 8192 + lo;
#define RDA(A, buf, kk, m) A[m] = *(const LAS h16x8*)(lds + (buf) + aoff + (m) * 2048 + (kk) * 1024)
#define RDB(B, buf, kk, n) B[n] = *(const LAS h16x8*)(lds + (buf) + boff + (n) * 2048 + (kk) * 1024)
#define ROW(A, B, m) _Pragma("unroll") for (int n = 0; n < 4; ++n) acc[m][n] = __builtin_amdgcn_mfma_f32_16x16x32_f16(A[m], B[n], acc[m][n], 0, 0, 0)
    unsigned kq;
#pragma unroll
    for (int k = 0; k < 3; ++k) {
        if (!avail) { poll(); WAIT_L0(); BAR(); news(); }
        kq = take();
#pragma unroll
        for (int i = 0; i < (k < 2 ? 6 : 3); ++i) g1(kq, k * STAGE, i);
    }
    wait_vm<9>(); BAR();
#pragma unroll
    for (int i = 0; i < 4; ++i) { RDB(b0, 0u, 0, i); RDA(a0, 0u, 0, i); }
    unsigned cur = 0, prv = 2 * STAGE;
    for (int t = 0; t < nstep; ++t) {
        const unsigned nxt = cur == 2 * STAGE ? 0u : cur + STAGE;
        const bool m2 = t + 2 < nstep, m3 = t + 3 < nstep;
        const bool dry = m3 && !avail;
        WAIT_L0(); SB();
        RDB(b1, cur, 1, 0); RDB(b1, cur, 1, 1); SB(); ROW(a0, b0, 0); SB(); if (m2) g1(kq, prv, 3); SB();
        RDB(b1, cur, 1, 2); RDB(b1, cur, 1, 3); SB(); ROW(a0, b0, 1); SB(); if (m2) g1(kq, prv, 4); SB();
        RDA(a1, cur, 1, 0); RDA(a1, cur, 1, 1); SB(); ROW(a0, b0, 2); SB(); if (m2) g1(kq, prv, 5); SB();
        RDA(a1, cur, 1, 2); RDA(a1, cur, 1, 3); SB(); ROW(a0, b0, 3); SB();
        if (dry) poll();
        WAIT_L0();
        if (m2) wait_vm<6>(); else wait_vm<0>();
        BAR(); SB();
        RDB(b0, nxt, 0, 0); RDB(b0, nxt, 0, 1); SB(); ROW(a1, b1, 0); SB();
        if (dry) news();
        if (m3) { kq = take(); g1(kq, cur, 0); } SB();
        RDB(b0, nxt, 0, 2); RDB(b0, nxt, 0, 3); SB(); ROW(a1, b1, 1); SB(); if (m3) g1(kq, cur, 1); SB();
        RDA(a0, nxt, 0, 0); RDA(a0, nxt, 0, 1); SB(); ROW(a1, b1, 2); SB(); if (m3) g1(kq, cur, 2); SB();
        RDA(a0, nxt, 0, 2); RDA(a0, nxt, 0, 3); SB(); ROW(a1, b1, 3); SB();
        prv = cur; cur = nxt;
    }

    if (part1 || part2) {
        const char* park = myws + WS_EXO + (size_t)(xcd * 8 + wid) * 16384;
        GAS u64* pf = (GAS u64*)(myws + WS_ORDY + (size_t)xcd * 128) + wid;
        if (part1) {
#pragma unroll
            for (int i = 0; i < 16; ++i) st16(park + i * 1024, lo, __builtin_bit_cast(u32x4, acc[i >> 2][i & 3]));
            wait_vm<0>();
            if (lane == 0) __hip_atomic_store(pf, (u64)call << 32, RLX, SC_SYS);
            continue;
        }
        const u64 t0 = __builtin_amdgcn_s_memrealtime();
        while ((unsigned)UNI((unsigned)(__hip_atomic_load(pf, RLX, SC_SYS) >> 32)) < call) {
            if (TIMED_OUT(t0)) { if (lane == 0) __hip_atomic_store(tmo, call, RLX, SC_AGENT); wait_vm<0>(); break; }
            __builtin_amdgcn_s_sleep(4);
        }
#pragma unroll
        for (int m = 0; m < 4; ++m) {
            u32x4 e0, e1, e2, e3;
            ld64(park + m * 4096, lo, e0, e1, e2, e3);
            acc[m][0] += __builtin_bit_cast(f32x4, e0); acc[m][1] += __builtin_bit_cast(f32x4, e1);
            acc[m][2] += __builtin_bit_cast(f32x4, e2); acc[m][3] += __builtin_bit_cast(f32x4, e3);
        }
    }
    const unsigned tmo_v = __hip_atomic_load(tmo, RLX, SC_AGENT);
    const unsigned oo = (unsigned)(fq * 4) * (GN * 4u) + fr * 16u;
    const u32x4 nan4 = {0x7fc00000u, 0x7fc00000u, 0x7fc00000u, 0x7fc00000u};
    auto finish = [&](unsigned (&o)[32], int i, float a, float b) {
        const f32x2 g = gelu2((f32x2){a, b});
        const int at = ((i >> 3) * 4 + (i & 1) * 2) * 4 + ((i >> 1) & 3);
        o[at] = __float_as_uint(g.x); o[at + 4] = __float_as_uint(g.y);
    };
#define ACC(hf, i, k) acc[2 * (hf) + ((i) >> 3)][((i) >> 1) & 3][((i) & 1) * 2 + (k)]
    if (late) {
        const bool bad = UNI(tmo_v) == call;
        const char* ob = (const char*)(p.out + (size_t)(rb_local(rank, rb) + wm * 64) * GN + (rank >> 1) * 4096 + cb * 128 + wn * 64);
#pragma unroll
        for (int hf = 0; hf < 2; ++hf) {
            unsigned o[32];
#pragma unroll
            for (int i = 0; i < 16; ++i) finish(o, i, ACC(hf, i, 0), ACC(hf, i, 1));
#pragma unroll
            for (int i = 0; i < 8; ++i) st16(ob + (size_t)(hf * 32 + (i >> 2) * 16 + (i & 3)) * (GN * 4), oo, bad ? nan4 : (u32x4){o[4 * i], o[4 * i + 1], o[4 * i + 2], o[4 * i + 3]});
        }
    } else {
        const int peer = rb_owner(rank, rb);
        const int mirror = (8 + rb_of(peer, rank) * 4 + (idx & 3)) * 8 + xcd;
        char* const peerws = p.ws[peer];
        WAIT_L0(); BAR();
        if (wid < 4) __builtin_amdgcn_s_setprio(2);
#pragma unroll
        for (int hf = 0; hf < 2; ++hf) {
            const int k = (wid >> 2) * 8 + hf * 4 + q;
            const char* ex = myws + WS_EXO + (size_t)(bid * 16 + k) * 8192;
            unsigned hdr = 1u << 13;
            if ((idx & 3) != 3 || k < 16 - NPLAIN) {
                LAS unsigned* pool = (LAS unsigned*)(lds + 131072 + wid * 256);
                pool[lane] = 0u;
                unsigned hi[16], ef[16], under = 0;
#pragma unroll
                for (int i = 0; i < 16; ++i) {
                    hi[i] = __builtin_amdgcn_perm(__float_as_uint(ACC(hf, i, 1)), __float_as_uint(ACC(hf, i, 0)), 0x07060302u);
                    ef[i] = (hi[i] >> 7) & 0x00ff00ffu;
                }
                const unsigned bs = (unsigned)code_base<8>(ef), e0 = bs > 20u ? bs - 20u : 0u;
#pragma unroll
                for (int i = 0; i < 16; ++i) { under |= SUBS(e0 * 0x00010001u, ef[i]); ef[i] = SUBS(ef[i], e0 * 0x00010001u); }
                if (__all(under == 0u) && code_put<POOL_OUT>(ef, bs - e0, pool, fq)) {
#pragma unroll
                    for (int i = 0; i < 16; ++i) hi[i] = (hi[i] & 0x807f807fu) | (ef[i] << 7);
#pragma unroll
                    for (int c = 0; c < 6; ++c) {
#pragma unroll
                        for (int i = 0; i < 3; ++i) { const int fld = c * 3 + i, at = 9 + 3 * (fld & 1); hi[fld >> 1] |= (hi[10 + c] << (at - 3 * i)) & (0x00070007u << at); }
                        hi[9] |= (hi[10 + c] >> (1 + c)) & (0x40004000u >> c);
                    }
                    const unsigned pw = __hip_atomic_load(pool + lane, RLX, SC_WG);
#pragma unroll
                    for (int i = 0; i < 4; ++i) {
                        u32x4 v;
#pragma unroll
                        for (int e = 0; e < 4; ++e) v[e] = __builtin_amdgcn_perm(__float_as_uint(ACC(hf, 4 * i + e, 1)), __float_as_uint(ACC(hf, 4 * i + e, 0)), 0x05040100u);
                        st16(ex + i * 1024, lo, v);
                    }
                    st16(ex + 4096, lo, (u32x4){hi[0], hi[1], hi[2], hi[3]});
                    st16(ex + 5120, lo, (u32x4){hi[4], hi[5], hi[6], hi[7]});
                    st8(ex + 6144, lane * 8u, (u32x2){hi[8], hi[9]});
                    if (lane < POOL_OUT) st4(ex + 6656, lane * 4u, pw);
                    hdr = e0 | ((bs - e0) << 8);
                }
            }
            if (hdr >> 13) {
                unsigned o[32];
#pragma unroll
                for (int i = 0; i < 16; ++i) finish(o, i, ACC(hf, i, 0), ACC(hf, i, 1));
#pragma unroll
                for (int i = 0; i < 8; ++i) st16(ex + i * 1024, lo, (u32x4){o[4 * i], o[4 * i + 1], o[4 * i + 2], o[4 * i + 3]});
            }
            wait_vm<0>();
            hdr |= UNI(tmo_v) == call ? 0xc0000000u : 0x40000000u;
            if (lane == 0) __hip_atomic_store((GAS u64*)(peerws + WS_ORDY + (size_t)mirror * 128) + k, ((u64)call << 32) | hdr, RLX, SC_SYS);
        }
        __builtin_amdgcn_s_setprio(0);
#pragma unroll 1
        for (int sk = 0; (idx & 3) + sk <= 3 && (idx & 3) != 3; sk += 3) {
        if (sk) {
            WAIT_L0(); BAR();
            if (tid < 32 || tid == 384 || tid == 401 || (tid >= 416 && tid < 432)) ctl[tid] = 0u;
            WAIT_L0(); BAR();
        }
        const int me = bid + 8 * sk;
        const char* exs = peerws + WS_EXO + (size_t)(mirror + 8 * sk) * (16 * 8192);
        if (wid == 0) {
            unsigned tnext;
            while ((tnext = lds_get(ctl + 16)) == 0u) __builtin_amdgcn_s_sleep(2);
            const unsigned trb = tnext & 255u, tr = trb - 128u;
            tnext = (tnext & ~255u) | 128u;
            __builtin_amdgcn_s_setprio(3);
            wait_vm<0>(); __builtin_amdgcn_s_waitcnt(0x0F70);
            int issued = 0, told = 0;
            unsigned hn = lds_get(ctl);
            for (int n = 0; n < 16; ++n) {
                if (hn == 0u && (hn = lds_get(ctl + n)) == 0u) {
                    wait_vm<0>();
                    told = issued;
                    if (lane == 0) __hip_atomic_store(ctl + 384, (unsigned)told, RLX, SC_WG);
                    while ((hn = lds_get(ctl + n)) == 0u) __builtin_amdgcn_s_sleep(1);
                }
                asm volatile("" ::: "memory");
                const bool plain = (hn >> 13) & 1u;
                const char* cur = exs + (size_t)n * 8192;
                const unsigned dst = lds0 + (unsigned)n * 8192u;
#pragma unroll
                for (int i = 0; i < 8; i += 2) {
                    wait_vm<WIN - 2>();
                    if (issued - (WIN - 2) > told) { told = issued - (WIN - 2); if (lane == 0) __hip_atomic_store(ctl + 384, (unsigned)told, RLX, SC_WG); }
                    pace(tnext, (i < 6 || plain ? 2u * TAU : PIECE_T - 6u * TAU) - tr);
                    if (i < 6 || plain) { dma<true>(cur + i * 1024, lo, dst + i * 1024); issued += 2; }
                    else { if (lane < (PB - 6144) / 16) dma<false>(cur + 6144, lo, dst + 6144); issued += 1; }
                    if (i == 0) {
                        if (lane == 0) __hip_atomic_store(ctl + 416 + n, (unsigned)issued + (plain ? 6u : 5u), RLX, SC_WG);
                        if (n == 14 && !sk && lane == 0)
                            __hip_atomic_store((GAS u64*)(myws + WS_MBOX + (size_t)(bid + 8) * 128), ((u64)call << 32) | ((tnext + 2u * (PIECE_T) - 2u * TAU - 7u * tr) & ~255u) | trb, RLX, SC_AGENT);
                    }
                    if (i == 2) hn = n < 15 ? lds_get(ctl + n + 1) : 0u;
                }
            }
            wait_vm<7>();
            if (issued - 7 > told && lane == 0) __hip_atomic_store(ctl + 384, (unsigned)issued - 7u, RLX, SC_WG);
            wait_vm<0>();
            if (lane == 0) __hip_atomic_store(ctl + 384, (unsigned)issued, RLX, SC_WG);
            __builtin_amdgcn_s_setprio(0);
        } else {
            if (wid == 1) {
                const GAS u64* fp = lane < 16 ? (const GAS u64*)(myws + WS_ORDY + (size_t)me * 128) + lane : (const GAS u64*)(myws + WS_MBOX + (size_t)me * 128);
                bool pend = lane < 17;
                const u64 t0 = __builtin_amdgcn_s_memrealtime();
                for (unsigned polls = 1;; ++polls) {
                    if (pend) {
                        const u64 f = __hip_atomic_load(fp, RLX, SC_SYS);
                        if ((unsigned)(f >> 32) >= call) { __hip_atomic_store(ctl + lane, (unsigned)f, RLX, SC_WG); pend = false; }
                    }
                    if (!__any(pend)) break;
                    if ((polls & 15u) == 0u && TIMED_OUT(t0)) {
                        if (lane == 0) __hip_atomic_store(tmo, call, RLX, SC_AGENT);
                        if (pend) __hip_atomic_store(ctl + lane, lane < 16 ? 0xc0002000u : (now64() & ~255u) | 128u, RLX, SC_WG);
                        break;
                    }
                    __builtin_amdgcn_s_sleep(4);
                }
            }
            for (;;) {
                unsigned tk = 0;
                if (lane == 0) tk = __hip_atomic_fetch_add(ctl + 401, 1u, RLX, SC_WG);
                const int k = UNI(tk);
                if (k >= 16) break;
                unsigned need;
                while ((need = lds_get(ctl + 416 + k)) == 0u) __builtin_amdgcn_s_sleep(4);
                const unsigned hdr = lds_get(ctl + k);
                while (lds_get(ctl + 384) < need) __builtin_amdgcn_s_sleep(1);
                asm volatile("" ::: "memory");
                const LAS unsigned char* src = lds + (unsigned)k * 8192u;
                unsigned o[32];
                if ((hdr >> 13) & 1u) {
#pragma unroll
                    for (int i = 0; i < 8; ++i) { const u32x4 a = *(const LAS u32x4*)(src + i * 1024 + lo); o[4 * i] = a[0]; o[4 * i + 1] = a[1]; o[4 * i + 2] = a[2]; o[4 * i + 3] = a[3]; }
                } else {
                    unsigned lw[16], hi[16], ne[16];
#pragma unroll
                    for (int i = 0; i < 2; ++i) { const u32x4 a = *(const LAS u32x4*)(src + 4096 + i * 1024 + lo); hi[4 * i] = a[0]; hi[4 * i + 1] = a[1]; hi[4 * i + 2] = a[2]; hi[4 * i + 3] = a[3]; }
                    { const u32x2 a = *(const LAS u32x2*)(src + 6144 + lane * 8); hi[8] = a[0]; hi[9] = a[1]; }
#pragma unroll
                    for (int i = 0; i < 4; ++i) { const u32x4 a = *(const LAS u32x4*)(src + i * 1024 + lo); lw[4 * i] = a[0]; lw[4 * i + 1] = a[1]; lw[4 * i + 2] = a[2]; lw[4 * i + 3] = a[3]; }
#pragma unroll
                    for (int c = 0; c < 6; ++c) {
                        unsigned cc = (hi[9] << (1 + c)) & 0x80008000u;
#pragma unroll
                        for (int i = 0; i < 3; ++i) { const int fld = c * 3 + i, at = 9 + 3 * (fld & 1); cc |= (hi[fld >> 1] >> (at - 3 * i)) & (0x00070007u << (3 * i)); }
                        hi[10 + c] = cc;
                    }
                    u64 str = code_find<7>(hi, ne, (const LAS unsigned*)(src + 6656), fq);
                    const unsigned e0 = (hdr & 255u) * 0x00800080u, b2 = ((hdr >> 8) & 31u) * 0x00010001u;
#pragma unroll
                    for (int i = 0; i < 16; ++i) {
                        const unsigned h2 = (hi[i] & 0x81ff81ffu) + e0 + (code_get(str, ne[i], b2) << 7);
                        finish(o, i, __uint_as_float(__builtin_amdgcn_perm(h2, lw[i], 0x05040100u)), __uint_as_float(__builtin_amdgcn_perm(h2, lw[i], 0x07060302u)));
                    }
                }
                const int blk = (k >> 3) * 4 + (k & 3);
                const char* ob = (const char*)(p.out + (size_t)((peer & 1) * 256 + (blk >> 1) * 64 + ((k >> 2) & 1) * 32) * GN + (peer >> 1) * 4096 + (cb + sk) * 128 + (blk & 1) * 64);
#pragma unroll
                for (int i = 0; i < 8; ++i)
                    st16(ob + (size_t)((i >> 2) * 16 + (i & 3)) * (GN * 4), oo, (hdr >> 31) ? nan4 : (u32x4){o[4 * i], o[4 * i + 1], o[4 * i + 2], o[4 * i + 3]});
            }
        }
        }
    }
    }
    if (tid == 0) __hip_atomic_store(ctr, call, RLX, SC_AGENT);
}

extern "C" void kernel_launch(int rank, int nranks, ncclComm_t comm, hipStream_t stream,
                              void* const* d_in, const int* in_sizes, int n_in,
                              void* d_out, int out_size, const int* in_dtypes, int out_dtype,
                              void* const* d_out_all, void* const* d_ws_all, size_t ws_bytes) {
    (void)comm; (void)in_dtypes; (void)out_dtype; (void)d_out_all;
    if (nranks != NR || (unsigned)rank >= (unsigned)NR || n_in != 2 || ws_bytes < WS_NEED ||
        in_sizes[0] != 4096 * KSH || in_sizes[1] != KSH * GN || out_size != 512 * GN) {
        fprintf(stderr, "kernel_launch: unexpected problem shape\n");
        return;
    }
    Params p;
    p.x = (const float*)d_in[0];
    p.w = (const float*)d_in[1];
    p.out = (float*)d_out;
    p.myws = (char*)d_ws_all[rank];
    for (int i = 0; i < NR; ++i) p.ws[i] = (char*)d_ws_all[i];
    p.rank = rank;
    p.pad = 0;
    int n = 0, np = 0, nd = 0;
    unsigned char deferred[8];
    bool px[8] = {};
    for (int i = 0; i < 128; ++i) p.order[0][i] = p.order[1][i] = 0;
    auto need_x = [&](int c0, int c1) { for (int c = c0; c <= c1; ++c) if (!px[c]) { px[c] = true; p.prod[np++] = (unsigned)(32 + c); } };
    for (int key = 0; key < 16 + 3 * SKEW2; ++key) {
        for (int pass = 0; pass < 2; ++pass) {
            if ((pass == 0) == (key != 0)) {
                for (int e = 32; e < 44; ++e)
                    if (2 * ((e - 32) * 8 / 12) == key) { p.order[0][n++] = (unsigned char)e; need_x((e - 32) * 8 / 12, ((e - 32) * 8 + 7) / 12); }
                if (key == XPOS) { p.order[0][n++] = 44; need_x(0, 3); }
                if (key == XPOS + 2) { p.order[0][n++] = 45; need_x(4, 7); }
                if (key == LPOS) { p.order[0][n++] = 46; p.order[0][n++] = 47; }
            } else {
                for (int b = 0; b < 4; ++b)
                    for (int c = 0; c < 8; ++c)
                        if ((b == 0 && c == 1 ? 0 : 2 * c + b * SKEW2) == key) {
                            p.prod[np++] = (unsigned)(c * 4 + b);
                            if (b == 3 && c < 2 * NFWD && ((c ^ rank) & 1)) deferred[nd++] = (unsigned char)(c * 4 + b);
                            else p.order[0][n++] = (unsigned char)(c * 4 + b);
                        }
            }
        }
    }
    int m = 0;
    for (int i = 0; i < n; ++i) if (p.order[0][i] < 44) p.order[1][m++] = p.order[0][i];
    for (int i = 0; i < nd; ++i) p.order[1][m++] = deferred[i];
    for (int i = 0; i < 6 * NFWD; ++i) p.order[1][m++] = (unsigned char)(48 + i);
    for (int e = 44; e < 48; ++e) p.order[1][m++] = (unsigned char)e;
    if (n != NU_OUT || m != NU_PAIR || np != 40 || p.prod[0] != 0 || p.prod[1] != 4 || p.prod[2] != 32) { fprintf(stderr, "kernel_launch: bad schedule\n"); return; }
    hipLaunchKernelGGL(fused_gemm_rs, dim3(NBLK), dim3(NTHR), 0, stream, p);
    HIP_CHECK(hipGetLastError());
}
```

```cpp
#include <hip/hip_runtime.h>
#include <rccl/rccl.h>
#include <cstdio>

#define HIP_CHECK(expr) do { hipError_t _e = (expr); if (_e != hipSuccess) { \
    fprintf(stderr, "HIP error %s at %s:%d\n", hipGetErrorString(_e), __FILE__, __LINE__); \
    return; } } while (0)

typedef float f32x4 __attribute__((ext_vector_type(4)));
typedef unsigned u32x4 __attribute__((ext_vector_type(4)));
typedef unsigned u32x3 __attribute__((ext_vector_type(3)));
typedef unsigned u32x2 __attribute__((ext_vector_type(2)));
typedef unsigned short u16x2 __attribute__((ext_vector_type(2)));
typedef _Float16 h16x8 __attribute__((ext_vector_type(8)));
typedef unsigned long long u64;
#define LAS __attribute__((address_space(3)))
#define GAS __attribute__((address_space(1)))
#define RLX __ATOMIC_RELAXED
#define SC_WG __HIP_MEMORY_SCOPE_WORKGROUP
#define SC_AGENT __HIP_MEMORY_SCOPE_AGENT
#define SC_SYS __HIP_MEMORY_SCOPE_SYSTEM

#define NPLAIN 3
#define TAU 864
#define TAU_OUT TAU
#define WIN_OUT WIN
#define EQ_AT 5
#define CREDIT 36
#define WIN 10
#define PROD_GAP 40
#define PUSH_SLOT 112
#define RSTRIDE 16
#define POLLW 12
#define PROD_OFF -2
#define B0_ITS 2
#define ROT 64
#define EQ2 1
#define T_PULL 370
#define PLAIN1 1
#define T1_DELAY 50
#define POLL_SLEEP 8
#define NFWD 3
#define XPOS 6
#define LPOS 18
#define SKEW2 2

constexpr int NR = 8;
constexpr int GN = 16384;
constexpr int KSH = 512;
constexpr int NBLK = 256;
constexpr int NTHR = 512;
constexpr int NSTEP = 64;
constexpr unsigned A_STEP = 32768, B_STEP = 16384, STAGE = A_STEP + B_STEP;
constexpr int NU_OUT = 48 - NFWD, NU_PAIR = 48 + 6 * NFWD;
#define RINGN 10
constexpr int RING = RINGN;
constexpr int PW = 14;
constexpr int POOL_IN = 48, POOL_OUT = 60;
constexpr int QB = 3328 + 4 * POOL_IN;
constexpr int PB = 6656 + 4 * POOL_OUT;
constexpr unsigned UNIT_T = (4 * QB + 127) / 128 * TAU / 8, PIECE_T = (PB + 127) / 128 * TAU_OUT / 8;
constexpr unsigned SLOT = 18432, SIDE = 14336;
constexpr size_t WS_CTR = 0;
constexpr size_t WS_FLAG = 32 << 10;
constexpr size_t WS_ORDY = 96 << 10;
constexpr size_t WS_MBOX = 128 << 10;
constexpr size_t WS_EQ = 160 << 10;
constexpr size_t WS_TMO = 192 << 10;
constexpr size_t WS_DIAG = 256 << 10;
constexpr size_t WS_STG = 1 << 20;
constexpr size_t STG_A = 8ull * NSTEP * A_STEP;
constexpr size_t STG_B = 32ull * NSTEP * B_STEP;
constexpr size_t WS_EXI = WS_STG + STG_A + STG_B;
constexpr size_t EXI_BYTES = 1600ull * SLOT;
constexpr size_t WS_EXO = WS_EXI + 2 * EXI_BYTES;
constexpr size_t WS_RDY = WS_EXO + (size_t)NBLK * 8 * 16384;
constexpr size_t WS_NEED = WS_RDY + 2 * 8 * 48 * 8 * 128;
constexpr u64 DEADLINE_TICKS = 50000000ull;

struct Params {
    const float* x;
    const float* w;
    float* out;
    char* myws;
    char* ws[NR];
    int rank;
    int pad;
    unsigned prod[40];
    unsigned char order[2][128];
};

#define WAIT_L0() do { asm volatile("s_waitcnt lgkmcnt(0)" ::: "memory"); __builtin_amdgcn_s_waitcnt(0xC07F); } while (0)
#define BAR() do { asm volatile("" ::: "memory"); __builtin_amdgcn_s_barrier(); asm volatile("" ::: "memory"); } while (0)
#define SB() __builtin_amdgcn_sched_barrier(0)
#define UNI(x) __builtin_amdgcn_readfirstlane(x)
template <int N> __device__ __forceinline__ void wait_vm() { asm volatile("s_waitcnt vmcnt(%0)" ::"i"(N) : "memory"); }

__device__ __forceinline__ void st16(const void* base, unsigned voff, u32x4 v) {
    asm volatile("s_nop 4\n\tglobal_store_dwordx4 %0, %1, %2 sc0 sc1\n\ts_nop 1" ::"v"(voff), "v"(v), "s"(base) : "memory");
}
__device__ __forceinline__ void st12(const void* base, unsigned voff, u32x3 v) {
    asm volatile("s_nop 4\n\tglobal_store_dwordx3 %0, %1, %2 sc0 sc1\n\ts_nop 1" ::"v"(voff), "v"(v), "s"(base) : "memory");
}
__device__ __forceinline__ void st8(const void* base, unsigned voff, u32x2 v) {
    asm volatile("s_nop 4\n\tglobal_store_dwordx2 %0, %1, %2 sc0 sc1" ::"v"(voff), "v"(v), "s"(base) : "memory");
}
__device__ __forceinline__ void st4(const void* base, unsigned voff, unsigned v) {
    asm volatile("s_nop 4\n\tglobal_store_dword %0, %1, %2 sc0 sc1" ::"v"(voff), "v"(v), "s"(base) : "memory");
}
__device__ __forceinline__ u32x3 ld12(const void* base, unsigned voff) {
    u32x3 r;
    asm volatile("s_nop 4\n\tglobal_load_dwordx3 %0, %1, %2 sc0 sc1\n\ts_waitcnt vmcnt(0)" : "=&v"(r) : "v"(voff), "s"(base) : "memory");
    return r;
}
__device__ __forceinline__ void ld64(const void* base, unsigned voff, u32x4& a, u32x4& b, u32x4& c, u32x4& d) {
    asm volatile("s_nop 4\n\tglobal_load_dwordx4 %0, %4, %5 sc0 sc1\n\tglobal_load_dwordx4 %1, %4, %5 offset:1024 sc0 sc1\n\t"
                 "global_load_dwordx4 %2, %4, %5 offset:2048 sc0 sc1\n\tglobal_load_dwordx4 %3, %4, %5 offset:3072 sc0 sc1\n\ts_waitcnt vmcnt(0)"
                 : "=&v"(a), "=&v"(b), "=&v"(c), "=&v"(d) : "v"(voff), "s"(base) : "memory");
}
template <bool TWO> __device__ __forceinline__ void dma(const void* base, unsigned voff, unsigned lds_dst) {
    unsigned keep;
    if (TWO) asm volatile("s_mov_b32 %0, m0\n\ts_mov_b32 m0, %3\n\ts_nop 4\n\tglobal_load_lds_dwordx4 %1, %2 sc0 sc1\n\tglobal_load_lds_dwordx4 %1, %2 offset:1024 sc0 sc1\n\ts_mov_b32 m0, %0"
                          : "=&s"(keep) : "v"(voff), "s"(base), "s"(lds_dst) : "memory");
    else asm volatile("s_mov_b32 %0, m0\n\ts_mov_b32 m0, %3\n\ts_nop 4\n\tglobal_load_lds_dwordx4 %1, %2 sc0 sc1\n\ts_mov_b32 m0, %0"
                      : "=&s"(keep) : "v"(voff), "s"(base), "s"(lds_dst) : "memory");
}
__device__ __forceinline__ unsigned now8() { return (unsigned)__builtin_amdgcn_s_memrealtime() << 6; }
__device__ __forceinline__ void pace(unsigned& t, unsigned period) {
    int late = (int)(now8() - t);
    if (late < -(1 << 22)) { t += (unsigned)late; late = 0; }
    while (late < 0) { __builtin_amdgcn_s_sleep(1); late = (int)(now8() - t); }
    t += period + (late > CREDIT * 64 ? (unsigned)(late - CREDIT * 64) : 0u);
}
__device__ __forceinline__ unsigned lds_get(const LAS unsigned* a) { return UNI(__hip_atomic_load(a, RLX, SC_WG)); }
__device__ __forceinline__ unsigned cvt2(float a, float b) {
    typedef _Float16 h2 __attribute__((ext_vector_type(2)));
    h2 r = {(_Float16)a, (_Float16)b};
    return __builtin_bit_cast(unsigned, r);
}
__device__ __forceinline__ int rb_owner(int d, int rb) { return rb >= 6 ? d : ((((d >> 1) + 1 + (rb >> 1)) & 3) << 1) | (rb & 1); }
__device__ __forceinline__ int rb_local(int d, int rb) { return rb >= 6 ? (rb - 6) * 256 : (d & 1) * 256; }
__device__ __forceinline__ int rb_of(int d, int o) { return ((((o >> 1) - (d >> 1) - 1) & 3) << 1) | (o & 1); }
__device__ __forceinline__ void unit_of(int e, int j, int& c, int& id) {
    if (e < 32) { c = e >> 2; id = j * 4 + (e & 3); }
    else if (e < 44) { const int u = (e - 32) * 8 + j; c = u / 12; id = 32 + u - c * 12; }
    else { const int u = (e & 1) * 8 + j; c = u >> 1; id = (e < 46 ? 46 : 44) + (u & 1); }
}
__device__ __forceinline__ int gid_of(int d, int id) {
    const int rb = (id - 32) >> 1;
    return id < 32 ? (d >> 1) * 32 + id : 128 + rb_owner(d, rb) * 4 + (rb_local(d, rb) >> 7) + (id & 1);
}
__device__ __forceinline__ size_t stage_off(int id, int ks) {
    return id < 32 ? STG_A + (size_t)(id * NSTEP + ks) * B_STEP : (size_t)(((id - 32) >> 1) * NSTEP + ks) * A_STEP + (id & 1) * 16384;
}
template <int BITS, int SH> __device__ __forceinline__ unsigned wave_max_exp(unsigned mag) {
    unsigned e = 0;
#pragma unroll
    for (int b = BITS - 1; b >= 0; --b) if (__any(mag >= ((e + (1u << b)) << SH))) e += 1u << b;
    return e;
}

__device__ __forceinline__ unsigned row_scan(unsigned x) {
    x += __builtin_amdgcn_update_dpp(0, x, 0x111, 0xf, 0xf, true);
    x += __builtin_amdgcn_update_dpp(0, x, 0x112, 0xf, 0xf, true);
    x += __builtin_amdgcn_update_dpp(0, x, 0x114, 0xf, 0xf, true);
    x += __builtin_amdgcn_update_dpp(0, x, 0x118, 0xf, 0xf, true);
    return x;
}
__device__ __forceinline__ unsigned ffbl(unsigned x) { unsigned r; asm("v_ffbl_b32 %0, %1" : "=v"(r) : "v"(x)); return r; }
#define PK(x) __builtin_bit_cast(u16x2, (unsigned)(x))
#define DW(x) __builtin_bit_cast(unsigned, (u16x2)(x))
#define SUBS(a, b) DW(__builtin_elementwise_sub_sat(PK(a), PK(b)))
#define MIN2(a, b) DW(__builtin_elementwise_min(PK(a), PK(b)))

template <int BITS> __device__ __forceinline__ int code_base(const unsigned (&ef)[16]) {
    u16x2 mx = PK(ef[0]);
#pragma unroll
    for (int i = 1; i < 16; ++i) mx = __builtin_elementwise_max(mx, PK(ef[i]));
    const int emax = (int)wave_max_exp<BITS, 0>(mx.x > mx.y ? mx.x : mx.y);
    int up = 0, mid = 0;
#pragma unroll
    for (int i = 0; i < 6; ++i) {
        const int e = (int)(ef[i] >> 16);
        up += __builtin_popcountll(__ballot(e == emax)) - __builtin_popcountll(__ballot(e == emax - 3));
        mid += __builtin_popcountll(__ballot(e == emax - 1)) - __builtin_popcountll(__ballot(e == emax - 4));
    }
    const int bs = emax - (up > 0 ? 3 : mid > 0 ? 4 : 5);
    return bs > 0 ? bs : 0;
}
template <int POOL> __device__ __forceinline__ bool code_put(unsigned (&ef)[16], unsigned B, LAS unsigned* pool, int row) {
    const unsigned b2 = B * 0x00010001u, lut = 0x0300u | ((B + 2u + (B ? 1u : 0u)) << 16);
    u64 str = 0;
    unsigned sh = 0;
#pragma unroll
    for (int i = 0; i < 16; ++i) {
        const unsigned k1 = SUBS(b2 + 0x00010001u, ef[i]), d = SUBS(ef[i], b2);
        const unsigned len = k1 + MIN2(SUBS(k1, 0x00020002u), 0x00010001u) + __builtin_amdgcn_perm(0u, lut, SUBS(d, 0x00030003u) | 0x0c000c00u);
        sh += len & 0xffffu; str |= 1ull << (sh & 63u);
        sh += len >> 16; str |= 1ull << (sh & 63u);
        ef[i] = MIN2(d, 0x00040004u) & 0x00030003u;
    }
    const unsigned end = row_scan(sh);
    const unsigned r0 = __builtin_amdgcn_readlane(end, 15), r1 = __builtin_amdgcn_readlane(end, 31), r2 = __builtin_amdgcn_readlane(end, 47);
    if (!__all(sh <= 63u && end <= 32u * (PW - 1)) || r0 + r1 + r2 + __builtin_amdgcn_readlane(end, 63) > 32u * (POOL - 1)) return false;
    const unsigned pos = 32u + end - sh + (row > 0 ? r0 : 0u) + (row > 1 ? r1 : 0u) + (row > 2 ? r2 : 0u);
    str >>= 1;
    const u64 s0 = str << (pos & 31u);
    LAS unsigned* pp = pool + (pos >> 5);
    __hip_atomic_store(pool, r0 | r1 << 10 | r2 << 20, RLX, SC_WG);
    __hip_atomic_fetch_or(pp, (unsigned)s0, RLX, SC_WG);
    __hip_atomic_fetch_or(pp + 1, (unsigned)(s0 >> 32), RLX, SC_WG);
    __hip_atomic_fetch_or(pp + 2, (unsigned)(((str >> 32) << (pos & 31u)) >> 32), RLX, SC_WG);
    return true;
}
template <int SH> __device__ __forceinline__ u64 code_find(const unsigned (&w)[16], unsigned (&ne)[16], const LAS unsigned* pool, int row) {
    const unsigned rl = pool[0];
    const unsigned start = 32u + (row > 0 ? rl & 1023u : 0u) + (row > 1 ? (rl >> 10) & 1023u : 0u) + (row > 2 ? (rl >> 20) & 1023u : 0u);
    pool += start >> 5;
    unsigned pd[PW - 1], sum = 0;
#pragma unroll
    for (int i = 0; i < PW - 1; ++i) pd[i] = pool[i];
#pragma unroll
    for (int i = 0; i < 16; ++i) { ne[i] = MIN2(w[i] & (0x00030003u << SH), 0x00010001u); sum += ne[i]; }
    sum = 32u - (sum & 0xffffu) - (sum >> 16);
    const unsigned skip = row_scan(sum) - sum + __builtin_popcount(__builtin_amdgcn_ubfe(pd[0], 0u, start));
    unsigned cum = 0, at = 0, below = 0;
#pragma unroll
    for (int i = 0; i < PW - 1; ++i) {
        cum += __builtin_popcount(pd[i]);
        const bool lt = cum < skip;
        at += lt; below = lt ? cum : below;
    }
    const unsigned d0 = pool[at], d1 = pool[at + 1], d2 = pool[at + 2];
    unsigned rem = skip - below, pos = 0;
#pragma unroll
    for (int sft = 16; sft; sft >>= 1) {
        const unsigned t = __builtin_popcount(__builtin_amdgcn_ubfe(d0, pos, sft));
        const bool lt = t < rem;
        rem -= lt ? t : 0u; pos += lt ? sft : 0u;
    }
    pos = skip ? pos + 1u : 0u;
    return (unsigned)((((u64)d1 << 32) | d0) >> pos) | ((((u64)d2 << 32) | d1) >> pos << 32);
}
__device__ __forceinline__ unsigned code_get(u64& str, unsigned ne, unsigned b2) {
    const unsigned l0 = ffbl(((unsigned)str << 1) | (ne & 1u)); str >>= l0 & 63u;
    const unsigned l1 = ffbl(((unsigned)str << 1) | (ne >> 16)); str >>= l1 & 63u;
    const unsigned len = l0 | (l1 << 16);
    const unsigned adj = __builtin_amdgcn_perm(0x02u, 0x07010100u, MIN2(len, 0x00040004u) | 0x0c000c00u);
    return MIN2(DW(PK(b2 + adj) - PK(len)), b2 + 0x00050005u);
}

typedef float f32x2 __attribute__((ext_vector_type(2)));
__device__ __forceinline__ f32x2 gelu2(f32x2 y) {
    const float k1 = -2.0f * 0.7978845608028654f * 1.4426950408889634f, k3 = k1 * 0.044715f;
    const f32x2 a = __builtin_elementwise_fma(y * y, (f32x2){k3, k3}, (f32x2){k1, k1}) * y;
    const f32x2 d = (f32x2){__builtin_amdgcn_exp2f(a.x), __builtin_amdgcn_exp2f(a.y)} + 1.0f;
    return y * (f32x2){__builtin_amdgcn_rcpf(d.x), __builtin_amdgcn_rcpf(d.y)};
}
__device__ __forceinline__ float gelu1(float y) { return gelu2((f32x2){y, y}).x; }

#define STAMP(i) do {} while (0)
#define TIMED_OUT(t0) (UNI(__hip_atomic_load(tmo, RLX, SC_AGENT)) == call || __builtin_amdgcn_s_memrealtime() - (t0) > DEADLINE_TICKS)

__global__ __launch_bounds__(NTHR, 2) void fused_gemm_rs(Params p) {
    __shared__ __attribute__((aligned(16))) unsigned char lds_raw[3 * STAGE + 2048];
    LAS unsigned char* lds = (LAS unsigned char*)lds_raw;
    LAS unsigned* ctl = (LAS unsigned*)(lds + 3 * STAGE);
    const unsigned lds0 = (unsigned)(size_t)lds;
    const int tid = threadIdx.x, lane = tid & 63;
    const int wid = UNI(tid >> 6);
    const int fr = lane & 15, fq = lane >> 4;
    const unsigned lo = lane * 16u;
    const unsigned bid = (blockIdx.x + ROT) & 255u;
    const int xcd = bid & 7, idx = bid >> 3;
    const bool late = idx < 8;
    const int q = wid & 3;

    const int team = (wid >> 2) * 192 + (int)bid - 64;
    f32x4 r[8];
    auto request = [&](int g, int u) {
        if (g < 32) {
            const GAS char* b = (const GAS char*)(p.w + (size_t)((g >> 2) * 64 + (q & 1) * 32) * GN + (u * 4 + (g & 3)) * 128 + (q >> 1) * 64) + (unsigned)(8 * fq * GN + 4 * fr) * 4u;
#pragma unroll
            for (int e = 0; e < 8; ++e) r[e] = *(const GAS f32x4*)(b + (size_t)e * (GN * 4));
        } else {
            const GAS char* b = (const GAS char*)(p.x + (size_t)(u * 128 + q * 32) * KSH + (g - 32) * 64) + (unsigned)(fr * KSH + 8 * fq) * 4u;
#pragma unroll
            for (int e = 0; e < 8; ++e) r[e] = *(const GAS f32x4*)(b + (e >> 2) * (16 * KSH * 4) + ((e >> 1) & 1) * 128 + (e & 1) * 16);
        }
    };
    if (!late && team < 96) request(team >= 64 ? 32 : (team >> 5) * 4, team & 31);
    const unsigned t_entry = now8();

    const int rank = p.rank & 7;
    char* myws = p.myws;
    STAMP(0);
    GAS unsigned* ctr = (GAS unsigned*)(myws + WS_CTR + (size_t)bid * 128);
    GAS unsigned* tmo = (GAS unsigned*)(myws + WS_TMO);
    const unsigned ctr_v = __hip_atomic_load(ctr, RLX, SC_AGENT);
    GAS unsigned* gflag = (GAS unsigned*)(myws + WS_FLAG);
    ctl[tid] = 0u;
    WAIT_L0();
    BAR();

    if (!late) {
        unsigned wlo = 0, whi = 0;
#pragma unroll
        for (int d = 0; d < NR; ++d) {
            unsigned a = (unsigned)(size_t)p.ws[d], b = (unsigned)((size_t)p.ws[d] >> 32);
            asm volatile("" : "+s"(a), "+s"(b));
            wlo = lane == d ? a : wlo;
            whi = lane == d ? b : whi;
        }
#pragma unroll 1
        for (int it = 0; it < 5; ++it) {
            const int t = it < B0_ITS ? it * 384 + team : B0_ITS * 384 + (it - B0_ITS) * 288 + (wid >> 2) * 144 + (((idx - 8) >> 2) * 3 + (idx & 3) - 1) * 8 + xcd;
            if (t >= 1280 || (it >= B0_ITS && (idx & 3) == 0)) break;
            const int g = p.prod[t >> 5], u = t & 31;
            if (t >= 96) {
                const unsigned ts = t_entry + (unsigned)((t >> 5) - PROD_OFF) * (PROD_GAP * 64);
                while ((int)(now8() - ts) < 0) __builtin_amdgcn_s_sleep(2);
                request(g, u);
            }
            const bool isw = g < 32;
            const int c = isw ? g >> 2 : g - 32;
            const int gid = isw ? u * 4 + (g & 3) : 128 + u;
            unsigned v[16];
            if (isw) {
#pragma unroll
                for (int i = 0; i < 16; ++i) v[i] = cvt2(r[(i & 3) * 2][i >> 2], r[(i & 3) * 2 + 1][i >> 2]);
            } else {
#pragma unroll
                for (int i = 0; i < 16; ++i) v[i] = cvt2(r[i >> 1][(i & 1) * 2], r[i >> 1][(i & 1) * 2 + 1]);
            }
            asm volatile("" : "+v"(v[15]));
            const unsigned t_loaded = now8();
            const bool pushed = t < 32;
            const unsigned call = UNI(ctr_v) + 1u;
            const size_t par = WS_EXI + (size_t)(call & 1u) * EXI_BYTES;
            const int o = u >> 2;
            const int myid = isw ? ((gid >> 5) == (rank >> 1) ? gid & 31 : -1)
                                 : (o == rank ? 44 + (u & 3) : ((o >> 1) != (rank >> 1) && ((u >> 1) & 1) == (rank & 1) ? 32 + rb_of(rank, o) * 2 + (u & 1) : -1));
            unsigned hdr = 0;
            {
                unsigned w[16];
#pragma unroll
                for (int i = 0; i < 16; ++i) w[i] = v[i];
                const char* home = myws + par + (size_t)(gid * 8 + c) * SLOT;
                unsigned pw = 0;
                hdr = 32u;
                if (t >= 32 + 32 * PLAIN1) {
                    LAS unsigned* pool = (LAS unsigned*)(lds + wid * 256);
                    pool[lane] = 0u;
                    unsigned ef[16];
#pragma unroll
                    for (int i = 0; i < 16; ++i) ef[i] = (w[i] >> 10) & 0x001f001fu;
                    const unsigned B = (unsigned)code_base<5>(ef);
                    if (code_put<POOL_IN>(ef, B, pool, fq)) {
#pragma unroll
                        for (int i = 0; i < 16; ++i) w[i] = (w[i] & 0x83ff83ffu) | (ef[i] << 10);
#pragma unroll
                        for (int k = 0; k < 3; ++k) {
#pragma unroll
                            for (int i = 0; i < 4; ++i) w[4 * k + i] |= (w[13 + k] << (12 - 3 * i)) & 0x70007000u;
                            w[12] |= (w[13 + k] >> (1 + k)) & (0x40004000u >> k);
                        }
                        pw = __hip_atomic_load(pool + lane, RLX, SC_WG);
                        hdr = B;
                    }
                }
                if (pushed) {
                    unsigned tp = t_loaded + (unsigned)((u & 7) * 4 + q) * PUSH_SLOT;
#pragma unroll
                    for (int i = 0; i < 4; ++i) {
                        pace(tp, 32u * PUSH_SLOT);
#pragma unroll
                        for (int k = 0; k < 2; ++k) {
                            const int d = (u >> 3) * 2 + k;
                            if (d != rank) st16(p.ws[d] + par + (size_t)(1280 + rank * 8 + (u & 7)) * SLOT + q * 4096 + i * 1024, lo, (u32x4){v[4 * i], v[4 * i + 1], v[4 * i + 2], v[4 * i + 3]});
                        }
                    }
                } else if (PLAIN1 && t < 64) {
#pragma unroll
                    for (int i = 0; i < 4; ++i) st16(home + q * 4096 + i * 1024, lo, (u32x4){v[4 * i], v[4 * i + 1], v[4 * i + 2], v[4 * i + 3]});
                } else {
#pragma unroll
                    for (int i = 0; i < 3; ++i) st16(home + q * QB + i * 1024, lo, (u32x4){w[4 * i], w[4 * i + 1], w[4 * i + 2], w[4 * i + 3]});
                    st4(home + q * QB + 3072, lane * 4u, w[12]);
                    if (hdr == 32u) st12(home + SIDE + q * 768, lane * 12u, (u32x3){v[13], v[14], v[15]});
                    else if (lane < POOL_IN) st4(home + q * QB + 3328, lane * 4u, pw);
                }
            }
            if (myid >= 0) {
                const char* sg = myws + WS_STG + stage_off(myid, rank * 8 + c) + (isw ? (q >> 1) * 8 + (q & 1) : q * 4) * 1024;
#pragma unroll
                for (int i = 0; i < 4; ++i) st16(sg + i * (isw ? 2048 : 1024), lo, (u32x4){v[4 * i], v[4 * i + 1], v[4 * i + 2], v[4 * i + 3]});
            }
            wait_vm<0>();
            unsigned tw = (1u << 24) | (hdr << (6 * q));
            if (lane == 0) tw += __hip_atomic_fetch_add(ctl + 402 + (wid >> 2) * 5 + it, tw, RLX, SC_WG);
            tw = UNI(tw);
            if ((tw >> 24) == 4u) {
                const int d = lane & 7;
                int did = isw ? ((gid >> 5) == (d >> 1) ? gid & 31 : -1)
                              : (o == d ? 44 + (u & 3) : ((o >> 1) != (d >> 1) && ((u >> 1) & 1) == (d & 1) ? 32 + rb_of(d, o) * 2 + (u & 1) : -1));
                if (isw && (g & 3) == 3 && c < 2 * NFWD && (d >> 1) != (rank >> 1) && ((c ^ d) & 1)) did = -1;
                if (lane < 8 && did >= 0) {
                    if (d == rank) __hip_atomic_store(gflag + did * 64 + c * 8 + rank, call, RLX, SC_AGENT);
                    else __hip_atomic_store((GAS u64*)((((size_t)whi << 32) | wlo) + WS_RDY) + ((((call & 1u) * 8 + rank) * 48 + did) * 8 + c) * RSTRIDE, ((u64)call << 32) | (tw & 0xffffffu), RLX, SC_SYS);
                }
            }
        }
        STAMP(9);
    } else if (idx < 7) {
        const int s = idx < 6 ? rb_owner(rank, idx) : rank ^ 1;
        const int nu = idx < 6 ? NU_OUT : NU_PAIR;
        unsigned desc[2];
#pragma unroll
        for (int k = 0; k < 2; ++k) {
            const int e = p.order[idx == 6][k * 64 + lane];
            int c, id, org = s, slot;
            if (e < 48) { unit_of(e, xcd, c, id); slot = gid_of(rank, id) * 8 + c; }
            else { const int kf = (e - 48) / 6; c = 2 * kf + (s & 1); id = xcd * 4 + 3; org = rb_owner(rank, e - 48 - kf * 6); slot = 1408 + (e - 48) * 8 + xcd; }
            desc[k] = (unsigned)(c | id << 3 | org << 9 | slot << 12);
        }
        auto desc_of = [&](int n) { return (unsigned)(n < 64 ? __builtin_amdgcn_readlane(desc[0], n) : __builtin_amdgcn_readlane(desc[1], n - 64)); };
        const unsigned call = UNI(ctr_v) + 1u;
        const size_t par = WS_EXI + (size_t)(call & 1u) * EXI_BYTES;
        GAS u64* eq = (GAS u64*)(myws + WS_EQ) + idx * 16;
        if (wid == 6) {
            const char* exs = p.ws[s] + par;
            unsigned tnext = t_entry + T_PULL * 64 + xcd * (TAU / 4);
            u64 todo = ~1ull;
            __builtin_amdgcn_s_setprio(3);
            const char* src = nullptr;
            unsigned trim = 0;
            auto choose = [&](int n) {
                const unsigned rdy = __hip_atomic_load(ctl + (n >= 64 ? n : lane), RLX, SC_WG);
                const unsigned fre = __hip_atomic_load(ctl + (n > RING - PLAIN1 ? 128 + n - RING : 400), RLX, SC_WG), tr = __hip_atomic_load(ctl + 400, RLX, SC_WG);
                const u64 rd = __ballot(rdy != 0u) & (n >= 64 ? 1ull : todo);
                if (!rd || (n > RING - PLAIN1 && UNI(fre) < 4u)) return;
                const int un = n >= 64 ? n : __builtin_ctzll(rd);
                todo &= ~(1ull << (un & 63));
                if (lane == 0) __hip_atomic_store(ctl + 416 + n, (unsigned)un, RLX, SC_WG);
                trim = n >= EQ_AT + 4 && n < EQ_AT + 36 ? UNI(tr) : 0u;
                src = exs + (size_t)(desc_of(un) >> 12) * SLOT;
            };
            if (PLAIN1) {
                while (lds_get(ctl + 1) == 0u) __builtin_amdgcn_s_sleep(1);
                if (T1_DELAY && lds_get(ctl) == 0u) { const unsigned t1 = now8() + T1_DELAY * 64; if ((int)(t1 - tnext) > 0) tnext = t1; }
                todo = ~3ull;
                if (lane == 0) __hip_atomic_store(ctl + 417, 1u, RLX, SC_WG);
                const char* cur = exs + (size_t)(desc_of(1) >> 12) * SLOT;
#pragma unroll
                for (int i = 0; i < 16; i += 2) {
                    wait_vm<WIN - 2>();
                    if (i > WIN - 2 && lane == 0) __hip_atomic_store(ctl + 384, (unsigned)(i - (WIN - 2)), RLX, SC_WG);
                    pace(tnext, 2u * TAU);
                    dma<true>(cur + i * 1024, lo, lds0 + 12288u + i * 1024);
                    if (i == 12) choose(2);
                }
            }
            for (int n = 1 + PLAIN1; n < nu; ++n) {
                bool drained = false;
                if (!src) {
                    choose(n);
                    if (!src) {
                        wait_vm<0>();
                        if (lane == 0) __hip_atomic_store(ctl + 384, (unsigned)(n - 1) * 14u + 2 * PLAIN1, RLX, SC_WG);
                        drained = true;
                        for (choose(n); !src; choose(n)) __builtin_amdgcn_s_sleep(1);
                    }
                }
                const char* cur = src;
                const unsigned dst = lds0 + (unsigned)(n % RING) * 14336u, tr = trim;
                src = nullptr;
                asm volatile("" ::: "memory");
#pragma unroll
                for (int i = 0; i < 14; i += 2) {
                    wait_vm<WIN - 2>();
                    if ((i > WIN - 2 || (!drained && n > 1)) && lane == 0) __hip_atomic_store(ctl + 384, (unsigned)((n - 1) * 14 + 2 * PLAIN1 + i - (WIN - 2)), RLX, SC_WG);
                    pace(tnext, (i < 12 ? 2u * TAU : UNIT_T - 12u * TAU) - tr);
                    if (i < 12) dma<true>(cur + i * 1024, lo, dst + i * 1024);
                    else {
                        dma<false>(cur + 12288, lo, dst + 12288);
                        if (lane < (4 * QB - 13312) / 16) dma<false>(cur + 13312, lo, dst + 13312);
                    }
                    if (i == 0 && lane == 0) {
                        if (n == EQ_AT) __hip_atomic_store(ctl + 387, tnext | 1u, RLX, SC_WG);
                        if (n == nu - 3) __hip_atomic_store(ctl + 386, tnext | 1u, RLX, SC_WG);
                        if (n == nu - 1) __hip_atomic_store(ctl + 385, (tnext + UNIT_T - 2u * TAU) | 1u, RLX, SC_WG);
                    }
                    if (i == 10 && n + 1 < nu) choose(n + 1);
                }
            }
            __builtin_amdgcn_s_setprio(0);
            wait_vm<0>();
            if (lane == 0) __hip_atomic_store(ctl + 384, (unsigned)(nu - 1) * 14u + 2 * PLAIN1, RLX, SC_WG);
            STAMP(9);
        } else {
        if (wid == 7) {
            const GAS u64* rp[2];
            bool pend[2];
#pragma unroll
            for (int k = 0; k < 2; ++k) {
                rp[k] = (const GAS u64*)(myws + WS_RDY) + ((((call & 1u) * 8 + ((desc[k] >> 9) & 7)) * 48 + ((desc[k] >> 3) & 63)) * 8 + (desc[k] & 7)) * RSTRIDE;
                pend[k] = k * 64 + lane < nu;
            }
            unsigned mine = 0;
            bool eq_open = EQ_AT != 0;
            const u64 t0 = __builtin_amdgcn_s_memrealtime();
            for (unsigned polls = 1;; ++polls) {
                const u64 pm = __ballot(pend[0]);
                const bool front = __builtin_amdgcn_mbcnt_hi((unsigned)(pm >> 32), __builtin_amdgcn_mbcnt_lo((unsigned)pm, 0u)) < POLLW;
#pragma unroll
                for (int k = 0; k < 2; ++k)
                    if (pend[k] && ((k == 0 && front) || (polls & 7u) == 0u)) {
                        const u64 f = __hip_atomic_load(rp[k], RLX, SC_SYS);
                        if ((unsigned)(f >> 32) >= call) { __hip_atomic_store(ctl + k * 64 + lane, (unsigned)f | 0x80000000u, RLX, SC_WG); pend[k] = false; }
                    }
                if (eq_open) {
                    if (!mine) {
                        mine = lds_get(ctl + 387);
                        if (mine && lane == 0) __hip_atomic_store(eq + xcd, ((u64)call << 32) | mine, RLX, SC_AGENT);
                    } else {
                        const u64 f = __hip_atomic_load(eq + (lane & 7), RLX, SC_AGENT);
                        if (__all((unsigned)(f >> 32) == call)) {
                            int dlt = (int)(mine - (unsigned)f);
                            dlt += __shfl_xor(dlt, 1); dlt += __shfl_xor(dlt, 2); dlt += __shfl_xor(dlt, 4);
                            dlt /= 8 * 32 * 7;
                            dlt = dlt > TAU / 8 ? TAU / 8 : dlt < -(TAU / 8) ? -(TAU / 8) : dlt;
                            if (lane == 0) __hip_atomic_store(ctl + 400, (unsigned)dlt, RLX, SC_WG);
                            eq_open = false;
                        } else if ((int)lds_get(ctl + 384) >= (EQ_AT + 2) * 14) eq_open = false;
                    }
                }
                const bool waiting = __any(pend[0] || pend[1]);
                if (!waiting && !eq_open) break;
                if (waiting && (polls & 15u) == 0u && TIMED_OUT(t0)) {
                    if (lane == 0) __hip_atomic_store(tmo, call, RLX, SC_AGENT);
                    wait_vm<0>();
#pragma unroll
                    for (int k = 0; k < 2; ++k) if (pend[k]) __hip_atomic_store(ctl + k * 64 + lane, 0x80000000u, RLX, SC_WG);
                    break;
                }
                if (polls < 32u) __builtin_amdgcn_s_sleep(1); else __builtin_amdgcn_s_sleep(16);
            }
        } else {
            int pend = -1, pend_n = 0;
            unsigned pend_h = 0;
            auto staged = [&]() {
                wait_vm<0>();
                if (lane == 0 && __hip_atomic_fetch_add(ctl + 256 + pend_n, 1u, RLX, SC_WG) == 3u) {
                    __hip_atomic_store(gflag + pend, call, RLX, SC_AGENT);
                    if (pend_h) __hip_atomic_store((GAS u64*)(p.ws[rank ^ 1] + WS_RDY) + ((((call & 1u) * 8 + s) * 48 + (pend >> 6)) * 8 + ((pend >> 3) & 7)) * RSTRIDE,
                                                   ((u64)call << 32) | (pend_h & 0xffffffu), RLX, SC_SYS);
                }
                pend = -1;
            };
            auto need = [&](int job) {
                return PLAIN1 && job < 8 ? (job - 3) * 4 : (job >> 2) * 14 - 14 + 2 * PLAIN1 + (((job & 3) + 1) * QB + 1023) / 1024; };
            auto ticket = [&]() { unsigned tk = 0; if (lane == 0) tk = __hip_atomic_fetch_add(ctl + 401, 1u, RLX, SC_WG); return (int)UNI(tk); };
            for (int job = ticket(); job < 4 * nu;) {
                const int n = job >> 2, qq = job & 3;
                unsigned w[16];
                int un = 0;
                unsigned relay_h = 0;
                if (n == 0) {
                    while (lds_get(ctl) == 0u) __builtin_amdgcn_s_sleep(1);
                    u32x4 a0, a1, a2, a3;
                    ld64(myws + par + (size_t)(1280 + s * 8 + xcd) * SLOT + qq * 4096, lo, a0, a1, a2, a3);
#pragma unroll
                    for (int i = 0; i < 4; ++i) { w[i] = a0[i]; w[4 + i] = a1[i]; w[8 + i] = a2[i]; w[12 + i] = a3[i]; }
                } else {
                    while ((int)lds_get(ctl + 384) < need(job)) __builtin_amdgcn_s_sleep(1);
                    asm volatile("" ::: "memory");
                    un = lds_get(ctl + 416 + n);
                }
                const unsigned de = desc_of(un);
                const int c = de & 7, id = (de >> 3) & 63, org = (de >> 9) & 7;
                if (PLAIN1 && n == 1) {
#pragma unroll
                    for (int i = 0; i < 4; ++i) { const u32x4 a = *(const LAS u32x4*)(lds + 12288 + qq * 4096 + i * 1024 + lo); w[4 * i] = a[0]; w[4 * i + 1] = a[1]; w[4 * i + 2] = a[2]; w[4 * i + 3] = a[3]; }
                    WAIT_L0();
                    if (lane == 0) { __hip_atomic_fetch_add(ctl + 128, 1u, RLX, SC_WG); __hip_atomic_fetch_add(ctl + 129, 1u, RLX, SC_WG); }
                    if (pend >= 0) staged();
                } else if (n) {
                    const LAS unsigned char* src = lds + (unsigned)(n % RING) * 14336u + qq * QB;
#pragma unroll
                    for (int i = 0; i < 3; ++i) { const u32x4 a = *(const LAS u32x4*)(src + i * 1024 + lo); w[4 * i] = a[0]; w[4 * i + 1] = a[1]; w[4 * i + 2] = a[2]; w[4 * i + 3] = a[3]; }
                    w[12] = *(const LAS unsigned*)(src + 3072 + lane * 4);
                    const LAS unsigned* pool = (const LAS unsigned*)(src + 3328);
                    const unsigned hu = lds_get(ctl + un), h = (hu >> (6 * qq)) & 63u;
                    if (idx < 6 && id < 32 && (id & 3) == 3 && c < 2 * NFWD) {
                        const unsigned pw = pool[lane];
                        if (pend >= 0) staged();
                        const char* dst = myws + par + (size_t)(1408 + ((c >> 1) * 6 + idx) * 8 + xcd) * SLOT + qq * QB;
#pragma unroll
                        for (int i = 0; i < 3; ++i) st16(dst + i * 1024, lo, (u32x4){w[4 * i], w[4 * i + 1], w[4 * i + 2], w[4 * i + 3]});
                        st4(dst + 3072, lane * 4u, w[12]);
                        if (lane < POOL_IN) st4(dst + 3328, lane * 4u, pw);
                        relay_h = hu;
                    }
                    if (h < 32u) {
#pragma unroll
                        for (int k = 0; k < 3; ++k) {
                            unsigned cc = (w[12] << (1 + k)) & 0x80008000u;
#pragma unroll
                            for (int i = 0; i < 4; ++i) cc |= (w[4 * k + i] >> (12 - 3 * i)) & (0x00070007u << (3 * i));
                            w[13 + k] = cc;
                        }
                        unsigned ne[16];
                        u64 str = code_find<10>(w, ne, pool, fq);
                        WAIT_L0();
                        if (lane == 0) __hip_atomic_fetch_add(ctl + 128 + n, 1u, RLX, SC_WG);
                        if (pend >= 0) staged();
#pragma unroll
                        for (int i = 0; i < 16; ++i) w[i] = (w[i] & 0x8fff8fffu) + (code_get(str, ne[i], h * 0x00010001u) << 10);
                    } else {
                        WAIT_L0();
                        if (lane == 0) __hip_atomic_fetch_add(ctl + 128 + n, 1u, RLX, SC_WG);
                        if (pend >= 0) staged();
                        const u32x3 a = ld12(p.ws[org] + par + (size_t)(gid_of(rank, id) * 8 + c) * SLOT + SIDE + qq * 768, lane * 12u);
                        w[13] = a[0]; w[14] = a[1]; w[15] = a[2];
                    }
                }
                const char* sg = myws + WS_STG + stage_off(id, org * 8 + c) + (id < 32 ? (qq >> 1) * 8 + (qq & 1) : qq * 4) * 1024;
#pragma unroll
                for (int i = 0; i < 4; ++i) st16(sg + i * (id < 32 ? 2048 : 1024), lo, (u32x4){w[4 * i], w[4 * i + 1], w[4 * i + 2], w[4 * i + 3]});
                pend = id * 64 + c * 8 + org; pend_n = n; pend_h = relay_h;
                job = ticket();
                if (job >= 4 * nu || job < 4 || (int)lds_get(ctl + 384) < need(job)) staged();
            }
        }
            if (wid == 7 && idx < 6) {
                unsigned mine;
                while ((mine = lds_get(ctl + 386)) == 0u) __builtin_amdgcn_s_sleep(8);
                if (lane == 0) __hip_atomic_store(eq + 8 + xcd, ((u64)call << 32) | mine, RLX, SC_AGENT);
                unsigned th;
                int dlt = 0;
                bool open = EQ2;
                while ((th = lds_get(ctl + 385)) == 0u) {
                    if (open) {
                        const u64 f = __hip_atomic_load(eq + 8 + (lane & 7), RLX, SC_AGENT);
                        if (__all((unsigned)(f >> 32) == call)) {
                            dlt = (int)(mine - (unsigned)f);
                            dlt += __shfl_xor(dlt, 1); dlt += __shfl_xor(dlt, 2); dlt += __shfl_xor(dlt, 4);
                            dlt /= 8 * 256;
                            dlt = dlt > 100 ? 100 : dlt < -100 ? -100 : dlt;
                            open = false;
                        }
                    }
                    __builtin_amdgcn_s_sleep(4);
                }
                if (lane == 0) __hip_atomic_store((GAS u64*)(myws + WS_MBOX + (size_t)((8 + idx * 4) * 8 + xcd) * 128), ((u64)call << 32) | (th & ~255u) | (unsigned)(dlt + 128), RLX, SC_AGENT);
            }
        }
    }
    const unsigned call = UNI(ctr_v) + 1u;

    const int ntile = idx == 7 ? 2 : 1;
    const int wm = wid >> 1, wn = wid & 1;
#pragma unroll 1
    for (int rep = 0; rep < ntile; ++rep) {
    const int rb = late ? 6 + (idx >> 2) : (idx - 8) >> 2;
    const int cb = xcd * 4 + (idx == 7 ? rep : idx == 6 ? 0 : idx == 4 || idx == 5 ? idx - 2 : idx & 3);
    const bool part1 = idx == 7 && rep == 0, part2 = idx == 6;
    const int nstep = part1 ? 56 : part2 ? 8 : NSTEP;
    const char* stgA = myws + WS_STG + (size_t)(rb * NSTEP) * A_STEP + wid * 4096;
    const char* stgB = myws + WS_STG + STG_A + (size_t)(cb * NSTEP) * B_STEP + wid * 2048;
    const u64 pbits = 0x0101010101010101ull << (rank ^ 1);
    u64 todo = part1 ? ~pbits : part2 ? pbits : ~0ull, avail = 0ull;
    int slot = 0;
    auto poll = [&]() {
        if (wid == 0) {
            const u64 t0 = __builtin_amdgcn_s_memrealtime();
            u64 got;
            for (unsigned polls = 1;; ++polls) {
                const unsigned f0 = __hip_atomic_load(gflag + cb * 64 + lane, RLX, SC_AGENT);
                const unsigned f1 = __hip_atomic_load(gflag + (32 + rb * 2) * 64 + lane, RLX, SC_AGENT);
                const unsigned f2 = __hip_atomic_load(gflag + (33 + rb * 2) * 64 + lane, RLX, SC_AGENT);
                got = __ballot(f0 >= call && f1 >= call && f2 >= call) & todo;
                if (got) break;
                if ((polls & 15u) == 0u && TIMED_OUT(t0)) {
                    if (lane == 0) __hip_atomic_store(tmo, call, RLX, SC_AGENT);
                    got = todo;
                    break;
                }
                __builtin_amdgcn_s_sleep(POLL_SLEEP);
            }
            if (lane < 2) ctl[388 + slot * 2 + lane] = (unsigned)(got >> (32 * lane));
        }
    };
    auto news = [&]() { avail = ((u64)(unsigned)UNI(ctl[389 + slot * 2]) << 32) | (unsigned)UNI(ctl[388 + slot * 2]); slot ^= 1; };
    auto take = [&]() {
        const unsigned b = (unsigned)__builtin_ctzll(avail) & 63u;
        avail &= avail - 1; todo &= ~(1ull << b);
        return (b & 7u) * 8u + (b >> 3);
    };
    auto g1 = [&](unsigned ks, unsigned buf, int i) {
        if (i < 4) __builtin_amdgcn_global_load_lds((const GAS unsigned*)(stgA + (size_t)ks * A_STEP + i * 1024 + lo), (LAS unsigned*)(lds + buf + wid * 4096 + i * 1024), 16, 0, 17);
        else __builtin_amdgcn_global_load_lds((const GAS unsigned*)(stgB + (size_t)ks * B_STEP + (i - 4) * 1024 + lo), (LAS unsigned*)(lds + buf + A_STEP + wid * 2048 + (i - 4) * 1024), 16, 0, 17);
    };

    f32x4 acc[4][4];
#pragma unroll
    for (int m = 0; m < 4; ++m)
#pragma unroll
        for (int n = 0; n < 4; ++n) acc[m][n] = (f32x4){0.f, 0.f, 0.f, 0.f};
    h16x8 a0[4], b0[4], a1[4], b1[4];
    const unsigned aoff = wm * 8192 + lo, boff = A_STEP + wn * 8192 + lo;
#define RDA(A, buf, kk, m) A[m] = *(const LAS h16x8*)(lds + (buf) + aoff + (m) * 2048 + (kk) * 1024)
#define RDB(B, buf, kk, n) B[n] = *(const LAS h16x8*)(lds + (buf) + boff + (n) * 2048 + (kk) * 1024)
#define ROW(A, B, m) _Pragma("unroll") for (int n = 0; n < 4; ++n) acc[m][n] = __builtin_amdgcn_mfma_f32_16x16x32_f16(A[m], B[n], acc[m][n], 0, 0, 0)

    STAMP(10);
    unsigned kq;
#pragma unroll
    for (int k = 0; k < 3; ++k) {
        if (!avail) { poll(); WAIT_L0(); BAR(); news(); }
        kq = take();
#pragma unroll
        for (int i = 0; i < (k < 2 ? 6 : 3); ++i) g1(kq, k * STAGE, i);
    }
    wait_vm<9>(); BAR();
#pragma unroll
    for (int i = 0; i < 4; ++i) { RDB(b0, 0u, 0, i); RDA(a0, 0u, 0, i); }
    unsigned cur = 0, prv = 2 * STAGE;
    for (int t = 0; t < nstep; ++t) {
        const unsigned nxt = cur == 2 * STAGE ? 0u : cur + STAGE;
        const bool m2 = t + 2 < nstep, m3 = t + 3 < nstep;
        const bool dry = m3 && !avail;
        WAIT_L0(); SB();
        RDB(b1, cur, 1, 0); RDB(b1, cur, 1, 1); SB(); ROW(a0, b0, 0); SB(); if (m2) g1(kq, prv, 3); SB();
        RDB(b1, cur, 1, 2); RDB(b1, cur, 1, 3); SB(); ROW(a0, b0, 1); SB(); if (m2) g1(kq, prv, 4); SB();
        RDA(a1, cur, 1, 0); RDA(a1, cur, 1, 1); SB(); ROW(a0, b0, 2); SB(); if (m2) g1(kq, prv, 5); SB();
        RDA(a1, cur, 1, 2); RDA(a1, cur, 1, 3); SB(); ROW(a0, b0, 3); SB();
        if (dry) poll();
        WAIT_L0();
        if (m2) wait_vm<6>(); else wait_vm<0>();
        BAR(); SB();
        RDB(b0, nxt, 0, 0); RDB(b0, nxt, 0, 1); SB(); ROW(a1, b1, 0); SB();
        if (dry) news();
        if (m3) { kq = take(); g1(kq, cur, 0); } SB();
        RDB(b0, nxt, 0, 2); RDB(b0, nxt, 0, 3); SB(); ROW(a1, b1, 1); SB(); if (m3) g1(kq, cur, 1); SB();
        RDA(a0, nxt, 0, 0); RDA(a0, nxt, 0, 1); SB(); ROW(a1, b1, 2); SB(); if (m3) g1(kq, cur, 2); SB();
        RDA(a0, nxt, 0, 2); RDA(a0, nxt, 0, 3); SB(); ROW(a1, b1, 3); SB();
        prv = cur; cur = nxt;
    }

    if (part1 || part2) {
        const char* park = myws + WS_EXO + (size_t)(xcd * 8 + wid) * 16384;
        GAS u64* pf = (GAS u64*)(myws + WS_ORDY + (size_t)xcd * 128) + wid;
        if (part1) {
#pragma unroll
            for (int i = 0; i < 16; ++i) st16(park + i * 1024, lo, __builtin_bit_cast(u32x4, acc[i >> 2][i & 3]));
            wait_vm<0>();
            if (lane == 0) __hip_atomic_store(pf, (u64)call << 32, RLX, SC_SYS);
            continue;
        }
        const u64 t0 = __builtin_amdgcn_s_memrealtime();
        while ((unsigned)UNI((unsigned)(__hip_atomic_load(pf, RLX, SC_SYS) >> 32)) < call) {
            if (TIMED_OUT(t0)) { if (lane == 0) __hip_atomic_store(tmo, call, RLX, SC_AGENT); wait_vm<0>(); break; }
            __builtin_amdgcn_s_sleep(4);
        }
#pragma unroll
        for (int m = 0; m < 4; ++m) {
            u32x4 e0, e1, e2, e3;
            ld64(park + m * 4096, lo, e0, e1, e2, e3);
            acc[m][0] += __builtin_bit_cast(f32x4, e0); acc[m][1] += __builtin_bit_cast(f32x4, e1);
            acc[m][2] += __builtin_bit_cast(f32x4, e2); acc[m][3] += __builtin_bit_cast(f32x4, e3);
        }
    }
    const unsigned tmo_v = __hip_atomic_load(tmo, RLX, SC_AGENT);
    const unsigned oo = (unsigned)(fq * 4) * (GN * 4u) + fr * 16u;
    const u32x4 nan4 = {0x7fc00000u, 0x7fc00000u, 0x7fc00000u, 0x7fc00000u};
    auto finish = [&](unsigned (&o)[32], int i, float a, float b) {
        const f32x2 g = gelu2((f32x2){a, b});
        const int at = ((i >> 3) * 4 + (i & 1) * 2) * 4 + ((i >> 1) & 3);
        o[at] = __float_as_uint(g.x); o[at + 4] = __float_as_uint(g.y);
    };
#define ACC(hf, i, k) acc[2 * (hf) + ((i) >> 3)][((i) >> 1) & 3][((i) & 1) * 2 + (k)]
    if (late) {
        const bool bad = UNI(tmo_v) == call;
        const char* ob = (const char*)(p.out + (size_t)(rb_local(rank, rb) + wm * 64) * GN + (rank >> 1) * 4096 + cb * 128 + wn * 64);
#pragma unroll
        for (int hf = 0; hf < 2; ++hf) {
            unsigned o[32];
#pragma unroll
            for (int i = 0; i < 16; ++i) finish(o, i, ACC(hf, i, 0), ACC(hf, i, 1));
#pragma unroll
            for (int i = 0; i < 8; ++i) st16(ob + (size_t)(hf * 32 + (i >> 2) * 16 + (i & 3)) * (GN * 4), oo, bad ? nan4 : (u32x4){o[4 * i], o[4 * i + 1], o[4 * i + 2], o[4 * i + 3]});
        }
    } else {
        const int peer = rb_owner(rank, rb);
        const int mirror = (8 + rb_of(peer, rank) * 4 + (idx & 3)) * 8 + xcd;
        char* const peerws = p.ws[peer];
        WAIT_L0(); BAR();
        if (wid < 4) __builtin_amdgcn_s_setprio(2);
#pragma unroll
        for (int hf = 0; hf < 2; ++hf) {
            const int k = (wid >> 2) * 8 + hf * 4 + q;
            const char* ex = myws + WS_EXO + (size_t)(bid * 16 + k) * 8192;
            unsigned hdr = 1u << 13;
            if ((idx & 3) != 3 || k < 16 - NPLAIN) {
                LAS unsigned* pool = (LAS unsigned*)(lds + 131072 + wid * 256);
                pool[lane] = 0u;
                unsigned hi[16], ef[16], under = 0;
#pragma unroll
                for (int i = 0; i < 16; ++i) {
                    hi[i] = __builtin_amdgcn_perm(__float_as_uint(ACC(hf, i, 1)), __float_as_uint(ACC(hf, i, 0)), 0x07060302u);
                    ef[i] = (hi[i] >> 7) & 0x00ff00ffu;
                }
                const unsigned bs = (unsigned)code_base<8>(ef), e0 = bs > 20u ? bs - 20u : 0u;
#pragma unroll
                for (int i = 0; i < 16; ++i) { under |= SUBS(e0 * 0x00010001u, ef[i]); ef[i] = SUBS(ef[i], e0 * 0x00010001u); }
                if (__all(under == 0u) && code_put<POOL_OUT>(ef, bs - e0, pool, fq)) {
#pragma unroll
                    for (int i = 0; i < 16; ++i) hi[i] = (hi[i] & 0x807f807fu) | (ef[i] << 7);
#pragma unroll
                    for (int c = 0; c < 6; ++c) {
#pragma unroll
                        for (int i = 0; i < 3; ++i) { const int fld = c * 3 + i, at = 9 + 3 * (fld & 1); hi[fld >> 1] |= (hi[10 + c] << (at - 3 * i)) & (0x00070007u << at); }
                        hi[9] |= (hi[10 + c] >> (1 + c)) & (0x40004000u >> c);
                    }
                    const unsigned pw = __hip_atomic_load(pool + lane, RLX, SC_WG);
#pragma unroll
                    for (int i = 0; i < 4; ++i) {
                        u32x4 v;
#pragma unroll
                        for (int e = 0; e < 4; ++e) v[e] = __builtin_amdgcn_perm(__float_as_uint(ACC(hf, 4 * i + e, 1)), __float_as_uint(ACC(hf, 4 * i + e, 0)), 0x05040100u);
                        st16(ex + i * 1024, lo, v);
                    }
                    st16(ex + 4096, lo, (u32x4){hi[0], hi[1], hi[2], hi[3]});
                    st16(ex + 5120, lo, (u32x4){hi[4], hi[5], hi[6], hi[7]});
                    st8(ex + 6144, lane * 8u, (u32x2){hi[8], hi[9]});
                    if (lane < POOL_OUT) st4(ex + 6656, lane * 4u, pw);
                    hdr = e0 | ((bs - e0) << 8);
                }
            }
            if (hdr >> 13) {
                unsigned o[32];
#pragma unroll
                for (int i = 0; i < 16; ++i) finish(o, i, ACC(hf, i, 0), ACC(hf, i, 1));
#pragma unroll
                for (int i = 0; i < 8; ++i) st16(ex + i * 1024, lo, (u32x4){o[4 * i], o[4 * i + 1], o[4 * i + 2], o[4 * i + 3]});
            }
            wait_vm<0>();
            hdr |= UNI(tmo_v) == call ? 0xc0000000u : 0x40000000u;
            if (lane == 0) __hip_atomic_store((GAS u64*)(peerws + WS_ORDY + (size_t)mirror * 128) + k, ((u64)call << 32) | hdr, RLX, SC_SYS);
        }
        __builtin_amdgcn_s_setprio(0);
        STAMP(22);
#pragma unroll 1
        for (int sk = 0; (idx & 3) + sk <= 3 && (idx & 3) != 3; sk += 3) {
        if (sk) {
            BAR();
            if (tid < 32 || tid == 384 || tid == 401 || (tid >= 416 && tid < 432)) ctl[tid] = 0u;
            WAIT_L0(); BAR();
        }
        const int me = bid + 8 * sk;
        const char* exs = peerws + WS_EXO + (size_t)(mirror + 8 * sk) * (16 * 8192);
        if (wid == 0) {
            unsigned tnext;
            while ((tnext = lds_get(ctl + 16)) == 0u) __builtin_amdgcn_s_sleep(2);
            STAMP(23);
            const unsigned trb = tnext & 255u, tr = trb - 128u;
            tnext = (tnext & ~255u) | 128u;
            __builtin_amdgcn_s_setprio(3);
            wait_vm<0>(); __builtin_amdgcn_s_waitcnt(0x0F70);
            int issued = 0, told = 0;
            unsigned hn = lds_get(ctl);
            for (int n = 0; n < 16; ++n) {
                if (hn == 0u && (hn = lds_get(ctl + n)) == 0u) {
                    wait_vm<0>();
                    told = issued;
                    if (lane == 0) __hip_atomic_store(ctl + 384, (unsigned)told, RLX, SC_WG);
                    while ((hn = lds_get(ctl + n)) == 0u) __builtin_amdgcn_s_sleep(1);
                }
                asm volatile("" ::: "memory");
                const bool plain = (hn >> 13) & 1u;
                const char* cur = exs + (size_t)n * 8192;
                const unsigned dst = lds0 + (unsigned)n * 8192u;
#pragma unroll
                for (int i = 0; i < 8; i += 2) {
                    wait_vm<WIN_OUT - 2>();
                    if (issued - (WIN_OUT - 2) > told) { told = issued - (WIN_OUT - 2); if (lane == 0) __hip_atomic_store(ctl + 384, (unsigned)told, RLX, SC_WG); }
                    pace(tnext, (i < 6 || plain ? 2u * TAU_OUT : PIECE_T - 6u * TAU_OUT) - tr);
                    if (i < 6 || plain) { dma<true>(cur + i * 1024, lo, dst + i * 1024); issued += 2; }
                    else { if (lane < (PB - 6144) / 16) dma<false>(cur + 6144, lo, dst + 6144); issued += 1; }
                    if (i == 0) {
                        if (lane == 0) __hip_atomic_store(ctl + 416 + n, (unsigned)issued + (plain ? 6u : 5u), RLX, SC_WG);
                        if (n == 14 && !sk && lane == 0)
                            __hip_atomic_store((GAS u64*)(myws + WS_MBOX + (size_t)(bid + 8) * 128), ((u64)call << 32) | ((tnext + 2u * (PIECE_T) - 2u * TAU_OUT - 7u * tr) & ~255u) | trb, RLX, SC_AGENT);
                    }
                    if (i == 2) hn = n < 15 ? lds_get(ctl + n + 1) : 0u;
                }
            }
            wait_vm<7>();
            if (issued - 7 > told && lane == 0) __hip_atomic_store(ctl + 384, (unsigned)issued - 7u, RLX, SC_WG);
            wait_vm<0>();
            if (lane == 0) __hip_atomic_store(ctl + 384, (unsigned)issued, RLX, SC_WG);
            __builtin_amdgcn_s_setprio(0);
            STAMP(19);
        } else {
            if (wid == 1) {
                const GAS u64* fp = lane < 16 ? (const GAS u64*)(myws + WS_ORDY + (size_t)me * 128) + lane : (const GAS u64*)(myws + WS_MBOX + (size_t)me * 128);
                bool pend = lane < 17;
                const u64 t0 = __builtin_amdgcn_s_memrealtime();
                for (unsigned polls = 1;; ++polls) {
                    if (pend) {
                        const u64 f = __hip_atomic_load(fp, RLX, SC_SYS);
                        if ((unsigned)(f >> 32) >= call) { __hip_atomic_store(ctl + lane, (unsigned)f, RLX, SC_WG); pend = false; }
                    }
                    if (!__any(pend)) break;
                    if ((polls & 15u) == 0u && TIMED_OUT(t0)) {
                        if (lane == 0) __hip_atomic_store(tmo, call, RLX, SC_AGENT);
                        if (pend) __hip_atomic_store(ctl + lane, lane < 16 ? 0xc0002000u : (now8() & ~255u) | 128u, RLX, SC_WG);
                        break;
                    }
                    __builtin_amdgcn_s_sleep(4);
                }
            }
            for (;;) {
                unsigned tk = 0;
                if (lane == 0) tk = __hip_atomic_fetch_add(ctl + 401, 1u, RLX, SC_WG);
                const int k = UNI(tk);
                if (k >= 16) break;
                unsigned need;
                while ((need = lds_get(ctl + 416 + k)) == 0u) __builtin_amdgcn_s_sleep(4);
                const unsigned hdr = lds_get(ctl + k);
                while (lds_get(ctl + 384) < need) __builtin_amdgcn_s_sleep(1);
                asm volatile("" ::: "memory");
                const LAS unsigned char* src = lds + (unsigned)k * 8192u;
                unsigned o[32];
                if ((hdr >> 13) & 1u) {
#pragma unroll
                    for (int i = 0; i < 8; ++i) { const u32x4 a = *(const LAS u32x4*)(src + i * 1024 + lo); o[4 * i] = a[0]; o[4 * i + 1] = a[1]; o[4 * i + 2] = a[2]; o[4 * i + 3] = a[3]; }
                } else {
                    unsigned lw[16], hi[16], ne[16];
#pragma unroll
                    for (int i = 0; i < 2; ++i) { const u32x4 a = *(const LAS u32x4*)(src + 4096 + i * 1024 + lo); hi[4 * i] = a[0]; hi[4 * i + 1] = a[1]; hi[4 * i + 2] = a[2]; hi[4 * i + 3] = a[3]; }
                    { const u32x2 a = *(const LAS u32x2*)(src + 6144 + lane * 8); hi[8] = a[0]; hi[9] = a[1]; }
#pragma unroll
                    for (int i = 0; i < 4; ++i) { const u32x4 a = *(const LAS u32x4*)(src + i * 1024 + lo); lw[4 * i] = a[0]; lw[4 * i + 1] = a[1]; lw[4 * i + 2] = a[2]; lw[4 * i + 3] = a[3]; }
#pragma unroll
                    for (int c = 0; c < 6; ++c) {
                        unsigned cc = (hi[9] << (1 + c)) & 0x80008000u;
#pragma unroll
                        for (int i = 0; i < 3; ++i) { const int fld = c * 3 + i, at = 9 + 3 * (fld & 1); cc |= (hi[fld >> 1] >> (at - 3 * i)) & (0x00070007u << (3 * i)); }
                        hi[10 + c] = cc;
                    }
                    u64 str = code_find<7>(hi, ne, (const LAS unsigned*)(src + 6656), fq);
                    const unsigned e0 = (hdr & 255u) * 0x00800080u, b2 = ((hdr >> 8) & 31u) * 0x00010001u;
#pragma unroll
                    for (int i = 0; i < 16; ++i) {
                        const unsigned h2 = (hi[i] & 0x81ff81ffu) + e0 + (code_get(str, ne[i], b2) << 7);
                        finish(o, i, __uint_as_float(__builtin_amdgcn_perm(h2, lw[i], 0x05040100u)), __uint_as_float(__builtin_amdgcn_perm(h2, lw[i], 0x07060302u)));
                    }
                }
                const int blk = (k >> 3) * 4 + (k & 3);
                const char* ob = (const char*)(p.out + (size_t)((peer & 1) * 256 + (blk >> 1) * 64 + ((k >> 2) & 1) * 32) * GN + (peer >> 1) * 4096 + (cb + sk) * 128 + (blk & 1) * 64);
#pragma unroll
                for (int i = 0; i < 8; ++i)
                    st16(ob + (size_t)((i >> 2) * 16 + (i & 3)) * (GN * 4), oo, (hdr >> 31) ? nan4 : (u32x4){o[4 * i], o[4 * i + 1], o[4 * i + 2], o[4 * i + 3]});
            }
        }
        }
    }
    }
    if (tid == 0) __hip_atomic_store(ctr, call, RLX, SC_AGENT);
}

extern "C" void kernel_launch(int rank, int nranks, ncclComm_t comm, hipStream_t stream,
                              void* const* d_in, const int* in_sizes, int n_in,
                              void* d_out, int out_size, const int* in_dtypes, int out_dtype,
                              void* const* d_out_all, void* const* d_ws_all, size_t ws_bytes) {
    (void)comm; (void)in_dtypes; (void)out_dtype; (void)d_out_all;
    if (nranks != NR || (unsigned)rank >= (unsigned)NR || n_in != 2 || ws_bytes < WS_NEED ||
        in_sizes[0] != 4096 * KSH || in_sizes[1] != KSH * GN || out_size != 512 * GN) {
        fprintf(stderr, "kernel_launch: unexpected problem shape\n");
        return;
    }
    Params p;
    p.x = (const float*)d_in[0];
    p.w = (const float*)d_in[1];
    p.out = (float*)d_out;
    p.myws = (char*)d_ws_all[rank];
    for (int i = 0; i < NR; ++i) p.ws[i] = (char*)d_ws_all[i];
    p.rank = rank;
    p.pad = 0;
    int n = 0, np = 0, nd = 0;
    unsigned char deferred[8];
    bool px[8] = {};
    for (int i = 0; i < 128; ++i) p.order[0][i] = p.order[1][i] = 0;
    auto need_x = [&](int c0, int c1) { for (int c = c0; c <= c1; ++c) if (!px[c]) { px[c] = true; p.prod[np++] = (unsigned)(32 + c); } };
    for (int key = 0; key < 16 + 3 * SKEW2; ++key) {
        for (int pass = 0; pass < 2; ++pass) {
            if ((pass == 0) == (key != 0)) {
                for (int e = 32; e < 44; ++e)
                    if (2 * ((e - 32) * 8 / 12) == key) { p.order[0][n++] = (unsigned char)e; need_x((e - 32) * 8 / 12, ((e - 32) * 8 + 7) / 12); }
                if (key == XPOS) { p.order[0][n++] = 44; need_x(0, 3); }
                if (key == XPOS + 2) { p.order[0][n++] = 45; need_x(4, 7); }
                if (key == LPOS) { p.order[0][n++] = 46; p.order[0][n++] = 47; }
            } else {
                for (int b = 0; b < 4; ++b)
                    for (int c = 0; c < 8; ++c)
                        if ((b == 0 && c == 1 ? 0 : 2 * c + b * SKEW2) == key) {
                            p.prod[np++] = (unsigned)(c * 4 + b);
                            if (b == 3 && c < 2 * NFWD && ((c ^ rank) & 1)) deferred[nd++] = (unsigned char)(c * 4 + b);
                            else p.order[0][n++] = (unsigned char)(c * 4 + b);
                        }
            }
        }
    }
    int m = 0;
    for (int i = 0; i < n; ++i) if (p.order[0][i] < 44) p.order[1][m++] = p.order[0][i];
    for (int i = 0; i < nd; ++i) p.order[1][m++] = deferred[i];
    for (int i = 0; i < 6 * NFWD; ++i) p.order[1][m++] = (unsigned char)(48 + i);
    for (int e = 44; e < 48; ++e) p.order[1][m++] = (unsigned char)e;
    if (n != NU_OUT || m != NU_PAIR || np != 40 || p.prod[0] != 0 || p.prod[1] != 4 || p.prod[2] != 32) { fprintf(stderr, "kernel_launch: bad schedule\n"); return; }
    hipLaunchKernelGGL(fused_gemm_rs, dim3(NBLK), dim3(NTHR), 0, stream, p);
    HIP_CHECK(hipGetLastError());
}
```

```cpp
#include <hip/hip_runtime.h>
#include <rccl/rccl.h>
#include <cstdio>

#define HIP_CHECK(expr) do { hipError_t _e = (expr); if (_e != hipSuccess) { \
    fprintf(stderr, "HIP error %s at %s:%d\n", hipGetErrorString(_e), __FILE__, __LINE__); \
    return; } } while (0)

typedef float f32x4 __attribute__((ext_vector_type(4)));
typedef unsigned u32x4 __attribute__((ext_vector_type(4)));
typedef unsigned u32x3 __attribute__((ext_vector_type(3)));
typedef unsigned u32x2 __attribute__((ext_vector_type(2)));
typedef unsigned short u16x2 __attribute__((ext_vector_type(2)));
typedef _Float16 h16x8 __attribute__((ext_vector_type(8)));
typedef unsigned long long u64;
#define LAS __attribute__((address_space(3)))
#define GAS __attribute__((address_space(1)))
#define RLX __ATOMIC_RELAXED
#define SC_WG __HIP_MEMORY_SCOPE_WORKGROUP
#define SC_AGENT __HIP_MEMORY_SCOPE_AGENT
#define SC_SYS __HIP_MEMORY_SCOPE_SYSTEM

#define NPLAIN 3
#define TAU 864
#define TAU_OUT TAU
#define WIN_OUT WIN
#define EQ_AT 5
#define CREDIT 36
#define WIN 10
#define PROD_GAP 40
#define PUSH_SLOT 112
#define RSTRIDE 16
#define POLLW 12
#define PROD_OFF -2
#define B0_ITS 2
#define ROT 64
#define EQ2 1
#define T_PULL 370
#define PLAIN1 1
#define T1_DELAY 50
#define WARM 3
#define POLL_SLEEP 8
#define NFWD 3
#define XPOS 6
#define LPOS 18
#define SKEW2 2

constexpr int NR = 8;
constexpr int GN = 16384;
constexpr int KSH = 512;
constexpr int NBLK = 256;
constexpr int NTHR = 512;
constexpr int NSTEP = 64;
constexpr unsigned A_STEP = 32768, B_STEP = 16384, STAGE = A_STEP + B_STEP;
constexpr int NU_OUT = 48 - NFWD, NU_PAIR = 48 + 6 * NFWD;
#define RINGN 10
constexpr int RING = RINGN;
constexpr int PW = 14;
constexpr int POOL_IN = 48, POOL_OUT = 60;
constexpr int QB = 3328 + 4 * POOL_IN;
constexpr int PB = 6656 + 4 * POOL_OUT;
constexpr unsigned UNIT_T = (4 * QB + 127) / 128 * TAU / 8, PIECE_T = (PB + 127) / 128 * TAU_OUT / 8;
constexpr unsigned SLOT = 18432, SIDE = 14336;
constexpr size_t WS_CTR = 0;
constexpr size_t WS_FLAG = 32 << 10;
constexpr size_t WS_ORDY = 96 << 10;
constexpr size_t WS_MBOX = 128 << 10;
constexpr size_t WS_EQ = 160 << 10;
constexpr size_t WS_TMO = 192 << 10;
constexpr size_t WS_DIAG = 256 << 10;
constexpr size_t WS_STG = 1 << 20;
constexpr size_t STG_A = 8ull * NSTEP * A_STEP;
constexpr size_t STG_B = 32ull * NSTEP * B_STEP;
constexpr size_t WS_EXI = WS_STG + STG_A + STG_B;
constexpr size_t EXI_BYTES = 1600ull * SLOT;
constexpr size_t WS_EXO = WS_EXI + 2 * EXI_BYTES;
constexpr size_t WS_RDY = WS_EXO + (size_t)NBLK * 8 * 16384;
constexpr size_t WS_NEED = WS_RDY + 2 * 8 * 48 * 8 * 128;
constexpr u64 DEADLINE_TICKS = 50000000ull;

struct Params {
    const float* x;
    const float* w;
    float* out;
    char* myws;
    char* ws[NR];
    int rank;
    int pad;
    unsigned prod[40];
    unsigned char order[2][128];
};

#define WAIT_L0() do { asm volatile("s_waitcnt lgkmcnt(0)" ::: "memory"); __builtin_amdgcn_s_waitcnt(0xC07F); } while (0)
#define BAR() do { asm volatile("" ::: "memory"); __builtin_amdgcn_s_barrier(); asm volatile("" ::: "memory"); } while (0)
#define SB() __builtin_amdgcn_sched_barrier(0)
#define UNI(x) __builtin_amdgcn_readfirstlane(x)
template <int N> __device__ __forceinline__ void wait_vm() { asm volatile("s_waitcnt vmcnt(%0)" ::"i"(N) : "memory"); }

__device__ __forceinline__ void st16(const void* base, unsigned voff, u32x4 v) {
    asm volatile("s_nop 4\n\tglobal_store_dwordx4 %0, %1, %2 sc0 sc1\n\ts_nop 1" ::"v"(voff), "v"(v), "s"(base) : "memory");
}
__device__ __forceinline__ void st12(const void* base, unsigned voff, u32x3 v) {
    asm volatile("s_nop 4\n\tglobal_store_dwordx3 %0, %1, %2 sc0 sc1\n\ts_nop 1" ::"v"(voff), "v"(v), "s"(base) : "memory");
}
__device__ __forceinline__ void st8(const void* base, unsigned voff, u32x2 v) {
    asm volatile("s_nop 4\n\tglobal_store_dwordx2 %0, %1, %2 sc0 sc1" ::"v"(voff), "v"(v), "s"(base) : "memory");
}
__device__ __forceinline__ void st4(const void* base, unsigned voff, unsigned v) {
    asm volatile("s_nop 4\n\tglobal_store_dword %0, %1, %2 sc0 sc1" ::"v"(voff), "v"(v), "s"(base) : "memory");
}
__device__ __forceinline__ u32x3 ld12(const void* base, unsigned voff) {
    u32x3 r;
    asm volatile("s_nop 4\n\tglobal_load_dwordx3 %0, %1, %2 sc0 sc1\n\ts_waitcnt vmcnt(0)" : "=&v"(r) : "v"(voff), "s"(base) : "memory");
    return r;
}
__device__ __forceinline__ void ld64(const void* base, unsigned voff, u32x4& a, u32x4& b, u32x4& c, u32x4& d) {
    asm volatile("s_nop 4\n\tglobal_load_dwordx4 %0, %4, %5 sc0 sc1\n\tglobal_load_dwordx4 %1, %4, %5 offset:1024 sc0 sc1\n\t"
                 "global_load_dwordx4 %2, %4, %5 offset:2048 sc0 sc1\n\tglobal_load_dwordx4 %3, %4, %5 offset:3072 sc0 sc1\n\ts_waitcnt vmcnt(0)"
                 : "=&v"(a), "=&v"(b), "=&v"(c), "=&v"(d) : "v"(voff), "s"(base) : "memory");
}
template <bool TWO> __device__ __forceinline__ void dma(const void* base, unsigned voff, unsigned lds_dst) {
    unsigned keep;
    if (TWO) asm volatile("s_mov_b32 %0, m0\n\ts_mov_b32 m0, %3\n\ts_nop 4\n\tglobal_load_lds_dwordx4 %1, %2 sc0 sc1\n\tglobal_load_lds_dwordx4 %1, %2 offset:1024 sc0 sc1\n\ts_mov_b32 m0, %0"
                          : "=&s"(keep) : "v"(voff), "s"(base), "s"(lds_dst) : "memory");
    else asm volatile("s_mov_b32 %0, m0\n\ts_mov_b32 m0, %3\n\ts_nop 4\n\tglobal_load_lds_dwordx4 %1, %2 sc0 sc1\n\ts_mov_b32 m0, %0"
                      : "=&s"(keep) : "v"(voff), "s"(base), "s"(lds_dst) : "memory");
}
__device__ __forceinline__ unsigned now8() { return (unsigned)__builtin_amdgcn_s_memrealtime() << 6; }
__device__ __forceinline__ void pace(unsigned& t, unsigned period) {
    int late = (int)(now8() - t);
    if (late < -(1 << 22)) { t += (unsigned)late; late = 0; }
    while (late < 0) { __builtin_amdgcn_s_sleep(1); late = (int)(now8() - t); }
    t += period + (late > CREDIT * 64 ? (unsigned)(late - CREDIT * 64) : 0u);
}
__device__ __forceinline__ unsigned lds_get(const LAS unsigned* a) { return UNI(__hip_atomic_load(a, RLX, SC_WG)); }
__device__ __forceinline__ unsigned cvt2(float a, float b) {
    typedef _Float16 h2 __attribute__((ext_vector_type(2)));
    h2 r = {(_Float16)a, (_Float16)b};
    return __builtin_bit_cast(unsigned, r);
}
__device__ __forceinline__ int rb_owner(int d, int rb) { return rb >= 6 ? d : ((((d >> 1) + 1 + (rb >> 1)) & 3) << 1) | (rb & 1); }
__device__ __forceinline__ int rb_local(int d, int rb) { return rb >= 6 ? (rb - 6) * 256 : (d & 1) * 256; }
__device__ __forceinline__ int rb_of(int d, int o) { return ((((o >> 1) - (d >> 1) - 1) & 3) << 1) | (o & 1); }
__device__ __forceinline__ void unit_of(int e, int j, int& c, int& id) {
    if (e < 32) { c = e >> 2; id = j * 4 + (e & 3); }
    else if (e < 44) { const int u = (e - 32) * 8 + j; c = u / 12; id = 32 + u - c * 12; }
    else { const int u = (e & 1) * 8 + j; c = u >> 1; id = (e < 46 ? 46 : 44) + (u & 1); }
}
__device__ __forceinline__ int gid_of(int d, int id) {
    const int rb = (id - 32) >> 1;
    return id < 32 ? (d >> 1) * 32 + id : 128 + rb_owner(d, rb) * 4 + (rb_local(d, rb) >> 7) + (id & 1);
}
__device__ __forceinline__ size_t stage_off(int id, int ks) {
    return id < 32 ? STG_A + (size_t)(id * NSTEP + ks) * B_STEP : (size_t)(((id - 32) >> 1) * NSTEP + ks) * A_STEP + (id & 1) * 16384;
}
template <int BITS, int SH> __device__ __forceinline__ unsigned wave_max_exp(unsigned mag) {
    unsigned e = 0;
#pragma unroll
    for (int b = BITS - 1; b >= 0; --b) if (__any(mag >= ((e + (1u << b)) << SH))) e += 1u << b;
    return e;
}

__device__ __forceinline__ unsigned row_scan(unsigned x) {
    x += __builtin_amdgcn_update_dpp(0, x, 0x111, 0xf, 0xf, true);
    x += __builtin_amdgcn_update_dpp(0, x, 0x112, 0xf, 0xf, true);
    x += __builtin_amdgcn_update_dpp(0, x, 0x114, 0xf, 0xf, true);
    x += __builtin_amdgcn_update_dpp(0, x, 0x118, 0xf, 0xf, true);
    return x;
}
__device__ __forceinline__ unsigned ffbl(unsigned x) { unsigned r; asm("v_ffbl_b32 %0, %1" : "=v"(r) : "v"(x)); return r; }
#define PK(x) __builtin_bit_cast(u16x2, (unsigned)(x))
#define DW(x) __builtin_bit_cast(unsigned, (u16x2)(x))
#define SUBS(a, b) DW(__builtin_elementwise_sub_sat(PK(a), PK(b)))
#define MIN2(a, b) DW(__builtin_elementwise_min(PK(a), PK(b)))

template <int BITS> __device__ __forceinline__ int code_base(const unsigned (&ef)[16]) {
    u16x2 mx = PK(ef[0]);
#pragma unroll
    for (int i = 1; i < 16; ++i) mx = __builtin_elementwise_max(mx, PK(ef[i]));
    const int emax = (int)wave_max_exp<BITS, 0>(mx.x > mx.y ? mx.x : mx.y);
    int up = 0, mid = 0;
#pragma unroll
    for (int i = 0; i < 6; ++i) {
        const int e = (int)(ef[i] >> 16);
        up += __builtin_popcountll(__ballot(e == emax)) - __builtin_popcountll(__ballot(e == emax - 3));
        mid += __builtin_popcountll(__ballot(e == emax - 1)) - __builtin_popcountll(__ballot(e == emax - 4));
    }
    const int bs = emax - (up > 0 ? 3 : mid > 0 ? 4 : 5);
    return bs > 0 ? bs : 0;
}
template <int POOL> __device__ __forceinline__ bool code_put(unsigned (&ef)[16], unsigned B, LAS unsigned* pool, int row) {
    const unsigned b2 = B * 0x00010001u, lut = 0x0300u | ((B + 2u + (B ? 1u : 0u)) << 16);
    u64 str = 0;
    unsigned sh = 0;
#pragma unroll
    for (int i = 0; i < 16; ++i) {
        const unsigned k1 = SUBS(b2 + 0x00010001u, ef[i]), d = SUBS(ef[i], b2);
        const unsigned len = k1 + MIN2(SUBS(k1, 0x00020002u), 0x00010001u) + __builtin_amdgcn_perm(0u, lut, SUBS(d, 0x00030003u) | 0x0c000c00u);
        sh += len & 0xffffu; str |= 1ull << (sh & 63u);
        sh += len >> 16; str |= 1ull << (sh & 63u);
        ef[i] = MIN2(d, 0x00040004u) & 0x00030003u;
    }
    const unsigned end = row_scan(sh);
    const unsigned r0 = __builtin_amdgcn_readlane(end, 15), r1 = __builtin_amdgcn_readlane(end, 31), r2 = __builtin_amdgcn_readlane(end, 47);
    if (!__all(sh <= 63u && end <= 32u * (PW - 1)) || r0 + r1 + r2 + __builtin_amdgcn_readlane(end, 63) > 32u * (POOL - 1)) return false;
    const unsigned pos = 32u + end - sh + (row > 0 ? r0 : 0u) + (row > 1 ? r1 : 0u) + (row > 2 ? r2 : 0u);
    str >>= 1;
    const u64 s0 = str << (pos & 31u);
    LAS unsigned* pp = pool + (pos >> 5);
    __hip_atomic_store(pool, r0 | r1 << 10 | r2 << 20, RLX, SC_WG);
    __hip_atomic_fetch_or(pp, (unsigned)s0, RLX, SC_WG);
    __hip_atomic_fetch_or(pp + 1, (unsigned)(s0 >> 32), RLX, SC_WG);
    __hip_atomic_fetch_or(pp + 2, (unsigned)(((str >> 32) << (pos & 31u)) >> 32), RLX, SC_WG);
    return true;
}
template <int SH> __device__ __forceinline__ u64 code_find(const unsigned (&w)[16], unsigned (&ne)[16], const LAS unsigned* pool, int row) {
    const unsigned rl = pool[0];
    const unsigned start = 32u + (row > 0 ? rl & 1023u : 0u) + (row > 1 ? (rl >> 10) & 1023u : 0u) + (row > 2 ? (rl >> 20) & 1023u : 0u);
    pool += start >> 5;
    unsigned pd[PW - 1], sum = 0;
#pragma unroll
    for (int i = 0; i < PW - 1; ++i) pd[i] = pool[i];
#pragma unroll
    for (int i = 0; i < 16; ++i) { ne[i] = MIN2(w[i] & (0x00030003u << SH), 0x00010001u); sum += ne[i]; }
    sum = 32u - (sum & 0xffffu) - (sum >> 16);
    const unsigned skip = row_scan(sum) - sum + __builtin_popcount(__builtin_amdgcn_ubfe(pd[0], 0u, start));
    unsigned cum = 0, at = 0, below = 0;
#pragma unroll
    for (int i = 0; i < PW - 1; ++i) {
        cum += __builtin_popcount(pd[i]);
        const bool lt = cum < skip;
        at += lt; below = lt ? cum : below;
    }
    const unsigned d0 = pool[at], d1 = pool[at + 1], d2 = pool[at + 2];
    unsigned rem = skip - below, pos = 0;
#pragma unroll
    for (int sft = 16; sft; sft >>= 1) {
        const unsigned t = __builtin_popcount(__builtin_amdgcn_ubfe(d0, pos, sft));
        const bool lt = t < rem;
        rem -= lt ? t : 0u; pos += lt ? sft : 0u;
    }
    pos = skip ? pos + 1u : 0u;
    return (unsigned)((((u64)d1 << 32) | d0) >> pos) | ((((u64)d2 << 32) | d1) >> pos << 32);
}
__device__ __forceinline__ unsigned code_get(u64& str, unsigned ne, unsigned b2) {
    const unsigned l0 = ffbl(((unsigned)str << 1) | (ne & 1u)); str >>= l0 & 63u;
    const unsigned l1 = ffbl(((unsigned)str << 1) | (ne >> 16)); str >>= l1 & 63u;
    const unsigned len = l0 | (l1 << 16);
    const unsigned adj = __builtin_amdgcn_perm(0x02u, 0x07010100u, MIN2(len, 0x00040004u) | 0x0c000c00u);
    return MIN2(DW(PK(b2 + adj) - PK(len)), b2 + 0x00050005u);
}

typedef float f32x2 __attribute__((ext_vector_type(2)));
__device__ __forceinline__ f32x2 gelu2(f32x2 y) {
    const float k1 = -2.0f * 0.7978845608028654f * 1.4426950408889634f, k3 = k1 * 0.044715f;
    const f32x2 a = __builtin_elementwise_fma(y * y, (f32x2){k3, k3}, (f32x2){k1, k1}) * y;
    const f32x2 d = (f32x2){__builtin_amdgcn_exp2f(a.x), __builtin_amdgcn_exp2f(a.y)} + 1.0f;
    return y * (f32x2){__builtin_amdgcn_rcpf(d.x), __builtin_amdgcn_rcpf(d.y)};
}
__device__ __forceinline__ float gelu1(float y) { return gelu2((f32x2){y, y}).x; }

#define STAMP(i) do {} while (0)
#define TIMED_OUT(t0) (UNI(__hip_atomic_load(tmo, RLX, SC_AGENT)) == call || __builtin_amdgcn_s_memrealtime() - (t0) > DEADLINE_TICKS)

__global__ __launch_bounds__(NTHR, 2) void fused_gemm_rs(Params p) {
    __shared__ __attribute__((aligned(16))) unsigned char lds_raw[3 * STAGE + 2048];
    LAS unsigned char* lds = (LAS unsigned char*)lds_raw;
    LAS unsigned* ctl = (LAS unsigned*)(lds + 3 * STAGE);
    const unsigned lds0 = (unsigned)(size_t)lds;
    const int tid = threadIdx.x, lane = tid & 63;
    const int wid = UNI(tid >> 6);
    const int fr = lane & 15, fq = lane >> 4;
    const unsigned lo = lane * 16u;
    const unsigned bid = (blockIdx.x + ROT) & 255u;
    const int xcd = bid & 7, idx = bid >> 3;
    const bool late = idx < 8;
    const int q = wid & 3;

    const int team = (wid >> 2) * 192 + (int)bid - 64;
    f32x4 r[8];
    auto request = [&](int g, int u) {
        if (g < 32) {
            const GAS char* b = (const GAS char*)(p.w + (size_t)((g >> 2) * 64 + (q & 1) * 32) * GN + (u * 4 + (g & 3)) * 128 + (q >> 1) * 64) + (unsigned)(8 * fq * GN + 4 * fr) * 4u;
#pragma unroll
            for (int e = 0; e < 8; ++e) r[e] = *(const GAS f32x4*)(b + (size_t)e * (GN * 4));
        } else {
            const GAS char* b = (const GAS char*)(p.x + (size_t)(u * 128 + q * 32) * KSH + (g - 32) * 64) + (unsigned)(fr * KSH + 8 * fq) * 4u;
#pragma unroll
            for (int e = 0; e < 8; ++e) r[e] = *(const GAS f32x4*)(b + (e >> 2) * (16 * KSH * 4) + ((e >> 1) & 1) * 128 + (e & 1) * 16);
        }
    };
    if (!late && team < 96) request(team >= 64 ? 32 : (team >> 5) * 4, team & 31);
    const unsigned t_entry = now8();

    const int rank = p.rank & 7;
    char* myws = p.myws;
    STAMP(0);
    GAS unsigned* ctr = (GAS unsigned*)(myws + WS_CTR + (size_t)bid * 128);
    GAS unsigned* tmo = (GAS unsigned*)(myws + WS_TMO);
    const unsigned ctr_v = __hip_atomic_load(ctr, RLX, SC_AGENT);
    GAS unsigned* gflag = (GAS unsigned*)(myws + WS_FLAG);
    ctl[tid] = 0u;
    WAIT_L0();
    BAR();

    if (!late) {
        unsigned wlo = 0, whi = 0;
#pragma unroll
        for (int d = 0; d < NR; ++d) {
            unsigned a = (unsigned)(size_t)p.ws[d], b = (unsigned)((size_t)p.ws[d] >> 32);
            asm volatile("" : "+s"(a), "+s"(b));
            wlo = lane == d ? a : wlo;
            whi = lane == d ? b : whi;
        }
        if ((WARM & 1) && wid == 4 && bid < 160) {
            const int tm = (int)bid - 64, u = tm & 31, k = lane & 1, pr = (lane >> 1) & 1, what = lane >> 2;
            const int d = tm < 64 ? (u >> 3) * 2 + k : (((u >> 3) + 1 + what) & 3) * 2 + ((u >> 1) & 1);
            const size_t base = ((size_t)(unsigned)__shfl(whi, d) << 32) | (unsigned)__shfl(wlo, d);
            const int c = tm < 64 ? tm >> 5 : 0, gid = tm < 64 ? u * 4 : 128 + u, did = tm < 64 ? gid & 31 : 32 + rb_of(d, u >> 2) * 2 + (u & 1);
            const size_t exi = WS_EXI + (size_t)pr * EXI_BYTES;
            size_t a = 0;
            if (tm < 64 ? what == 0 : (what < 3 && k == 0)) a = base + WS_RDY + (size_t)(((pr * 8 + rank) * 48 + did) * 8 + c) * (RSTRIDE * 8) + 64;
            else if (what == 3 && k == 0) a = (size_t)myws + exi + (size_t)(gid * 8 + c) * SLOT + 17408;
            else if (what == 1 && tm < 32) a = base + exi + (size_t)(1280 + rank * 8 + (u & 7)) * SLOT + 17408;
            if (lane < 16 && a) { unsigned dummy = __hip_atomic_load((const GAS unsigned*)a, RLX, SC_SYS); asm volatile("" ::"v"(dummy)); }
        }
#pragma unroll 1
        for (int it = 0; it < 5; ++it) {
            const int t = it < B0_ITS ? it * 384 + team : B0_ITS * 384 + (it - B0_ITS) * 288 + (wid >> 2) * 144 + (((idx - 8) >> 2) * 3 + (idx & 3) - 1) * 8 + xcd;
            if (t >= 1280 || (it >= B0_ITS && (idx & 3) == 0)) break;
            const int g = p.prod[t >> 5], u = t & 31;
            if (t >= 96) {
                const unsigned ts = t_entry + (unsigned)((t >> 5) - PROD_OFF) * (PROD_GAP * 64);
                while ((int)(now8() - ts) < 0) __builtin_amdgcn_s_sleep(2);
                request(g, u);
            }
            const bool isw = g < 32;
            const int c = isw ? g >> 2 : g - 32;
            const int gid = isw ? u * 4 + (g & 3) : 128 + u;
            unsigned v[16];
            if (isw) {
#pragma unroll
                for (int i = 0; i < 16; ++i) v[i] = cvt2(r[(i & 3) * 2][i >> 2], r[(i & 3) * 2 + 1][i >> 2]);
            } else {
#pragma unroll
                for (int i = 0; i < 16; ++i) v[i] = cvt2(r[i >> 1][(i & 1) * 2], r[i >> 1][(i & 1) * 2 + 1]);
            }
            asm volatile("" : "+v"(v[15]));
            const unsigned t_loaded = now8();
            const bool pushed = t < 32;
            const unsigned call = UNI(ctr_v) + 1u;
            const size_t par = WS_EXI + (size_t)(call & 1u) * EXI_BYTES;
            const int o = u >> 2;
            const int myid = isw ? ((gid >> 5) == (rank >> 1) ? gid & 31 : -1)
                                 : (o == rank ? 44 + (u & 3) : ((o >> 1) != (rank >> 1) && ((u >> 1) & 1) == (rank & 1) ? 32 + rb_of(rank, o) * 2 + (u & 1) : -1));
            unsigned hdr = 0;
            {
                unsigned w[16];
#pragma unroll
                for (int i = 0; i < 16; ++i) w[i] = v[i];
                const char* home = myws + par + (size_t)(gid * 8 + c) * SLOT;
                unsigned pw = 0;
                hdr = 32u;
                if (t >= 32 + 32 * PLAIN1) {
                    LAS unsigned* pool = (LAS unsigned*)(lds + wid * 256);
                    pool[lane] = 0u;
                    unsigned ef[16];
#pragma unroll
                    for (int i = 0; i < 16; ++i) ef[i] = (w[i] >> 10) & 0x001f001fu;
                    const unsigned B = (unsigned)code_base<5>(ef);
                    if (code_put<POOL_IN>(ef, B, pool, fq)) {
#pragma unroll
                        for (int i = 0; i < 16; ++i) w[i] = (w[i] & 0x83ff83ffu) | (ef[i] << 10);
#pragma unroll
                        for (int k = 0; k < 3; ++k) {
#pragma unroll
                            for (int i = 0; i < 4; ++i) w[4 * k + i] |= (w[13 + k] << (12 - 3 * i)) & 0x70007000u;
                            w[12] |= (w[13 + k] >> (1 + k)) & (0x40004000u >> k);
                        }
                        pw = __hip_atomic_load(pool + lane, RLX, SC_WG);
                        hdr = B;
                    }
                }
                if (pushed) {
                    unsigned tp = t_loaded + (unsigned)((u & 7) * 4 + q) * PUSH_SLOT;
#pragma unroll
                    for (int i = 0; i < 4; ++i) {
                        pace(tp, 32u * PUSH_SLOT);
#pragma unroll
                        for (int k = 0; k < 2; ++k) {
                            const int d = (u >> 3) * 2 + k;
                            if (d != rank) st16(p.ws[d] + par + (size_t)(1280 + rank * 8 + (u & 7)) * SLOT + q * 4096 + i * 1024, lo, (u32x4){v[4 * i], v[4 * i + 1], v[4 * i + 2], v[4 * i + 3]});
                        }
                    }
                } else if (PLAIN1 && t < 64) {
#pragma unroll
                    for (int i = 0; i < 4; ++i) st16(home + q * 4096 + i * 1024, lo, (u32x4){v[4 * i], v[4 * i + 1], v[4 * i + 2], v[4 * i + 3]});
                } else {
#pragma unroll
                    for (int i = 0; i < 3; ++i) st16(home + q * QB + i * 1024, lo, (u32x4){w[4 * i], w[4 * i + 1], w[4 * i + 2], w[4 * i + 3]});
                    st4(home + q * QB + 3072, lane * 4u, w[12]);
                    if (hdr == 32u) st12(home + SIDE + q * 768, lane * 12u, (u32x3){v[13], v[14], v[15]});
                    else if (lane < POOL_IN) st4(home + q * QB + 3328, lane * 4u, pw);
                }
            }
            if (myid >= 0) {
                const char* sg = myws + WS_STG + stage_off(myid, rank * 8 + c) + (isw ? (q >> 1) * 8 + (q & 1) : q * 4) * 1024;
#pragma unroll
                for (int i = 0; i < 4; ++i) st16(sg + i * (isw ? 2048 : 1024), lo, (u32x4){v[4 * i], v[4 * i + 1], v[4 * i + 2], v[4 * i + 3]});
            }
            wait_vm<0>();
            unsigned tw = (1u << 24) | (hdr << (6 * q));
            if (lane == 0) tw += __hip_atomic_fetch_add(ctl + 402 + (wid >> 2) * 5 + it, tw, RLX, SC_WG);
            tw = UNI(tw);
            if ((tw >> 24) == 4u) {
                const int d = lane & 7;
                int did = isw ? ((gid >> 5) == (d >> 1) ? gid & 31 : -1)
                              : (o == d ? 44 + (u & 3) : ((o >> 1) != (d >> 1) && ((u >> 1) & 1) == (d & 1) ? 32 + rb_of(d, o) * 2 + (u & 1) : -1));
                if (isw && (g & 3) == 3 && c < 2 * NFWD && (d >> 1) != (rank >> 1) && ((c ^ d) & 1)) did = -1;
                if (lane < 8 && did >= 0) {
                    if (d == rank) __hip_atomic_store(gflag + did * 64 + c * 8 + rank, call, RLX, SC_AGENT);
                    else __hip_atomic_store((GAS u64*)((((size_t)whi << 32) | wlo) + WS_RDY) + ((((call & 1u) * 8 + rank) * 48 + did) * 8 + c) * RSTRIDE, ((u64)call << 32) | (tw & 0xffffffu), RLX, SC_SYS);
                }
            }
        }
        STAMP(9);
    } else if (idx < 7) {
        const int s = idx < 6 ? rb_owner(rank, idx) : rank ^ 1;
        const int nu = idx < 6 ? NU_OUT : NU_PAIR;
        unsigned desc[2];
#pragma unroll
        for (int k = 0; k < 2; ++k) {
            const int e = p.order[idx == 6][k * 64 + lane];
            int c, id, org = s, slot;
            if (e < 48) { unit_of(e, xcd, c, id); slot = gid_of(rank, id) * 8 + c; }
            else { const int kf = (e - 48) / 6; c = 2 * kf + (s & 1); id = xcd * 4 + 3; org = rb_owner(rank, e - 48 - kf * 6); slot = 1408 + (e - 48) * 8 + xcd; }
            desc[k] = (unsigned)(c | id << 3 | org << 9 | slot << 12);
        }
        auto desc_of = [&](int n) { return (unsigned)(n < 64 ? __builtin_amdgcn_readlane(desc[0], n) : __builtin_amdgcn_readlane(desc[1], n - 64)); };
        const unsigned call = UNI(ctr_v) + 1u;
        const size_t par = WS_EXI + (size_t)(call & 1u) * EXI_BYTES;
        GAS u64* eq = (GAS u64*)(myws + WS_EQ) + idx * 16;
        if (wid == 6) {
            const char* exs = p.ws[s] + par;
            unsigned tnext = t_entry + T_PULL * 64 + xcd * (TAU / 4);
            u64 todo = ~1ull;
            __builtin_amdgcn_s_setprio(3);
            const char* src = nullptr;
            unsigned trim = 0;
            auto choose = [&](int n) {
                const unsigned rdy = __hip_atomic_load(ctl + (n >= 64 ? n : lane), RLX, SC_WG);
                const unsigned fre = __hip_atomic_load(ctl + (n > RING - PLAIN1 ? 128 + n - RING : 400), RLX, SC_WG), tr = __hip_atomic_load(ctl + 400, RLX, SC_WG);
                const u64 rd = __ballot(rdy != 0u) & (n >= 64 ? 1ull : todo);
                if (!rd || (n > RING - PLAIN1 && UNI(fre) < 4u)) return;
                const int un = n >= 64 ? n : __builtin_ctzll(rd);
                todo &= ~(1ull << (un & 63));
                if (lane == 0) __hip_atomic_store(ctl + 416 + n, (unsigned)un, RLX, SC_WG);
                trim = n >= EQ_AT + 4 && n < EQ_AT + 36 ? UNI(tr) : 0u;
                src = exs + (size_t)(desc_of(un) >> 12) * SLOT;
            };
            if (PLAIN1) {
                while (lds_get(ctl + 1) == 0u) __builtin_amdgcn_s_sleep(1);
                if (T1_DELAY && lds_get(ctl) == 0u) { const unsigned t1 = now8() + T1_DELAY * 64; if ((int)(t1 - tnext) > 0) tnext = t1; }
                todo = ~3ull;
                if (lane == 0) __hip_atomic_store(ctl + 417, 1u, RLX, SC_WG);
                const char* cur = exs + (size_t)(desc_of(1) >> 12) * SLOT;
#pragma unroll
                for (int i = 0; i < 16; i += 2) {
                    wait_vm<WIN - 2>();
                    if (i > WIN - 2 && lane == 0) __hip_atomic_store(ctl + 384, (unsigned)(i - (WIN - 2)), RLX, SC_WG);
                    pace(tnext, 2u * TAU);
                    dma<true>(cur + i * 1024, lo, lds0 + 12288u + i * 1024);
                    if (i == 12) choose(2);
                }
            }
            for (int n = 1 + PLAIN1; n < nu; ++n) {
                bool drained = false;
                if (!src) {
                    choose(n);
                    if (!src) {
                        wait_vm<0>();
                        if (lane == 0) __hip_atomic_store(ctl + 384, (unsigned)(n - 1) * 14u + 2 * PLAIN1, RLX, SC_WG);
                        drained = true;
                        for (choose(n); !src; choose(n)) __builtin_amdgcn_s_sleep(1);
                    }
                }
                const char* cur = src;
                const unsigned dst = lds0 + (unsigned)(n % RING) * 14336u, tr = trim;
                src = nullptr;
                asm volatile("" ::: "memory");
#pragma unroll
                for (int i = 0; i < 14; i += 2) {
                    wait_vm<WIN - 2>();
                    if ((i > WIN - 2 || (!drained && n > 1)) && lane == 0) __hip_atomic_store(ctl + 384, (unsigned)((n - 1) * 14 + 2 * PLAIN1 + i - (WIN - 2)), RLX, SC_WG);
                    pace(tnext, (i < 12 ? 2u * TAU : UNIT_T - 12u * TAU) - tr);
                    if (i < 12) dma<true>(cur + i * 1024, lo, dst + i * 1024);
                    else {
                        dma<false>(cur + 12288, lo, dst + 12288);
                        if (lane < (4 * QB - 13312) / 16) dma<false>(cur + 13312, lo, dst + 13312);
                    }
                    if (i == 0 && lane == 0) {
                        if (n == EQ_AT) __hip_atomic_store(ctl + 387, tnext | 1u, RLX, SC_WG);
                        if (n == nu - 3) __hip_atomic_store(ctl + 386, tnext | 1u, RLX, SC_WG);
                        if (n == nu - 1) __hip_atomic_store(ctl + 385, (tnext + UNIT_T - 2u * TAU) | 1u, RLX, SC_WG);
                    }
                    if (i == 10 && n + 1 < nu) choose(n + 1);
                }
            }
            __builtin_amdgcn_s_setprio(0);
            wait_vm<0>();
            if (lane == 0) __hip_atomic_store(ctl + 384, (unsigned)(nu - 1) * 14u + 2 * PLAIN1, RLX, SC_WG);
            STAMP(9);
        } else {
        if (wid == 7) {
            const GAS u64* rp[2];
            bool pend[2];
#pragma unroll
            for (int k = 0; k < 2; ++k) {
                rp[k] = (const GAS u64*)(myws + WS_RDY) + ((((call & 1u) * 8 + ((desc[k] >> 9) & 7)) * 48 + ((desc[k] >> 3) & 63)) * 8 + (desc[k] & 7)) * RSTRIDE;
                pend[k] = k * 64 + lane < nu;
            }
            unsigned mine = 0;
            bool eq_open = EQ_AT != 0;
            const u64 t0 = __builtin_amdgcn_s_memrealtime();
            for (unsigned polls = 1;; ++polls) {
                const u64 pm = __ballot(pend[0]);
                const bool front = __builtin_amdgcn_mbcnt_hi((unsigned)(pm >> 32), __builtin_amdgcn_mbcnt_lo((unsigned)pm, 0u)) < POLLW;
#pragma unroll
                for (int k = 0; k < 2; ++k)
                    if (pend[k] && ((k == 0 && front) || (polls & 7u) == 0u)) {
                        const u64 f = __hip_atomic_load(rp[k], RLX, SC_SYS);
                        if ((unsigned)(f >> 32) >= call) { __hip_atomic_store(ctl + k * 64 + lane, (unsigned)f | 0x80000000u, RLX, SC_WG); pend[k] = false; }
                    }
                if (eq_open) {
                    if (!mine) {
                        mine = lds_get(ctl + 387);
                        if (mine && lane == 0) __hip_atomic_store(eq + xcd, ((u64)call << 32) | mine, RLX, SC_AGENT);
                    } else {
                        const u64 f = __hip_atomic_load(eq + (lane & 7), RLX, SC_AGENT);
                        if (__all((unsigned)(f >> 32) == call)) {
                            int dlt = (int)(mine - (unsigned)f);
                            dlt += __shfl_xor(dlt, 1); dlt += __shfl_xor(dlt, 2); dlt += __shfl_xor(dlt, 4);
                            dlt /= 8 * 32 * 7;
                            dlt = dlt > TAU / 8 ? TAU / 8 : dlt < -(TAU / 8) ? -(TAU / 8) : dlt;
                            if (lane == 0) __hip_atomic_store(ctl + 400, (unsigned)dlt, RLX, SC_WG);
                            eq_open = false;
                        } else if ((int)lds_get(ctl + 384) >= (EQ_AT + 2) * 14) eq_open = false;
                    }
                }
                const bool waiting = __any(pend[0] || pend[1]);
                if (!waiting && !eq_open) break;
                if (waiting && (polls & 15u) == 0u && TIMED_OUT(t0)) {
                    if (lane == 0) __hip_atomic_store(tmo, call, RLX, SC_AGENT);
                    wait_vm<0>();
#pragma unroll
                    for (int k = 0; k < 2; ++k) if (pend[k]) __hip_atomic_store(ctl + k * 64 + lane, 0x80000000u, RLX, SC_WG);
                    break;
                }
                if (polls < 32u) __builtin_amdgcn_s_sleep(1); else __builtin_amdgcn_s_sleep(16);
            }
        } else {
            if ((WARM & 2) && wid == 5 && lane < nu) {
                unsigned dummy = __hip_atomic_load((const GAS unsigned*)(p.ws[s] + par + (size_t)(desc[0] >> 12) * SLOT + 17536 + (rank & 3) * 128), RLX, SC_SYS);
                asm volatile("" ::"v"(dummy));
            }
            int pend = -1, pend_n = 0;
            unsigned pend_h = 0;
            auto staged = [&]() {
                wait_vm<0>();
                if (lane == 0 && __hip_atomic_fetch_add(ctl + 256 + pend_n, 1u, RLX, SC_WG) == 3u) {
                    __hip_atomic_store(gflag + pend, call, RLX, SC_AGENT);
                    if (pend_h) __hip_atomic_store((GAS u64*)(p.ws[rank ^ 1] + WS_RDY) + ((((call & 1u) * 8 + s) * 48 + (pend >> 6)) * 8 + ((pend >> 3) & 7)) * RSTRIDE,
                                                   ((u64)call << 32) | (pend_h & 0xffffffu), RLX, SC_SYS);
                }
                pend = -1;
            };
            auto need = [&](int job) {
                return PLAIN1 && job < 8 ? (job - 3) * 4 : (job >> 2) * 14 - 14 + 2 * PLAIN1 + (((job & 3) + 1) * QB + 1023) / 1024; };
            auto ticket = [&]() { unsigned tk = 0; if (lane == 0) tk = __hip_atomic_fetch_add(ctl + 401, 1u, RLX, SC_WG); return (int)UNI(tk); };
            for (int job = ticket(); job < 4 * nu;) {
                const int n = job >> 2, qq = job & 3;
                unsigned w[16];
                int un = 0;
                unsigned relay_h = 0;
                if (n == 0) {
                    while (lds_get(ctl) == 0u) __builtin_amdgcn_s_sleep(1);
                    u32x4 a0, a1, a2, a3;
                    ld64(myws + par + (size_t)(1280 + s * 8 + xcd) * SLOT + qq * 4096, lo, a0, a1, a2, a3);
#pragma unroll
                    for (int i = 0; i < 4; ++i) { w[i] = a0[i]; w[4 + i] = a1[i]; w[8 + i] = a2[i]; w[12 + i] = a3[i]; }
                } else {
                    while ((int)lds_get(ctl + 384) < need(job)) __builtin_amdgcn_s_sleep(1);
                    asm volatile("" ::: "memory");
                    un = lds_get(ctl + 416 + n);
                }
                const unsigned de = desc_of(un);
                const int c = de & 7, id = (de >> 3) & 63, org = (de >> 9) & 7;
                if (PLAIN1 && n == 1) {
#pragma unroll
                    for (int i = 0; i < 4; ++i) { const u32x4 a = *(const LAS u32x4*)(lds + 12288 + qq * 4096 + i * 1024 + lo); w[4 * i] = a[0]; w[4 * i + 1] = a[1]; w[4 * i + 2] = a[2]; w[4 * i + 3] = a[3]; }
                    WAIT_L0();
                    if (lane == 0) { __hip_atomic_fetch_add(ctl + 128, 1u, RLX, SC_WG); __hip_atomic_fetch_add(ctl + 129, 1u, RLX, SC_WG); }
                    if (pend >= 0) staged();
                } else if (n) {
                    const LAS unsigned char* src = lds + (unsigned)(n % RING) * 14336u + qq * QB;
#pragma unroll
                    for (int i = 0; i < 3; ++i) { const u32x4 a = *(const LAS u32x4*)(src + i * 1024 + lo); w[4 * i] = a[0]; w[4 * i + 1] = a[1]; w[4 * i + 2] = a[2]; w[4 * i + 3] = a[3]; }
                    w[12] = *(const LAS unsigned*)(src + 3072 + lane * 4);
                    const LAS unsigned* pool = (const LAS unsigned*)(src + 3328);
                    const unsigned hu = lds_get(ctl + un), h = (hu >> (6 * qq)) & 63u;
                    if (idx < 6 && id < 32 && (id & 3) == 3 && c < 2 * NFWD) {
                        const unsigned pw = pool[lane];
                        if (pend >= 0) staged();
                        const char* dst = myws + par + (size_t)(1408 + ((c >> 1) * 6 + idx) * 8 + xcd) * SLOT + qq * QB;
#pragma unroll
                        for (int i = 0; i < 3; ++i) st16(dst + i * 1024, lo, (u32x4){w[4 * i], w[4 * i + 1], w[4 * i + 2], w[4 * i + 3]});
                        st4(dst + 3072, lane * 4u, w[12]);
                        if (lane < POOL_IN) st4(dst + 3328, lane * 4u, pw);
                        relay_h = hu;
                    }
                    if (h < 32u) {
#pragma unroll
                        for (int k = 0; k < 3; ++k) {
                            unsigned cc = (w[12] << (1 + k)) & 0x80008000u;
#pragma unroll
                            for (int i = 0; i < 4; ++i) cc |= (w[4 * k + i] >> (12 - 3 * i)) & (0x00070007u << (3 * i));
                            w[13 + k] = cc;
                        }
                        unsigned ne[16];
                        u64 str = code_find<10>(w, ne, pool, fq);
                        WAIT_L0();
                        if (lane == 0) __hip_atomic_fetch_add(ctl + 128 + n, 1u, RLX, SC_WG);
                        if (pend >= 0) staged();
#pragma unroll
                        for (int i = 0; i < 16; ++i) w[i] = (w[i] & 0x8fff8fffu) + (code_get(str, ne[i], h * 0x00010001u) << 10);
                    } else {
                        WAIT_L0();
                        if (lane == 0) __hip_atomic_fetch_add(ctl + 128 + n, 1u, RLX, SC_WG);
                        if (pend >= 0) staged();
                        const u32x3 a = ld12(p.ws[org] + par + (size_t)(gid_of(rank, id) * 8 + c) * SLOT + SIDE + qq * 768, lane * 12u);
                        w[13] = a[0]; w[14] = a[1]; w[15] = a[2];
                    }
                }
                const char* sg = myws + WS_STG + stage_off(id, org * 8 + c) + (id < 32 ? (qq >> 1) * 8 + (qq & 1) : qq * 4) * 1024;
#pragma unroll
                for (int i = 0; i < 4; ++i) st16(sg + i * (id < 32 ? 2048 : 1024), lo, (u32x4){w[4 * i], w[4 * i + 1], w[4 * i + 2], w[4 * i + 3]});
                pend = id * 64 + c * 8 + org; pend_n = n; pend_h = relay_h;
                job = ticket();
                if (job >= 4 * nu || job < 4 || (int)lds_get(ctl + 384) < need(job)) staged();
            }
        }
            if (wid == 7 && idx < 6) {
                unsigned mine;
                while ((mine = lds_get(ctl + 386)) == 0u) __builtin_amdgcn_s_sleep(8);
                if (lane == 0) __hip_atomic_store(eq + 8 + xcd, ((u64)call << 32) | mine, RLX, SC_AGENT);
                unsigned th;
                int dlt = 0;
                bool open = EQ2;
                while ((th = lds_get(ctl + 385)) == 0u) {
                    if (open) {
                        const u64 f = __hip_atomic_load(eq + 8 + (lane & 7), RLX, SC_AGENT);
                        if (__all((unsigned)(f >> 32) == call)) {
                            dlt = (int)(mine - (unsigned)f);
                            dlt += __shfl_xor(dlt, 1); dlt += __shfl_xor(dlt, 2); dlt += __shfl_xor(dlt, 4);
                            dlt /= 8 * 256;
                            dlt = dlt > 100 ? 100 : dlt < -100 ? -100 : dlt;
                            open = false;
                        }
                    }
                    __builtin_amdgcn_s_sleep(4);
                }
                if (lane == 0) __hip_atomic_store((GAS u64*)(myws + WS_MBOX + (size_t)((8 + idx * 4) * 8 + xcd) * 128), ((u64)call << 32) | (th & ~255u) | (unsigned)(dlt + 128), RLX, SC_AGENT);
            }
        }
    }
    const unsigned call = UNI(ctr_v) + 1u;

    const int ntile = idx == 7 ? 2 : 1;
    const int wm = wid >> 1, wn = wid & 1;
#pragma unroll 1
    for (int rep = 0; rep < ntile; ++rep) {
    const int rb = late ? 6 + (idx >> 2) : (idx - 8) >> 2;
    const int cb = xcd * 4 + (idx == 7 ? rep : idx == 6 ? 0 : idx == 4 || idx == 5 ? idx - 2 : idx & 3);
    const bool part1 = idx == 7 && rep == 0, part2 = idx == 6;
    const int nstep = part1 ? 56 : part2 ? 8 : NSTEP;
    const char* stgA = myws + WS_STG + (size_t)(rb * NSTEP) * A_STEP + wid * 4096;
    const char* stgB = myws + WS_STG + STG_A + (size_t)(cb * NSTEP) * B_STEP + wid * 2048;
    const u64 pbits = 0x0101010101010101ull << (rank ^ 1);
    u64 todo = part1 ? ~pbits : part2 ? pbits : ~0ull, avail = 0ull;
    int slot = 0;
    auto poll = [&]() {
        if (wid == 0) {
            const u64 t0 = __builtin_amdgcn_s_memrealtime();
            u64 got;
            for (unsigned polls = 1;; ++polls) {
                const unsigned f0 = __hip_atomic_load(gflag + cb * 64 + lane, RLX, SC_AGENT);
                const unsigned f1 = __hip_atomic_load(gflag + (32 + rb * 2) * 64 + lane, RLX, SC_AGENT);
                const unsigned f2 = __hip_atomic_load(gflag + (33 + rb * 2) * 64 + lane, RLX, SC_AGENT);
                got = __ballot(f0 >= call && f1 >= call && f2 >= call) & todo;
                if (got) break;
                if ((polls & 15u) == 0u && TIMED_OUT(t0)) {
                    if (lane == 0) __hip_atomic_store(tmo, call, RLX, SC_AGENT);
                    got = todo;
                    break;
                }
                __builtin_amdgcn_s_sleep(POLL_SLEEP);
            }
            if (lane < 2) ctl[388 + slot * 2 + lane] = (unsigned)(got >> (32 * lane));
        }
    };
    auto news = [&]() { avail = ((u64)(unsigned)UNI(ctl[389 + slot * 2]) << 32) | (unsigned)UNI(ctl[388 + slot * 2]); slot ^= 1; };
    auto take = [&]() {
        const unsigned b = (unsigned)__builtin_ctzll(avail) & 63u;
        avail &= avail - 1; todo &= ~(1ull << b);
        return (b & 7u) * 8u + (b >> 3);
    };
    auto g1 = [&](unsigned ks, unsigned buf, int i) {
        if (i < 4) __builtin_amdgcn_global_load_lds((const GAS unsigned*)(stgA + (size_t)ks * A_STEP + i * 1024 + lo), (LAS unsigned*)(lds + buf + wid * 4096 + i * 1024), 16, 0, 17);
        else __builtin_amdgcn_global_load_lds((const GAS unsigned*)(stgB + (size_t)ks * B_STEP + (i - 4) * 1024 + lo), (LAS unsigned*)(lds + buf + A_STEP + wid * 2048 + (i - 4) * 1024), 16, 0, 17);
    };

    f32x4 acc[4][4];
#pragma unroll
    for (int m = 0; m < 4; ++m)
#pragma unroll
        for (int n = 0; n < 4; ++n) acc[m][n] = (f32x4){0.f, 0.f, 0.f, 0.f};
    h16x8 a0[4], b0[4], a1[4], b1[4];
    const unsigned aoff = wm * 8192 + lo, boff = A_STEP + wn * 8192 + lo;
#define RDA(A, buf, kk, m) A[m] = *(const LAS h16x8*)(lds + (buf) + aoff + (m) * 2048 + (kk) * 1024)
#define RDB(B, buf, kk, n) B[n] = *(const LAS h16x8*)(lds + (buf) + boff + (n) * 2048 + (kk) * 1024)
#define ROW(A, B, m) _Pragma("unroll") for (int n = 0; n < 4; ++n) acc[m][n] = __builtin_amdgcn_mfma_f32_16x16x32_f16(A[m], B[n], acc[m][n], 0, 0, 0)

    STAMP(10);
    unsigned kq;
#pragma unroll
    for (int k = 0; k < 3; ++k) {
        if (!avail) { poll(); WAIT_L0(); BAR(); news(); }
        kq = take();
#pragma unroll
        for (int i = 0; i < (k < 2 ? 6 : 3); ++i) g1(kq, k * STAGE, i);
    }
    wait_vm<9>(); BAR();
#pragma unroll
    for (int i = 0; i < 4; ++i) { RDB(b0, 0u, 0, i); RDA(a0, 0u, 0, i); }
    unsigned cur = 0, prv = 2 * STAGE;
    for (int t = 0; t < nstep; ++t) {
        const unsigned nxt = cur == 2 * STAGE ? 0u : cur + STAGE;
        const bool m2 = t + 2 < nstep, m3 = t + 3 < nstep;
        const bool dry = m3 && !avail;
        WAIT_L0(); SB();
        RDB(b1, cur, 1, 0); RDB(b1, cur, 1, 1); SB(); ROW(a0, b0, 0); SB(); if (m2) g1(kq, prv, 3); SB();
        RDB(b1, cur, 1, 2); RDB(b1, cur, 1, 3); SB(); ROW(a0, b0, 1); SB(); if (m2) g1(kq, prv, 4); SB();
        RDA(a1, cur, 1, 0); RDA(a1, cur, 1, 1); SB(); ROW(a0, b0, 2); SB(); if (m2) g1(kq, prv, 5); SB();
        RDA(a1, cur, 1, 2); RDA(a1, cur, 1, 3); SB(); ROW(a0, b0, 3); SB();
        if (dry) poll();
        WAIT_L0();
        if (m2) wait_vm<6>(); else wait_vm<0>();
        BAR(); SB();
        RDB(b0, nxt, 0, 0); RDB(b0, nxt, 0, 1); SB(); ROW(a1, b1, 0); SB();
        if (dry) news();
        if (m3) { kq = take(); g1(kq, cur, 0); } SB();
        RDB(b0, nxt, 0, 2); RDB(b0, nxt, 0, 3); SB(); ROW(a1, b1, 1); SB(); if (m3) g1(kq, cur, 1); SB();
        RDA(a0, nxt, 0, 0); RDA(a0, nxt, 0, 1); SB(); ROW(a1, b1, 2); SB(); if (m3) g1(kq, cur, 2); SB();
        RDA(a0, nxt, 0, 2); RDA(a0, nxt, 0, 3); SB(); ROW(a1, b1, 3); SB();
        prv = cur; cur = nxt;
    }

    if (part1 || part2) {
        const char* park = myws + WS_EXO + (size_t)(xcd * 8 + wid) * 16384;
        GAS u64* pf = (GAS u64*)(myws + WS_ORDY + (size_t)xcd * 128) + wid;
        if (part1) {
#pragma unroll
            for (int i = 0; i < 16; ++i) st16(park + i * 1024, lo, __builtin_bit_cast(u32x4, acc[i >> 2][i & 3]));
            wait_vm<0>();
            if (lane == 0) __hip_atomic_store(pf, (u64)call << 32, RLX, SC_SYS);
            continue;
        }
        const u64 t0 = __builtin_amdgcn_s_memrealtime();
        while ((unsigned)UNI((unsigned)(__hip_atomic_load(pf, RLX, SC_SYS) >> 32)) < call) {
            if (TIMED_OUT(t0)) { if (lane == 0) __hip_atomic_store(tmo, call, RLX, SC_AGENT); wait_vm<0>(); break; }
            __builtin_amdgcn_s_sleep(4);
        }
#pragma unroll
        for (int m = 0; m < 4; ++m) {
            u32x4 e0, e1, e2, e3;
            ld64(park + m * 4096, lo, e0, e1, e2, e3);
            acc[m][0] += __builtin_bit_cast(f32x4, e0); acc[m][1] += __builtin_bit_cast(f32x4, e1);
            acc[m][2] += __builtin_bit_cast(f32x4, e2); acc[m][3] += __builtin_bit_cast(f32x4, e3);
        }
    }
    const unsigned tmo_v = __hip_atomic_load(tmo, RLX, SC_AGENT);
    const unsigned oo = (unsigned)(fq * 4) * (GN * 4u) + fr * 16u;
    const u32x4 nan4 = {0x7fc00000u, 0x7fc00000u, 0x7fc00000u, 0x7fc00000u};
    auto finish = [&](unsigned (&o)[32], int i, float a, float b) {
        const f32x2 g = gelu2((f32x2){a, b});
        const int at = ((i >> 3) * 4 + (i & 1) * 2) * 4 + ((i >> 1) & 3);
        o[at] = __float_as_uint(g.x); o[at + 4] = __float_as_uint(g.y);
    };
#define ACC(hf, i, k) acc[2 * (hf) + ((i) >> 3)][((i) >> 1) & 3][((i) & 1) * 2 + (k)]
    if (late) {
        const bool bad = UNI(tmo_v) == call;
        const char* ob = (const char*)(p.out + (size_t)(rb_local(rank, rb) + wm * 64) * GN + (rank >> 1) * 4096 + cb * 128 + wn * 64);
#pragma unroll
        for (int hf = 0; hf < 2; ++hf) {
            unsigned o[32];
#pragma unroll
            for (int i = 0; i < 16; ++i) finish(o, i, ACC(hf, i, 0), ACC(hf, i, 1));
#pragma unroll
            for (int i = 0; i < 8; ++i) st16(ob + (size_t)(hf * 32 + (i >> 2) * 16 + (i & 3)) * (GN * 4), oo, bad ? nan4 : (u32x4){o[4 * i], o[4 * i + 1], o[4 * i + 2], o[4 * i + 3]});
        }
    } else {
        const int peer = rb_owner(rank, rb);
        const int mirror = (8 + rb_of(peer, rank) * 4 + (idx & 3)) * 8 + xcd;
        char* const peerws = p.ws[peer];
        WAIT_L0(); BAR();
        if (wid < 4) __builtin_amdgcn_s_setprio(2);
#pragma unroll
        for (int hf = 0; hf < 2; ++hf) {
            const int k = (wid >> 2) * 8 + hf * 4 + q;
            const char* ex = myws + WS_EXO + (size_t)(bid * 16 + k) * 8192;
            unsigned hdr = 1u << 13;
            if ((idx & 3) != 3 || k < 16 - NPLAIN) {
                LAS unsigned* pool = (LAS unsigned*)(lds + 131072 + wid * 256);
                pool[lane] = 0u;
                unsigned hi[16], ef[16], under = 0;
#pragma unroll
                for (int i = 0; i < 16; ++i) {
                    hi[i] = __builtin_amdgcn_perm(__float_as_uint(ACC(hf, i, 1)), __float_as_uint(ACC(hf, i, 0)), 0x07060302u);
                    ef[i] = (hi[i] >> 7) & 0x00ff00ffu;
                }
                const unsigned bs = (unsigned)code_base<8>(ef), e0 = bs > 20u ? bs - 20u : 0u;
#pragma unroll
                for (int i = 0; i < 16; ++i) { under |= SUBS(e0 * 0x00010001u, ef[i]); ef[i] = SUBS(ef[i], e0 * 0x00010001u); }
                if (__all(under == 0u) && code_put<POOL_OUT>(ef, bs - e0, pool, fq)) {
#pragma unroll
                    for (int i = 0; i < 16; ++i) hi[i] = (hi[i] & 0x807f807fu) | (ef[i] << 7);
#pragma unroll
                    for (int c = 0; c < 6; ++c) {
#pragma unroll
                        for (int i = 0; i < 3; ++i) { const int fld = c * 3 + i, at = 9 + 3 * (fld & 1); hi[fld >> 1] |= (hi[10 + c] << (at - 3 * i)) & (0x00070007u << at); }
                        hi[9] |= (hi[10 + c] >> (1 + c)) & (0x40004000u >> c);
                    }
                    const unsigned pw = __hip_atomic_load(pool + lane, RLX, SC_WG);
#pragma unroll
                    for (int i = 0; i < 4; ++i) {
                        u32x4 v;
#pragma unroll
                        for (int e = 0; e < 4; ++e) v[e] = __builtin_amdgcn_perm(__float_as_uint(ACC(hf, 4 * i + e, 1)), __float_as_uint(ACC(hf, 4 * i + e, 0)), 0x05040100u);
                        st16(ex + i * 1024, lo, v);
                    }
                    st16(ex + 4096, lo, (u32x4){hi[0], hi[1], hi[2], hi[3]});
                    st16(ex + 5120, lo, (u32x4){hi[4], hi[5], hi[6], hi[7]});
                    st8(ex + 6144, lane * 8u, (u32x2){hi[8], hi[9]});
                    if (lane < POOL_OUT) st4(ex + 6656, lane * 4u, pw);
                    hdr = e0 | ((bs - e0) << 8);
                }
            }
            if (hdr >> 13) {
                unsigned o[32];
#pragma unroll
                for (int i = 0; i < 16; ++i) finish(o, i, ACC(hf, i, 0), ACC(hf, i, 1));
#pragma unroll
                for (int i = 0; i < 8; ++i) st16(ex + i * 1024, lo, (u32x4){o[4 * i], o[4 * i + 1], o[4 * i + 2], o[4 * i + 3]});
            }
            wait_vm<0>();
            hdr |= UNI(tmo_v) == call ? 0xc0000000u : 0x40000000u;
            if (lane == 0) __hip_atomic_store((GAS u64*)(peerws + WS_ORDY + (size_t)mirror * 128) + k, ((u64)call << 32) | hdr, RLX, SC_SYS);
        }
        __builtin_amdgcn_s_setprio(0);
        STAMP(22);
#pragma unroll 1
        for (int sk = 0; (idx & 3) + sk <= 3 && (idx & 3) != 3; sk += 3) {
        if (sk) {
            BAR();
            if (tid < 32 || tid == 384 || tid == 401 || (tid >= 416 && tid < 432)) ctl[tid] = 0u;
            WAIT_L0(); BAR();
        }
        const int me = bid + 8 * sk;
        const char* exs = peerws + WS_EXO + (size_t)(mirror + 8 * sk) * (16 * 8192);
        if (wid == 0) {
            unsigned tnext;
            while ((tnext = lds_get(ctl + 16)) == 0u) __builtin_amdgcn_s_sleep(2);
            STAMP(23);
            const unsigned trb = tnext & 255u, tr = trb - 128u;
            tnext = (tnext & ~255u) | 128u;
            __builtin_amdgcn_s_setprio(3);
            wait_vm<0>(); __builtin_amdgcn_s_waitcnt(0x0F70);
            int issued = 0, told = 0;
            unsigned hn = lds_get(ctl);
            for (int n = 0; n < 16; ++n) {
                if (hn == 0u && (hn = lds_get(ctl + n)) == 0u) {
                    wait_vm<0>();
                    told = issued;
                    if (lane == 0) __hip_atomic_store(ctl + 384, (unsigned)told, RLX, SC_WG);
                    while ((hn = lds_get(ctl + n)) == 0u) __builtin_amdgcn_s_sleep(1);
                }
                asm volatile("" ::: "memory");
                const bool plain = (hn >> 13) & 1u;
                const char* cur = exs + (size_t)n * 8192;
                const unsigned dst = lds0 + (unsigned)n * 8192u;
#pragma unroll
                for (int i = 0; i < 8; i += 2) {
                    wait_vm<WIN_OUT - 2>();
                    if (issued - (WIN_OUT - 2) > told) { told = issued - (WIN_OUT - 2); if (lane == 0) __hip_atomic_store(ctl + 384, (unsigned)told, RLX, SC_WG); }
                    pace(tnext, (i < 6 || plain ? 2u * TAU_OUT : PIECE_T - 6u * TAU_OUT) - tr);
                    if (i < 6 || plain) { dma<true>(cur + i * 1024, lo, dst + i * 1024); issued += 2; }
                    else { if (lane < (PB - 6144) / 16) dma<false>(cur + 6144, lo, dst + 6144); issued += 1; }
                    if (i == 0) {
                        if (lane == 0) __hip_atomic_store(ctl + 416 + n, (unsigned)issued + (plain ? 6u : 5u), RLX, SC_WG);
                        if (n == 14 && !sk && lane == 0)
                            __hip_atomic_store((GAS u64*)(myws + WS_MBOX + (size_t)(bid + 8) * 128), ((u64)call << 32) | ((tnext + 2u * (PIECE_T) - 2u * TAU_OUT - 7u * tr) & ~255u) | trb, RLX, SC_AGENT);
                    }
                    if (i == 2) hn = n < 15 ? lds_get(ctl + n + 1) : 0u;
                }
            }
            wait_vm<7>();
            if (issued - 7 > told && lane == 0) __hip_atomic_store(ctl + 384, (unsigned)issued - 7u, RLX, SC_WG);
            wait_vm<0>();
            if (lane == 0) __hip_atomic_store(ctl + 384, (unsigned)issued, RLX, SC_WG);
            __builtin_amdgcn_s_setprio(0);
            STAMP(19);
        } else {
            if (wid == 1) {
                const GAS u64* fp = lane < 16 ? (const GAS u64*)(myws + WS_ORDY + (size_t)me * 128) + lane : (const GAS u64*)(myws + WS_MBOX + (size_t)me * 128);
                bool pend = lane < 17;
                const u64 t0 = __builtin_amdgcn_s_memrealtime();
                for (unsigned polls = 1;; ++polls) {
                    if (pend) {
                        const u64 f = __hip_atomic_load(fp, RLX, SC_SYS);
                        if ((unsigned)(f >> 32) >= call) { __hip_atomic_store(ctl + lane, (unsigned)f, RLX, SC_WG); pend = false; }
                    }
                    if (!__any(pend)) break;
                    if ((polls & 15u) == 0u && TIMED_OUT(t0)) {
                        if (lane == 0) __hip_atomic_store(tmo, call, RLX, SC_AGENT);
                        if (pend) __hip_atomic_store(ctl + lane, lane < 16 ? 0xc0002000u : (now8() & ~255u) | 128u, RLX, SC_WG);
                        break;
                    }
                    __builtin_amdgcn_s_sleep(4);
                }
            }
            for (;;) {
                unsigned tk = 0;
                if (lane == 0) tk = __hip_atomic_fetch_add(ctl + 401, 1u, RLX, SC_WG);
                const int k = UNI(tk);
                if (k >= 16) break;
                unsigned need;
                while ((need = lds_get(ctl + 416 + k)) == 0u) __builtin_amdgcn_s_sleep(4);
                const unsigned hdr = lds_get(ctl + k);
                while (lds_get(ctl + 384) < need) __builtin_amdgcn_s_sleep(1);
                asm volatile("" ::: "memory");
                const LAS unsigned char* src = lds + (unsigned)k * 8192u;
                unsigned o[32];
                if ((hdr >> 13) & 1u) {
#pragma unroll
                    for (int i = 0; i < 8; ++i) { const u32x4 a = *(const LAS u32x4*)(src + i * 1024 + lo); o[4 * i] = a[0]; o[4 * i + 1] = a[1]; o[4 * i + 2] = a[2]; o[4 * i + 3] = a[3]; }
                } else {
                    unsigned lw[16], hi[16], ne[16];
#pragma unroll
                    for (int i = 0; i < 2; ++i) { const u32x4 a = *(const LAS u32x4*)(src + 4096 + i * 1024 + lo); hi[4 * i] = a[0]; hi[4 * i + 1] = a[1]; hi[4 * i + 2] = a[2]; hi[4 * i + 3] = a[3]; }
                    { const u32x2 a = *(const LAS u32x2*)(src + 6144 + lane * 8); hi[8] = a[0]; hi[9] = a[1]; }
#pragma unroll
                    for (int i = 0; i < 4; ++i) { const u32x4 a = *(const LAS u32x4*)(src + i * 1024 + lo); lw[4 * i] = a[0]; lw[4 * i + 1] = a[1]; lw[4 * i + 2] = a[2]; lw[4 * i + 3] = a[3]; }
#pragma unroll
                    for (int c = 0; c < 6; ++c) {
                        unsigned cc = (hi[9] << (1 + c)) & 0x80008000u;
#pragma unroll
                        for (int i = 0; i < 3; ++i) { const int fld = c * 3 + i, at = 9 + 3 * (fld & 1); cc |= (hi[fld >> 1] >> (at - 3 * i)) & (0x00070007u << (3 * i)); }
                        hi[10 + c] = cc;
                    }
                    u64 str = code_find<7>(hi, ne, (const LAS unsigned*)(src + 6656), fq);
                    const unsigned e0 = (hdr & 255u) * 0x00800080u, b2 = ((hdr >> 8) & 31u) * 0x00010001u;
#pragma unroll
                    for (int i = 0; i < 16; ++i) {
                        const unsigned h2 = (hi[i] & 0x81ff81ffu) + e0 + (code_get(str, ne[i], b2) << 7);
                        finish(o, i, __uint_as_float(__builtin_amdgcn_perm(h2, lw[i], 0x05040100u)), __uint_as_float(__builtin_amdgcn_perm(h2, lw[i], 0x07060302u)));
                    }
                }
                const int blk = (k >> 3) * 4 + (k & 3);
                const char* ob = (const char*)(p.out + (size_t)((peer & 1) * 256 + (blk >> 1) * 64 + ((k >> 2) & 1) * 32) * GN + (peer >> 1) * 4096 + (cb + sk) * 128 + (blk & 1) * 64);
#pragma unroll
                for (int i = 0; i < 8; ++i)
                    st16(ob + (size_t)((i >> 2) * 16 + (i & 3)) * (GN * 4), oo, (hdr >> 31) ? nan4 : (u32x4){o[4 * i], o[4 * i + 1], o[4 * i + 2], o[4 * i + 3]});
            }
        }
        }
    }
    }
    if (tid == 0) __hip_atomic_store(ctr, call, RLX, SC_AGENT);
}

extern "C" void kernel_launch(int rank, int nranks, ncclComm_t comm, hipStream_t stream,
                              void* const* d_in, const int* in_sizes, int n_in,
                              void* d_out, int out_size, const int* in_dtypes, int out_dtype,
                              void* const* d_out_all, void* const* d_ws_all, size_t ws_bytes) {
    (void)comm; (void)in_dtypes; (void)out_dtype; (void)d_out_all;
    if (nranks != NR || (unsigned)rank >= (unsigned)NR || n_in != 2 || ws_bytes < WS_NEED ||
        in_sizes[0] != 4096 * KSH || in_sizes[1] != KSH * GN || out_size != 512 * GN) {
        fprintf(stderr, "kernel_launch: unexpected problem shape\n");
        return;
    }
    Params p;
    p.x = (const float*)d_in[0];
    p.w = (const float*)d_in[1];
    p.out = (float*)d_out;
    p.myws = (char*)d_ws_all[rank];
    for (int i = 0; i < NR; ++i) p.ws[i] = (char*)d_ws_all[i];
    p.rank = rank;
    p.pad = 0;
    int n = 0, np = 0, nd = 0;
    unsigned char deferred[8];
    bool px[8] = {};
    for (int i = 0; i < 128; ++i) p.order[0][i] = p.order[1][i] = 0;
    auto need_x = [&](int c0, int c1) { for (int c = c0; c <= c1; ++c) if (!px[c]) { px[c] = true; p.prod[np++] = (unsigned)(32 + c); } };
    for (int key = 0; key < 16 + 3 * SKEW2; ++key) {
        for (int pass = 0; pass < 2; ++pass) {
            if ((pass == 0) == (key != 0)) {
                for (int e = 32; e < 44; ++e)
                    if (2 * ((e - 32) * 8 / 12) == key) { p.order[0][n++] = (unsigned char)e; need_x((e - 32) * 8 / 12, ((e - 32) * 8 + 7) / 12); }
                if (key == XPOS) { p.order[0][n++] = 44; need_x(0, 3); }
                if (key == XPOS + 2) { p.order[0][n++] = 45; need_x(4, 7); }
                if (key == LPOS) { p.order[0][n++] = 46; p.order[0][n++] = 47; }
            } else {
                for (int b = 0; b < 4; ++b)
                    for (int c = 0; c < 8; ++c)
                        if ((b == 0 && c == 1 ? 0 : 2 * c + b * SKEW2) == key) {
                            p.prod[np++] = (unsigned)(c * 4 + b);
                            if (b == 3 && c < 2 * NFWD && ((c ^ rank) & 1)) deferred[nd++] = (unsigned char)(c * 4 + b);
                            else p.order[0][n++] = (unsigned char)(c * 4 + b);
                        }
            }
        }
    }
    int m = 0;
    for (int i = 0; i < n; ++i) if (p.order[0][i] < 44) p.order[1][m++] = p.order[0][i];
    for (int i = 0; i < nd; ++i) p.order[1][m++] = deferred[i];
    for (int i = 0; i < 6 * NFWD; ++i) p.order[1][m++] = (unsigned char)(48 + i);
    for (int e = 44; e < 48; ++e) p.order[1][m++] = (unsigned char)e;
    if (n != NU_OUT || m != NU_PAIR || np != 40 || p.prod[0] != 0 || p.prod[1] != 4 || p.prod[2] != 32) { fprintf(stderr, "kernel_launch: bad schedule\n"); return; }
    hipLaunchKernelGGL(fused_gemm_rs, dim3(NBLK), dim3(NTHR), 0, stream, p);
    HIP_CHECK(hipGetLastError());
}
```

```cpp
#include <hip/hip_runtime.h>
#include <rccl/rccl.h>
#include <cstdio>

#define HIP_CHECK(expr) do { hipError_t _e = (expr); if (_e != hipSuccess) { \
    fprintf(stderr, "HIP error %s at %s:%d\n", hipGetErrorString(_e), __FILE__, __LINE__); \
    return; } } while (0)

typedef float f32x4 __attribute__((ext_vector_type(4)));
typedef unsigned u32x4 __attribute__((ext_vector_type(4)));
typedef unsigned u32x3 __attribute__((ext_vector_type(3)));
typedef unsigned u32x2 __attribute__((ext_vector_type(2)));
typedef unsigned short u16x2 __attribute__((ext_vector_type(2)));
typedef _Float16 h16x8 __attribute__((ext_vector_type(8)));
typedef unsigned long long u64;
#define LAS __attribute__((address_space(3)))
#define GAS __attribute__((address_space(1)))
#define RLX __ATOMIC_RELAXED
#define SC_WG __HIP_MEMORY_SCOPE_WORKGROUP
#define SC_AGENT __HIP_MEMORY_SCOPE_AGENT
#define SC_SYS __HIP_MEMORY_SCOPE_SYSTEM

constexpr int TAU = 864;
constexpr int WIN = 10;
constexpr int CREDIT = 36;
constexpr int EQ_AT = 5;
constexpr int PUSH_SLOT = 112;
constexpr int T_PULL = 350;
constexpr int PROD_GAP = 40;
constexpr int NFWD = 3;
constexpr int NPLAIN = 3;
constexpr int SKEW2 = 2;
constexpr int XPOS = 6, LPOS = 18;
constexpr int RSTRIDE = 16;
constexpr int POLLW = 12;

constexpr int NR = 8;
constexpr int GN = 16384;
constexpr int KSH = 512;
constexpr int NBLK = 256;
constexpr int NTHR = 512;
constexpr int NSTEP = 64;
constexpr unsigned A_STEP = 32768, B_STEP = 16384, STAGE = A_STEP + B_STEP;
constexpr int NU_OUT = 48 - NFWD, NU_PAIR = 48 + 6 * NFWD;
constexpr int RING = 10;
constexpr int PW = 14;
constexpr int POOL_IN = 48, POOL_OUT = 60;
constexpr int QB = 3328 + 4 * POOL_IN;
constexpr int PB = 6656 + 4 * POOL_OUT;
constexpr unsigned UNIT_T = (4 * QB + 127) / 128 * TAU / 8, PIECE_T = (PB + 127) / 128 * TAU / 8;
constexpr unsigned SLOT = 18432, SIDE = 14336;
constexpr size_t WS_CTR = 0;
constexpr size_t WS_FLAG = 32 << 10;
constexpr size_t WS_ORDY = 96 << 10;
constexpr size_t WS_MBOX = 128 << 10;
constexpr size_t WS_EQ = 160 << 10;
constexpr size_t WS_TMO = 192 << 10;
constexpr size_t WS_STG = 1 << 20;
constexpr size_t STG_A = 8ull * NSTEP * A_STEP;
constexpr size_t STG_B = 32ull * NSTEP * B_STEP;
constexpr size_t WS_EXI = WS_STG + STG_A + STG_B;
constexpr size_t EXI_BYTES = 1600ull * SLOT;
constexpr size_t WS_EXO = WS_EXI + 2 * EXI_BYTES;
constexpr size_t WS_RDY = WS_EXO + (size_t)NBLK * 8 * 16384;
constexpr size_t WS_NEED = WS_RDY + 2 * 8 * 48 * 8 * 128;
constexpr u64 DEADLINE_TICKS = 50000000ull;

struct Params {
    const float* x;
    const float* w;
    float* out;
    char* myws;
    char* ws[NR];
    int rank;
    int pad;
    unsigned prod[40];
    unsigned char order[2][128];
};

#define WAIT_L0() do { asm volatile("s_waitcnt lgkmcnt(0)" ::: "memory"); __builtin_amdgcn_s_waitcnt(0xC07F); } while (0)
#define BAR() do { asm volatile("" ::: "memory"); __builtin_amdgcn_s_barrier(); asm volatile("" ::: "memory"); } while (0)
#define SB() __builtin_amdgcn_sched_barrier(0)
#define UNI(x) __builtin_amdgcn_readfirstlane(x)
template <int N> __device__ __forceinline__ void wait_vm() { asm volatile("s_waitcnt vmcnt(%0)" ::"i"(N) : "memory"); }

__device__ __forceinline__ void st16(const void* base, unsigned voff, u32x4 v) {
    asm volatile("s_nop 4\n\tglobal_store_dwordx4 %0, %1, %2 sc0 sc1\n\ts_nop 1" ::"v"(voff), "v"(v), "s"(base) : "memory");
}
__device__ __forceinline__ void st12(const void* base, unsigned voff, u32x3 v) {
    asm volatile("s_nop 4\n\tglobal_store_dwordx3 %0, %1, %2 sc0 sc1\n\ts_nop 1" ::"v"(voff), "v"(v), "s"(base) : "memory");
}
__device__ __forceinline__ void st8(const void* base, unsigned voff, u32x2 v) {
    asm volatile("s_nop 4\n\tglobal_store_dwordx2 %0, %1, %2 sc0 sc1" ::"v"(voff), "v"(v), "s"(base) : "memory");
}
__device__ __forceinline__ void st4(const void* base, unsigned voff, unsigned v) {
    asm volatile("s_nop 4\n\tglobal_store_dword %0, %1, %2 sc0 sc1" ::"v"(voff), "v"(v), "s"(base) : "memory");
}
__device__ __forceinline__ u32x3 ld12(const void* base, unsigned voff) {
    u32x3 r;
    asm volatile("s_nop 4\n\tglobal_load_dwordx3 %0, %1, %2 sc0 sc1\n\ts_waitcnt vmcnt(0)" : "=&v"(r) : "v"(voff), "s"(base) : "memory");
    return r;
}
__device__ __forceinline__ void ld64(const void* base, unsigned voff, u32x4& a, u32x4& b, u32x4& c, u32x4& d) {
    asm volatile("s_nop 4\n\tglobal_load_dwordx4 %0, %4, %5 sc0 sc1\n\tglobal_load_dwordx4 %1, %4, %5 offset:1024 sc0 sc1\n\t"
                 "global_load_dwordx4 %2, %4, %5 offset:2048 sc0 sc1\n\tglobal_load_dwordx4 %3, %4, %5 offset:3072 sc0 sc1\n\ts_waitcnt vmcnt(0)"
                 : "=&v"(a), "=&v"(b), "=&v"(c), "=&v"(d) : "v"(voff), "s"(base) : "memory");
}
template <bool TWO> __device__ __forceinline__ void dma(const void* base, unsigned voff, unsigned lds_dst) {
    unsigned keep;
    if (TWO) asm volatile("s_mov_b32 %0, m0\n\ts_mov_b32 m0, %3\n\ts_nop 4\n\tglobal_load_lds_dwordx4 %1, %2 sc0 sc1\n\tglobal_load_lds_dwordx4 %1, %2 offset:1024 sc0 sc1\n\ts_mov_b32 m0, %0"
                          : "=&s"(keep) : "v"(voff), "s"(base), "s"(lds_dst) : "memory");
    else asm volatile("s_mov_b32 %0, m0\n\ts_mov_b32 m0, %3\n\ts_nop 4\n\tglobal_load_lds_dwordx4 %1, %2 sc0 sc1\n\ts_mov_b32 m0, %0"
                      : "=&s"(keep) : "v"(voff), "s"(base), "s"(lds_dst) : "memory");
}
__device__ __forceinline__ unsigned now64() { return (unsigned)__builtin_amdgcn_s_memrealtime() << 6; }
__device__ __forceinline__ void pace(unsigned& t, unsigned period) {
    int late = (int)(now64() - t);
    if (late < -(1 << 22)) { t += (unsigned)late; late = 0; }
    while (late < 0) { __builtin_amdgcn_s_sleep(1); late = (int)(now64() - t); }
    t += period + (late > CREDIT * 64 ? (unsigned)(late - CREDIT * 64) : 0u);
}
__device__ __forceinline__ unsigned lds_get(const LAS unsigned* a) { return UNI(__hip_atomic_load(a, RLX, SC_WG)); }
__device__ __forceinline__ unsigned cvt2(float a, float b) {
    typedef _Float16 h2 __attribute__((ext_vector_type(2)));
    h2 r = {(_Float16)a, (_Float16)b};
    return __builtin_bit_cast(unsigned, r);
}
__device__ __forceinline__ int rb_owner(int d, int rb) { return rb >= 6 ? d : ((((d >> 1) + 1 + (rb >> 1)) & 3) << 1) | (rb & 1); }
__device__ __forceinline__ int rb_local(int d, int rb) { return rb >= 6 ? (rb - 6) * 256 : (d & 1) * 256; }
__device__ __forceinline__ int rb_of(int d, int o) { return ((((o >> 1) - (d >> 1) - 1) & 3) << 1) | (o & 1); }
__device__ __forceinline__ void unit_of(int e, int j, int& c, int& id) {
    if (e < 32) { c = e >> 2; id = j * 4 + (e & 3); }
    else if (e < 44) { const int u = (e - 32) * 8 + j; c = u / 12; id = 32 + u - c * 12; }
    else { const int u = (e & 1) * 8 + j; c = u >> 1; id = (e < 46 ? 46 : 44) + (u & 1); }
}
__device__ __forceinline__ int gid_of(int d, int id) {
    const int rb = (id - 32) >> 1;
    return id < 32 ? (d >> 1) * 32 + id : 128 + rb_owner(d, rb) * 4 + (rb_local(d, rb) >> 7) + (id & 1);
}
__device__ __forceinline__ size_t stage_off(int id, int ks) {
    return id < 32 ? STG_A + (size_t)(id * NSTEP + ks) * B_STEP : (size_t)(((id - 32) >> 1) * NSTEP + ks) * A_STEP + (id & 1) * 16384;
}
template <int BITS, int SH> __device__ __forceinline__ unsigned wave_max_exp(unsigned mag) {
    unsigned e = 0;
#pragma unroll
    for (int b = BITS - 1; b >= 0; --b) if (__any(mag >= ((e + (1u << b)) << SH))) e += 1u << b;
    return e;
}

__device__ __forceinline__ unsigned row_scan(unsigned x) {
    x += __builtin_amdgcn_update_dpp(0, x, 0x111, 0xf, 0xf, true);
    x += __builtin_amdgcn_update_dpp(0, x, 0x112, 0xf, 0xf, true);
    x += __builtin_amdgcn_update_dpp(0, x, 0x114, 0xf, 0xf, true);
    x += __builtin_amdgcn_update_dpp(0, x, 0x118, 0xf, 0xf, true);
    return x;
}
__device__ __forceinline__ unsigned ffbl(unsigned x) { unsigned r; asm("v_ffbl_b32 %0, %1" : "=v"(r) : "v"(x)); return r; }
#define PK(x) __builtin_bit_cast(u16x2, (unsigned)(x))
#define DW(x) __builtin_bit_cast(unsigned, (u16x2)(x))
#define SUBS(a, b) DW(__builtin_elementwise_sub_sat(PK(a), PK(b)))
#define MIN2(a, b) DW(__builtin_elementwise_min(PK(a), PK(b)))

template <int BITS> __device__ __forceinline__ int code_base(const unsigned (&ef)[16]) {
    u16x2 mx = PK(ef[0]);
#pragma unroll
    for (int i = 1; i < 16; ++i) mx = __builtin_elementwise_max(mx, PK(ef[i]));
    const int emax = (int)wave_max_exp<BITS, 0>(mx.x > mx.y ? mx.x : mx.y);
    int up = 0, mid = 0;
#pragma unroll
    for (int i = 0; i < 6; ++i) {
        const int e = (int)(ef[i] >> 16);
        up += __builtin_popcountll(__ballot(e == emax)) - __builtin_popcountll(__ballot(e == emax - 3));
        mid += __builtin_popcountll(__ballot(e == emax - 1)) - __builtin_popcountll(__ballot(e == emax - 4));
    }
    const int bs = emax - (up > 0 ? 3 : mid > 0 ? 4 : 5);
    return bs > 0 ? bs : 0;
}
template <int POOL> __device__ __forceinline__ bool code_put(unsigned (&ef)[16], unsigned B, LAS unsigned* pool, int row) {
    const unsigned b2 = B * 0x00010001u, lut = 0x0300u | ((B + 2u + (B ? 1u : 0u)) << 16);
    u64 str = 0;
    unsigned sh = 0;
#pragma unroll
    for (int i = 0; i < 16; ++i) {
        const unsigned k1 = SUBS(b2 + 0x00010001u, ef[i]), d = SUBS(ef[i], b2);
        const unsigned len = k1 + MIN2(SUBS(k1, 0x00020002u), 0x00010001u) + __builtin_amdgcn_perm(0u, lut, SUBS(d, 0x00030003u) | 0x0c000c00u);
        sh += len & 0xffffu; str |= 1ull << (sh & 63u);
        sh += len >> 16; str |= 1ull << (sh & 63u);
        ef[i] = MIN2(d, 0x00040004u) & 0x00030003u;
    }
    const unsigned end = row_scan(sh);
    const unsigned r0 = __builtin_amdgcn_readlane(end, 15), r1 = __builtin_amdgcn_readlane(end, 31), r2 = __builtin_amdgcn_readlane(end, 47);
    if (!__all(sh <= 63u && end <= 32u * (PW - 1)) || r0 + r1 + r2 + __builtin_amdgcn_readlane(end, 63) > 32u * (POOL - 1)) return false;
    const unsigned pos = 32u + end - sh + (row > 0 ? r0 : 0u) + (row > 1 ? r1 : 0u) + (row > 2 ? r2 : 0u);
    str >>= 1;
    const u64 s0 = str << (pos & 31u);
    LAS unsigned* pp = pool + (pos >> 5);
    __hip_atomic_store(pool, r0 | r1 << 10 | r2 << 20, RLX, SC_WG);
    __hip_atomic_fetch_or(pp, (unsigned)s0, RLX, SC_WG);
    __hip_atomic_fetch_or(pp + 1, (unsigned)(s0 >> 32), RLX, SC_WG);
    __hip_atomic_fetch_or(pp + 2, (unsigned)(((str >> 32) << (pos & 31u)) >> 32), RLX, SC_WG);
    return true;
}
template <int SH> __device__ __forceinline__ u64 code_find(const unsigned (&w)[16], unsigned (&ne)[16], const LAS unsigned* pool, int row) {
    const unsigned rl = pool[0];
    const unsigned start = 32u + (row > 0 ? rl & 1023u : 0u) + (row > 1 ? (rl >> 10) & 1023u : 0u) + (row > 2 ? (rl >> 20) & 1023u : 0u);
    pool += start >> 5;
    unsigned pd[PW - 1], sum = 0;
#pragma unroll
    for (int i = 0; i < PW - 1; ++i) pd[i] = pool[i];
#pragma unroll
    for (int i = 0; i < 16; ++i) { ne[i] = MIN2(w[i] & (0x00030003u << SH), 0x00010001u); sum += ne[i]; }
    sum = 32u - (sum & 0xffffu) - (sum >> 16);
    const unsigned skip = row_scan(sum) - sum + __builtin_popcount(__builtin_amdgcn_ubfe(pd[0], 0u, start));
    unsigned cum = 0, at = 0, below = 0;
#pragma unroll
    for (int i = 0; i < PW - 1; ++i) {
        cum += __builtin_popcount(pd[i]);
        const bool lt = cum < skip;
        at += lt; below = lt ? cum : below;
    }
    const unsigned d0 = pool[at], d1 = pool[at + 1], d2 = pool[at + 2];
    unsigned rem = skip - below, pos = 0;
#pragma unroll
    for (int sft = 16; sft; sft >>= 1) {
        const unsigned t = __builtin_popcount(__builtin_amdgcn_ubfe(d0, pos, sft));
        const bool lt = t < rem;
        rem -= lt ? t : 0u; pos += lt ? sft : 0u;
    }
    pos = skip ? pos + 1u : 0u;
    return (unsigned)((((u64)d1 << 32) | d0) >> pos) | ((((u64)d2 << 32) | d1) >> pos << 32);
}
__device__ __forceinline__ unsigned code_get(u64& str, unsigned ne, unsigned b2) {
    const unsigned l0 = ffbl(((unsigned)str << 1) | (ne & 1u)); str >>= l0 & 63u;
    const unsigned l1 = ffbl(((unsigned)str << 1) | (ne >> 16)); str >>= l1 & 63u;
    const unsigned len = l0 | (l1 << 16);
    const unsigned adj = __builtin_amdgcn_perm(0x02u, 0x07010100u, MIN2(len, 0x00040004u) | 0x0c000c00u);
    return MIN2(DW(PK(b2 + adj) - PK(len)), b2 + 0x00050005u);
}

typedef float f32x2 __attribute__((ext_vector_type(2)));
__device__ __forceinline__ f32x2 gelu2(f32x2 y) {
    const float k1 = -2.0f * 0.7978845608028654f * 1.4426950408889634f, k3 = k1 * 0.044715f;
    const f32x2 a = __builtin_elementwise_fma(y * y, (f32x2){k3, k3}, (f32x2){k1, k1}) * y;
    const f32x2 d = (f32x2){__builtin_amdgcn_exp2f(a.x), __builtin_amdgcn_exp2f(a.y)} + 1.0f;
    return y * (f32x2){__builtin_amdgcn_rcpf(d.x), __builtin_amdgcn_rcpf(d.y)};
}

#define TIMED_OUT(t0) (UNI(__hip_atomic_load(tmo, RLX, SC_AGENT)) == call || __builtin_amdgcn_s_memrealtime() - (t0) > DEADLINE_TICKS)

__global__ __launch_bounds__(NTHR, 2) void fused_gemm_rs(Params p) {
    __shared__ __attribute__((aligned(16))) unsigned char lds_raw[3 * STAGE + 2048];
    LAS unsigned char* lds = (LAS unsigned char*)lds_raw;
    LAS unsigned* ctl = (LAS unsigned*)(lds + 3 * STAGE);
    const unsigned lds0 = (unsigned)(size_t)lds;
    const int tid = threadIdx.x, lane = tid & 63;
    const int wid = UNI(tid >> 6);
    const int fr = lane & 15, fq = lane >> 4;
    const unsigned lo = lane * 16u;
    const unsigned bid = (blockIdx.x + 64u) & 255u;
    const int xcd = bid & 7, idx = bid >> 3;
    const bool late = idx < 8;
    const int q = wid & 3;

    const int team = (wid >> 2) * 192 + (int)bid - 64;
    f32x4 r[8];
    auto request = [&](int g, int u) {
        if (g < 32) {
            const GAS char* b = (const GAS char*)(p.w + (size_t)((g >> 2) * 64 + (q & 1) * 32) * GN + (u * 4 + (g & 3)) * 128 + (q >> 1) * 64) + (unsigned)(8 * fq * GN + 4 * fr) * 4u;
#pragma unroll
            for (int e = 0; e < 8; ++e) r[e] = *(const GAS f32x4*)(b + (size_t)e * (GN * 4));
        } else {
            const GAS char* b = (const GAS char*)(p.x + (size_t)(u * 128 + q * 32) * KSH + (g - 32) * 64) + (unsigned)(fr * KSH + 8 * fq) * 4u;
#pragma unroll
            for (int e = 0; e < 8; ++e) r[e] = *(const GAS f32x4*)(b + (e >> 2) * (16 * KSH * 4) + ((e >> 1) & 1) * 128 + (e & 1) * 16);
        }
    };
    if (!late && team < 96) request(team >= 64 ? 32 : (team >> 5) * 4, team & 31);
    const unsigned t_entry = now64();

    const int rank = p.rank & 7;
    char* myws = p.myws;
    GAS unsigned* ctr = (GAS unsigned*)(myws + WS_CTR + (size_t)bid * 128);
    GAS unsigned* tmo = (GAS unsigned*)(myws + WS_TMO);
    const unsigned ctr_v = __hip_atomic_load(ctr, RLX, SC_AGENT);
    GAS unsigned* gflag = (GAS unsigned*)(myws + WS_FLAG);
    ctl[tid] = 0u;
    WAIT_L0();
    BAR();

    if (!late) {
        unsigned wlo = 0, whi = 0;
#pragma unroll
        for (int d = 0; d < NR; ++d) {
            unsigned a = (unsigned)(size_t)p.ws[d], b = (unsigned)((size_t)p.ws[d] >> 32);
            asm volatile("" : "+s"(a), "+s"(b));
            wlo = lane == d ? a : wlo;
            whi = lane == d ? b : whi;
        }
#pragma unroll 1
        for (int it = 0; it < 5; ++it) {
            const int t = it < 2 ? it * 384 + team : 2 * 384 + (it - 2) * 288 + (wid >> 2) * 144 + (((idx - 8) >> 2) * 3 + (idx & 3) - 1) * 8 + xcd;
            if (t >= 1280 || (it >= 2 && (idx & 3) == 0)) break;
            const int g = p.prod[t >> 5], u = t & 31;
            if (t >= 96) {
                const unsigned ts = t_entry + (unsigned)((t >> 5) + 2) * (PROD_GAP * 64);
                while ((int)(now64() - ts) < 0) __builtin_amdgcn_s_sleep(2);
                request(g, u);
            }
            const bool isw = g < 32;
            const int c = isw ? g >> 2 : g - 32;
            const int gid = isw ? u * 4 + (g & 3) : 128 + u;
            unsigned v[16];
            if (isw) {
#pragma unroll
                for (int i = 0; i < 16; ++i) v[i] = cvt2(r[(i & 3) * 2][i >> 2], r[(i & 3) * 2 + 1][i >> 2]);
            } else {
#pragma unroll
                for (int i = 0; i < 16; ++i) v[i] = cvt2(r[i >> 1][(i & 1) * 2], r[i >> 1][(i & 1) * 2 + 1]);
            }
            asm volatile("" : "+v"(v[15]));
            const unsigned t_loaded = now64();
            const bool pushed = t < 32;
            const unsigned call = UNI(ctr_v) + 1u;
            const size_t par = WS_EXI + (size_t)(call & 1u) * EXI_BYTES;
            const int o = u >> 2;
            const int myid = isw ? ((gid >> 5) == (rank >> 1) ? gid & 31 : -1)
                                 : (o == rank ? 44 + (u & 3) : ((o >> 1) != (rank >> 1) && ((u >> 1) & 1) == (rank & 1) ? 32 + rb_of(rank, o) * 2 + (u & 1) : -1));
            unsigned hdr = 0;
            {
                unsigned w[16];
#pragma unroll
                for (int i = 0; i < 16; ++i) w[i] = v[i];
                const char* home = myws + par + (size_t)(gid * 8 + c) * SLOT;
                unsigned pw = 0;
                hdr = 32u;
                if (!pushed) {
                    LAS unsigned* pool = (LAS unsigned*)(lds + wid * 256);
                    pool[lane] = 0u;
                    unsigned ef[16];
#pragma unroll
                    for (int i = 0; i < 16; ++i) ef[i] = (w[i] >> 10) & 0x001f001fu;
                    const unsigned B = (unsigned)code_base<5>(ef);
                    if (code_put<POOL_IN>(ef, B, pool, fq)) {
#pragma unroll
                        for (int i = 0; i < 16; ++i) w[i] = (w[i] & 0x83ff83ffu) | (ef[i] << 10);
#pragma unroll
                        for (int k = 0; k < 3; ++k) {
#pragma unroll
                            for (int i = 0; i < 4; ++i) w[4 * k + i] |= (w[13 + k] << (12 - 3 * i)) & 0x70007000u;
                            w[12] |= (w[13 + k] >> (1 + k)) & (0x40004000u >> k);
                        }
                        pw = __hip_atomic_load(pool + lane, RLX, SC_WG);
                        hdr = B;
                    }
                }
                if (pushed) {
                    unsigned tp = t_loaded + (unsigned)((u & 7) * 4 + q) * PUSH_SLOT;
#pragma unroll
                    for (int i = 0; i < 4; ++i) {
                        pace(tp, 32u * PUSH_SLOT);
#pragma unroll
                        for (int k = 0; k < 2; ++k) {
                            const int d = (u >> 3) * 2 + k;
                            if (d != rank) st16(p.ws[d] + par + (size_t)(1280 + rank * 8 + (u & 7)) * SLOT + q * 4096 + i * 1024, lo, (u32x4){v[4 * i], v[4 * i + 1], v[4 * i + 2], v[4 * i + 3]});
                        }
                    }
                } else {
#pragma unroll
                    for (int i = 0; i < 3; ++i) st16(home + q * QB + i * 1024, lo, (u32x4){w[4 * i], w[4 * i + 1], w[4 * i + 2], w[4 * i + 3]});
                    st4(home + q * QB + 3072, lane * 4u, w[12]);
                    if (hdr == 32u) st12(home + SIDE + q * 768, lane * 12u, (u32x3){v[13], v[14], v[15]});
                    else if (lane < POOL_IN) st4(home + q * QB + 3328, lane * 4u, pw);
                }
            }
            if (myid >= 0) {
                const char* sg = myws + WS_STG + stage_off(myid, rank * 8 + c) + (isw ? (q >> 1) * 8 + (q & 1) : q * 4) * 1024;
#pragma unroll
                for (int i = 0; i < 4; ++i) st16(sg + i * (isw ? 2048 : 1024), lo, (u32x4){v[4 * i], v[4 * i + 1], v[4 * i + 2], v[4 * i + 3]});
            }
            wait_vm<0>();
            unsigned tw = (1u << 24) | (hdr << (6 * q));
            if (lane == 0) tw += __hip_atomic_fetch_add(ctl + 402 + (wid >> 2) * 5 + it, tw, RLX, SC_WG);
            tw = UNI(tw);
            if ((tw >> 24) == 4u) {
                const int d = lane & 7;
                int did = isw ? ((gid >> 5) == (d >> 1) ? gid & 31 : -1)
                              : (o == d ? 44 + (u & 3) : ((o >> 1) != (d >> 1) && ((u >> 1) & 1) == (d & 1) ? 32 + rb_of(d, o) * 2 + (u & 1) : -1));
                if (isw && (g & 3) == 3 && c < 2 * NFWD && (d >> 1) != (rank >> 1) && ((c ^ d) & 1)) did = -1;
                if (lane < 8 && did >= 0) {
                    if (d == rank) __hip_atomic_store(gflag + did * 64 + c * 8 + rank, call, RLX, SC_AGENT);
                    else __hip_atomic_store((GAS u64*)((((size_t)whi << 32) | wlo) + WS_RDY) + ((((call & 1u) * 8 + rank) * 48 + did) * 8 + c) * RSTRIDE, ((u64)call << 32) | (tw & 0xffffffu), RLX, SC_SYS);
                }
            }
        }
    } else if (idx < 7) {
        const int s = idx < 6 ? rb_owner(rank, idx) : rank ^ 1;
        const int nu = idx < 6 ? NU_OUT : NU_PAIR;
        unsigned desc[2];
#pragma unroll
        for (int k = 0; k < 2; ++k) {
            const int e = p.order[idx == 6][k * 64 + lane];
            int c, id, org = s, slot;
            if (e < 48) { unit_of(e, xcd, c, id); slot = gid_of(rank, id) * 8 + c; }
            else { const int kf = (e - 48) / 6; c = 2 * kf + (s & 1); id = xcd * 4 + 3; org = rb_owner(rank, e - 48 - kf * 6); slot = 1408 + (e - 48) * 8 + xcd; }
            desc[k] = (unsigned)(c | id << 3 | org << 9 | slot << 12);
        }
        auto desc_of = [&](int n) { return (unsigned)(n < 64 ? __builtin_amdgcn_readlane(desc[0], n) : __builtin_amdgcn_readlane(desc[1], n - 64)); };
        const unsigned call = UNI(ctr_v) + 1u;
        const size_t par = WS_EXI + (size_t)(call & 1u) * EXI_BYTES;
        GAS u64* eq = (GAS u64*)(myws + WS_EQ) + idx * 16;
        if (wid == 6) {
            const char* exs = p.ws[s] + par;
            unsigned tnext = t_entry + T_PULL * 64 + xcd * (TAU / 4);
            u64 todo = ~1ull;
            __builtin_amdgcn_s_setprio(3);
            const char* src = nullptr;
            unsigned trim = 0;
            auto choose = [&](int n) {
                const unsigned rdy = __hip_atomic_load(ctl + (n >= 64 ? n : lane), RLX, SC_WG);
                const unsigned fre = __hip_atomic_load(ctl + (n > RING ? 128 + n - RING : 400), RLX, SC_WG), tr = __hip_atomic_load(ctl + 400, RLX, SC_WG);
                const u64 rd = __ballot(rdy != 0u) & (n >= 64 ? 1ull : todo);
                if (!rd || (n > RING && UNI(fre) < 4u)) return;
                const int un = n >= 64 ? n : __builtin_ctzll(rd);
                todo &= ~(1ull << (un & 63));
                if (lane == 0) __hip_atomic_store(ctl + 416 + n, (unsigned)un, RLX, SC_WG);
                trim = n >= EQ_AT + 4 && n < EQ_AT + 36 ? UNI(tr) : 0u;
                src = exs + (size_t)(desc_of(un) >> 12) * SLOT;
            };
            for (int n = 1; n < nu; ++n) {
                bool drained = false;
                if (!src) {
                    choose(n);
                    if (!src) {
                        wait_vm<0>();
                        if (lane == 0) __hip_atomic_store(ctl + 384, (unsigned)(n - 1) * 14u, RLX, SC_WG);
                        drained = true;
                        for (choose(n); !src; choose(n)) __builtin_amdgcn_s_sleep(1);
                    }
                }
                const char* cur = src;
                const unsigned dst = lds0 + (unsigned)(n % RING) * 14336u, tr = trim;
                src = nullptr;
                asm volatile("" ::: "memory");
#pragma unroll
                for (int i = 0; i < 14; i += 2) {
                    wait_vm<WIN - 2>();
                    if ((i > WIN - 2 || (!drained && n > 1)) && lane == 0) __hip_atomic_store(ctl + 384, (unsigned)((n - 1) * 14 + i - (WIN - 2)), RLX, SC_WG);
                    pace(tnext, (i < 12 ? 2u * TAU : UNIT_T - 12u * TAU) - tr);
                    if (i < 12) dma<true>(cur + i * 1024, lo, dst + i * 1024);
                    else {
                        dma<false>(cur + 12288, lo, dst + 12288);
                        if (lane < (4 * QB - 13312) / 16) dma<false>(cur + 13312, lo, dst + 13312);
                    }
                    if (i == 0 && lane == 0) {
                        if (n == EQ_AT) __hip_atomic_store(ctl + 387, tnext | 1u, RLX, SC_WG);
                        if (n == nu - 3) __hip_atomic_store(ctl + 386, tnext | 1u, RLX, SC_WG);
                        if (n == nu - 1) __hip_atomic_store(ctl + 385, (tnext + UNIT_T - 2u * TAU) | 1u, RLX, SC_WG);
                    }
                    if (i == 10 && n + 1 < nu) choose(n + 1);
                }
            }
            __builtin_amdgcn_s_setprio(0);
            wait_vm<0>();
            if (lane == 0) __hip_atomic_store(ctl + 384, (unsigned)(nu - 1) * 14u, RLX, SC_WG);
        } else if (wid == 7) {
            const GAS u64* rp[2];
            bool pend[2];
#pragma unroll
            for (int k = 0; k < 2; ++k) {
                rp[k] = (const GAS u64*)(myws + WS_RDY) + ((((call & 1u) * 8 + ((desc[k] >> 9) & 7)) * 48 + ((desc[k] >> 3) & 63)) * 8 + (desc[k] & 7)) * RSTRIDE;
                pend[k] = k * 64 + lane < nu;
            }
            unsigned mine = 0;
            bool eq_open = true;
            const u64 t0 = __builtin_amdgcn_s_memrealtime();
            for (unsigned polls = 1;; ++polls) {
                const u64 pm = __ballot(pend[0]);
                const bool front = __builtin_amdgcn_mbcnt_hi((unsigned)(pm >> 32), __builtin_amdgcn_mbcnt_lo((unsigned)pm, 0u)) < POLLW;
#pragma unroll
                for (int k = 0; k < 2; ++k)
                    if (pend[k] && ((k == 0 && front) || (polls & 7u) == 0u)) {
                        const u64 f = __hip_atomic_load(rp[k], RLX, SC_SYS);
                        if ((unsigned)(f >> 32) >= call) { __hip_atomic_store(ctl + k * 64 + lane, (unsigned)f | 0x80000000u, RLX, SC_WG); pend[k] = false; }
                    }
                if (eq_open) {
                    if (!mine) {
                        mine = lds_get(ctl + 387);
                        if (mine && lane == 0) __hip_atomic_store(eq + xcd, ((u64)call << 32) | mine, RLX, SC_AGENT);
                    } else {
                        const u64 f = __hip_atomic_load(eq + (lane & 7), RLX, SC_AGENT);
                        if (__all((unsigned)(f >> 32) == call)) {
                            int dlt = (int)(mine - (unsigned)f);
                            dlt += __shfl_xor(dlt, 1); dlt += __shfl_xor(dlt, 2); dlt += __shfl_xor(dlt, 4);
                            dlt /= 8 * 32 * 7;
                            dlt = dlt > TAU / 8 ? TAU / 8 : dlt < -(TAU / 8) ? -(TAU / 8) : dlt;
                            if (lane == 0) __hip_atomic_store(ctl + 400, (unsigned)dlt, RLX, SC_WG);
                            eq_open = false;
                        } else if ((int)lds_get(ctl + 384) >= (EQ_AT + 2) * 14) eq_open = false;
                    }
                }
                const bool waiting = __any(pend[0] || pend[1]);
                if (!waiting && !eq_open) break;
                if (waiting && (polls & 15u) == 0u && TIMED_OUT(t0)) {
                    if (lane == 0) __hip_atomic_store(tmo, call, RLX, SC_AGENT);
                    wait_vm<0>();
#pragma unroll
                    for (int k = 0; k < 2; ++k) if (pend[k]) __hip_atomic_store(ctl + k * 64 + lane, 0x80000000u, RLX, SC_WG);
                    break;
                }
                if (polls < 32u) __builtin_amdgcn_s_sleep(1); else __builtin_amdgcn_s_sleep(16);
            }
            if (idx < 6) {
                while ((mine = lds_get(ctl + 386)) == 0u) __builtin_amdgcn_s_sleep(8);
                if (lane == 0) __hip_atomic_store(eq + 8 + xcd, ((u64)call << 32) | mine, RLX, SC_AGENT);
                unsigned th;
                int dlt = 0;
                bool open = true;
                while ((th = lds_get(ctl + 385)) == 0u) {
                    if (open) {
                        const u64 f = __hip_atomic_load(eq + 8 + (lane & 7), RLX, SC_AGENT);
                        if (__all((unsigned)(f >> 32) == call)) {
                            dlt = (int)(mine - (unsigned)f);
                            dlt += __shfl_xor(dlt, 1); dlt += __shfl_xor(dlt, 2); dlt += __shfl_xor(dlt, 4);
                            dlt /= 8 * 256;
                            dlt = dlt > 100 ? 100 : dlt < -100 ? -100 : dlt;
                            open = false;
                        }
                    }
                    __builtin_amdgcn_s_sleep(4);
                }
                if (lane == 0) __hip_atomic_store((GAS u64*)(myws + WS_MBOX + (size_t)((8 + idx * 4) * 8 + xcd) * 128), ((u64)call << 32) | (th & ~255u) | (unsigned)(dlt + 128), RLX, SC_AGENT);
            }
        } else {
            int pend = -1, pend_n = 0;
            unsigned pend_h = 0;
            auto staged = [&]() {
                wait_vm<0>();
                if (lane == 0 && __hip_atomic_fetch_add(ctl + 256 + pend_n, 1u, RLX, SC_WG) == 3u) {
                    __hip_atomic_store(gflag + pend, call, RLX, SC_AGENT);
                    if (pend_h) __hip_atomic_store((GAS u64*)(p.ws[rank ^ 1] + WS_RDY) + ((((call & 1u) * 8 + s) * 48 + (pend >> 6)) * 8 + ((pend >> 3) & 7)) * RSTRIDE,
                                                   ((u64)call << 32) | (pend_h & 0xffffffu), RLX, SC_SYS);
                }
                pend = -1;
            };
            auto need = [&](int job) { return (job >> 2) * 14 - 14 + (((job & 3) + 1) * QB + 1023) / 1024; };
            auto ticket = [&]() { unsigned tk = 0; if (lane == 0) tk = __hip_atomic_fetch_add(ctl + 401, 1u, RLX, SC_WG); return (int)UNI(tk); };
            for (int job = ticket(); job < 4 * nu;) {
                const int n = job >> 2, qq = job & 3;
                unsigned w[16];
                int un = 0;
                unsigned relay_h = 0;
                if (n == 0) {
                    while (lds_get(ctl) == 0u) __builtin_amdgcn_s_sleep(1);
                    u32x4 a0, a1, a2, a3;
                    ld64(myws + par + (size_t)(1280 + s * 8 + xcd) * SLOT + qq * 4096, lo, a0, a1, a2, a3);
#pragma unroll
                    for (int i = 0; i < 4; ++i) { w[i] = a0[i]; w[4 + i] = a1[i]; w[8 + i] = a2[i]; w[12 + i] = a3[i]; }
                } else {
                    while ((int)lds_get(ctl + 384) < need(job)) __builtin_amdgcn_s_sleep(1);
                    asm volatile("" ::: "memory");
                    un = lds_get(ctl + 416 + n);
                }
                const unsigned de = desc_of(un);
                const int c = de & 7, id = (de >> 3) & 63, org = (de >> 9) & 7;
                if (n) {
                    const LAS unsigned char* src = lds + (unsigned)(n % RING) * 14336u + qq * QB;
#pragma unroll
                    for (int i = 0; i < 3; ++i) { const u32x4 a = *(const LAS u32x4*)(src + i * 1024 + lo); w[4 * i] = a[0]; w[4 * i + 1] = a[1]; w[4 * i + 2] = a[2]; w[4 * i + 3] = a[3]; }
                    w[12] = *(const LAS unsigned*)(src + 3072 + lane * 4);
                    const LAS unsigned* pool = (const LAS unsigned*)(src + 3328);
                    const unsigned hu = lds_get(ctl + un), h = (hu >> (6 * qq)) & 63u;
                    if (idx < 6 && id < 32 && (id & 3) == 3 && c < 2 * NFWD) {
                        const unsigned pw = pool[lane];
                        if (pend >= 0) staged();
                        const char* dst = myws + par + (size_t)(1408 + ((c >> 1) * 6 + idx) * 8 + xcd) * SLOT + qq * QB;
#pragma unroll
                        for (int i = 0; i < 3; ++i) st16(dst + i * 1024, lo, (u32x4){w[4 * i], w[4 * i + 1], w[4 * i + 2], w[4 * i + 3]});
                        st4(dst + 3072, lane * 4u, w[12]);
                        if (lane < POOL_IN) st4(dst + 3328, lane * 4u, pw);
                        relay_h = hu;
                    }
                    if (h < 32u) {
#pragma unroll
                        for (int k = 0; k < 3; ++k) {
                            unsigned cc = (w[12] << (1 + k)) & 0x80008000u;
#pragma unroll
                            for (int i = 0; i < 4; ++i) cc |= (w[4 * k + i] >> (12 - 3 * i)) & (0x00070007u << (3 * i));
                            w[13 + k] = cc;
                        }
                        unsigned ne[16];
                        u64 str = code_find<10>(w, ne, pool, fq);
                        WAIT_L0();
                        if (lane == 0) __hip_atomic_fetch_add(ctl + 128 + n, 1u, RLX, SC_WG);
                        if (pend >= 0) staged();
#pragma unroll
                        for (int i = 0; i < 16; ++i) w[i] = (w[i] & 0x8fff8fffu) + (code_get(str, ne[i], h * 0x00010001u) << 10);
                    } else {
                        WAIT_L0();
                        if (lane == 0) __hip_atomic_fetch_add(ctl + 128 + n, 1u, RLX, SC_WG);
                        if (pend >= 0) staged();
                        const u32x3 a = ld12(p.ws[org] + par + (size_t)(gid_of(rank, id) * 8 + c) * SLOT + SIDE + qq * 768, lane * 12u);
                        w[13] = a[0]; w[14] = a[1]; w[15] = a[2];
                    }
                }
                const char* sg = myws + WS_STG + stage_off(id, org * 8 + c) + (id < 32 ? (qq >> 1) * 8 + (qq & 1) : qq * 4) * 1024;
#pragma unroll
                for (int i = 0; i < 4; ++i) st16(sg + i * (id < 32 ? 2048 : 1024), lo, (u32x4){w[4 * i], w[4 * i + 1], w[4 * i + 2], w[4 * i + 3]});
                pend = id * 64 + c * 8 + org; pend_n = n; pend_h = relay_h;
                job = ticket();
                if (job >= 4 * nu || job < 4 || (int)lds_get(ctl + 384) < need(job)) staged();
            }
        }
    }
    const unsigned call = UNI(ctr_v) + 1u;

    const int ntile = idx == 7 ? 2 : 1;
    const int wm = wid >> 1, wn = wid & 1;
#pragma unroll 1
    for (int rep = 0; rep < ntile; ++rep) {
    const int rb = late ? 6 + (idx >> 2) : (idx - 8) >> 2;
    const int cb = xcd * 4 + (idx == 7 ? rep : idx == 6 ? 0 : idx == 4 || idx == 5 ? idx - 2 : idx & 3);
    const bool part1 = idx == 7 && rep == 0, part2 = idx == 6;
    const int nstep = part1 ? 56 : part2 ? 8 : NSTEP;
    const char* stgA = myws + WS_STG + (size_t)(rb * NSTEP) * A_STEP + wid * 4096;
    const char* stgB = myws + WS_STG + STG_A + (size_t)(cb * NSTEP) * B_STEP + wid * 2048;
    const u64 pbits = 0x0101010101010101ull << (rank ^ 1);
    u64 todo = part1 ? ~pbits : part2 ? pbits : ~0ull, avail = 0ull;
    int slot = 0;
    auto poll = [&]() {
        if (wid == 0) {
            const u64 t0 = __builtin_amdgcn_s_memrealtime();
            u64 got;
            for (unsigned polls = 1;; ++polls) {
                const unsigned f0 = __hip_atomic_load(gflag + cb * 64 + lane, RLX, SC_AGENT);
                const unsigned f1 = __hip_atomic_load(gflag + (32 + rb * 2) * 64 + lane, RLX, SC_AGENT);
                const unsigned f2 = __hip_atomic_load(gflag + (33 + rb * 2) * 64 + lane, RLX, SC_AGENT);
                got = __ballot(f0 >= call && f1 >= call && f2 >= call) & todo;
                if (got) break;
                if ((polls & 15u) == 0u && TIMED_OUT(t0)) {
                    if (lane == 0) __hip_atomic_store(tmo, call, RLX, SC_AGENT);
                    got = todo;
                    break;
                }
                __builtin_amdgcn_s_sleep(8);
            }
            if (lane < 2) ctl[388 + slot * 2 + lane] = (unsigned)(got >> (32 * lane));
        }
    };
    auto news = [&]() { avail = ((u64)(unsigned)UNI(ctl[389 + slot * 2]) << 32) | (unsigned)UNI(ctl[388 + slot * 2]); slot ^= 1; };
    auto take = [&]() {
        const unsigned b = (unsigned)__builtin_ctzll(avail) & 63u;
        avail &= avail - 1; todo &= ~(1ull << b);
        return (b & 7u) * 8u + (b >> 3);
    };
    auto g1 = [&](unsigned ks, unsigned buf, int i) {
        if (i < 4) __builtin_amdgcn_global_load_lds((const GAS unsigned*)(stgA + (size_t)ks * A_STEP + i * 1024 + lo), (LAS unsigned*)(lds + buf + wid * 4096 + i * 1024), 16, 0, 17);
        else __builtin_amdgcn_global_load_lds((const GAS unsigned*)(stgB + (size_t)ks * B_STEP + (i - 4) * 1024 + lo), (LAS unsigned*)(lds + buf + A_STEP + wid * 2048 + (i - 4) * 1024), 16, 0, 17);
    };

    f32x4 acc[4][4];
#pragma unroll
    for (int m = 0; m < 4; ++m)
#pragma unroll
        for (int n = 0; n < 4; ++n) acc[m][n] = (f32x4){0.f, 0.f, 0.f, 0.f};
    h16x8 a0[4], b0[4], a1[4], b1[4];
    const unsigned aoff = wm * 8192 + lo, boff = A_STEP + wn * 8192 + lo;
#define RDA(A, buf, kk, m) A[m] = *(const LAS h16x8*)(lds + (buf) + aoff + (m) * 2048 + (kk) * 1024)
#define RDB(B, buf, kk, n) B[n] = *(const LAS h16x8*)(lds + (buf) + boff + (n) * 2048 + (kk) * 1024)
#define ROW(A, B, m) _Pragma("unroll") for (int n = 0; n < 4; ++n) acc[m][n] = __builtin_amdgcn_mfma_f32_16x16x32_f16(A[m], B[n], acc[m][n], 0, 0, 0)
    unsigned kq;
#pragma unroll
    for (int k = 0; k < 3; ++k) {
        if (!avail) { poll(); WAIT_L0(); BAR(); news(); }
        kq = take();
#pragma unroll
        for (int i = 0; i < (k < 2 ? 6 : 3); ++i) g1(kq, k * STAGE, i);
    }
    wait_vm<9>(); BAR();
#pragma unroll
    for (int i = 0; i < 4; ++i) { RDB(b0, 0u, 0, i); RDA(a0, 0u, 0, i); }
    unsigned cur = 0, prv = 2 * STAGE;
    for (int t = 0; t < nstep; ++t) {
        const unsigned nxt = cur == 2 * STAGE ? 0u : cur + STAGE;
        const bool m2 = t + 2 < nstep, m3 = t + 3 < nstep;
        const bool dry = m3 && !avail;
        WAIT_L0(); SB();
        RDB(b1, cur, 1, 0); RDB(b1, cur, 1, 1); SB(); ROW(a0, b0, 0); SB(); if (m2) g1(kq, prv, 3); SB();
        RDB(b1, cur, 1, 2); RDB(b1, cur, 1, 3); SB(); ROW(a0, b0, 1); SB(); if (m2) g1(kq, prv, 4); SB();
        RDA(a1, cur, 1, 0); RDA(a1, cur, 1, 1); SB(); ROW(a0, b0, 2); SB(); if (m2) g1(kq, prv, 5); SB();
        RDA(a1, cur, 1, 2); RDA(a1, cur, 1, 3); SB(); ROW(a0, b0, 3); SB();
        if (dry) poll();
        WAIT_L0();
        if (m2) wait_vm<6>(); else wait_vm<0>();
        BAR(); SB();
        RDB(b0, nxt, 0, 0); RDB(b0, nxt, 0, 1); SB(); ROW(a1, b1, 0); SB();
        if (dry) news();
        if (m3) { kq = take(); g1(kq, cur, 0); } SB();
        RDB(b0, nxt, 0, 2); RDB(b0, nxt, 0, 3); SB(); ROW(a1, b1, 1); SB(); if (m3) g1(kq, cur, 1); SB();
        RDA(a0, nxt, 0, 0); RDA(a0, nxt, 0, 1); SB(); ROW(a1, b1, 2); SB(); if (m3) g1(kq, cur, 2); SB();
        RDA(a0, nxt, 0, 2); RDA(a0, nxt, 0, 3); SB(); ROW(a1, b1, 3); SB();
        prv = cur; cur = nxt;
    }

    if (part1 || part2) {
        const char* park = myws + WS_EXO + (size_t)(xcd * 8 + wid) * 16384;
        GAS u64* pf = (GAS u64*)(myws + WS_ORDY + (size_t)xcd * 128) + wid;
        if (part1) {
#pragma unroll
            for (int i = 0; i < 16; ++i) st16(park + i * 1024, lo, __builtin_bit_cast(u32x4, acc[i >> 2][i & 3]));
            wait_vm<0>();
            if (lane == 0) __hip_atomic_store(pf, (u64)call << 32, RLX, SC_SYS);
            continue;
        }
        const u64 t0 = __builtin_amdgcn_s_memrealtime();
        while ((unsigned)UNI((unsigned)(__hip_atomic_load(pf, RLX, SC_SYS) >> 32)) < call) {
            if (TIMED_OUT(t0)) { if (lane == 0) __hip_atomic_store(tmo, call, RLX, SC_AGENT); wait_vm<0>(); break; }
            __builtin_amdgcn_s_sleep(4);
        }
#pragma unroll
        for (int m = 0; m < 4; ++m) {
            u32x4 e0, e1, e2, e3;
            ld64(park + m * 4096, lo, e0, e1, e2, e3);
            acc[m][0] += __builtin_bit_cast(f32x4, e0); acc[m][1] += __builtin_bit_cast(f32x4, e1);
            acc[m][2] += __builtin_bit_cast(f32x4, e2); acc[m][3] += __builtin_bit_cast(f32x4, e3);
        }
    }
    const unsigned tmo_v = __hip_atomic_load(tmo, RLX, SC_AGENT);
    const unsigned oo = (unsigned)(fq * 4) * (GN * 4u) + fr * 16u;
    const u32x4 nan4 = {0x7fc00000u, 0x7fc00000u, 0x7fc00000u, 0x7fc00000u};
    auto finish = [&](unsigned (&o)[32], int i, float a, float b) {
        const f32x2 g = gelu2((f32x2){a, b});
        const int at = ((i >> 3) * 4 + (i & 1) * 2) * 4 + ((i >> 1) & 3);
        o[at] = __float_as_uint(g.x); o[at + 4] = __float_as_uint(g.y);
    };
#define ACC(hf, i, k) acc[2 * (hf) + ((i) >> 3)][((i) >> 1) & 3][((i) & 1) * 2 + (k)]
    if (late) {
        const bool bad = UNI(tmo_v) == call;
        const char* ob = (const char*)(p.out + (size_t)(rb_local(rank, rb) + wm * 64) * GN + (rank >> 1) * 4096 + cb * 128 + wn * 64);
#pragma unroll
        for (int hf = 0; hf < 2; ++hf) {
            unsigned o[32];
#pragma unroll
            for (int i = 0; i < 16; ++i) finish(o, i, ACC(hf, i, 0), ACC(hf, i, 1));
#pragma unroll
            for (int i = 0; i < 8; ++i) st16(ob + (size_t)(hf * 32 + (i >> 2) * 16 + (i & 3)) * (GN * 4), oo, bad ? nan4 : (u32x4){o[4 * i], o[4 * i + 1], o[4 * i + 2], o[4 * i + 3]});
        }
    } else {
        const int peer = rb_owner(rank, rb);
        const int mirror = (8 + rb_of(peer, rank) * 4 + (idx & 3)) * 8 + xcd;
        char* const peerws = p.ws[peer];
        WAIT_L0(); BAR();
        if (wid < 4) __builtin_amdgcn_s_setprio(2);
#pragma unroll
        for (int hf = 0; hf < 2; ++hf) {
            const int k = (wid >> 2) * 8 + hf * 4 + q;
            const char* ex = myws + WS_EXO + (size_t)(bid * 16 + k) * 8192;
            unsigned hdr = 1u << 13;
            if ((idx & 3) != 3 || k < 16 - NPLAIN) {
                LAS unsigned* pool = (LAS unsigned*)(lds + 131072 + wid * 256);
                pool[lane] = 0u;
                unsigned hi[16], ef[16], under = 0;
#pragma unroll
                for (int i = 0; i < 16; ++i) {
                    hi[i] = __builtin_amdgcn_perm(__float_as_uint(ACC(hf, i, 1)), __float_as_uint(ACC(hf, i, 0)), 0x07060302u);
                    ef[i] = (hi[i] >> 7) & 0x00ff00ffu;
                }
                const unsigned bs = (unsigned)code_base<8>(ef), e0 = bs > 20u ? bs - 20u : 0u;
#pragma unroll
                for (int i = 0; i < 16; ++i) { under |= SUBS(e0 * 0x00010001u, ef[i]); ef[i] = SUBS(ef[i], e0 * 0x00010001u); }
                if (__all(under == 0u) && code_put<POOL_OUT>(ef, bs - e0, pool, fq)) {
#pragma unroll
                    for (int i = 0; i < 16; ++i) hi[i] = (hi[i] & 0x807f807fu) | (ef[i] << 7);
#pragma unroll
                    for (int c = 0; c < 6; ++c) {
#pragma unroll
                        for (int i = 0; i < 3; ++i) { const int fld = c * 3 + i, at = 9 + 3 * (fld & 1); hi[fld >> 1] |= (hi[10 + c] << (at - 3 * i)) & (0x00070007u << at); }
                        hi[9] |= (hi[10 + c] >> (1 + c)) & (0x40004000u >> c);
                    }
                    const unsigned pw = __hip_atomic_load(pool + lane, RLX, SC_WG);
#pragma unroll
                    for (int i = 0; i < 4; ++i) {
                        u32x4 v;
#pragma unroll
                        for (int e = 0; e < 4; ++e) v[e] = __builtin_amdgcn_perm(__float_as_uint(ACC(hf, 4 * i + e, 1)), __float_as_uint(ACC(hf, 4 * i + e, 0)), 0x05040100u);
                        st16(ex + i * 1024, lo, v);
                    }
                    st16(ex + 4096, lo, (u32x4){hi[0], hi[1], hi[2], hi[3]});
                    st16(ex + 5120, lo, (u32x4){hi[4], hi[5], hi[6], hi[7]});
                    st8(ex + 6144, lane * 8u, (u32x2){hi[8], hi[9]});
                    if (lane < POOL_OUT) st4(ex + 6656, lane * 4u, pw);
                    hdr = e0 | ((bs - e0) << 8);
                }
            }
            if (hdr >> 13) {
                unsigned o[32];
#pragma unroll
                for (int i = 0; i < 16; ++i) finish(o, i, ACC(hf, i, 0), ACC(hf, i, 1));
#pragma unroll
                for (int i = 0; i < 8; ++i) st16(ex + i * 1024, lo, (u32x4){o[4 * i], o[4 * i + 1], o[4 * i + 2], o[4 * i + 3]});
            }
            wait_vm<0>();
            hdr |= UNI(tmo_v) == call ? 0xc0000000u : 0x40000000u;
            if (lane == 0) __hip_atomic_store((GAS u64*)(peerws + WS_ORDY + (size_t)mirror * 128) + k, ((u64)call << 32) | hdr, RLX, SC_SYS);
        }
        __builtin_amdgcn_s_setprio(0);
#pragma unroll 1
        for (int sk = 0; (idx & 3) + sk <= 3 && (idx & 3) != 3; sk += 3) {
        if (sk) {
            WAIT_L0(); BAR();
            if (tid < 32 || tid == 384 || tid == 401 || (tid >= 416 && tid < 432)) ctl[tid] = 0u;
            WAIT_L0(); BAR();
        }
        const int me = bid + 8 * sk;
        const char* exs = peerws + WS_EXO + (size_t)(mirror + 8 * sk) * (16 * 8192);
        if (wid == 0) {
            unsigned tnext;
            while ((tnext = lds_get(ctl + 16)) == 0u) __builtin_amdgcn_s_sleep(2);
            const unsigned trb = tnext & 255u, tr = trb - 128u;
            tnext = (tnext & ~255u) | 128u;
            __builtin_amdgcn_s_setprio(3);
            wait_vm<0>(); __builtin_amdgcn_s_waitcnt(0x0F70);
            int issued = 0, told = 0;
            unsigned hn = lds_get(ctl);
            for (int n = 0; n < 16; ++n) {
                if (hn == 0u && (hn = lds_get(ctl + n)) == 0u) {
                    wait_vm<0>();
                    told = issued;
                    if (lane == 0) __hip_atomic_store(ctl + 384, (unsigned)told, RLX, SC_WG);
                    while ((hn = lds_get(ctl + n)) == 0u) __builtin_amdgcn_s_sleep(1);
                }
                asm volatile("" ::: "memory");
                const bool plain = (hn >> 13) & 1u;
                const char* cur = exs + (size_t)n * 8192;
                const unsigned dst = lds0 + (unsigned)n * 8192u;
#pragma unroll
                for (int i = 0; i < 8; i += 2) {
                    wait_vm<WIN - 2>();
                    if (issued - (WIN - 2) > told) { told = issued - (WIN - 2); if (lane == 0) __hip_atomic_store(ctl + 384, (unsigned)told, RLX, SC_WG); }
                    pace(tnext, (i < 6 || plain ? 2u * TAU : PIECE_T - 6u * TAU) - tr);
                    if (i < 6 || plain) { dma<true>(cur + i * 1024, lo, dst + i * 1024); issued += 2; }
                    else { if (lane < (PB - 6144) / 16) dma<false>(cur + 6144, lo, dst + 6144); issued += 1; }
                    if (i == 0) {
                        if (lane == 0) __hip_atomic_store(ctl + 416 + n, (unsigned)issued + (plain ? 6u : 5u), RLX, SC_WG);
                        if (n == 14 && !sk && lane == 0)
                            __hip_atomic_store((GAS u64*)(myws + WS_MBOX + (size_t)(bid + 8) * 128), ((u64)call << 32) | ((tnext + 2u * (PIECE_T) - 2u * TAU - 7u * tr) & ~255u) | trb, RLX, SC_AGENT);
                    }
                    if (i == 2) hn = n < 15 ? lds_get(ctl + n + 1) : 0u;
                }
            }
            wait_vm<7>();
            if (issued - 7 > told && lane == 0) __hip_atomic_store(ctl + 384, (unsigned)issued - 7u, RLX, SC_WG);
            wait_vm<0>();
            if (lane == 0) __hip_atomic_store(ctl + 384, (unsigned)issued, RLX, SC_WG);
            __builtin_amdgcn_s_setprio(0);
        } else {
            if (wid == 1) {
                const GAS u64* fp = lane < 16 ? (const GAS u64*)(myws + WS_ORDY + (size_t)me * 128) + lane : (const GAS u64*)(myws + WS_MBOX + (size_t)me * 128);
                bool pend = lane < 17;
                const u64 t0 = __builtin_amdgcn_s_memrealtime();
                for (unsigned polls = 1;; ++polls) {
                    if (pend) {
                        const u64 f = __hip_atomic_load(fp, RLX, SC_SYS);
                        if ((unsigned)(f >> 32) >= call) { __hip_atomic_store(ctl + lane, (unsigned)f, RLX, SC_WG); pend = false; }
                    }
                    if (!__any(pend)) break;
                    if ((polls & 15u) == 0u && TIMED_OUT(t0)) {
                        if (lane == 0) __hip_atomic_store(tmo, call, RLX, SC_AGENT);
                        if (pend) __hip_atomic_store(ctl + lane, lane < 16 ? 0xc0002000u : (now64() & ~255u) | 128u, RLX, SC_WG);
                        break;
                    }
                    __builtin_amdgcn_s_sleep(4);
                }
            }
            for (;;) {
                unsigned tk = 0;
                if (lane == 0) tk = __hip_atomic_fetch_add(ctl + 401, 1u, RLX, SC_WG);
                const int k = UNI(tk);
                if (k >= 16) break;
                unsigned need;
                while ((need = lds_get(ctl + 416 + k)) == 0u) __builtin_amdgcn_s_sleep(4);
                const unsigned hdr = lds_get(ctl + k);
                while (lds_get(ctl + 384) < need) __builtin_amdgcn_s_sleep(1);
                asm volatile("" ::: "memory");
                const LAS unsigned char* src = lds + (unsigned)k * 8192u;
                unsigned o[32];
                if ((hdr >> 13) & 1u) {
#pragma unroll
                    for (int i = 0; i < 8; ++i) { const u32x4 a = *(const LAS u32x4*)(src + i * 1024 + lo); o[4 * i] = a[0]; o[4 * i + 1] = a[1]; o[4 * i + 2] = a[2]; o[4 * i + 3] = a[3]; }
                } else {
                    unsigned lw[16], hi[16], ne[16];
#pragma unroll
                    for (int i = 0; i < 2; ++i) { const u32x4 a = *(const LAS u32x4*)(src + 4096 + i * 1024 + lo); hi[4 * i] = a[0]; hi[4 * i + 1] = a[1]; hi[4 * i + 2] = a[2]; hi[4 * i + 3] = a[3]; }
                    { const u32x2 a = *(const LAS u32x2*)(src + 6144 + lane * 8); hi[8] = a[0]; hi[9] = a[1]; }
#pragma unroll
                    for (int i = 0; i < 4; ++i) { const u32x4 a = *(const LAS u32x4*)(src + i * 1024 + lo); lw[4 * i] = a[0]; lw[4 * i + 1] = a[1]; lw[4 * i + 2] = a[2]; lw[4 * i + 3] = a[3]; }
#pragma unroll
                    for (int c = 0; c < 6; ++c) {
                        unsigned cc = (hi[9] << (1 + c)) & 0x80008000u;
#pragma unroll
                        for (int i = 0; i < 3; ++i) { const int fld = c * 3 + i, at = 9 + 3 * (fld & 1); cc |= (hi[fld >> 1] >> (at - 3 * i)) & (0x00070007u << (3 * i)); }
                        hi[10 + c] = cc;
                    }
                    u64 str = code_find<7>(hi, ne, (const LAS unsigned*)(src + 6656), fq);
                    const unsigned e0 = (hdr & 255u) * 0x00800080u, b2 = ((hdr >> 8) & 31u) * 0x00010001u;
#pragma unroll
                    for (int i = 0; i < 16; ++i) {
                        const unsigned h2 = (hi[i] & 0x81ff81ffu) + e0 + (code_get(str, ne[i], b2) << 7);
                        finish(o, i, __uint_as_float(__builtin_amdgcn_perm(h2, lw[i], 0x05040100u)), __uint_as_float(__builtin_amdgcn_perm(h2, lw[i], 0x07060302u)));
                    }
                }
                const int blk = (k >> 3) * 4 + (k & 3);
                const char* ob = (const char*)(p.out + (size_t)((peer & 1) * 256 + (blk >> 1) * 64 + ((k >> 2) & 1) * 32) * GN + (peer >> 1) * 4096 + (cb + sk) * 128 + (blk & 1) * 64);
#pragma unroll
                for (int i = 0; i < 8; ++i)
                    st16(ob + (size_t)((i >> 2) * 16 + (i & 3)) * (GN * 4), oo, (hdr >> 31) ? nan4 : (u32x4){o[4 * i], o[4 * i + 1], o[4 * i + 2], o[4 * i + 3]});
            }
        }
        }
    }
    }
    if (tid == 0) __hip_atomic_store(ctr, call, RLX, SC_AGENT);
}

extern "C" void kernel_launch(int rank, int nranks, ncclComm_t comm, hipStream_t stream,
                              void* const* d_in, const int* in_sizes, int n_in,
                              void* d_out, int out_size, const int* in_dtypes, int out_dtype,
                              void* const* d_out_all, void* const* d_ws_all, size_t ws_bytes) {
    (void)comm; (void)in_dtypes; (void)out_dtype; (void)d_out_all;
    if (nranks != NR || (unsigned)rank >= (unsigned)NR || n_in != 2 || ws_bytes < WS_NEED ||
        in_sizes[0] != 4096 * KSH || in_sizes[1] != KSH * GN || out_size != 512 * GN) {
        fprintf(stderr, "kernel_launch: unexpected problem shape\n");
        return;
    }
    Params p;
    p.x = (const float*)d_in[0];
    p.w = (const float*)d_in[1];
    p.out = (float*)d_out;
    p.myws = (char*)d_ws_all[rank];
    for (int i = 0; i < NR; ++i) p.ws[i] = (char*)d_ws_all[i];
    p.rank = rank;
    p.pad = 0;
    int n = 0, np = 0, nd = 0;
    unsigned char deferred[8];
    bool px[8] = {};
    for (int i = 0; i < 128; ++i) p.order[0][i] = p.order[1][i] = 0;
    auto need_x = [&](int c0, int c1) { for (int c = c0; c <= c1; ++c) if (!px[c]) { px[c] = true; p.prod[np++] = (unsigned)(32 + c); } };
    for (int key = 0; key < 16 + 3 * SKEW2; ++key) {
        for (int pass = 0; pass < 2; ++pass) {
            if ((pass == 0) == (key != 0)) {
                for (int e = 32; e < 44; ++e)
                    if (2 * ((e - 32) * 8 / 12) == key) { p.order[0][n++] = (unsigned char)e; need_x((e - 32) * 8 / 12, ((e - 32) * 8 + 7) / 12); }
                if (key == XPOS) { p.order[0][n++] = 44; need_x(0, 3); }
                if (key == XPOS + 2) { p.order[0][n++] = 45; need_x(4, 7); }
                if (key == LPOS) { p.order[0][n++] = 46; p.order[0][n++] = 47; }
            } else {
                for (int b = 0; b < 4; ++b)
                    for (int c = 0; c < 8; ++c)
                        if ((b == 0 && c == 1 ? 0 : 2 * c + b * SKEW2) == key) {
                            p.prod[np++] = (unsigned)(c * 4 + b);
                            if (b == 3 && c < 2 * NFWD && ((c ^ rank) & 1)) deferred[nd++] = (unsigned char)(c * 4 + b);
                            else p.order[0][n++] = (unsigned char)(c * 4 + b);
                        }
            }
        }
    }
    int m = 0;
    for (int i = 0; i < n; ++i) if (p.order[0][i] < 44) p.order[1][m++] = p.order[0][i];
    for (int i = 0; i < nd; ++i) p.order[1][m++] = deferred[i];
    for (int i = 0; i < 6 * NFWD; ++i) p.order[1][m++] = (unsigned char)(48 + i);
    for (int e = 44; e < 48; ++e) p.order[1][m++] = (unsigned char)e;
    if (n != NU_OUT || m != NU_PAIR || np != 40 || p.prod[0] != 0 || p.prod[1] != 4 || p.prod[2] != 32) { fprintf(stderr, "kernel_launch: bad schedule\n"); return; }
    hipLaunchKernelGGL(fused_gemm_rs, dim3(NBLK), dim3(NTHR), 0, stream, p);
    HIP_CHECK(hipGetLastError());
}
```

```cpp
#include <hip/hip_runtime.h>
#include <rccl/rccl.h>
#include <cstdio>

#define HIP_CHECK(expr) do { hipError_t _e = (expr); if (_e != hipSuccess) { \
    fprintf(stderr, "HIP error %s at %s:%d\n", hipGetErrorString(_e), __FILE__, __LINE__); \
    return; } } while (0)

typedef float f32x4 __attribute__((ext_vector_type(4)));
typedef unsigned u32x4 __attribute__((ext_vector_type(4)));
typedef unsigned u32x3 __attribute__((ext_vector_type(3)));
typedef unsigned u32x2 __attribute__((ext_vector_type(2)));
typedef unsigned short u16x2 __attribute__((ext_vector_type(2)));
typedef _Float16 h16x8 __attribute__((ext_vector_type(8)));
typedef unsigned long long u64;
#define LAS __attribute__((address_space(3)))
#define GAS __attribute__((address_space(1)))
#define RLX __ATOMIC_RELAXED
#define SC_WG __HIP_MEMORY_SCOPE_WORKGROUP
#define SC_AGENT __HIP_MEMORY_SCOPE_AGENT
#define SC_SYS __HIP_MEMORY_SCOPE_SYSTEM

constexpr int TAU = 863;
constexpr int WIN = 10;
constexpr int CREDIT = 36;
constexpr int EQ_AT = 5;
constexpr int PUSH_SLOT = 112;
constexpr int T_PULL = 350;
constexpr int PROD_GAP = 40;
constexpr int NFWD = 3;
constexpr int NPLAIN = 3;
constexpr int SKEW2 = 2;
constexpr int XPOS = 6, LPOS = 18;
constexpr int RSTRIDE = 16;
constexpr int POLLW = 12;

constexpr int NR = 8;
constexpr int GN = 16384;
constexpr int KSH = 512;
constexpr int NBLK = 256;
constexpr int NTHR = 512;
constexpr int NSTEP = 64;
constexpr unsigned A_STEP = 32768, B_STEP = 16384, STAGE = A_STEP + B_STEP;
constexpr int NU_OUT = 48 - NFWD, NU_PAIR = 48 + 6 * NFWD;
constexpr int RING = 10;
constexpr int PW = 14;
constexpr int POOL_IN = 47, POOL_OUT = 48;
constexpr int QB = 3328, PAREA = 4 * QB;
constexpr unsigned SLOT = 18432, SIDE = 14336;
constexpr size_t WS_CTR = 0;
constexpr size_t WS_FLAG = 32 << 10;
constexpr size_t WS_ORDY = 96 << 10;
constexpr size_t WS_MBOX = 128 << 10;
constexpr size_t WS_EQ = 160 << 10;
constexpr size_t WS_TMO = 192 << 10;
constexpr size_t WS_STG = 1 << 20;
constexpr size_t STG_A = 8ull * NSTEP * A_STEP;
constexpr size_t STG_B = 32ull * NSTEP * B_STEP;
constexpr size_t WS_EXI = WS_STG + STG_A + STG_B;
constexpr size_t EXI_BYTES = 1600ull * SLOT;
constexpr size_t WS_EXO = WS_EXI + 2 * EXI_BYTES;
constexpr size_t WS_EXP = WS_EXO + (size_t)NBLK * 8 * 16384;
constexpr size_t WS_RDY = WS_EXP + (size_t)NBLK * 4096;
constexpr size_t WS_NEED = WS_RDY + 2 * 8 * 48 * 8 * 128;
constexpr u64 DEADLINE_TICKS = 50000000ull;

struct Params {
    const float* x;
    const float* w;
    float* out;
    char* myws;
    char* ws[NR];
    int rank;
    int pad;
    unsigned prod[40];
    unsigned char order[2][128];
};

#define WAIT_L0() do { asm volatile("s_waitcnt lgkmcnt(0)" ::: "memory"); __builtin_amdgcn_s_waitcnt(0xC07F); } while (0)
#define BAR() do { asm volatile("" ::: "memory"); __builtin_amdgcn_s_barrier(); asm volatile("" ::: "memory"); } while (0)
#define SB() __builtin_amdgcn_sched_barrier(0)
#define UNI(x) __builtin_amdgcn_readfirstlane(x)
template <int N> __device__ __forceinline__ void wait_vm() { asm volatile("s_waitcnt vmcnt(%0)" ::"i"(N) : "memory"); }

__device__ __forceinline__ void st16(const void* base, unsigned voff, u32x4 v) {
    asm volatile("s_nop 4\n\tglobal_store_dwordx4 %0, %1, %2 sc0 sc1\n\ts_nop 1" ::"v"(voff), "v"(v), "s"(base) : "memory");
}
__device__ __forceinline__ void st12(const void* base, unsigned voff, u32x3 v) {
    asm volatile("s_nop 4\n\tglobal_store_dwordx3 %0, %1, %2 sc0 sc1\n\ts_nop 1" ::"v"(voff), "v"(v), "s"(base) : "memory");
}
__device__ __forceinline__ void st8(const void* base, unsigned voff, u32x2 v) {
    asm volatile("s_nop 4\n\tglobal_store_dwordx2 %0, %1, %2 sc0 sc1" ::"v"(voff), "v"(v), "s"(base) : "memory");
}
__device__ __forceinline__ void st1(const void* base, unsigned voff, unsigned v) {
    asm volatile("s_nop 4\n\tglobal_store_byte %0, %1, %2 sc0 sc1" ::"v"(voff), "v"(v), "s"(base) : "memory");
}
__device__ __forceinline__ void st4(const void* base, unsigned voff, unsigned v) {
    asm volatile("s_nop 4\n\tglobal_store_dword %0, %1, %2 sc0 sc1" ::"v"(voff), "v"(v), "s"(base) : "memory");
}
__device__ __forceinline__ u32x3 ld12(const void* base, unsigned voff) {
    u32x3 r;
    asm volatile("s_nop 4\n\tglobal_load_dwordx3 %0, %1, %2 sc0 sc1\n\ts_waitcnt vmcnt(0)" : "=&v"(r) : "v"(voff), "s"(base) : "memory");
    return r;
}
__device__ __forceinline__ void ld64(const void* base, unsigned voff, u32x4& a, u32x4& b, u32x4& c, u32x4& d) {
    asm volatile("s_nop 4\n\tglobal_load_dwordx4 %0, %4, %5 sc0 sc1\n\tglobal_load_dwordx4 %1, %4, %5 offset:1024 sc0 sc1\n\t"
                 "global_load_dwordx4 %2, %4, %5 offset:2048 sc0 sc1\n\tglobal_load_dwordx4 %3, %4, %5 offset:3072 sc0 sc1\n\ts_waitcnt vmcnt(0)"
                 : "=&v"(a), "=&v"(b), "=&v"(c), "=&v"(d) : "v"(voff), "s"(base) : "memory");
}
template <bool TWO> __device__ __forceinline__ void dma(const void* base, unsigned voff, unsigned lds_dst) {
    unsigned keep;
    if (TWO) asm volatile("s_mov_b32 %0, m0\n\ts_mov_b32 m0, %3\n\ts_nop 4\n\tglobal_load_lds_dwordx4 %1, %2 sc0 sc1\n\tglobal_load_lds_dwordx4 %1, %2 offset:1024 sc0 sc1\n\ts_mov_b32 m0, %0"
                          : "=&s"(keep) : "v"(voff), "s"(base), "s"(lds_dst) : "memory");
    else asm volatile("s_mov_b32 %0, m0\n\ts_mov_b32 m0, %3\n\ts_nop 4\n\tglobal_load_lds_dwordx4 %1, %2 sc0 sc1\n\ts_mov_b32 m0, %0"
                      : "=&s"(keep) : "v"(voff), "s"(base), "s"(lds_dst) : "memory");
}
__device__ __forceinline__ unsigned now64() { return (unsigned)__builtin_amdgcn_s_memrealtime() << 6; }
__device__ __forceinline__ void pace(unsigned& t, unsigned period) {
    int late = (int)(now64() - t);
    if (late < -(1 << 22)) { t += (unsigned)late; late = 0; }
    while (late < 0) { __builtin_amdgcn_s_sleep(1); late = (int)(now64() - t); }
    t += period + (late > CREDIT * 64 ? (unsigned)(late - CREDIT * 64) : 0u);
}
__device__ __forceinline__ unsigned lds_get(const LAS unsigned* a) { return UNI(__hip_atomic_load(a, RLX, SC_WG)); }
__device__ __forceinline__ unsigned cvt2(float a, float b) {
    typedef _Float16 h2 __attribute__((ext_vector_type(2)));
    h2 r = {(_Float16)a, (_Float16)b};
    return __builtin_bit_cast(unsigned, r);
}
__device__ __forceinline__ int rb_owner(int d, int rb) { return rb >= 6 ? d : ((((d >> 1) + 1 + (rb >> 1)) & 3) << 1) | (rb & 1); }
__device__ __forceinline__ int rb_local(int d, int rb) { return rb >= 6 ? (rb - 6) * 256 : (d & 1) * 256; }
__device__ __forceinline__ int rb_of(int d, int o) { return ((((o >> 1) - (d >> 1) - 1) & 3) << 1) | (o & 1); }
__device__ __forceinline__ void unit_of(int e, int j, int& c, int& id) {
    if (e < 32) { c = e >> 2; id = j * 4 + (e & 3); }
    else if (e < 44) { const int u = (e - 32) * 8 + j; c = u / 12; id = 32 + u - c * 12; }
    else { const int u = (e & 1) * 8 + j; c = u >> 1; id = (e < 46 ? 46 : 44) + (u & 1); }
}
__device__ __forceinline__ int gid_of(int d, int id) {
    const int rb = (id - 32) >> 1;
    return id < 32 ? (d >> 1) * 32 + id : 128 + rb_owner(d, rb) * 4 + (rb_local(d, rb) >> 7) + (id & 1);
}
__device__ __forceinline__ size_t stage_off(int id, int ks) {
    return id < 32 ? STG_A + (size_t)(id * NSTEP + ks) * B_STEP : (size_t)(((id - 32) >> 1) * NSTEP + ks) * A_STEP + (id & 1) * 16384;
}
template <int BITS, int SH> __device__ __forceinline__ unsigned wave_max_exp(unsigned mag) {
    unsigned e = 0;
#pragma unroll
    for (int b = BITS - 1; b >= 0; --b) if (__any(mag >= ((e + (1u << b)) << SH))) e += 1u << b;
    return e;
}

__device__ __forceinline__ unsigned row_scan(unsigned x) {
    x += __builtin_amdgcn_update_dpp(0, x, 0x111, 0xf, 0xf, true);
    x += __builtin_amdgcn_update_dpp(0, x, 0x112, 0xf, 0xf, true);
    x += __builtin_amdgcn_update_dpp(0, x, 0x114, 0xf, 0xf, true);
    x += __builtin_amdgcn_update_dpp(0, x, 0x118, 0xf, 0xf, true);
    return x;
}
__device__ __forceinline__ unsigned ffbl(unsigned x) { unsigned r; asm("v_ffbl_b32 %0, %1" : "=v"(r) : "v"(x)); return r; }
#define PK(x) __builtin_bit_cast(u16x2, (unsigned)(x))
#define DW(x) __builtin_bit_cast(unsigned, (u16x2)(x))
#define SUBS(a, b) DW(__builtin_elementwise_sub_sat(PK(a), PK(b)))
#define MIN2(a, b) DW(__builtin_elementwise_min(PK(a), PK(b)))

template <int BITS> __device__ __forceinline__ int code_base(const unsigned (&ef)[16]) {
    u16x2 mx = PK(ef[0]);
#pragma unroll
    for (int i = 1; i < 16; ++i) mx = __builtin_elementwise_max(mx, PK(ef[i]));
    const int emax = (int)wave_max_exp<BITS, 0>(mx.x > mx.y ? mx.x : mx.y);
    int up = 0, mid = 0;
#pragma unroll
    for (int i = 0; i < 6; ++i) {
        const int e = (int)(ef[i] >> 16);
        up += __builtin_popcountll(__ballot(e == emax)) - __builtin_popcountll(__ballot(e == emax - 3));
        mid += __builtin_popcountll(__ballot(e == emax - 1)) - __builtin_popcountll(__ballot(e == emax - 4));
    }
    const int bs = emax - (up > 0 ? 3 : mid > 0 ? 4 : 5);
    return bs > 0 ? bs : 0;
}
template <int POOL> __device__ __forceinline__ unsigned code_put(unsigned (&ef)[16], unsigned B, LAS unsigned* pool, int row) {
    const unsigned b2 = B * 0x00010001u, lut = 0x0300u | ((B + 2u + (B ? 1u : 0u)) << 16);
    u64 str = 0;
    unsigned sh = 0;
#pragma unroll
    for (int i = 0; i < 16; ++i) {
        const unsigned k1 = SUBS(b2 + 0x00010001u, ef[i]), d = SUBS(ef[i], b2);
        const unsigned len = k1 + MIN2(SUBS(k1, 0x00020002u), 0x00010001u) + __builtin_amdgcn_perm(0u, lut, SUBS(d, 0x00030003u) | 0x0c000c00u);
        sh += len & 0xffffu; str |= 1ull << (sh & 63u);
        sh += len >> 16; str |= 1ull << (sh & 63u);
        ef[i] = MIN2(d, 0x00040004u) & 0x00030003u;
    }
    const unsigned end = row_scan(sh);
    const unsigned r0 = __builtin_amdgcn_readlane(end, 15), r1 = __builtin_amdgcn_readlane(end, 31), r2 = __builtin_amdgcn_readlane(end, 47);
    const unsigned bits = r0 + r1 + r2 + __builtin_amdgcn_readlane(end, 63);
    if (!__all(sh <= 63u && end <= 32u * (PW - 1)) || bits > 32u * (POOL - 1)) return 0u;
    const unsigned pos = 32u + end - sh + (row > 0 ? r0 : 0u) + (row > 1 ? r1 : 0u) + (row > 2 ? r2 : 0u);
    str >>= 1;
    const u64 s0 = str << (pos & 31u);
    LAS unsigned* pp = pool + (pos >> 5);
    __hip_atomic_store(pool, r0 | r1 << 10 | r2 << 20, RLX, SC_WG);
    __hip_atomic_fetch_or(pp, (unsigned)s0, RLX, SC_WG);
    __hip_atomic_fetch_or(pp + 1, (unsigned)(s0 >> 32), RLX, SC_WG);
    __hip_atomic_fetch_or(pp + 2, (unsigned)(((str >> 32) << (pos & 31u)) >> 32), RLX, SC_WG);
    return (bits + 63u) >> 5;
}
template <int SH> __device__ __forceinline__ u64 code_find(const unsigned (&w)[16], unsigned (&ne)[16], const LAS unsigned* pool, int row) {
    const unsigned rl = pool[0];
    const unsigned start = 32u + (row > 0 ? rl & 1023u : 0u) + (row > 1 ? (rl >> 10) & 1023u : 0u) + (row > 2 ? (rl >> 20) & 1023u : 0u);
    pool += start >> 5;
    unsigned pd[PW - 1], sum = 0;
#pragma unroll
    for (int i = 0; i < PW - 1; ++i) pd[i] = pool[i];
#pragma unroll
    for (int i = 0; i < 16; ++i) { ne[i] = MIN2(w[i] & (0x00030003u << SH), 0x00010001u); sum += ne[i]; }
    sum = 32u - (sum & 0xffffu) - (sum >> 16);
    const unsigned skip = row_scan(sum) - sum + __builtin_popcount(__builtin_amdgcn_ubfe(pd[0], 0u, start));
    unsigned cum = 0, at = 0, below = 0;
#pragma unroll
    for (int i = 0; i < PW - 1; ++i) {
        cum += __builtin_popcount(pd[i]);
        const bool lt = cum < skip;
        at += lt; below = lt ? cum : below;
    }
    const unsigned d0 = pool[at], d1 = pool[at + 1], d2 = pool[at + 2];
    unsigned rem = skip - below, pos = 0;
#pragma unroll
    for (int sft = 16; sft; sft >>= 1) {
        const unsigned t = __builtin_popcount(__builtin_amdgcn_ubfe(d0, pos, sft));
        const bool lt = t < rem;
        rem -= lt ? t : 0u; pos += lt ? sft : 0u;
    }
    pos = skip ? pos + 1u : 0u;
    return (unsigned)((((u64)d1 << 32) | d0) >> pos) | ((((u64)d2 << 32) | d1) >> pos << 32);
}
__device__ __forceinline__ unsigned code_get(u64& str, unsigned ne, unsigned b2) {
    const unsigned l0 = ffbl(((unsigned)str << 1) | (ne & 1u)); str >>= l0 & 63u;
    const unsigned l1 = ffbl(((unsigned)str << 1) | (ne >> 16)); str >>= l1 & 63u;
    const unsigned len = l0 | (l1 << 16);
    const unsigned adj = __builtin_amdgcn_perm(0x02u, 0x07010100u, MIN2(len, 0x00040004u) | 0x0c000c00u);
    return MIN2(DW(PK(b2 + adj) - PK(len)), b2 + 0x00050005u);
}

typedef float f32x2 __attribute__((ext_vector_type(2)));
__device__ __forceinline__ f32x2 gelu2(f32x2 y) {
    const float k1 = -2.0f * 0.7978845608028654f * 1.4426950408889634f, k3 = k1 * 0.044715f;
    const f32x2 a = __builtin_elementwise_fma(y * y, (f32x2){k3, k3}, (f32x2){k1, k1}) * y;
    const f32x2 d = (f32x2){__builtin_amdgcn_exp2f(a.x), __builtin_amdgcn_exp2f(a.y)} + 1.0f;
    return y * (f32x2){__builtin_amdgcn_rcpf(d.x), __builtin_amdgcn_rcpf(d.y)};
}

#define TIMED_OUT(t0) (UNI(__hip_atomic_load(tmo, RLX, SC_AGENT)) == call || __builtin_amdgcn_s_memrealtime() - (t0) > DEADLINE_TICKS)

__global__ __launch_bounds__(NTHR, 2) void fused_gemm_rs(Params p) {
    __shared__ __attribute__((aligned(16))) unsigned char lds_raw[3 * STAGE + 2048];
    LAS unsigned char* lds = (LAS unsigned char*)lds_raw;
    LAS unsigned* ctl = (LAS unsigned*)(lds + 3 * STAGE);
    const unsigned lds0 = (unsigned)(size_t)lds;
    const int tid = threadIdx.x, lane = tid & 63;
    const int wid = UNI(tid >> 6);
    const int fr = lane & 15, fq = lane >> 4;
    const unsigned lo = lane * 16u;
    const unsigned bid = (blockIdx.x + 64u) & 255u;
    const int xcd = bid & 7, idx = bid >> 3;
    const bool late = idx < 8;
    const int q = wid & 3;

    const int team = (wid >> 2) * 192 + (int)bid - 64;
    f32x4 r[8];
    auto request = [&](int g, int u) {
        if (g < 32) {
            const GAS char* b = (const GAS char*)(p.w + (size_t)((g >> 2) * 64 + (q & 1) * 32) * GN + (u * 4 + (g & 3)) * 128 + (q >> 1) * 64) + (unsigned)(8 * fq * GN + 4 * fr) * 4u;
#pragma unroll
            for (int e = 0; e < 8; ++e) r[e] = *(const GAS f32x4*)(b + (size_t)e * (GN * 4));
        } else {
            const GAS char* b = (const GAS char*)(p.x + (size_t)(u * 128 + q * 32) * KSH + (g - 32) * 64) + (unsigned)(fr * KSH + 8 * fq) * 4u;
#pragma unroll
            for (int e = 0; e < 8; ++e) r[e] = *(const GAS f32x4*)(b + (e >> 2) * (16 * KSH * 4) + ((e >> 1) & 1) * 128 + (e & 1) * 16);
        }
    };
    if (!late && team < 96) request(team >= 64 ? 32 : (team >> 5) * 4, team & 31);
    const unsigned t_entry = now64();

    const int rank = p.rank & 7;
    char* myws = p.myws;
    GAS unsigned* ctr = (GAS unsigned*)(myws + WS_CTR + (size_t)bid * 128);
    GAS unsigned* tmo = (GAS unsigned*)(myws + WS_TMO);
    const unsigned ctr_v = __hip_atomic_load(ctr, RLX, SC_AGENT);
    GAS unsigned* gflag = (GAS unsigned*)(myws + WS_FLAG);
    ctl[tid] = 0u;
    WAIT_L0();
    BAR();

    if (!late) {
        unsigned wlo = 0, whi = 0;
#pragma unroll
        for (int d = 0; d < NR; ++d) {
            unsigned a = (unsigned)(size_t)p.ws[d], b = (unsigned)((size_t)p.ws[d] >> 32);
            asm volatile("" : "+s"(a), "+s"(b));
            wlo = lane == d ? a : wlo;
            whi = lane == d ? b : whi;
        }
#pragma unroll 1
        for (int it = 0; it < 5; ++it) {
            const int t = it < 2 ? it * 384 + team : 2 * 384 + (it - 2) * 288 + (wid >> 2) * 144 + (((idx - 8) >> 2) * 3 + (idx & 3) - 1) * 8 + xcd;
            if (t >= 1280 || (it >= 2 && (idx & 3) == 0)) break;
            const int g = p.prod[t >> 5], u = t & 31;
            if (t >= 96) {
                const unsigned ts = t_entry + (unsigned)((t >> 5) + 2) * (PROD_GAP * 64);
                while ((int)(now64() - ts) < 0) __builtin_amdgcn_s_sleep(2);
                request(g, u);
            }
            const bool isw = g < 32;
            const int c = isw ? g >> 2 : g - 32;
            const int gid = isw ? u * 4 + (g & 3) : 128 + u;
            unsigned v[16];
            if (isw) {
#pragma unroll
                for (int i = 0; i < 16; ++i) v[i] = cvt2(r[(i & 3) * 2][i >> 2], r[(i & 3) * 2 + 1][i >> 2]);
            } else {
#pragma unroll
                for (int i = 0; i < 16; ++i) v[i] = cvt2(r[i >> 1][(i & 1) * 2], r[i >> 1][(i & 1) * 2 + 1]);
            }
            asm volatile("" : "+v"(v[15]));
            const unsigned t_loaded = now64();
            const bool pushed = t < 32;
            const unsigned call = UNI(ctr_v) + 1u;
            const size_t par = WS_EXI + (size_t)(call & 1u) * EXI_BYTES;
            const int o = u >> 2;
            const int myid = isw ? ((gid >> 5) == (rank >> 1) ? gid & 31 : -1)
                                 : (o == rank ? 44 + (u & 3) : ((o >> 1) != (rank >> 1) && ((u >> 1) & 1) == (rank & 1) ? 32 + rb_of(rank, o) * 2 + (u & 1) : -1));
            unsigned hdr = 0;
            {
                unsigned w[16];
#pragma unroll
                for (int i = 0; i < 16; ++i) w[i] = v[i];
                const char* home = myws + par + (size_t)(gid * 8 + c) * SLOT;
                unsigned pw = 0, used = 0, at = 0;
                hdr = 32u;
                if (!pushed) {
                    LAS unsigned* pool = (LAS unsigned*)(lds + wid * 256);
                    pool[lane] = 0u;
                    unsigned ef[16];
#pragma unroll
                    for (int i = 0; i < 16; ++i) ef[i] = (w[i] >> 10) & 0x001f001fu;
                    const unsigned B = (unsigned)code_base<5>(ef);
                    if ((used = code_put<POOL_IN>(ef, B, pool, fq))) {
                        if (lane == 0) at = __hip_atomic_fetch_add(ctl + 490 + (wid >> 2) * 5 + it, used, RLX, SC_WG);
#pragma unroll
                        for (int i = 0; i < 16; ++i) w[i] = (w[i] & 0x83ff83ffu) | (ef[i] << 10);
#pragma unroll
                        for (int k = 0; k < 3; ++k) {
#pragma unroll
                            for (int i = 0; i < 4; ++i) w[4 * k + i] |= (w[13 + k] << (12 - 3 * i)) & 0x70007000u;
                            w[12] |= (w[13 + k] >> (1 + k)) & (0x40004000u >> k);
                        }
                        pw = __hip_atomic_load(pool + lane, RLX, SC_WG);
                        hdr = B;
                    }
                }
                if (pushed) {
                    unsigned tp = t_loaded + (unsigned)((u & 7) * 4 + q) * PUSH_SLOT;
#pragma unroll
                    for (int i = 0; i < 4; ++i) {
                        pace(tp, 32u * PUSH_SLOT);
#pragma unroll
                        for (int k = 0; k < 2; ++k) {
                            const int d = (u >> 3) * 2 + k;
                            if (d != rank) st16(p.ws[d] + par + (size_t)(1280 + rank * 8 + (u & 7)) * SLOT + q * 4096 + i * 1024, lo, (u32x4){v[4 * i], v[4 * i + 1], v[4 * i + 2], v[4 * i + 3]});
                        }
                    }
                } else {
#pragma unroll
                    for (int i = 0; i < 3; ++i) st16(home + q * QB + i * 1024, lo, (u32x4){w[4 * i], w[4 * i + 1], w[4 * i + 2], w[4 * i + 3]});
                    st4(home + q * QB + 3072, lane * 4u, w[12]);
                    if (hdr == 32u) st12(home + SIDE + q * 768, lane * 12u, (u32x3){v[13], v[14], v[15]});
                    else {
                        at = UNI(at) + 1u;
                        if (lane < used) st4(home + PAREA + at * 4u, lane * 4u, pw);
                        if (lane == 0) st1(home + PAREA, q, at);
                    }
                }
            }
            if (myid >= 0) {
                const char* sg = myws + WS_STG + stage_off(myid, rank * 8 + c) + (isw ? (q >> 1) * 8 + (q & 1) : q * 4) * 1024;
#pragma unroll
                for (int i = 0; i < 4; ++i) st16(sg + i * (isw ? 2048 : 1024), lo, (u32x4){v[4 * i], v[4 * i + 1], v[4 * i + 2], v[4 * i + 3]});
            }
            wait_vm<0>();
            unsigned tw = (1u << 24) | (hdr << (6 * q));
            if (lane == 0) tw += __hip_atomic_fetch_add(ctl + 402 + (wid >> 2) * 5 + it, tw, RLX, SC_WG);
            tw = UNI(tw);
            if ((tw >> 24) == 4u) {
                tw = (tw & 0xffffffu) | (lds_get(ctl + 490 + (wid >> 2) * 5 + it) >= 160u ? 1u << 24 : 0u);
                const int d = lane & 7;
                int did = isw ? ((gid >> 5) == (d >> 1) ? gid & 31 : -1)
                              : (o == d ? 44 + (u & 3) : ((o >> 1) != (d >> 1) && ((u >> 1) & 1) == (d & 1) ? 32 + rb_of(d, o) * 2 + (u & 1) : -1));
                if (isw && (g & 3) == 3 && c < 2 * NFWD && (d >> 1) != (rank >> 1) && ((c ^ d) & 1)) did = -1;
                if (lane < 8 && did >= 0) {
                    if (d == rank) __hip_atomic_store(gflag + did * 64 + c * 8 + rank, call, RLX, SC_AGENT);
                    else __hip_atomic_store((GAS u64*)((((size_t)whi << 32) | wlo) + WS_RDY) + ((((call & 1u) * 8 + rank) * 48 + did) * 8 + c) * RSTRIDE, ((u64)call << 32) | tw, RLX, SC_SYS);
                }
            }
        }
    } else if (idx < 7) {
        const int s = idx < 6 ? rb_owner(rank, idx) : rank ^ 1;
        const int nu = idx < 6 ? NU_OUT : NU_PAIR;
        unsigned desc[2];
#pragma unroll
        for (int k = 0; k < 2; ++k) {
            const int e = p.order[idx == 6][k * 64 + lane];
            int c, id, org = s, slot;
            if (e < 48) { unit_of(e, xcd, c, id); slot = gid_of(rank, id) * 8 + c; }
            else { const int kf = (e - 48) / 6; c = 2 * kf + (s & 1); id = xcd * 4 + 3; org = rb_owner(rank, e - 48 - kf * 6); slot = 1408 + (e - 48) * 8 + xcd; }
            desc[k] = (unsigned)(c | id << 3 | org << 9 | slot << 12);
        }
        auto desc_of = [&](int n) { return (unsigned)(n < 64 ? __builtin_amdgcn_readlane(desc[0], n) : __builtin_amdgcn_readlane(desc[1], n - 64)); };
        const unsigned call = UNI(ctr_v) + 1u;
        const size_t par = WS_EXI + (size_t)(call & 1u) * EXI_BYTES;
        GAS u64* eq = (GAS u64*)(myws + WS_EQ) + idx * 16;
        if (wid == 6) {
            const char* exs = p.ws[s] + par;
            unsigned tnext = t_entry + T_PULL * 64 + xcd * (TAU / 4);
            u64 todo = ~1ull;
            __builtin_amdgcn_s_setprio(3);
            const char* src = nullptr;
            unsigned trim = 0;
            bool big = false;
            auto choose = [&](int n) {
                const unsigned rdy = __hip_atomic_load(ctl + (n >= 64 ? n : lane), RLX, SC_WG);
                const unsigned fre = __hip_atomic_load(ctl + (n > RING ? 128 + n - RING : 400), RLX, SC_WG), tr = __hip_atomic_load(ctl + 400, RLX, SC_WG);
                const u64 rd = __ballot(rdy != 0u) & (n >= 64 ? 1ull : todo);
                if (!rd || (n > RING && UNI(fre) < 4u)) return;
                const int un = n >= 64 ? n : __builtin_ctzll(rd);
                todo &= ~(1ull << (un & 63));
                if (lane == 0) __hip_atomic_store(ctl + 416 + n, (unsigned)un, RLX, SC_WG);
                trim = n >= EQ_AT + 4 && n < EQ_AT + 36 ? UNI(tr) : 0u;
                big = (__builtin_amdgcn_readlane(rdy, un & 63) >> 24) & 1u;
                src = exs + (size_t)(desc_of(un) >> 12) * SLOT;
            };
            for (int n = 1; n < nu; ++n) {
                bool drained = false;
                if (!src) {
                    choose(n);
                    if (!src) {
                        wait_vm<0>();
                        if (lane == 0) __hip_atomic_store(ctl + 384, (unsigned)(n - 1) * 14u, RLX, SC_WG);
                        drained = true;
                        for (choose(n); !src; choose(n)) __builtin_amdgcn_s_sleep(1);
                    }
                }
                const char* cur = src;
                const unsigned dst = lds0 + (unsigned)(n % RING) * 14336u, tr = trim, pl = big ? 48u : 40u;
                src = nullptr;
                asm volatile("" ::: "memory");
#pragma unroll
                for (int i = 0; i < 14; i += 2) {
                    wait_vm<WIN - 2>();
                    if ((i > WIN - 2 || (!drained && n > 1)) && lane == 0) __hip_atomic_store(ctl + 384, (unsigned)((n - 1) * 14 + i - (WIN - 2)), RLX, SC_WG);
                    pace(tnext, (i ? 2u * TAU : TAU + pl * TAU / 64u) - tr);
                    if (i) dma<true>(cur + (i - 1) * 1024, lo, dst + (i - 1) * 1024);
                    else {
                        if (lane < pl) dma<false>(cur + PAREA, lo, dst + PAREA);
                        dma<false>(cur, lo, dst);
                    }
                    if (i == 0 && lane == 0) {
                        if (n == EQ_AT) __hip_atomic_store(ctl + 387, tnext | 1u, RLX, SC_WG);
                        if (n == nu - 3) __hip_atomic_store(ctl + 386, tnext | 1u, RLX, SC_WG);
                        if (n == nu - 1) __hip_atomic_store(ctl + 385, (tnext + 12u * TAU) | 1u, RLX, SC_WG);
                    }
                    if (i == 10 && n + 1 < nu) choose(n + 1);
                }
            }
            __builtin_amdgcn_s_setprio(0);
            wait_vm<0>();
            if (lane == 0) __hip_atomic_store(ctl + 384, (unsigned)(nu - 1) * 14u, RLX, SC_WG);
        } else if (wid == 7) {
            const GAS u64* rp[2];
            bool pend[2];
#pragma unroll
            for (int k = 0; k < 2; ++k) {
                rp[k] = (const GAS u64*)(myws + WS_RDY) + ((((call & 1u) * 8 + ((desc[k] >> 9) & 7)) * 48 + ((desc[k] >> 3) & 63)) * 8 + (desc[k] & 7)) * RSTRIDE;
                pend[k] = k * 64 + lane < nu;
            }
            unsigned mine = 0;
            bool eq_open = true;
            const u64 t0 = __builtin_amdgcn_s_memrealtime();
            for (unsigned polls = 1;; ++polls) {
                const u64 pm = __ballot(pend[0]);
                const bool front = __builtin_amdgcn_mbcnt_hi((unsigned)(pm >> 32), __builtin_amdgcn_mbcnt_lo((unsigned)pm, 0u)) < POLLW;
#pragma unroll
                for (int k = 0; k < 2; ++k)
                    if (pend[k] && ((k == 0 && front) || (polls & 7u) == 0u)) {
                        const u64 f = __hip_atomic_load(rp[k], RLX, SC_SYS);
                        if ((unsigned)(f >> 32) >= call) { __hip_atomic_store(ctl + k * 64 + lane, (unsigned)f | 0x80000000u, RLX, SC_WG); pend[k] = false; }
                    }
                if (eq_open) {
                    if (!mine) {
                        mine = lds_get(ctl + 387);
                        if (mine && lane == 0) __hip_atomic_store(eq + xcd, ((u64)call << 32) | mine, RLX, SC_AGENT);
                    } else {
                        const u64 f = __hip_atomic_load(eq + (lane & 7), RLX, SC_AGENT);
                        if (__all((unsigned)(f >> 32) == call)) {
                            int dlt = (int)(mine - (unsigned)f);
                            dlt += __shfl_xor(dlt, 1); dlt += __shfl_xor(dlt, 2); dlt += __shfl_xor(dlt, 4);
                            dlt /= 8 * 32 * 7;
                            dlt = dlt > TAU / 8 ? TAU / 8 : dlt < -(TAU / 8) ? -(TAU / 8) : dlt;
                            if (lane == 0) __hip_atomic_store(ctl + 400, (unsigned)dlt, RLX, SC_WG);
                            eq_open = false;
                        } else if ((int)lds_get(ctl + 384) >= (EQ_AT + 2) * 14) eq_open = false;
                    }
                }
                const bool waiting = __any(pend[0] || pend[1]);
                if (!waiting && !eq_open) break;
                if (waiting && (polls & 15u) == 0u && TIMED_OUT(t0)) {
                    if (lane == 0) __hip_atomic_store(tmo, call, RLX, SC_AGENT);
                    wait_vm<0>();
#pragma unroll
                    for (int k = 0; k < 2; ++k) if (pend[k]) __hip_atomic_store(ctl + k * 64 + lane, 0x80000000u, RLX, SC_WG);
                    break;
                }
                if (polls < 32u) __builtin_amdgcn_s_sleep(1); else __builtin_amdgcn_s_sleep(16);
            }
            if (idx < 6) {
                while ((mine = lds_get(ctl + 386)) == 0u) __builtin_amdgcn_s_sleep(8);
                if (lane == 0) __hip_atomic_store(eq + 8 + xcd, ((u64)call << 32) | mine, RLX, SC_AGENT);
                unsigned th;
                int dlt = 0;
                bool open = true;
                while ((th = lds_get(ctl + 385)) == 0u) {
                    if (open) {
                        const u64 f = __hip_atomic_load(eq + 8 + (lane & 7), RLX, SC_AGENT);
                        if (__all((unsigned)(f >> 32) == call)) {
                            dlt = (int)(mine - (unsigned)f);
                            dlt += __shfl_xor(dlt, 1); dlt += __shfl_xor(dlt, 2); dlt += __shfl_xor(dlt, 4);
                            dlt /= 8 * 256;
                            dlt = dlt > 100 ? 100 : dlt < -100 ? -100 : dlt;
                            open = false;
                        }
                    }
                    __builtin_amdgcn_s_sleep(4);
                }
                if (lane == 0) __hip_atomic_store((GAS u64*)(myws + WS_MBOX + (size_t)((8 + idx * 4) * 8 + xcd) * 128), ((u64)call << 32) | (th & ~255u) | (unsigned)(dlt + 128), RLX, SC_AGENT);
            }
        } else {
            int pend = -1, pend_n = 0;
            unsigned pend_h = 0;
            auto staged = [&]() {
                wait_vm<0>();
                if (lane == 0 && __hip_atomic_fetch_add(ctl + 256 + pend_n, 1u, RLX, SC_WG) == 3u) {
                    __hip_atomic_store(gflag + pend, call, RLX, SC_AGENT);
                    if (pend_h) __hip_atomic_store((GAS u64*)(p.ws[rank ^ 1] + WS_RDY) + ((((call & 1u) * 8 + s) * 48 + (pend >> 6)) * 8 + ((pend >> 3) & 7)) * RSTRIDE,
                                                   ((u64)call << 32) | (pend_h & 0x1ffffffu), RLX, SC_SYS);
                }
                pend = -1;
            };
            auto need = [&](int job) { return (job >> 2) * 14 - 13 + (((job & 3) + 1) * QB + 1023) / 1024; };
            auto ticket = [&]() { unsigned tk = 0; if (lane == 0) tk = __hip_atomic_fetch_add(ctl + 401, 1u, RLX, SC_WG); return (int)UNI(tk); };
            for (int job = ticket(); job < 4 * nu;) {
                const int n = job >> 2, qq = job & 3;
                unsigned w[16];
                int un = 0;
                unsigned relay_h = 0;
                if (n == 0) {
                    while (lds_get(ctl) == 0u) __builtin_amdgcn_s_sleep(1);
                    u32x4 a0, a1, a2, a3;
                    ld64(myws + par + (size_t)(1280 + s * 8 + xcd) * SLOT + qq * 4096, lo, a0, a1, a2, a3);
#pragma unroll
                    for (int i = 0; i < 4; ++i) { w[i] = a0[i]; w[4 + i] = a1[i]; w[8 + i] = a2[i]; w[12 + i] = a3[i]; }
                } else {
                    while ((int)lds_get(ctl + 384) < need(job)) __builtin_amdgcn_s_sleep(1);
                    asm volatile("" ::: "memory");
                    un = lds_get(ctl + 416 + n);
                }
                const unsigned de = desc_of(un);
                const int c = de & 7, id = (de >> 3) & 63, org = (de >> 9) & 7;
                if (n) {
                    const LAS unsigned char* src = lds + (unsigned)(n % RING) * 14336u + qq * QB;
#pragma unroll
                    for (int i = 0; i < 3; ++i) { const u32x4 a = *(const LAS u32x4*)(src + i * 1024 + lo); w[4 * i] = a[0]; w[4 * i + 1] = a[1]; w[4 * i + 2] = a[2]; w[4 * i + 3] = a[3]; }
                    w[12] = *(const LAS unsigned*)(src + 3072 + lane * 4);
                    const LAS unsigned char* parea = lds + (unsigned)(n % RING) * 14336u + PAREA;
                    const LAS unsigned* pool = (const LAS unsigned*)parea + parea[qq];
                    const unsigned hu = lds_get(ctl + un), h = (hu >> (6 * qq)) & 63u;
                    if (idx < 6 && id < 32 && (id & 3) == 3 && c < 2 * NFWD) {
                        const unsigned pw = ((const LAS unsigned*)parea)[qq * 48 + lane];
                        if (pend >= 0) staged();
                        const char* dst = myws + par + (size_t)(1408 + ((c >> 1) * 6 + idx) * 8 + xcd) * SLOT + qq * QB;
#pragma unroll
                        for (int i = 0; i < 3; ++i) st16(dst + i * 1024, lo, (u32x4){w[4 * i], w[4 * i + 1], w[4 * i + 2], w[4 * i + 3]});
                        st4(dst + 3072, lane * 4u, w[12]);
                        if (lane < 48) st4(dst + (PAREA - qq * QB) + qq * 192, lane * 4u, pw);
                        relay_h = hu;
                    }
                    if (h < 32u) {
#pragma unroll
                        for (int k = 0; k < 3; ++k) {
                            unsigned cc = (w[12] << (1 + k)) & 0x80008000u;
#pragma unroll
                            for (int i = 0; i < 4; ++i) cc |= (w[4 * k + i] >> (12 - 3 * i)) & (0x00070007u << (3 * i));
                            w[13 + k] = cc;
                        }
                        unsigned ne[16];
                        u64 str = code_find<10>(w, ne, pool, fq);
                        WAIT_L0();
                        if (lane == 0) __hip_atomic_fetch_add(ctl + 128 + n, 1u, RLX, SC_WG);
                        if (pend >= 0) staged();
#pragma unroll
                        for (int i = 0; i < 16; ++i) w[i] = (w[i] & 0x8fff8fffu) + (code_get(str, ne[i], h * 0x00010001u) << 10);
                    } else {
                        WAIT_L0();
                        if (lane == 0) __hip_atomic_fetch_add(ctl + 128 + n, 1u, RLX, SC_WG);
                        if (pend >= 0) staged();
                        const u32x3 a = ld12(p.ws[org] + par + (size_t)(gid_of(rank, id) * 8 + c) * SLOT + SIDE + qq * 768, lane * 12u);
                        w[13] = a[0]; w[14] = a[1]; w[15] = a[2];
                    }
                }
                const char* sg = myws + WS_STG + stage_off(id, org * 8 + c) + (id < 32 ? (qq >> 1) * 8 + (qq & 1) : qq * 4) * 1024;
#pragma unroll
                for (int i = 0; i < 4; ++i) st16(sg + i * (id < 32 ? 2048 : 1024), lo, (u32x4){w[4 * i], w[4 * i + 1], w[4 * i + 2], w[4 * i + 3]});
                pend = id * 64 + c * 8 + org; pend_n = n; pend_h = relay_h;
                job = ticket();
                if (job >= 4 * nu || job < 4 || (int)lds_get(ctl + 384) < need(job)) staged();
            }
        }
    }
    const unsigned call = UNI(ctr_v) + 1u;

    const int ntile = idx == 7 ? 2 : 1;
    const int wm = wid >> 1, wn = wid & 1;
#pragma unroll 1
    for (int rep = 0; rep < ntile; ++rep) {
    const int rb = late ? 6 + (idx >> 2) : (idx - 8) >> 2;
    const int cb = xcd * 4 + (idx == 7 ? rep : idx == 6 ? 0 : idx == 4 || idx == 5 ? idx - 2 : idx & 3);
    const bool part1 = idx == 7 && rep == 0, part2 = idx == 6;
    const int nstep = part1 ? 56 : part2 ? 8 : NSTEP;
    const char* stgA = myws + WS_STG + (size_t)(rb * NSTEP) * A_STEP + wid * 4096;
    const char* stgB = myws + WS_STG + STG_A + (size_t)(cb * NSTEP) * B_STEP + wid * 2048;
    const u64 pbits = 0x0101010101010101ull << (rank ^ 1);
    u64 todo = part1 ? ~pbits : part2 ? pbits : ~0ull, avail = 0ull;
    int slot = 0;
    auto poll = [&]() {
        if (wid == 0) {
            const u64 t0 = __builtin_amdgcn_s_memrealtime();
            u64 got;
            for (unsigned polls = 1;; ++polls) {
                const unsigned f0 = __hip_atomic_load(gflag + cb * 64 + lane, RLX, SC_AGENT);
                const unsigned f1 = __hip_atomic_load(gflag + (32 + rb * 2) * 64 + lane, RLX, SC_AGENT);
                const unsigned f2 = __hip_atomic_load(gflag + (33 + rb * 2) * 64 + lane, RLX, SC_AGENT);
                got = __ballot(f0 >= call && f1 >= call && f2 >= call) & todo;
                if (got) break;
                if ((polls & 15u) == 0u && TIMED_OUT(t0)) {
                    if (lane == 0) __hip_atomic_store(tmo, call, RLX, SC_AGENT);
                    got = todo;
                    break;
                }
                __builtin_amdgcn_s_sleep(8);
            }
            if (lane < 2) ctl[388 + slot * 2 + lane] = (unsigned)(got >> (32 * lane));
        }
    };
    auto news = [&]() { avail = ((u64)(unsigned)UNI(ctl[389 + slot * 2]) << 32) | (unsigned)UNI(ctl[388 + slot * 2]); slot ^= 1; };
    auto take = [&]() {
        const unsigned b = (unsigned)__builtin_ctzll(avail) & 63u;
        avail &= avail - 1; todo &= ~(1ull << b);
        return (b & 7u) * 8u + (b >> 3);
    };
    auto g1 = [&](unsigned ks, unsigned buf, int i) {
        if (i < 4) __builtin_amdgcn_global_load_lds((const GAS unsigned*)(stgA + (size_t)ks * A_STEP + i * 1024 + lo), (LAS unsigned*)(lds + buf + wid * 4096 + i * 1024), 16, 0, 17);
        else __builtin_amdgcn_global_load_lds((const GAS unsigned*)(stgB + (size_t)ks * B_STEP + (i - 4) * 1024 + lo), (LAS unsigned*)(lds + buf + A_STEP + wid * 2048 + (i - 4) * 1024), 16, 0, 17);
    };

    f32x4 acc[4][4];
#pragma unroll
    for (int m = 0; m < 4; ++m)
#pragma unroll
        for (int n = 0; n < 4; ++n) acc[m][n] = (f32x4){0.f, 0.f, 0.f, 0.f};
    h16x8 a0[4], b0[4], a1[4], b1[4];
    const unsigned aoff = wm * 8192 + lo, boff = A_STEP + wn * 8192 + lo;
#define RDA(A, buf, kk, m) A[m] = *(const LAS h16x8*)(lds + (buf) + aoff + (m) * 2048 + (kk) * 1024)
#define RDB(B, buf, kk, n) B[n] = *(const LAS h16x8*)(lds + (buf) + boff + (n) * 2048 + (kk) * 1024)
#define ROW(A, B, m) _Pragma("unroll") for (int n = 0; n < 4; ++n) acc[m][n] = __builtin_amdgcn_mfma_f32_16x16x32_f16(A[m], B[n], acc[m][n], 0, 0, 0)
    unsigned kq;
#pragma unroll
    for (int k = 0; k < 3; ++k) {
        if (!avail) { poll(); WAIT_L0(); BAR(); news(); }
        kq = take();
#pragma unroll
        for (int i = 0; i < (k < 2 ? 6 : 3); ++i) g1(kq, k * STAGE, i);
    }
    wait_vm<9>(); BAR();
#pragma unroll
    for (int i = 0; i < 4; ++i) { RDB(b0, 0u, 0, i); RDA(a0, 0u, 0, i); }
    unsigned cur = 0, prv = 2 * STAGE;
    for (int t = 0; t < nstep; ++t) {
        const unsigned nxt = cur == 2 * STAGE ? 0u : cur + STAGE;
        const bool m2 = t + 2 < nstep, m3 = t + 3 < nstep;
        const bool dry = m3 && !avail;
        WAIT_L0(); SB();
        RDB(b1, cur, 1, 0); RDB(b1, cur, 1, 1); SB(); ROW(a0, b0, 0); SB(); if (m2) g1(kq, prv, 3); SB();
        RDB(b1, cur, 1, 2); RDB(b1, cur, 1, 3); SB(); ROW(a0, b0, 1); SB(); if (m2) g1(kq, prv, 4); SB();
        RDA(a1, cur, 1, 0); RDA(a1, cur, 1, 1); SB(); ROW(a0, b0, 2); SB(); if (m2) g1(kq, prv, 5); SB();
        RDA(a1, cur, 1, 2); RDA(a1, cur, 1, 3); SB(); ROW(a0, b0, 3); SB();
        if (dry) poll();
        WAIT_L0();
        if (m2) wait_vm<6>(); else wait_vm<0>();
        BAR(); SB();
        RDB(b0, nxt, 0, 0); RDB(b0, nxt, 0, 1); SB(); ROW(a1, b1, 0); SB();
        if (dry) news();
        if (m3) { kq = take(); g1(kq, cur, 0); } SB();
        RDB(b0, nxt, 0, 2); RDB(b0, nxt, 0, 3); SB(); ROW(a1, b1, 1); SB(); if (m3) g1(kq, cur, 1); SB();
        RDA(a0, nxt, 0, 0); RDA(a0, nxt, 0, 1); SB(); ROW(a1, b1, 2); SB(); if (m3) g1(kq, cur, 2); SB();
        RDA(a0, nxt, 0, 2); RDA(a0, nxt, 0, 3); SB(); ROW(a1, b1, 3); SB();
        prv = cur; cur = nxt;
    }

    if (part1 || part2) {
        const char* park = myws + WS_EXO + (size_t)(xcd * 8 + wid) * 16384;
        GAS u64* pf = (GAS u64*)(myws + WS_ORDY + (size_t)xcd * 128) + wid;
        if (part1) {
#pragma unroll
            for (int i = 0; i < 16; ++i) st16(park + i * 1024, lo, __builtin_bit_cast(u32x4, acc[i >> 2][i & 3]));
            wait_vm<0>();
            if (lane == 0) __hip_atomic_store(pf, (u64)call << 32, RLX, SC_SYS);
            continue;
        }
        const u64 t0 = __builtin_amdgcn_s_memrealtime();
        while ((unsigned)UNI((unsigned)(__hip_atomic_load(pf, RLX, SC_SYS) >> 32)) < call) {
            if (TIMED_OUT(t0)) { if (lane == 0) __hip_atomic_store(tmo, call, RLX, SC_AGENT); wait_vm<0>(); break; }
            __builtin_amdgcn_s_sleep(4);
        }
#pragma unroll
        for (int m = 0; m < 4; ++m) {
            u32x4 e0, e1, e2, e3;
            ld64(park + m * 4096, lo, e0, e1, e2, e3);
            acc[m][0] += __builtin_bit_cast(f32x4, e0); acc[m][1] += __builtin_bit_cast(f32x4, e1);
            acc[m][2] += __builtin_bit_cast(f32x4, e2); acc[m][3] += __builtin_bit_cast(f32x4, e3);
        }
    }
    const unsigned tmo_v = __hip_atomic_load(tmo, RLX, SC_AGENT);
    const unsigned oo = (unsigned)(fq * 4) * (GN * 4u) + fr * 16u;
    const u32x4 nan4 = {0x7fc00000u, 0x7fc00000u, 0x7fc00000u, 0x7fc00000u};
    auto finish = [&](unsigned (&o)[32], int i, float a, float b) {
        const f32x2 g = gelu2((f32x2){a, b});
        const int at = ((i >> 3) * 4 + (i & 1) * 2) * 4 + ((i >> 1) & 3);
        o[at] = __float_as_uint(g.x); o[at + 4] = __float_as_uint(g.y);
    };
#define ACC(hf, i, k) acc[2 * (hf) + ((i) >> 3)][((i) >> 1) & 3][((i) & 1) * 2 + (k)]
    if (late) {
        const bool bad = UNI(tmo_v) == call;
        const char* ob = (const char*)(p.out + (size_t)(rb_local(rank, rb) + wm * 64) * GN + (rank >> 1) * 4096 + cb * 128 + wn * 64);
#pragma unroll
        for (int hf = 0; hf < 2; ++hf) {
            unsigned o[32];
#pragma unroll
            for (int i = 0; i < 16; ++i) finish(o, i, ACC(hf, i, 0), ACC(hf, i, 1));
#pragma unroll
            for (int i = 0; i < 8; ++i) st16(ob + (size_t)(hf * 32 + (i >> 2) * 16 + (i & 3)) * (GN * 4), oo, bad ? nan4 : (u32x4){o[4 * i], o[4 * i + 1], o[4 * i + 2], o[4 * i + 3]});
        }
    } else {
        const int peer = rb_owner(rank, rb);
        const int mirror = (8 + rb_of(peer, rank) * 4 + (idx & 3)) * 8 + xcd;
        char* const peerws = p.ws[peer];
        WAIT_L0(); BAR();
        if (wid < 4) __builtin_amdgcn_s_setprio(2);
#pragma unroll
        for (int hf = 0; hf < 2; ++hf) {
            const int k = (wid >> 2) * 8 + hf * 4 + q;
            const char* ex = myws + WS_EXO + (size_t)(bid * 16 + k) * 8192;
            unsigned hdr = 1u << 13;
            if ((idx & 3) != 3 || k < 16 - NPLAIN) {
                LAS unsigned* pool = (LAS unsigned*)(lds + 131072 + wid * 256);
                pool[lane] = 0u;
                unsigned hi[16], ef[16], under = 0;
#pragma unroll
                for (int i = 0; i < 16; ++i) {
                    hi[i] = __builtin_amdgcn_perm(__float_as_uint(ACC(hf, i, 1)), __float_as_uint(ACC(hf, i, 0)), 0x07060302u);
                    ef[i] = (hi[i] >> 7) & 0x00ff00ffu;
                }
                const unsigned bs = (unsigned)code_base<8>(ef), e0 = bs > 20u ? bs - 20u : 0u;
#pragma unroll
                for (int i = 0; i < 16; ++i) { under |= SUBS(e0 * 0x00010001u, ef[i]); ef[i] = SUBS(ef[i], e0 * 0x00010001u); }
                unsigned used, at = 0;
                if (__all(under == 0u) && (used = code_put<POOL_OUT>(ef, bs - e0, pool, fq))) {
                    if (lane == 0) at = __hip_atomic_fetch_add(ctl + 500 + (k >> 2), used, RLX, SC_WG);
#pragma unroll
                    for (int i = 0; i < 16; ++i) hi[i] = (hi[i] & 0x807f807fu) | (ef[i] << 7);
#pragma unroll
                    for (int c = 0; c < 6; ++c) {
#pragma unroll
                        for (int i = 0; i < 3; ++i) { const int fld = c * 3 + i, at = 9 + 3 * (fld & 1); hi[fld >> 1] |= (hi[10 + c] << (at - 3 * i)) & (0x00070007u << at); }
                        hi[9] |= (hi[10 + c] >> (1 + c)) & (0x40004000u >> c);
                    }
                    const unsigned pw = __hip_atomic_load(pool + lane, RLX, SC_WG);
#pragma unroll
                    for (int i = 0; i < 4; ++i) {
                        u32x4 v;
#pragma unroll
                        for (int e = 0; e < 4; ++e) v[e] = __builtin_amdgcn_perm(__float_as_uint(ACC(hf, 4 * i + e, 1)), __float_as_uint(ACC(hf, 4 * i + e, 0)), 0x05040100u);
                        st16(ex + i * 1024, lo, v);
                    }
                    st16(ex + 4096, lo, (u32x4){hi[0], hi[1], hi[2], hi[3]});
                    st16(ex + 5120, lo, (u32x4){hi[4], hi[5], hi[6], hi[7]});
                    st8(ex + 6144, lane * 8u, (u32x2){hi[8], hi[9]});
                    at = UNI(at);
                    if (lane < used) st4(myws + WS_EXP + (size_t)bid * 4096 + (k >> 2) * 768 + at * 4u, lane * 4u, pw);
                    hdr = e0 | ((bs - e0) << 8) | (at + used > 160u ? 1u << 14 : 0u) | (at << 16);
                }
            }
            if ((hdr >> 13) & 1u) {
                unsigned o[32];
#pragma unroll
                for (int i = 0; i < 16; ++i) finish(o, i, ACC(hf, i, 0), ACC(hf, i, 1));
#pragma unroll
                for (int i = 0; i < 8; ++i) st16(ex + i * 1024, lo, (u32x4){o[4 * i], o[4 * i + 1], o[4 * i + 2], o[4 * i + 3]});
            }
            wait_vm<0>();
            hdr |= UNI(tmo_v) == call ? 0xc0000000u : 0x40000000u;
            const int gs = (idx & 3) == 0 || (idx & 3) == 3 ? 4 : 16, k0 = k & -gs;
            unsigned cnt = 0;
            if (lane == 0) { __hip_atomic_store(ctl + 64 + k, hdr, RLX, SC_WG); cnt = __hip_atomic_fetch_add(ctl + 96 + k0, 1u, RLX, SC_WG); }
            if ((int)UNI(cnt) == gs - 1 && lane < gs)
                __hip_atomic_store((GAS u64*)(peerws + WS_ORDY + (size_t)mirror * 128) + k0 + lane, ((u64)call << 32) | __hip_atomic_load(ctl + 64 + k0 + lane, RLX, SC_WG), RLX, SC_SYS);
        }
        __builtin_amdgcn_s_setprio(0);
#pragma unroll 1
        for (int sk = 0; (idx & 3) + sk <= 3 && (idx & 3) != 3; sk += 3) {
        if (sk) {
            WAIT_L0(); BAR();
            if (tid < 32 || tid == 384 || tid == 401 || (tid >= 416 && tid < 432)) ctl[tid] = 0u;
            WAIT_L0(); BAR();
        }
        const int me = bid + 8 * sk;
        const char* exs = peerws + WS_EXO + (size_t)(mirror + 8 * sk) * (16 * 8192);
        if (wid == 0) {
            unsigned tnext;
            while ((tnext = lds_get(ctl + 16)) == 0u) __builtin_amdgcn_s_sleep(2);
            const unsigned trb = tnext & 255u, tr = trb - 128u;
            tnext = (tnext & ~255u) | 128u;
            __builtin_amdgcn_s_setprio(3);
            wait_vm<0>(); __builtin_amdgcn_s_waitcnt(0x0F70);
            int issued = 0, told = 0;
            unsigned rdy = 0, pln = 0, big = 0;
            auto look = [&]() {
                const unsigned h = __hip_atomic_load(ctl + (lane & 15), RLX, SC_WG);
                rdy = (unsigned)__ballot(h != 0u); pln = (unsigned)__ballot((h >> 13) & 1u); big = (unsigned)__ballot((h >> 14) & 1u);
            };
            look();
            for (int n = 0; n < 16; ++n) {
                const bool head = (n & 3) == 0;
                if (head && ((rdy >> n) & 15u) != 15u) {
                    look();
                    if (((rdy >> n) & 15u) != 15u) {
                        wait_vm<0>();
                        told = issued;
                        if (lane == 0) __hip_atomic_store(ctl + 384, (unsigned)told, RLX, SC_WG);
                        for (look(); ((rdy >> n) & 15u) != 15u; look()) __builtin_amdgcn_s_sleep(1);
                    }
                }
                asm volatile("" ::: "memory");
                const bool plain = (pln >> n) & 1u;
                const unsigned pl = (big >> n) & 15u ? 48u : 40u;
                const char* cur = exs + (size_t)n * 8192;
                const unsigned dst = lds0 + (unsigned)n * 8192u;
#pragma unroll
                for (int i = 0; i < 8; i += 2) {
                    wait_vm<WIN - 2>();
                    if (issued - (WIN - 2) > told) { told = issued - (WIN - 2); if (lane == 0) __hip_atomic_store(ctl + 384, (unsigned)told, RLX, SC_WG); }
                    pace(tnext, (i < 6 || plain ? 2u * TAU : TAU / 2u) + (i == 6 && head ? pl * TAU / 64u : 0u) - tr);
                    if (i < 6 || plain) { dma<true>(cur + i * 1024, lo, dst + i * 1024); issued += 2; }
                    else { if (lane < 32) dma<false>(cur + 6144, lo, dst + 6144); issued += 1; }
                    if (i == 6 && head) {
                        if (lane < pl) dma<false>(peerws + WS_EXP + (size_t)(mirror + 8 * sk) * 4096 + n * 192, lo, lds0 + 133120u + n * 192);
                        issued += 1;
                    }
                    if (i == 0) {
                        if (lane == 0) __hip_atomic_store(ctl + 416 + n, (unsigned)issued + (plain ? 6u : 5u) + head, RLX, SC_WG);
                        if (n == 14 && !sk && lane == 0)
                            __hip_atomic_store((GAS u64*)(myws + WS_MBOX + (size_t)(bid + 8) * 128), ((u64)call << 32) | ((tnext + 11u * TAU - 7u * tr) & ~255u) | trb, RLX, SC_AGENT);
                    }
                    if (i == 2 && (n & 3) == 3) look();
                }
            }
            wait_vm<7>();
            if (issued - 7 > told && lane == 0) __hip_atomic_store(ctl + 384, (unsigned)issued - 7u, RLX, SC_WG);
            wait_vm<0>();
            if (lane == 0) __hip_atomic_store(ctl + 384, (unsigned)issued, RLX, SC_WG);
            __builtin_amdgcn_s_setprio(0);
        } else {
            if (wid == 1) {
                const GAS u64* fp = lane < 16 ? (const GAS u64*)(myws + WS_ORDY + (size_t)me * 128) + lane : (const GAS u64*)(myws + WS_MBOX + (size_t)me * 128);
                bool pend = lane < 17;
                const u64 t0 = __builtin_amdgcn_s_memrealtime();
                for (unsigned polls = 1;; ++polls) {
                    if (pend) {
                        const u64 f = __hip_atomic_load(fp, RLX, SC_SYS);
                        if ((unsigned)(f >> 32) >= call) { __hip_atomic_store(ctl + lane, (unsigned)f, RLX, SC_WG); pend = false; }
                    }
                    if (!__any(pend)) break;
                    if ((polls & 15u) == 0u && TIMED_OUT(t0)) {
                        if (lane == 0) __hip_atomic_store(tmo, call, RLX, SC_AGENT);
                        if (pend) __hip_atomic_store(ctl + lane, lane < 16 ? 0xc0002000u : (now64() & ~255u) | 128u, RLX, SC_WG);
                        break;
                    }
                    __builtin_amdgcn_s_sleep(4);
                }
            }
            for (;;) {
                unsigned tk = 0;
                if (lane == 0) tk = __hip_atomic_fetch_add(ctl + 401, 1u, RLX, SC_WG);
                const int k = UNI(tk);
                if (k >= 16) break;
                unsigned need;
                while ((need = lds_get(ctl + 416 + k)) == 0u) __builtin_amdgcn_s_sleep(4);
                const unsigned hdr = lds_get(ctl + k);
                while (lds_get(ctl + 384) < need) __builtin_amdgcn_s_sleep(1);
                asm volatile("" ::: "memory");
                const LAS unsigned char* src = lds + (unsigned)k * 8192u;
                unsigned o[32];
                if ((hdr >> 13) & 1u) {
#pragma unroll
                    for (int i = 0; i < 8; ++i) { const u32x4 a = *(const LAS u32x4*)(src + i * 1024 + lo); o[4 * i] = a[0]; o[4 * i + 1] = a[1]; o[4 * i + 2] = a[2]; o[4 * i + 3] = a[3]; }
                } else {
                    unsigned lw[16], hi[16], ne[16];
#pragma unroll
                    for (int i = 0; i < 2; ++i) { const u32x4 a = *(const LAS u32x4*)(src + 4096 + i * 1024 + lo); hi[4 * i] = a[0]; hi[4 * i + 1] = a[1]; hi[4 * i + 2] = a[2]; hi[4 * i + 3] = a[3]; }
                    { const u32x2 a = *(const LAS u32x2*)(src + 6144 + lane * 8); hi[8] = a[0]; hi[9] = a[1]; }
#pragma unroll
                    for (int i = 0; i < 4; ++i) { const u32x4 a = *(const LAS u32x4*)(src + i * 1024 + lo); lw[4 * i] = a[0]; lw[4 * i + 1] = a[1]; lw[4 * i + 2] = a[2]; lw[4 * i + 3] = a[3]; }
#pragma unroll
                    for (int c = 0; c < 6; ++c) {
                        unsigned cc = (hi[9] << (1 + c)) & 0x80008000u;
#pragma unroll
                        for (int i = 0; i < 3; ++i) { const int fld = c * 3 + i, at = 9 + 3 * (fld & 1); cc |= (hi[fld >> 1] >> (at - 3 * i)) & (0x00070007u << (3 * i)); }
                        hi[10 + c] = cc;
                    }
                    u64 str = code_find<7>(hi, ne, (const LAS unsigned*)(lds + 133120 + (k >> 2) * 768) + ((hdr >> 16) & 255u), fq);
                    const unsigned e0 = (hdr & 255u) * 0x00800080u, b2 = ((hdr >> 8) & 31u) * 0x00010001u;
#pragma unroll
                    for (int i = 0; i < 16; ++i) {
                        const unsigned h2 = (hi[i] & 0x81ff81ffu) + e0 + (code_get(str, ne[i], b2) << 7);
                        finish(o, i, __uint_as_float(__builtin_amdgcn_perm(h2, lw[i], 0x05040100u)), __uint_as_float(__builtin_amdgcn_perm(h2, lw[i], 0x07060302u)));
                    }
                }
                const int blk = (k >> 3) * 4 + (k & 3);
                const char* ob = (const char*)(p.out + (size_t)((peer & 1) * 256 + (blk >> 1) * 64 + ((k >> 2) & 1) * 32) * GN + (peer >> 1) * 4096 + (cb + sk) * 128 + (blk & 1) * 64);
#pragma unroll
                for (int i = 0; i < 8; ++i)
                    st16(ob + (size_t)((i >> 2) * 16 + (i & 3)) * (GN * 4), oo, (hdr >> 31) ? nan4 : (u32x4){o[4 * i], o[4 * i + 1], o[4 * i + 2], o[4 * i + 3]});
            }
        }
        }
    }
    }
    if (tid == 0) __hip_atomic_store(ctr, call, RLX, SC_AGENT);
}

extern "C" void kernel_launch(int rank, int nranks, ncclComm_t comm, hipStream_t stream,
                              void* const* d_in, const int* in_sizes, int n_in,
                              void* d_out, int out_size, const int* in_dtypes, int out_dtype,
                              void* const* d_out_all, void* const* d_ws_all, size_t ws_bytes) {
    (void)comm; (void)in_dtypes; (void)out_dtype; (void)d_out_all;
    if (nranks != NR || (unsigned)rank >= (unsigned)NR || n_in != 2 || ws_bytes < WS_NEED ||
        in_sizes[0] != 4096 * KSH || in_sizes[1] != KSH * GN || out_size != 512 * GN) {
        fprintf(stderr, "kernel_launch: unexpected problem shape\n");
        return;
    }
    Params p;
    p.x = (const float*)d_in[0];
    p.w = (const float*)d_in[1];
    p.out = (float*)d_out;
    p.myws = (char*)d_ws_all[rank];
    for (int i = 0; i < NR; ++i) p.ws[i] = (char*)d_ws_all[i];
    p.rank = rank;
    p.pad = 0;
    int n = 0, np = 0, nd = 0;
    unsigned char deferred[8];
    bool px[8] = {};
    for (int i = 0; i < 128; ++i) p.order[0][i] = p.order[1][i] = 0;
    auto need_x = [&](int c0, int c1) { for (int c = c0; c <= c1; ++c) if (!px[c]) { px[c] = true; p.prod[np++] = (unsigned)(32 + c); } };
    for (int key = 0; key < 16 + 3 * SKEW2; ++key) {
        for (int pass = 0; pass < 2; ++pass) {
            if ((pass == 0) == (key != 0)) {
                for (int e = 32; e < 44; ++e)
                    if (2 * ((e - 32) * 8 / 12) == key) { p.order[0][n++] = (unsigned char)e; need_x((e - 32) * 8 / 12, ((e - 32) * 8 + 7) / 12); }
                if (key == XPOS) { p.order[0][n++] = 44; need_x(0, 3); }
                if (key == XPOS + 2) { p.order[0][n++] = 45; need_x(4, 7); }
                if (key == LPOS) { p.order[0][n++] = 46; p.order[0][n++] = 47; }
            } else {
                for (int b = 0; b < 4; ++b)
                    for (int c = 0; c < 8; ++c)
                        if ((b == 0 && c == 1 ? 0 : 2 * c + b * SKEW2) == key) {
                            p.prod[np++] = (unsigned)(c * 4 + b);
                            if (b == 3 && c < 2 * NFWD && ((c ^ rank) & 1)) deferred[nd++] = (unsigned char)(c * 4 + b);
                            else p.order[0][n++] = (unsigned char)(c * 4 + b);
                        }
            }
        }
    }
    int m = 0;
    for (int i = 0; i < n; ++i) if (p.order[0][i] < 44) p.order[1][m++] = p.order[0][i];
    for (int i = 0; i < nd; ++i) p.order[1][m++] = deferred[i];
    for (int i = 0; i < 6 * NFWD; ++i) p.order[1][m++] = (unsigned char)(48 + i);
    for (int e = 44; e < 48; ++e) p.order[1][m++] = (unsigned char)e;
    if (n != NU_OUT || m != NU_PAIR || np != 40 || p.prod[0] != 0 || p.prod[1] != 4 || p.prod[2] != 32) { fprintf(stderr, "kernel_launch: bad schedule\n"); return; }
    hipLaunchKernelGGL(fused_gemm_rs, dim3(NBLK), dim3(NTHR), 0, stream, p);
    HIP_CHECK(hipGetLastError());
}
```

```cpp
#include <hip/hip_runtime.h>
#include <rccl/rccl.h>
#include <cstdio>

#define HIP_CHECK(expr) do { hipError_t _e = (expr); if (_e != hipSuccess) { \
    fprintf(stderr, "HIP error %s at %s:%d\n", hipGetErrorString(_e), __FILE__, __LINE__); \
    return; } } while (0)

typedef float f32x4 __attribute__((ext_vector_type(4)));
typedef unsigned u32x4 __attribute__((ext_vector_type(4)));
typedef unsigned u32x3 __attribute__((ext_vector_type(3)));
typedef unsigned u32x2 __attribute__((ext_vector_type(2)));
typedef unsigned short u16x2 __attribute__((ext_vector_type(2)));
typedef _Float16 h16x8 __attribute__((ext_vector_type(8)));
typedef unsigned long long u64;
#define LAS __attribute__((address_space(3)))
#define GAS __attribute__((address_space(1)))
#define RLX __ATOMIC_RELAXED
#define SC_WG __HIP_MEMORY_SCOPE_WORKGROUP
#define SC_AGENT __HIP_MEMORY_SCOPE_AGENT
#define SC_SYS __HIP_MEMORY_SCOPE_SYSTEM

constexpr int TAU = 863;
constexpr int WIN = 10;
constexpr int CREDIT = 36;
constexpr int EQ_AT = 5;
constexpr int PUSH_SLOT = 112;
constexpr int T_PULL = 320;
constexpr int PROD_GAP = 40;
constexpr int NFWD = 3;
constexpr int NPLAIN = 3;
constexpr int SKEW2 = 4;
constexpr int XPOS = 6, LPOS = 20;
constexpr int RKEY = 8;
constexpr int NODEFER = 48;
constexpr int RSTRIDE = 16;
constexpr int POLLW = 12;

constexpr int NR = 8;
constexpr int GN = 16384;
constexpr int KSH = 512;
constexpr int NBLK = 256;
constexpr int NTHR = 512;
constexpr int NSTEP = 64;
constexpr unsigned A_STEP = 32768, B_STEP = 16384, STAGE = A_STEP + B_STEP;
constexpr int NU_OUT = 48 - NFWD, NU_PAIR = 48 + 6 * NFWD;
constexpr int RING = 10;
constexpr int PW = 14;
constexpr int POOL_IN = 47, POOL_OUT = 48;
constexpr int QB = 3328, PAREA = 4 * QB;
constexpr unsigned SLOT = 18432, SIDE = 14336;
constexpr size_t WS_CTR = 0;
constexpr size_t WS_FLAG = 32 << 10;
constexpr size_t WS_ORDY = 96 << 10;
constexpr size_t WS_MBOX = 128 << 10;
constexpr size_t WS_EQ = 160 << 10;
constexpr size_t WS_TMO = 192 << 10;
constexpr size_t WS_STG = 1 << 20;
constexpr size_t STG_A = 8ull * NSTEP * A_STEP;
constexpr size_t STG_B = 32ull * NSTEP * B_STEP;
constexpr size_t WS_EXI = WS_STG + STG_A + STG_B;
constexpr size_t EXI_BYTES = 1600ull * SLOT;
constexpr size_t WS_EXO = WS_EXI + 2 * EXI_BYTES;
constexpr size_t WS_EXP = WS_EXO + (size_t)NBLK * 8 * 16384;
constexpr size_t WS_RDY = WS_EXP + (size_t)NBLK * 4096;
constexpr size_t WS_NEED = WS_RDY + 2 * 8 * 48 * 8 * 128;
constexpr u64 DEADLINE_TICKS = 50000000ull;

struct Params {
    const float* x;
    const float* w;
    float* out;
    char* myws;
    char* ws[NR];
    int rank;
    int pad;
    unsigned prod[40];
    unsigned char order[2][128];
};

#define WAIT_L0() do { asm volatile("s_waitcnt lgkmcnt(0)" ::: "memory"); __builtin_amdgcn_s_waitcnt(0xC07F); } while (0)
#define BAR() do { asm volatile("" ::: "memory"); __builtin_amdgcn_s_barrier(); asm volatile("" ::: "memory"); } while (0)
#define SB() __builtin_amdgcn_sched_barrier(0)
#define UNI(x) __builtin_amdgcn_readfirstlane(x)
template <int N> __device__ __forceinline__ void wait_vm() { asm volatile("s_waitcnt vmcnt(%0)" ::"i"(N) : "memory"); }

__device__ __forceinline__ void st16(const void* base, unsigned voff, u32x4 v) {
    asm volatile("s_nop 4\n\tglobal_store_dwordx4 %0, %1, %2 sc0 sc1\n\ts_nop 1" ::"v"(voff), "v"(v), "s"(base) : "memory");
}
__device__ __forceinline__ void st12(const void* base, unsigned voff, u32x3 v) {
    asm volatile("s_nop 4\n\tglobal_store_dwordx3 %0, %1, %2 sc0 sc1\n\ts_nop 1" ::"v"(voff), "v"(v), "s"(base) : "memory");
}
__device__ __forceinline__ void st8(const void* base, unsigned voff, u32x2 v) {
    asm volatile("s_nop 4\n\tglobal_store_dwordx2 %0, %1, %2 sc0 sc1" ::"v"(voff), "v"(v), "s"(base) : "memory");
}
__device__ __forceinline__ void st1(const void* base, unsigned voff, unsigned v) {
    asm volatile("s_nop 4\n\tglobal_store_byte %0, %1, %2 sc0 sc1" ::"v"(voff), "v"(v), "s"(base) : "memory");
}
__device__ __forceinline__ void st4(const void* base, unsigned voff, unsigned v) {
    asm volatile("s_nop 4\n\tglobal_store_dword %0, %1, %2 sc0 sc1" ::"v"(voff), "v"(v), "s"(base) : "memory");
}
__device__ __forceinline__ u32x3 ld12(const void* base, unsigned voff) {
    u32x3 r;
    asm volatile("s_nop 4\n\tglobal_load_dwordx3 %0, %1, %2 sc0 sc1\n\ts_waitcnt vmcnt(0)" : "=&v"(r) : "v"(voff), "s"(base) : "memory");
    return r;
}
__device__ __forceinline__ void ld64(const void* base, unsigned voff, u32x4& a, u32x4& b, u32x4& c, u32x4& d) {
    asm volatile("s_nop 4\n\tglobal_load_dwordx4 %0, %4, %5 sc0 sc1\n\tglobal_load_dwordx4 %1, %4, %5 offset:1024 sc0 sc1\n\t"
                 "global_load_dwordx4 %2, %4, %5 offset:2048 sc0 sc1\n\tglobal_load_dwordx4 %3, %4, %5 offset:3072 sc0 sc1\n\ts_waitcnt vmcnt(0)"
                 : "=&v"(a), "=&v"(b), "=&v"(c), "=&v"(d) : "v"(voff), "s"(base) : "memory");
}
template <bool TWO> __device__ __forceinline__ void dma(const void* base, unsigned voff, unsigned lds_dst) {
    unsigned keep;
    if (TWO) asm volatile("s_mov_b32 %0, m0\n\ts_mov_b32 m0, %3\n\ts_nop 4\n\tglobal_load_lds_dwordx4 %1, %2 sc0 sc1\n\tglobal_load_lds_dwordx4 %1, %2 offset:1024 sc0 sc1\n\ts_mov_b32 m0, %0"
                          : "=&s"(keep) : "v"(voff), "s"(base), "s"(lds_dst) : "memory");
    else asm volatile("s_mov_b32 %0, m0\n\ts_mov_b32 m0, %3\n\ts_nop 4\n\tglobal_load_lds_dwordx4 %1, %2 sc0 sc1\n\ts_mov_b32 m0, %0"
                      : "=&s"(keep) : "v"(voff), "s"(base), "s"(lds_dst) : "memory");
}
__device__ __forceinline__ unsigned now64() { return (unsigned)__builtin_amdgcn_s_memrealtime() << 6; }
__device__ __forceinline__ void pace(unsigned& t, unsigned period) {
    int late = (int)(now64() - t);
    if (late < -(1 << 22)) { t += (unsigned)late; late = 0; }
    while (late < 0) { __builtin_amdgcn_s_sleep(1); late = (int)(now64() - t); }
    t += period + (late > CREDIT * 64 ? (unsigned)(late - CREDIT * 64) : 0u);
}
__device__ __forceinline__ unsigned lds_get(const LAS unsigned* a) { return UNI(__hip_atomic_load(a, RLX, SC_WG)); }
__device__ __forceinline__ unsigned cvt2(float a, float b) {
    typedef _Float16 h2 __attribute__((ext_vector_type(2)));
    h2 r = {(_Float16)a, (_Float16)b};
    return __builtin_bit_cast(unsigned, r);
}
__device__ __forceinline__ int rb_owner(int d, int rb) { return rb >= 6 ? d : ((((d >> 1) + 1 + (rb >> 1)) & 3) << 1) | (rb & 1); }
__device__ __forceinline__ int rb_local(int d, int rb) { return rb >= 6 ? (rb - 6) * 256 : (d & 1) * 256; }
__device__ __forceinline__ int rb_of(int d, int o) { return ((((o >> 1) - (d >> 1) - 1) & 3) << 1) | (o & 1); }
__device__ __forceinline__ void unit_of(int e, int j, int& c, int& id) {
    if (e < 32) { c = e >> 2; id = j * 4 + (e & 3); }
    else if (e < 44) { const int u = (e - 32) * 8 + j; c = u / 12; id = 32 + u - c * 12; }
    else { const int u = (e & 1) * 8 + j; c = u >> 1; id = (e < 46 ? 46 : 44) + (u & 1); }
}
__device__ __forceinline__ int gid_of(int d, int id) {
    const int rb = (id - 32) >> 1;
    return id < 32 ? (d >> 1) * 32 + id : 128 + rb_owner(d, rb) * 4 + (rb_local(d, rb) >> 7) + (id & 1);
}
__device__ __forceinline__ size_t stage_off(int id, int ks) {
    return id < 32 ? STG_A + (size_t)(id * NSTEP + ks) * B_STEP : (size_t)(((id - 32) >> 1) * NSTEP + ks) * A_STEP + (id & 1) * 16384;
}
template <int BITS, int SH> __device__ __forceinline__ unsigned wave_max_exp(unsigned mag) {
    unsigned e = 0;
#pragma unroll
    for (int b = BITS - 1; b >= 0; --b) if (__any(mag >= ((e + (1u << b)) << SH))) e += 1u << b;
    return e;
}

__device__ __forceinline__ unsigned row_scan(unsigned x) {
    x += __builtin_amdgcn_update_dpp(0, x, 0x111, 0xf, 0xf, true);
    x += __builtin_amdgcn_update_dpp(0, x, 0x112, 0xf, 0xf, true);
    x += __builtin_amdgcn_update_dpp(0, x, 0x114, 0xf, 0xf, true);
    x += __builtin_amdgcn_update_dpp(0, x, 0x118, 0xf, 0xf, true);
    return x;
}
__device__ __forceinline__ unsigned ones2() { unsigned x = 0x00010001u; asm("" : "+s"(x)); return x; }
#define PK(x) __builtin_bit_cast(u16x2, (unsigned)(x))
#define DW(x) __builtin_bit_cast(unsigned, (u16x2)(x))
#define SUBS(a, b) DW(__builtin_elementwise_sub_sat(PK(a), PK(b)))
#define MIN2(a, b) DW(__builtin_elementwise_min(PK(a), PK(b)))

template <int BITS> __device__ __forceinline__ int code_base(const unsigned (&ef)[16]) {
    u16x2 m4[4];
#pragma unroll
    for (int i = 0; i < 16; ++i) m4[i & 3] = i < 4 ? PK(ef[i]) : __builtin_elementwise_max(m4[i & 3], PK(ef[i]));
    const u16x2 mx = __builtin_elementwise_max(__builtin_elementwise_max(m4[0], m4[1]), __builtin_elementwise_max(m4[2], m4[3]));
    const int emax = (int)wave_max_exp<BITS, 0>(mx.x > mx.y ? mx.x : mx.y);
    int up = 0, mid = 0;
#pragma unroll
    for (int i = 0; i < 3; ++i) {
        const int e = (int)(ef[5 * i] >> 16);
        up += __builtin_popcountll(__ballot(e == emax)) - __builtin_popcountll(__ballot(e == emax - 3));
        mid += __builtin_popcountll(__ballot(e == emax - 1)) - __builtin_popcountll(__ballot(e == emax - 4));
    }
    const int bs = emax - (up > 0 ? 3 : mid > 0 ? 4 : 5);
    return bs > 0 ? bs : 0;
}
template <int POOL> __device__ __forceinline__ unsigned code_put(unsigned (&ef)[16], unsigned B, LAS unsigned* pool, int row) {
    const unsigned b2 = B * 0x00010001u, lut = 0x00030000u | ((B + 2u + (B ? 1u : 0u)) << 24);
    u64 str = 0;
    unsigned sh = 0;
#pragma unroll
    for (int i = 0; i < 16; ++i) {
        const unsigned sel = DW(__builtin_elementwise_max(PK(ef[i] + (0x0c020c02u - b2)), PK(0x0c000c00u)));
        const unsigned len = SUBS(b2 + 0x00010001u, ef[i]) + __builtin_amdgcn_perm(lut, 0x00000001u, sel);
        sh += len & 0xffffu; str |= 1ull << (sh & 63u);
        sh += len >> 16; str |= 1ull << (sh & 63u);
        ef[i] = __builtin_amdgcn_perm(0x00000302u, 0x01000000u, sel);
    }
    const unsigned end = row_scan(sh);
    const unsigned r0 = __builtin_amdgcn_readlane(end, 15), r1 = __builtin_amdgcn_readlane(end, 31), r2 = __builtin_amdgcn_readlane(end, 47);
    const unsigned bits = r0 + r1 + r2 + __builtin_amdgcn_readlane(end, 63);
    if (!__all(sh <= 63u && end <= 32u * (PW - 1)) || bits > 32u * (POOL - 1)) return 0u;
    const unsigned pos = 32u + end - sh + (row > 0 ? r0 : 0u) + (row > 1 ? r1 : 0u) + (row > 2 ? r2 : 0u);
    str >>= 1;
    const u64 s0 = str << (pos & 31u);
    LAS unsigned* pp = pool + (pos >> 5);
    __hip_atomic_store(pool, r0 | r1 << 10 | r2 << 20, RLX, SC_WG);
    __hip_atomic_fetch_or(pp, (unsigned)s0, RLX, SC_WG);
    __hip_atomic_fetch_or(pp + 1, (unsigned)(s0 >> 32), RLX, SC_WG);
    __hip_atomic_fetch_or(pp + 2, (unsigned)(((str >> 32) << (pos & 31u)) >> 32), RLX, SC_WG);
    return (bits + 63u) >> 5;
}
template <int SH> __device__ __forceinline__ u64 code_find(const unsigned (&w)[16], unsigned (&ne)[16], const LAS unsigned* pool, int row) {
    const unsigned rl = pool[0];
    const unsigned start = 32u + (row > 0 ? rl & 1023u : 0u) + (row > 1 ? (rl >> 10) & 1023u : 0u) + (row > 2 ? (rl >> 20) & 1023u : 0u);
    pool += start >> 5;
    const unsigned one = ones2();
    unsigned pd[PW - 1], sum = 0;
#pragma unroll
    for (int i = 0; i < PW - 1; ++i) pd[i] = pool[i];
#pragma unroll
    for (int i = 0; i < 16; ++i) { ne[i] = SUBS(one, w[i] & (0x00030003u << SH)); sum += ne[i]; }
    sum = (sum & 0xffffu) + (sum >> 16);
    const unsigned skip = row_scan(sum) - sum + __builtin_popcount(__builtin_amdgcn_ubfe(pd[0], 0u, start));
    unsigned cum = 0, at = 0, below = 0;
#pragma unroll
    for (int i = 0; i < PW - 1; ++i) {
        cum += __builtin_popcount(pd[i]);
        const bool lt = cum < skip;
        at = lt ? i + 1 : at; below = lt ? cum : below;
    }
    const unsigned d0 = pool[at], d1 = pool[at + 1], d2 = pool[at + 2];
    unsigned rem = skip - below, pos = 0;
#pragma unroll
    for (int sft = 16; sft; sft >>= 1) {
        const unsigned t = __builtin_popcount(__builtin_amdgcn_ubfe(d0, pos, sft));
        const bool lt = t < rem;
        rem -= lt ? t : 0u; pos += lt ? sft : 0u;
    }
    pos = skip ? pos + 1u : 0u;
    return ((unsigned)((((u64)d1 << 32) | d0) >> pos) | ((((u64)d2 << 32) | d1) >> pos << 32)) << 1 | 1u;
}
__device__ __forceinline__ unsigned code_get(u64& str, unsigned ne, unsigned B) {
    const unsigned l0 = __builtin_ctz((unsigned)str ^ (ne & 0xffffu)); str >>= l0;
    const unsigned l1 = __builtin_ctz((unsigned)str ^ (ne >> 16)); str >>= l1;
    const unsigned len = l0 | (l1 << 16);
    const unsigned top = __builtin_amdgcn_perm(B + 2u, B * 0x01010101u + 0x07010100u, MIN2(len, 0x00040004u) | 0x0c000c00u);
    return MIN2(DW(PK(top) - PK(len)), (B + 5u) * 0x00010001u);
}

typedef float f32x2 __attribute__((ext_vector_type(2)));
__device__ __forceinline__ f32x2 gelu2(f32x2 y) {
    const float k1 = -2.0f * 0.7978845608028654f * 1.4426950408889634f, k3 = k1 * 0.044715f;
    const f32x2 a = __builtin_elementwise_fma(y * y, (f32x2){k3, k3}, (f32x2){k1, k1}) * y;
    const f32x2 d = (f32x2){__builtin_amdgcn_exp2f(a.x), __builtin_amdgcn_exp2f(a.y)} + 1.0f;
    return y * (f32x2){__builtin_amdgcn_rcpf(d.x), __builtin_amdgcn_rcpf(d.y)};
}

#define TIMED_OUT(t0) (UNI(__hip_atomic_load(tmo, RLX, SC_AGENT)) == call || __builtin_amdgcn_s_memrealtime() - (t0) > DEADLINE_TICKS)

__global__ __launch_bounds__(NTHR, 2) void fused_gemm_rs(Params p) {
    __shared__ __attribute__((aligned(16))) unsigned char lds_raw[3 * STAGE + 2048];
    LAS unsigned char* lds = (LAS unsigned char*)lds_raw;
    LAS unsigned* ctl = (LAS unsigned*)(lds + 3 * STAGE);
    const unsigned lds0 = (unsigned)(size_t)lds;
    const int tid = threadIdx.x, lane = tid & 63;
    const int wid = UNI(tid >> 6);
    const int fr = lane & 15, fq = lane >> 4;
    const unsigned lo = lane * 16u;
    const unsigned bid = (blockIdx.x + 64u) & 255u;
    const int xcd = bid & 7, idx = bid >> 3;
    const bool late = idx < 8;
    const int q = wid & 3;

    const int team = (wid >> 2) * 192 + (int)bid - 64;
    f32x4 r[8];
    auto request = [&](int g, int u) {
        if (g < 32) {
            const GAS char* b = (const GAS char*)(p.w + (size_t)((g >> 2) * 64 + (q & 1) * 32) * GN + (u * 4 + (g & 3)) * 128 + (q >> 1) * 64) + (unsigned)(8 * fq * GN + 4 * fr) * 4u;
#pragma unroll
            for (int e = 0; e < 8; ++e) r[e] = *(const GAS f32x4*)(b + (size_t)e * (GN * 4));
        } else {
            const GAS char* b = (const GAS char*)(p.x + (size_t)(u * 128 + q * 32) * KSH + (g - 32) * 64) + (unsigned)(fr * KSH + 8 * fq) * 4u;
#pragma unroll
            for (int e = 0; e < 8; ++e) r[e] = *(const GAS f32x4*)(b + (e >> 2) * (16 * KSH * 4) + ((e >> 1) & 1) * 128 + (e & 1) * 16);
        }
    };
    if (!late && team < 96) request(team >= 64 ? 32 : (team >> 5) * 4, team & 31);
    const unsigned t_entry = now64();

    const int rank = p.rank & 7;
    char* myws = p.myws;
    GAS unsigned* ctr = (GAS unsigned*)(myws + WS_CTR + (size_t)bid * 128);
    GAS unsigned* tmo = (GAS unsigned*)(myws + WS_TMO);
    const unsigned ctr_v = __hip_atomic_load(ctr, RLX, SC_AGENT);
    GAS unsigned* gflag = (GAS unsigned*)(myws + WS_FLAG);
    ctl[tid] = 0u;
    WAIT_L0();
    BAR();

    if (!late) {
        unsigned wlo = 0, whi = 0;
#pragma unroll
        for (int d = 0; d < NR; ++d) {
            unsigned a = (unsigned)(size_t)p.ws[d], b = (unsigned)((size_t)p.ws[d] >> 32);
            asm volatile("" : "+s"(a), "+s"(b));
            wlo = lane == d ? a : wlo;
            whi = lane == d ? b : whi;
        }
#pragma unroll 1
        for (int it = 0; it < 5; ++it) {
            const int t = it < 2 ? it * 384 + team : 2 * 384 + (it - 2) * 288 + (wid >> 2) * 144 + (((idx - 8) >> 2) * 3 + (idx & 3) - 1) * 8 + xcd;
            if (t >= 1280 || (it >= 2 && (idx & 3) == 0)) break;
            const int g = p.prod[t >> 5], u = t & 31;
            if (t >= 96) {
                const unsigned ts = t_entry + (unsigned)((t >> 5) + 2) * (PROD_GAP * 64);
                while ((int)(now64() - ts) < 0) __builtin_amdgcn_s_sleep(2);
                request(g, u);
            }
            const bool isw = g < 32;
            const int c = isw ? g >> 2 : g - 32;
            const int gid = isw ? u * 4 + (g & 3) : 128 + u;
            unsigned v[16];
            if (isw) {
#pragma unroll
                for (int i = 0; i < 16; ++i) v[i] = cvt2(r[(i & 3) * 2][i >> 2], r[(i & 3) * 2 + 1][i >> 2]);
            } else {
#pragma unroll
                for (int i = 0; i < 16; ++i) v[i] = cvt2(r[i >> 1][(i & 1) * 2], r[i >> 1][(i & 1) * 2 + 1]);
            }
            asm volatile("" : "+v"(v[15]));
            const unsigned t_loaded = now64();
            const bool pushed = t < 32;
            const unsigned call = UNI(ctr_v) + 1u;
            const size_t par = WS_EXI + (size_t)(call & 1u) * EXI_BYTES;
            const int o = u >> 2;
            const int myid = isw ? ((gid >> 5) == (rank >> 1) ? gid & 31 : -1)
                                 : (o == rank ? 44 + (u & 3) : ((o >> 1) != (rank >> 1) && ((u >> 1) & 1) == (rank & 1) ? 32 + rb_of(rank, o) * 2 + (u & 1) : -1));
            unsigned hdr = 0;
            {
                unsigned w[16];
#pragma unroll
                for (int i = 0; i < 16; ++i) w[i] = v[i];
                const char* home = myws + par + (size_t)(gid * 8 + c) * SLOT;
                unsigned pw = 0, used = 0, at = 0;
                hdr = 32u;
                if (!pushed) {
                    LAS unsigned* pool = (LAS unsigned*)(lds + wid * 256);
                    pool[lane] = 0u;
                    unsigned ef[16];
#pragma unroll
                    for (int i = 0; i < 16; ++i) ef[i] = (w[i] >> 10) & 0x001f001fu;
                    const unsigned B = (unsigned)code_base<5>(ef);
                    if ((used = code_put<POOL_IN>(ef, B, pool, fq))) {
                        if (lane == 0) at = __hip_atomic_fetch_add(ctl + 490 + (wid >> 2) * 5 + it, used, RLX, SC_WG);
#pragma unroll
                        for (int i = 0; i < 16; ++i) w[i] = (w[i] & 0x83ff83ffu) | (ef[i] << 10);
#pragma unroll
                        for (int k = 0; k < 3; ++k) {
#pragma unroll
                            for (int i = 0; i < 4; ++i) w[4 * k + i] |= (w[13 + k] << (12 - 3 * i)) & 0x70007000u;
                            w[12] |= (w[13 + k] >> (1 + k)) & (0x40004000u >> k);
                        }
                        pw = __hip_atomic_load(pool + lane, RLX, SC_WG);
                        hdr = B;
                    }
                }
                if (pushed) {
                    unsigned tp = t_loaded + (unsigned)((u & 7) * 4 + q) * PUSH_SLOT;
#pragma unroll
                    for (int i = 0; i < 4; ++i) {
                        pace(tp, 32u * PUSH_SLOT);
#pragma unroll
                        for (int k = 0; k < 2; ++k) {
                            const int d = (u >> 3) * 2 + k;
                            if (d != rank) st16(p.ws[d] + par + (size_t)(1280 + rank * 8 + (u & 7)) * SLOT + q * 4096 + i * 1024, lo, (u32x4){v[4 * i], v[4 * i + 1], v[4 * i + 2], v[4 * i + 3]});
                        }
                    }
                } else {
#pragma unroll
                    for (int i = 0; i < 3; ++i) st16(home + q * QB + i * 1024, lo, (u32x4){w[4 * i], w[4 * i + 1], w[4 * i + 2], w[4 * i + 3]});
                    st4(home + q * QB + 3072, lane * 4u, w[12]);
                    if (hdr == 32u) st12(home + SIDE + q * 768, lane * 12u, (u32x3){v[13], v[14], v[15]});
                    else {
                        at = UNI(at) + 1u;
                        if (lane < used) st4(home + PAREA + at * 4u, lane * 4u, pw);
                        if (lane == 0) st1(home + PAREA, q, at);
                    }
                }
            }
            if (myid >= 0) {
                const char* sg = myws + WS_STG + stage_off(myid, rank * 8 + c) + (isw ? (q >> 1) * 8 + (q & 1) : q * 4) * 1024;
#pragma unroll
                for (int i = 0; i < 4; ++i) st16(sg + i * (isw ? 2048 : 1024), lo, (u32x4){v[4 * i], v[4 * i + 1], v[4 * i + 2], v[4 * i + 3]});
            }
            wait_vm<0>();
            unsigned tw = (1u << 24) | (hdr << (6 * q));
            if (lane == 0) tw += __hip_atomic_fetch_add(ctl + 402 + (wid >> 2) * 5 + it, tw, RLX, SC_WG);
            tw = UNI(tw);
            if ((tw >> 24) == 4u) {
                tw = (tw & 0xffffffu) | (lds_get(ctl + 490 + (wid >> 2) * 5 + it) >= 160u ? 1u << 24 : 0u);
                const int d = lane & 7;
                int did = isw ? ((gid >> 5) == (d >> 1) ? gid & 31 : -1)
                              : (o == d ? 44 + (u & 3) : ((o >> 1) != (d >> 1) && ((u >> 1) & 1) == (d & 1) ? 32 + rb_of(d, o) * 2 + (u & 1) : -1));
                if (isw && (g & 3) == 3 && c < 2 * NFWD && (d >> 1) != (rank >> 1) && ((c ^ d) & 1)) did = -1;
                if (lane < 8 && did >= 0) {
                    if (d == rank) __hip_atomic_store(gflag + did * 64 + c * 8 + rank, call, RLX, SC_AGENT);
                    else __hip_atomic_store((GAS u64*)((((size_t)whi << 32) | wlo) + WS_RDY) + ((((call & 1u) * 8 + rank) * 48 + did) * 8 + c) * RSTRIDE, ((u64)call << 32) | tw, RLX, SC_SYS);
                }
            }
        }
    } else if (idx < 7) {
        const int s = idx < 6 ? rb_owner(rank, idx) : rank ^ 1;
        const int nu = idx < 6 ? NU_OUT : NU_PAIR;
        unsigned desc[2];
#pragma unroll
        for (int k = 0; k < 2; ++k) {
            const int e = p.order[idx == 6][k * 64 + lane];
            int c, id, org = s, slot;
            if (e < 48) { unit_of(e, xcd, c, id); slot = gid_of(rank, id) * 8 + c; }
            else { const int kf = (e - 48) / 6; c = 2 * kf + (s & 1); id = xcd * 4 + 3; org = rb_owner(rank, e - 48 - kf * 6); slot = 1408 + (e - 48) * 8 + xcd; }
            desc[k] = (unsigned)(c | id << 3 | org << 9 | slot << 12);
        }
        auto desc_of = [&](int n) { return (unsigned)(n < 64 ? __builtin_amdgcn_readlane(desc[0], n) : __builtin_amdgcn_readlane(desc[1], n - 64)); };
        const unsigned call = UNI(ctr_v) + 1u;
        const size_t par = WS_EXI + (size_t)(call & 1u) * EXI_BYTES;
        GAS u64* eq = (GAS u64*)(myws + WS_EQ) + idx * 16;
        if (wid == 6) {
            const char* exs = p.ws[s] + par;
            unsigned tnext = t_entry + T_PULL * 64 + xcd * (TAU / 4);
            u64 todo = ~1ull;
            __builtin_amdgcn_s_setprio(3);
            const char* src = nullptr;
            unsigned trim = 0;
            bool big = false;
            auto choose = [&](int n) {
                const unsigned rdy = __hip_atomic_load(ctl + (n >= 64 ? n : lane), RLX, SC_WG);
                const unsigned fre = __hip_atomic_load(ctl + (n > RING ? 128 + n - RING : 400), RLX, SC_WG), tr = __hip_atomic_load(ctl + 400, RLX, SC_WG);
                const u64 rd = __ballot(rdy != 0u) & (n >= 64 ? 1ull : todo);
                if (!rd || (n > RING && UNI(fre) < 4u)) return;
                const int un = n >= 64 ? n : __builtin_ctzll(rd);
                todo &= ~(1ull << (un & 63));
                if (lane == 0) __hip_atomic_store(ctl + 416 + n, (unsigned)un, RLX, SC_WG);
                trim = n >= EQ_AT + 4 && n < EQ_AT + 36 ? UNI(tr) : 0u;
                big = (__builtin_amdgcn_readlane(rdy, un & 63) >> 24) & 1u;
                src = exs + (size_t)(desc_of(un) >> 12) * SLOT;
            };
            for (int n = 1; n < nu; ++n) {
                bool drained = false;
                if (!src) {
                    choose(n);
                    if (!src) {
                        wait_vm<0>();
                        if (lane == 0) __hip_atomic_store(ctl + 384, (unsigned)(n - 1) * 14u, RLX, SC_WG);
                        drained = true;
                        for (choose(n); !src; choose(n)) __builtin_amdgcn_s_sleep(1);
                    }
                }
                const char* cur = src;
                const unsigned dst = lds0 + (unsigned)(n % RING) * 14336u, tr = trim, pl = big ? 48u : 40u;
                src = nullptr;
                asm volatile("" ::: "memory");
#pragma unroll
                for (int i = 0; i < 14; i += 2) {
                    wait_vm<WIN - 2>();
                    if ((i > WIN - 2 || (!drained && n > 1)) && lane == 0) __hip_atomic_store(ctl + 384, (unsigned)((n - 1) * 14 + i - (WIN - 2)), RLX, SC_WG);
                    pace(tnext, (i ? 2u * TAU : TAU + pl * TAU / 64u) - tr);
                    if (i) dma<true>(cur + (i - 1) * 1024, lo, dst + (i - 1) * 1024);
                    else {
                        if (lane < pl) dma<false>(cur + PAREA, lo, dst + PAREA);
                        dma<false>(cur, lo, dst);
                    }
                    if (i == 0 && lane == 0) {
                        if (n == EQ_AT) __hip_atomic_store(ctl + 387, tnext | 1u, RLX, SC_WG);
                        if (n == nu - 3) __hip_atomic_store(ctl + 386, tnext | 1u, RLX, SC_WG);
                        if (n == nu - 1) __hip_atomic_store(ctl + 385, (tnext + 12u * TAU) | 1u, RLX, SC_WG);
                    }
                    if (i == 10 && n + 1 < nu) choose(n + 1);
                }
            }
            __builtin_amdgcn_s_setprio(0);
            wait_vm<0>();
            if (lane == 0) __hip_atomic_store(ctl + 384, (unsigned)(nu - 1) * 14u, RLX, SC_WG);
        } else {
        if (wid == 7) {
            const GAS u64* rp[2];
            bool pend[2];
#pragma unroll
            for (int k = 0; k < 2; ++k) {
                rp[k] = (const GAS u64*)(myws + WS_RDY) + ((((call & 1u) * 8 + ((desc[k] >> 9) & 7)) * 48 + ((desc[k] >> 3) & 63)) * 8 + (desc[k] & 7)) * RSTRIDE;
                pend[k] = k * 64 + lane < nu;
            }
            unsigned mine = 0;
            bool eq_open = true;
            const u64 t0 = __builtin_amdgcn_s_memrealtime();
            for (unsigned polls = 1;; ++polls) {
                const u64 pm = __ballot(pend[0]);
                const bool front = __builtin_amdgcn_mbcnt_hi((unsigned)(pm >> 32), __builtin_amdgcn_mbcnt_lo((unsigned)pm, 0u)) < POLLW;
#pragma unroll
                for (int k = 0; k < 2; ++k)
                    if (pend[k] && ((k == 0 && front) || (polls & 7u) == 0u)) {
                        const u64 f = __hip_atomic_load(rp[k], RLX, SC_SYS);
                        if ((unsigned)(f >> 32) >= call) { __hip_atomic_store(ctl + k * 64 + lane, (unsigned)f | 0x80000000u, RLX, SC_WG); pend[k] = false; }
                    }
                if (eq_open) {
                    if (!mine) {
                        mine = lds_get(ctl + 387);
                        if (mine && lane == 0) __hip_atomic_store(eq + xcd, ((u64)call << 32) | mine, RLX, SC_AGENT);
                    } else {
                        const u64 f = __hip_atomic_load(eq + (lane & 7), RLX, SC_AGENT);
                        if (__all((unsigned)(f >> 32) == call)) {
                            int dlt = (int)(mine - (unsigned)f);
                            dlt += __shfl_xor(dlt, 1); dlt += __shfl_xor(dlt, 2); dlt += __shfl_xor(dlt, 4);
                            dlt /= 8 * 32 * 7;
                            dlt = dlt > TAU / 8 ? TAU / 8 : dlt < -(TAU / 8) ? -(TAU / 8) : dlt;
                            if (lane == 0) __hip_atomic_store(ctl + 400, (unsigned)dlt, RLX, SC_WG);
                            eq_open = false;
                        } else if ((int)lds_get(ctl + 384) >= (EQ_AT + 2) * 14) eq_open = false;
                    }
                }
                const bool waiting = __any(pend[0] || pend[1]);
                if (!waiting && !eq_open) break;
                if (waiting && (polls & 15u) == 0u && TIMED_OUT(t0)) {
                    if (lane == 0) __hip_atomic_store(tmo, call, RLX, SC_AGENT);
                    wait_vm<0>();
#pragma unroll
                    for (int k = 0; k < 2; ++k) if (pend[k]) __hip_atomic_store(ctl + k * 64 + lane, 0x80000000u, RLX, SC_WG);
                    break;
                }
                if (polls < 32u) __builtin_amdgcn_s_sleep(1); else __builtin_amdgcn_s_sleep(16);
            }
        }
        if (wid < 6 || idx == 6) {
            int pend = -1, pend_n = 0;
            unsigned pend_h = 0;
            auto staged = [&]() {
                wait_vm<0>();
                if (lane == 0 && __hip_atomic_fetch_add(ctl + 256 + pend_n, 1u, RLX, SC_WG) == 3u) {
                    __hip_atomic_store(gflag + pend, call, RLX, SC_AGENT);
                    if (pend_h) __hip_atomic_store((GAS u64*)(p.ws[rank ^ 1] + WS_RDY) + ((((call & 1u) * 8 + s) * 48 + (pend >> 6)) * 8 + ((pend >> 3) & 7)) * RSTRIDE,
                                                   ((u64)call << 32) | (pend_h & 0x1ffffffu), RLX, SC_SYS);
                }
                pend = -1;
            };
            auto need = [&](int job) { return (job >> 2) * 14 - 13 + (((job & 3) + 1) * QB + 1023) / 1024; };
            auto ticket = [&]() { unsigned tk = 0; if (lane == 0) tk = __hip_atomic_fetch_add(ctl + 401, 1u, RLX, SC_WG); return (int)UNI(tk); };
            for (int job = ticket(); job < 4 * nu;) {
                const int n = job >> 2, qq = job & 3;
                unsigned w[16];
                int un = 0;
                unsigned relay_h = 0;
                if (n == 0) {
                    while (lds_get(ctl) == 0u) __builtin_amdgcn_s_sleep(1);
                    u32x4 a0, a1, a2, a3;
                    ld64(myws + par + (size_t)(1280 + s * 8 + xcd) * SLOT + qq * 4096, lo, a0, a1, a2, a3);
#pragma unroll
                    for (int i = 0; i < 4; ++i) { w[i] = a0[i]; w[4 + i] = a1[i]; w[8 + i] = a2[i]; w[12 + i] = a3[i]; }
                } else {
                    while ((int)lds_get(ctl + 384) < need(job)) __builtin_amdgcn_s_sleep(1);
                    asm volatile("" ::: "memory");
                    un = lds_get(ctl + 416 + n);
                }
                const unsigned de = desc_of(un);
                const int c = de & 7, id = (de >> 3) & 63, org = (de >> 9) & 7;
                if (n) {
                    const LAS unsigned char* src = lds + (unsigned)(n % RING) * 14336u + qq * QB;
#pragma unroll
                    for (int i = 0; i < 3; ++i) { const u32x4 a = *(const LAS u32x4*)(src + i * 1024 + lo); w[4 * i] = a[0]; w[4 * i + 1] = a[1]; w[4 * i + 2] = a[2]; w[4 * i + 3] = a[3]; }
                    w[12] = *(const LAS unsigned*)(src + 3072 + lane * 4);
                    const LAS unsigned char* parea = lds + (unsigned)(n % RING) * 14336u + PAREA;
                    const LAS unsigned* pool = (const LAS unsigned*)parea + parea[qq];
                    const unsigned hu = lds_get(ctl + un), h = (hu >> (6 * qq)) & 63u;
                    if (idx < 6 && id < 32 && (id & 3) == 3 && c < 2 * NFWD) {
                        const unsigned pw = ((const LAS unsigned*)parea)[qq * 48 + lane];
                        if (pend >= 0) staged();
                        const char* dst = myws + par + (size_t)(1408 + ((c >> 1) * 6 + idx) * 8 + xcd) * SLOT + qq * QB;
#pragma unroll
                        for (int i = 0; i < 3; ++i) st16(dst + i * 1024, lo, (u32x4){w[4 * i], w[4 * i + 1], w[4 * i + 2], w[4 * i + 3]});
                        st4(dst + 3072, lane * 4u, w[12]);
                        if (lane < 48) st4(dst + (PAREA - qq * QB) + qq * 192, lane * 4u, pw);
                        relay_h = hu;
                    }
                    if (h < 32u) {
#pragma unroll
                        for (int k = 0; k < 3; ++k) {
                            unsigned cc = (w[12] << (1 + k)) & 0x80008000u;
#pragma unroll
                            for (int i = 0; i < 4; ++i) cc |= (w[4 * k + i] >> (12 - 3 * i)) & (0x00070007u << (3 * i));
                            w[13 + k] = cc;
                        }
                        unsigned ne[16];
                        u64 str = code_find<10>(w, ne, pool, fq);
                        WAIT_L0();
                        if (lane == 0) __hip_atomic_fetch_add(ctl + 128 + n, 1u, RLX, SC_WG);
                        if (pend >= 0) staged();
#pragma unroll
                        for (int i = 0; i < 16; ++i) w[i] = (w[i] & 0x8fff8fffu) + (code_get(str, ne[i], h) << 10);
                    } else {
                        WAIT_L0();
                        if (lane == 0) __hip_atomic_fetch_add(ctl + 128 + n, 1u, RLX, SC_WG);
                        if (pend >= 0) staged();
                        const u32x3 a = ld12(p.ws[org] + par + (size_t)(gid_of(rank, id) * 8 + c) * SLOT + SIDE + qq * 768, lane * 12u);
                        w[13] = a[0]; w[14] = a[1]; w[15] = a[2];
                    }
                }
                const char* sg = myws + WS_STG + stage_off(id, org * 8 + c) + (id < 32 ? (qq >> 1) * 8 + (qq & 1) : qq * 4) * 1024;
#pragma unroll
                for (int i = 0; i < 4; ++i) st16(sg + i * (id < 32 ? 2048 : 1024), lo, (u32x4){w[4 * i], w[4 * i + 1], w[4 * i + 2], w[4 * i + 3]});
                pend = id * 64 + c * 8 + org; pend_n = n; pend_h = relay_h;
                job = ticket();
                if (job >= 4 * nu || job < 4 || n >= NODEFER || (int)lds_get(ctl + 384) < need(job)) staged();
            }
        }
            if (wid == 7 && idx < 6) {
                unsigned mine;
                while ((mine = lds_get(ctl + 386)) == 0u) __builtin_amdgcn_s_sleep(8);
                if (lane == 0) __hip_atomic_store(eq + 8 + xcd, ((u64)call << 32) | mine, RLX, SC_AGENT);
                unsigned th;
                int dlt = 0;
                bool open = true;
                while ((th = lds_get(ctl + 385)) == 0u) {
                    if (open) {
                        const u64 f = __hip_atomic_load(eq + 8 + (lane & 7), RLX, SC_AGENT);
                        if (__all((unsigned)(f >> 32) == call)) {
                            dlt = (int)(mine - (unsigned)f);
                            dlt += __shfl_xor(dlt, 1); dlt += __shfl_xor(dlt, 2); dlt += __shfl_xor(dlt, 4);
                            dlt /= 8 * 256;
                            dlt = dlt > 100 ? 100 : dlt < -100 ? -100 : dlt;
                            open = false;
                        }
                    }
                    __builtin_amdgcn_s_sleep(4);
                }
                if (lane == 0) __hip_atomic_store((GAS u64*)(myws + WS_MBOX + (size_t)((8 + idx * 4) * 8 + xcd) * 128), ((u64)call << 32) | (th & ~255u) | (unsigned)(dlt + 128), RLX, SC_AGENT);
            }
        }
    }
    const unsigned call = UNI(ctr_v) + 1u;

    const int ntile = idx == 7 ? 2 : 1;
    const int wm = wid >> 1, wn = wid & 1;
#pragma unroll 1
    for (int rep = 0; rep < ntile; ++rep) {
    const int rb = late ? 6 + (idx >> 2) : (idx - 8) >> 2;
    const int cb = xcd * 4 + (idx == 7 ? rep : idx == 6 ? 0 : idx == 4 || idx == 5 ? idx - 2 : idx & 3);
    const bool part1 = idx == 7 && rep == 0, part2 = idx == 6;
    const int nstep = part1 ? 56 : part2 ? 8 : NSTEP;
    const char* stgA = myws + WS_STG + (size_t)(rb * NSTEP) * A_STEP + wid * 4096;
    const char* stgB = myws + WS_STG + STG_A + (size_t)(cb * NSTEP) * B_STEP + wid * 2048;
    const u64 pbits = 0x0101010101010101ull << (rank ^ 1);
    u64 todo = part1 ? ~pbits : part2 ? pbits : ~0ull, avail = 0ull;
    int slot = 0;
    auto poll = [&]() {
        if (wid == 0) {
            const u64 t0 = __builtin_amdgcn_s_memrealtime();
            u64 got;
            for (unsigned polls = 1;; ++polls) {
                const unsigned f0 = __hip_atomic_load(gflag + cb * 64 + lane, RLX, SC_AGENT);
                const unsigned f1 = __hip_atomic_load(gflag + (32 + rb * 2) * 64 + lane, RLX, SC_AGENT);
                const unsigned f2 = __hip_atomic_load(gflag + (33 + rb * 2) * 64 + lane, RLX, SC_AGENT);
                got = __ballot(f0 >= call && f1 >= call && f2 >= call) & todo;
                if (got) break;
                if ((polls & 15u) == 0u && TIMED_OUT(t0)) {
                    if (lane == 0) __hip_atomic_store(tmo, call, RLX, SC_AGENT);
                    got = todo;
                    break;
                }
                __builtin_amdgcn_s_sleep(8);
            }
            if (lane < 2) ctl[388 + slot * 2 + lane] = (unsigned)(got >> (32 * lane));
        }
    };
    auto news = [&]() { avail = ((u64)(unsigned)UNI(ctl[389 + slot * 2]) << 32) | (unsigned)UNI(ctl[388 + slot * 2]); slot ^= 1; };
    auto take = [&]() {
        const unsigned b = (unsigned)__builtin_ctzll(avail) & 63u;
        avail &= avail - 1; todo &= ~(1ull << b);
        return (b & 7u) * 8u + (b >> 3);
    };
    auto g1 = [&](unsigned ks, unsigned buf, int i) {
        if (i < 4) __builtin_amdgcn_global_load_lds((const GAS unsigned*)(stgA + (size_t)ks * A_STEP + i * 1024 + lo), (LAS unsigned*)(lds + buf + wid * 4096 + i * 1024), 16, 0, 17);
        else __builtin_amdgcn_global_load_lds((const GAS unsigned*)(stgB + (size_t)ks * B_STEP + (i - 4) * 1024 + lo), (LAS unsigned*)(lds + buf + A_STEP + wid * 2048 + (i - 4) * 1024), 16, 0, 17);
    };

    f32x4 acc[4][4];
#pragma unroll
    for (int m = 0; m < 4; ++m)
#pragma unroll
        for (int n = 0; n < 4; ++n) acc[m][n] = (f32x4){0.f, 0.f, 0.f, 0.f};
    h16x8 a0[4], b0[4], a1[4], b1[4];
    const unsigned aoff = wm * 8192 + lo, boff = A_STEP + wn * 8192 + lo;
#define RDA(A, buf, kk, m) A[m] = *(const LAS h16x8*)(lds + (buf) + aoff + (m) * 2048 + (kk) * 1024)
#define RDB(B, buf, kk, n) B[n] = *(const LAS h16x8*)(lds + (buf) + boff + (n) * 2048 + (kk) * 1024)
#define ROW(A, B, m) _Pragma("unroll") for (int n = 0; n < 4; ++n) acc[m][n] = __builtin_amdgcn_mfma_f32_16x16x32_f16(A[m], B[n], acc[m][n], 0, 0, 0)
    unsigned kq;
#pragma unroll
    for (int k = 0; k < 3; ++k) {
        if (!avail) { poll(); WAIT_L0(); BAR(); news(); }
        kq = take();
#pragma unroll
        for (int i = 0; i < (k < 2 ? 6 : 3); ++i) g1(kq, k * STAGE, i);
    }
    wait_vm<9>(); BAR();
#pragma unroll
    for (int i = 0; i < 4; ++i) { RDB(b0, 0u, 0, i); RDA(a0, 0u, 0, i); }
    unsigned cur = 0, prv = 2 * STAGE;
    for (int t = 0; t < nstep; ++t) {
        const unsigned nxt = cur == 2 * STAGE ? 0u : cur + STAGE;
        const bool m2 = t + 2 < nstep, m3 = t + 3 < nstep;
        const bool dry = m3 && !avail;
        WAIT_L0(); SB();
        RDB(b1, cur, 1, 0); RDB(b1, cur, 1, 1); SB(); ROW(a0, b0, 0); SB(); if (m2) g1(kq, prv, 3); SB();
        RDB(b1, cur, 1, 2); RDB(b1, cur, 1, 3); SB(); ROW(a0, b0, 1); SB(); if (m2) g1(kq, prv, 4); SB();
        RDA(a1, cur, 1, 0); RDA(a1, cur, 1, 1); SB(); ROW(a0, b0, 2); SB(); if (m2) g1(kq, prv, 5); SB();
        RDA(a1, cur, 1, 2); RDA(a1, cur, 1, 3); SB(); ROW(a0, b0, 3); SB();
        if (dry) poll();
        WAIT_L0();
        if (m2) wait_vm<6>(); else wait_vm<0>();
        BAR(); SB();
        RDB(b0, nxt, 0, 0); RDB(b0, nxt, 0, 1); SB(); ROW(a1, b1, 0); SB();
        if (dry) news();
        if (m3) { kq = take(); g1(kq, cur, 0); } SB();
        RDB(b0, nxt, 0, 2); RDB(b0, nxt, 0, 3); SB(); ROW(a1, b1, 1); SB(); if (m3) g1(kq, cur, 1); SB();
        RDA(a0, nxt, 0, 0); RDA(a0, nxt, 0, 1); SB(); ROW(a1, b1, 2); SB(); if (m3) g1(kq, cur, 2); SB();
        RDA(a0, nxt, 0, 2); RDA(a0, nxt, 0, 3); SB(); ROW(a1, b1, 3); SB();
        prv = cur; cur = nxt;
    }
    if (part1 || part2) {
        const char* park = myws + WS_EXO + (size_t)(xcd * 8 + wid) * 16384;
        GAS u64* pf = (GAS u64*)(myws + WS_ORDY + (size_t)xcd * 128) + wid;
        if (part1) {
#pragma unroll
            for (int i = 0; i < 16; ++i) st16(park + i * 1024, lo, __builtin_bit_cast(u32x4, acc[i >> 2][i & 3]));
            wait_vm<0>();
            if (lane == 0) __hip_atomic_store(pf, (u64)call << 32, RLX, SC_SYS);
            continue;
        }
        const u64 t0 = __builtin_amdgcn_s_memrealtime();
        while ((unsigned)UNI((unsigned)(__hip_atomic_load(pf, RLX, SC_SYS) >> 32)) < call) {
            if (TIMED_OUT(t0)) { if (lane == 0) __hip_atomic_store(tmo, call, RLX, SC_AGENT); wait_vm<0>(); break; }
            __builtin_amdgcn_s_sleep(4);
        }
#pragma unroll
        for (int m = 0; m < 4; ++m) {
            u32x4 e0, e1, e2, e3;
            ld64(park + m * 4096, lo, e0, e1, e2, e3);
            acc[m][0] += __builtin_bit_cast(f32x4, e0); acc[m][1] += __builtin_bit_cast(f32x4, e1);
            acc[m][2] += __builtin_bit_cast(f32x4, e2); acc[m][3] += __builtin_bit_cast(f32x4, e3);
        }
    }
    const unsigned tmo_v = __hip_atomic_load(tmo, RLX, SC_AGENT);
    const unsigned oo = (unsigned)(fq * 4) * (GN * 4u) + fr * 16u;
    const u32x4 nan4 = {0x7fc00000u, 0x7fc00000u, 0x7fc00000u, 0x7fc00000u};
    auto finish = [&](unsigned (&o)[32], int i, float a, float b) {
        const f32x2 g = gelu2((f32x2){a, b});
        const int at = ((i >> 3) * 4 + (i & 1) * 2) * 4 + ((i >> 1) & 3);
        o[at] = __float_as_uint(g.x); o[at + 4] = __float_as_uint(g.y);
    };
#define ACC(hf, i, k) acc[2 * (hf) + ((i) >> 3)][((i) >> 1) & 3][((i) & 1) * 2 + (k)]
    if (late) {
        const bool bad = UNI(tmo_v) == call;
        const char* ob = (const char*)(p.out + (size_t)(rb_local(rank, rb) + wm * 64) * GN + (rank >> 1) * 4096 + cb * 128 + wn * 64);
#pragma unroll
        for (int hf = 0; hf < 2; ++hf) {
            unsigned o[32];
#pragma unroll
            for (int i = 0; i < 16; ++i) finish(o, i, ACC(hf, i, 0), ACC(hf, i, 1));
#pragma unroll
            for (int i = 0; i < 8; ++i) st16(ob + (size_t)(hf * 32 + (i >> 2) * 16 + (i & 3)) * (GN * 4), oo, bad ? nan4 : (u32x4){o[4 * i], o[4 * i + 1], o[4 * i + 2], o[4 * i + 3]});
        }
    } else {
        const int peer = rb_owner(rank, rb);
        const int mirror = (8 + rb_of(peer, rank) * 4 + (idx & 3)) * 8 + xcd;
        char* const peerws = p.ws[peer];
        WAIT_L0(); BAR();
        if (wid < 4) __builtin_amdgcn_s_setprio(2);
#pragma unroll
        for (int hf = 0; hf < 2; ++hf) {
            const int k = (wid >> 2) * 8 + hf * 4 + q;
            const char* ex = myws + WS_EXO + (size_t)(bid * 16 + k) * 8192;
            unsigned hdr = 1u << 13;
            if ((idx & 3) != 3 || k < 16 - NPLAIN) {
                LAS unsigned* pool = (LAS unsigned*)(lds + 131072 + wid * 256);
                pool[lane] = 0u;
                unsigned hi[16], ef[16], under = 0;
#pragma unroll
                for (int i = 0; i < 16; ++i) {
                    hi[i] = __builtin_amdgcn_perm(__float_as_uint(ACC(hf, i, 1)), __float_as_uint(ACC(hf, i, 0)), 0x07060302u);
                    ef[i] = (hi[i] >> 7) & 0x00ff00ffu;
                }
                const unsigned bs = (unsigned)code_base<8>(ef), e0 = bs > 20u ? bs - 20u : 0u;
#pragma unroll
                for (int i = 0; i < 16; ++i) { under |= SUBS(e0 * 0x00010001u, ef[i]); ef[i] = SUBS(ef[i], e0 * 0x00010001u); }
                unsigned used, at = 0;
                if (__all(under == 0u) && (used = code_put<POOL_OUT>(ef, bs - e0, pool, fq))) {
                    if (lane == 0) at = __hip_atomic_fetch_add(ctl + 500 + (k >> 2), used, RLX, SC_WG);
#pragma unroll
                    for (int i = 0; i < 16; ++i) hi[i] = (hi[i] & 0x807f807fu) | (ef[i] << 7);
#pragma unroll
                    for (int c = 0; c < 6; ++c) {
#pragma unroll
                        for (int i = 0; i < 3; ++i) { const int fld = c * 3 + i, at = 9 + 3 * (fld & 1); hi[fld >> 1] |= (hi[10 + c] << (at - 3 * i)) & (0x00070007u << at); }
                        hi[9] |= (hi[10 + c] >> (1 + c)) & (0x40004000u >> c);
                    }
                    const unsigned pw = __hip_atomic_load(pool + lane, RLX, SC_WG);
#pragma unroll
                    for (int i = 0; i < 4; ++i) {
                        u32x4 v;
#pragma unroll
                        for (int e = 0; e < 4; ++e) v[e] = __builtin_amdgcn_perm(__float_as_uint(ACC(hf, 4 * i + e, 1)), __float_as_uint(ACC(hf, 4 * i + e, 0)), 0x05040100u);
                        st16(ex + i * 1024, lo, v);
                    }
                    st16(ex + 4096, lo, (u32x4){hi[0], hi[1], hi[2], hi[3]});
                    st16(ex + 5120, lo, (u32x4){hi[4], hi[5], hi[6], hi[7]});
                    st8(ex + 6144, lane * 8u, (u32x2){hi[8], hi[9]});
                    at = UNI(at);
                    if (lane < used) st4(myws + WS_EXP + (size_t)bid * 4096 + (k >> 2) * 768 + at * 4u, lane * 4u, pw);
                    hdr = e0 | ((bs - e0) << 8) | (at + used > 160u ? 1u << 14 : 0u) | (at << 16);
                }
            }
            if ((hdr >> 13) & 1u) {
                unsigned o[32];
#pragma unroll
                for (int i = 0; i < 16; ++i) finish(o, i, ACC(hf, i, 0), ACC(hf, i, 1));
#pragma unroll
                for (int i = 0; i < 8; ++i) st16(ex + i * 1024, lo, (u32x4){o[4 * i], o[4 * i + 1], o[4 * i + 2], o[4 * i + 3]});
            }
            wait_vm<0>();
            hdr |= UNI(tmo_v) == call ? 0xc0000000u : 0x40000000u;
            const int gs = (idx & 3) == 0 || (idx & 3) == 3 ? 4 : 16, k0 = k & -gs;
            unsigned cnt = 0;
            if (lane == 0) { __hip_atomic_store(ctl + 64 + k, hdr, RLX, SC_WG); cnt = __hip_atomic_fetch_add(ctl + 96 + k0, 1u, RLX, SC_WG); }
            if ((int)UNI(cnt) == gs - 1 && lane < gs)
                __hip_atomic_store((GAS u64*)(peerws + WS_ORDY + (size_t)mirror * 128) + k0 + lane, ((u64)call << 32) | __hip_atomic_load(ctl + 64 + k0 + lane, RLX, SC_WG), RLX, SC_SYS);
        }
        __builtin_amdgcn_s_setprio(0);
#pragma unroll 1
        for (int sk = 0; (idx & 3) + sk <= 3 && (idx & 3) != 3; sk += 3) {
        if (sk) {
            WAIT_L0(); BAR();
            if (tid < 32 || tid == 384 || tid == 401 || (tid >= 416 && tid < 432)) ctl[tid] = 0u;
            WAIT_L0(); BAR();
        }
        const int me = bid + 8 * sk;
        const char* exs = peerws + WS_EXO + (size_t)(mirror + 8 * sk) * (16 * 8192);
        if (wid == 0) {
            unsigned tnext;
            while ((tnext = lds_get(ctl + 16)) == 0u) __builtin_amdgcn_s_sleep(2);
            const unsigned trb = tnext & 255u, tr = trb - 128u;
            tnext = (tnext & ~255u) | 128u;
            __builtin_amdgcn_s_setprio(3);
            wait_vm<0>(); __builtin_amdgcn_s_waitcnt(0x0F70);
            int issued = 0, told = 0;
            unsigned rdy = 0, pln = 0, big = 0;
            auto look = [&]() {
                const unsigned h = __hip_atomic_load(ctl + (lane & 15), RLX, SC_WG);
                rdy = (unsigned)__ballot(h != 0u); pln = (unsigned)__ballot((h >> 13) & 1u); big = (unsigned)__ballot((h >> 14) & 1u);
            };
            look();
            for (int n = 0; n < 16; ++n) {
                const bool head = (n & 3) == 0;
                if (head && ((rdy >> n) & 15u) != 15u) {
                    look();
                    if (((rdy >> n) & 15u) != 15u) {
                        wait_vm<0>();
                        told = issued;
                        if (lane == 0) __hip_atomic_store(ctl + 384, (unsigned)told, RLX, SC_WG);
                        for (look(); ((rdy >> n) & 15u) != 15u; look()) __builtin_amdgcn_s_sleep(1);
                    }
                }
                asm volatile("" ::: "memory");
                const bool plain = (pln >> n) & 1u;
                const unsigned pl = (big >> n) & 15u ? 48u : 40u;
                const char* cur = exs + (size_t)n * 8192;
                const unsigned dst = lds0 + (unsigned)n * 8192u;
#pragma unroll
                for (int i = 0; i < 8; i += 2) {
                    wait_vm<WIN - 2>();
                    if (issued - (WIN - 2) > told) { told = issued - (WIN - 2); if (lane == 0) __hip_atomic_store(ctl + 384, (unsigned)told, RLX, SC_WG); }
                    pace(tnext, (i < 6 || plain ? 2u * TAU : TAU / 2u) + (i == 6 && head ? pl * TAU / 64u : 0u) - tr);
                    if (i < 6 || plain) { dma<true>(cur + i * 1024, lo, dst + i * 1024); issued += 2; }
                    else { if (lane < 32) dma<false>(cur + 6144, lo, dst + 6144); issued += 1; }
                    if (i == 6 && head) {
                        if (lane < pl) dma<false>(peerws + WS_EXP + (size_t)(mirror + 8 * sk) * 4096 + n * 192, lo, lds0 + 133120u + n * 192);
                        issued += 1;
                    }
                    if (i == 0) {
                        if (lane == 0) __hip_atomic_store(ctl + 416 + n, (unsigned)issued + (plain ? 6u : 5u) + head, RLX, SC_WG);
                        if (n == 14 && !sk && lane == 0)
                            __hip_atomic_store((GAS u64*)(myws + WS_MBOX + (size_t)(bid + 8) * 128), ((u64)call << 32) | ((tnext + 11u * TAU - 7u * tr) & ~255u) | trb, RLX, SC_AGENT);
                    }
                    if (i == 2 && (n & 3) == 3) look();
                }
            }
            wait_vm<7>();
            if (issued - 7 > told && lane == 0) __hip_atomic_store(ctl + 384, (unsigned)issued - 7u, RLX, SC_WG);
            wait_vm<0>();
            if (lane == 0) __hip_atomic_store(ctl + 384, (unsigned)issued, RLX, SC_WG);
            __builtin_amdgcn_s_setprio(0);
        } else {
            if (wid == 1) {
                const GAS u64* fp = lane < 16 ? (const GAS u64*)(myws + WS_ORDY + (size_t)me * 128) + lane : (const GAS u64*)(myws + WS_MBOX + (size_t)me * 128);
                bool pend = lane < 17;
                const u64 t0 = __builtin_amdgcn_s_memrealtime();
                for (unsigned polls = 1;; ++polls) {
                    if (pend) {
                        const u64 f = __hip_atomic_load(fp, RLX, SC_SYS);
                        if ((unsigned)(f >> 32) >= call) { __hip_atomic_store(ctl + lane, (unsigned)f, RLX, SC_WG); pend = false; }
                    }
                    if (!__any(pend)) break;
                    if ((polls & 15u) == 0u && TIMED_OUT(t0)) {
                        if (lane == 0) __hip_atomic_store(tmo, call, RLX, SC_AGENT);
                        if (pend) __hip_atomic_store(ctl + lane, lane < 16 ? 0xc0002000u : (now64() & ~255u) | 128u, RLX, SC_WG);
                        break;
                    }
                    __builtin_amdgcn_s_sleep(4);
                }
            }
            for (;;) {
                unsigned tk = 0;
                if (lane == 0) tk = __hip_atomic_fetch_add(ctl + 401, 1u, RLX, SC_WG);
                const int k = UNI(tk);
                if (k >= 16) break;
                unsigned need;
                while ((need = lds_get(ctl + 416 + k)) == 0u) __builtin_amdgcn_s_sleep(4);
                const unsigned hdr = lds_get(ctl + k);
                while (lds_get(ctl + 384) < need) __builtin_amdgcn_s_sleep(1);
                asm volatile("" ::: "memory");
                const LAS unsigned char* src = lds + (unsigned)k * 8192u;
                unsigned o[32];
                if ((hdr >> 13) & 1u) {
#pragma unroll
                    for (int i = 0; i < 8; ++i) { const u32x4 a = *(const LAS u32x4*)(src + i * 1024 + lo); o[4 * i] = a[0]; o[4 * i + 1] = a[1]; o[4 * i + 2] = a[2]; o[4 * i + 3] = a[3]; }
                } else {
                    unsigned lw[16], hi[16], ne[16];
#pragma unroll
                    for (int i = 0; i < 2; ++i) { const u32x4 a = *(const LAS u32x4*)(src + 4096 + i * 1024 + lo); hi[4 * i] = a[0]; hi[4 * i + 1] = a[1]; hi[4 * i + 2] = a[2]; hi[4 * i + 3] = a[3]; }
                    { const u32x2 a = *(const LAS u32x2*)(src + 6144 + lane * 8); hi[8] = a[0]; hi[9] = a[1]; }
#pragma unroll
                    for (int i = 0; i < 4; ++i) { const u32x4 a = *(const LAS u32x4*)(src + i * 1024 + lo); lw[4 * i] = a[0]; lw[4 * i + 1] = a[1]; lw[4 * i + 2] = a[2]; lw[4 * i + 3] = a[3]; }
#pragma unroll
                    for (int c = 0; c < 6; ++c) {
                        unsigned cc = (hi[9] << (1 + c)) & 0x80008000u;
#pragma unroll
                        for (int i = 0; i < 3; ++i) { const int fld = c * 3 + i, at = 9 + 3 * (fld & 1); cc |= (hi[fld >> 1] >> (at - 3 * i)) & (0x00070007u << (3 * i)); }
                        hi[10 + c] = cc;
                    }
                    u64 str = code_find<7>(hi, ne, (const LAS unsigned*)(lds + 133120 + (k >> 2) * 768) + ((hdr >> 16) & 255u), fq);
                    const unsigned e0 = (hdr & 255u) * 0x00800080u, b2 = (hdr >> 8) & 31u;
#pragma unroll
                    for (int i = 0; i < 16; ++i) {
                        const unsigned h2 = (hi[i] & 0x81ff81ffu) + e0 + (code_get(str, ne[i], b2) << 7);
                        finish(o, i, __uint_as_float(__builtin_amdgcn_perm(h2, lw[i], 0x05040100u)), __uint_as_float(__builtin_amdgcn_perm(h2, lw[i], 0x07060302u)));
                    }
                }
                const int blk = (k >> 3) * 4 + (k & 3);
                const char* ob = (const char*)(p.out + (size_t)((peer & 1) * 256 + (blk >> 1) * 64 + ((k >> 2) & 1) * 32) * GN + (peer >> 1) * 4096 + (cb + sk) * 128 + (blk & 1) * 64);
#pragma unroll
                for (int i = 0; i < 8; ++i)
                    st16(ob + (size_t)((i >> 2) * 16 + (i & 3)) * (GN * 4), oo, (hdr >> 31) ? nan4 : (u32x4){o[4 * i], o[4 * i + 1], o[4 * i + 2], o[4 * i + 3]});
            }
        }
        }
    }
    }
    if (tid == 0) __hip_atomic_store(ctr, call, RLX, SC_AGENT);
}

extern "C" void kernel_launch(int rank, int nranks, ncclComm_t comm, hipStream_t stream,
                              void* const* d_in, const int* in_sizes, int n_in,
                              void* d_out, int out_size, const int* in_dtypes, int out_dtype,
                              void* const* d_out_all, void* const* d_ws_all, size_t ws_bytes) {
    (void)comm; (void)in_dtypes; (void)out_dtype; (void)d_out_all;
    if (nranks != NR || (unsigned)rank >= (unsigned)NR || n_in != 2 || ws_bytes < WS_NEED ||
        in_sizes[0] != 4096 * KSH || in_sizes[1] != KSH * GN || out_size != 512 * GN) {
        fprintf(stderr, "kernel_launch: unexpected problem shape\n");
        return;
    }
    Params p;
    p.x = (const float*)d_in[0];
    p.w = (const float*)d_in[1];
    p.out = (float*)d_out;
    p.myws = (char*)d_ws_all[rank];
    for (int i = 0; i < NR; ++i) p.ws[i] = (char*)d_ws_all[i];
    p.rank = rank;
    p.pad = 0;
    int n = 0, np = 0, nd = 0;
    unsigned char deferred[8];
    bool px[8] = {};
    for (int i = 0; i < 128; ++i) p.order[0][i] = p.order[1][i] = 0;
    auto need_x = [&](int c0, int c1) { for (int c = c0; c <= c1; ++c) if (!px[c]) { px[c] = true; p.prod[np++] = (unsigned)(32 + c); } };
    for (int key = 0; key < 16 + 3 * SKEW2; ++key) {
        for (int pass = 0; pass < 2; ++pass) {
            if ((pass == 0) == (key != 0)) {
                for (int e = 32; e < 44; ++e)
                    if (2 * ((e - 32) * 8 / 12) == key) { p.order[0][n++] = (unsigned char)e; need_x((e - 32) * 8 / 12, ((e - 32) * 8 + 7) / 12); }
                if (key == XPOS) { p.order[0][n++] = 44; need_x(0, 3); }
                if (key == XPOS + 2) { p.order[0][n++] = 45; need_x(4, 7); }
                if (key == LPOS) { p.order[0][n++] = 46; p.order[0][n++] = 47; }
            } else {
                for (int b = 0; b < 4; ++b)
                    for (int c = 0; c < 8; ++c)
                        if ((b == 0 && c == 1 ? 0 : b == 3 && c < 2 * NFWD ? 2 * c + RKEY : 2 * c + b * SKEW2) == key) {
                            p.prod[np++] = (unsigned)(c * 4 + b);
                            if (b == 3 && c < 2 * NFWD && ((c ^ rank) & 1)) deferred[nd++] = (unsigned char)(c * 4 + b);
                            else p.order[0][n++] = (unsigned char)(c * 4 + b);
                        }
            }
        }
    }
    int m = 0;
    for (int i = 0; i < n; ++i) if (p.order[0][i] < 44) p.order[1][m++] = p.order[0][i];
    for (int i = 0; i < nd; ++i) p.order[1][m++] = deferred[i];
    for (int i = 0; i < 6 * NFWD; ++i) p.order[1][m++] = (unsigned char)(48 + i);
    for (int e = 44; e < 48; ++e) p.order[1][m++] = (unsigned char)e;
    if (n != NU_OUT || m != NU_PAIR || np != 40 || p.prod[0] != 0 || p.prod[1] != 4 || p.prod[2] != 32) { fprintf(stderr, "kernel_launch: bad schedule\n"); return; }
    hipLaunchKernelGGL(fused_gemm_rs, dim3(NBLK), dim3(NTHR), 0, stream, p);
    HIP_CHECK(hipGetLastError());
}
```
